```python
import numpy as np

kernel_name = "gpu_a2a_gemm_m4096_k4096_n32768_f32_gelu"


ABSMAX_RTOL = 0.001

COMPUTE_DTYPE = "float32"
EPILOGUE = "gelu"
IN_QUANT = None
OUT_QUANT = None
IN_DTYPES = {"x": "float32", "w_mat": "float32"}
WORLD_SIZE = 8

def setup_inputs():

    x = np.random.randn(4096, 4096).astype(np.float32)
    w_mat = (np.random.randn(4096, 32768) / np.sqrt(4096)).astype(np.float32)
    return {"x": x, "w_mat": w_mat}

def _epilogue(y):
    c = 0.7978845608028654
    return (0.5 * y * (1.0 + np.tanh(c * (y + 0.044715 * y ** 3)))).astype(np.float32)

def reference(x, w_mat):
    y = (x @ w_mat).astype(np.float32)
    return _epilogue(y)
```

```cpp
#include <hip/hip_runtime.h>
#include <rccl/rccl.h>
#include <cstdio>
#include <cstdint>

#define HIP_CHECK(expr) do { hipError_t _e = (expr); if (_e != hipSuccess) { \
    fprintf(stderr, "HIP error %s at %s:%d\n", hipGetErrorString(_e), __FILE__, __LINE__); \
    return; } } while (0)

constexpr int NR = 8;
constexpr int MB = 512;
constexpr int KT = 4096;
constexpr int NT = 32768;
constexpr int KB = 512;
constexpr int BN = 128;
constexpr int NWG = NT / BN;
constexpr int NTHR = 256;

constexpr size_t WS_TICKET = 0;
constexpr size_t WS_TMO    = 128 * 1024;
constexpr size_t WS_FLAGS  = 132 * 1024;
constexpr size_t WS_A      = 1024 * 1024;
constexpr size_t A_SLOT    = (size_t)MB * KT * 2;
constexpr size_t WS_NEED   = WS_A + 2 * A_SLOT;

struct Params {
    const float* x;
    const float* w;
    float* out;
    char* ws[NR];
    int rank;
};

__global__ __launch_bounds__(NTHR, 1) void fused_a2a_gemm(Params p) {
    extern __shared__ __attribute__((aligned(16))) char lds[];
    const int tid = threadIdx.x, lane = tid & 63;
    const int wv = __builtin_amdgcn_readfirstlane(tid >> 6);
    const int g = blockIdx.x;
    const int rank = p.rank;
    char* wsl = p.ws[rank];
    const __amdgpu_buffer_rsrc_t rw = __builtin_amdgcn_make_buffer_rsrc((void*)p.w, 0, (int)((size_t)KT * NT * 4), 0x00020000);
    const __amdgpu_buffer_rsrc_t rc = __builtin_amdgcn_make_buffer_rsrc((void*)wsl, 0, (int)WS_NEED, 0x00020000);
    const __amdgpu_buffer_rsrc_t rs = __builtin_amdgcn_make_buffer_rsrc((void*)p.x, 0, (int)((size_t)NR * MB * KB * 4), 0x00020000);
    const __amdgpu_buffer_rsrc_t ro = __builtin_amdgcn_make_buffer_rsrc((void*)p.out, 0, (int)((size_t)MB * NT * 4), 0x00020000);
    const unsigned long long rx0 = (unsigned long long)(wsl + WS_A);
    const int n4 = lane & 31, kg = 2 * wv + (lane >> 5);
    const int li = lane & 15, lq = lane >> 4;
    const unsigned lb = (unsigned)(size_t)lds;
    constexpr unsigned CMUL = 3;
    auto chunk = [&](unsigned q) { return (CMUL * (4u * g + q)) & 1023u; };
    const int vwoff = kg * 8 * (NT * 4) + chunk(n4 >> 3) * 128 + (n4 & 7) * 16;
    const int vxoff = wv * 8192 + lane * 16;
    const int vlw = lb + n4 * 128 + ((kg ^ (n4 & 7)) * 16);
    const int vlr0 = lb + li * 128 + (((0 + lq) ^ (li & 7)) * 16);
    const int vlr1 = lb + li * 128 + (((4 + lq) ^ (li & 7)) * 16);
    const int pkk = (g >> 2) & 15, pmb = (g >> 6) * 8 + (g & 3) * 2 + (wv >> 1), phalf = wv & 1;
    const int ppc = lane & 1, pr8 = (lane >> 1) & 7;
    const int vps = ((pmb * 16 + phalf * 8 + pr8) * KB + pkk * 32 + lq * 8 + ppc * 4) * 4;
    int vpd = (int)WS_A + ((rank * 16 + pkk) * 32 + pmb) * 1024 + lq * 256 + phalf * 128 + pr8 * 16 + ppc * 8;
    const int vpf = (int)WS_FLAGS + rank * 1024 + g * 4;
    const int vst = lb + wv * 512 + lane * 8;
    int vtk = (int)WS_TICKET + (g * 4 + wv) * 128;
    const int vfl = (int)WS_FLAGS + wv * 256 + lane * 4;
    const int vout0 = (wv * 128 + lq * 4) * NT * 4 + chunk(0 + (li >> 3)) * 128 + (li & 7) * 16;
    const int vout1 = (wv * 128 + lq * 4) * NT * 4 + chunk(2 + (li >> 3)) * 128 + (li & 7) * 16;
    int sd[7]; unsigned long long pd[7];
#pragma unroll
    for (int k = 0; k < 7; ++k) {
        const int i = 1 + (k + g) % 7;
        sd[k] = (rank - i) & 7;
        pd[k] = (unsigned long long)p.ws[sd[k]];
    }
    asm volatile(
        "s_mov_b64 s[56:57], exec\n"
        "s_mov_b64 exec, 1\n"
        "v_mov_b32 v243, 1\n"
        "buffer_atomic_add v243, %[vtk], %[rc], 0 offen sc0\n"
        "s_mov_b64 exec, s[56:57]\n"
        "s_lshl_b32 s44, %[srank], 20\n"
        "buffer_load_dwordx4 v[0:3], %[vps], %[rs], s44 offen\n"
        "s_mov_b32 s41, 0\n"
        "s_lshl_b32 s43, %[srank], 26\n"
        "s_lshl_b32 s42, %[srank], 19\n"
        "s_add_u32 s58, s42, 0x8000\n"
        "s_add_u32 s88, s42, 0x1000\n"
        "s_add_u32 s89, s58, 0x1000\n"
        "v_mov_b32 v241, 0x20000\n"
        "v_accvgpr_write_b32 a0, 0\n"
        "v_accvgpr_write_b32 a1, 0\n"
        "v_accvgpr_write_b32 a2, 0\n"
        "v_accvgpr_write_b32 a3, 0\n"
        "v_accvgpr_write_b32 a4, 0\n"
        "v_accvgpr_write_b32 a5, 0\n"
        "v_accvgpr_write_b32 a6, 0\n"
        "v_accvgpr_write_b32 a7, 0\n"
        "v_accvgpr_write_b32 a8, 0\n"
        "v_accvgpr_write_b32 a9, 0\n"
        "v_accvgpr_write_b32 a10, 0\n"
        "v_accvgpr_write_b32 a11, 0\n"
        "v_accvgpr_write_b32 a12, 0\n"
        "v_accvgpr_write_b32 a13, 0\n"
        "v_accvgpr_write_b32 a14, 0\n"
        "v_accvgpr_write_b32 a15, 0\n"
        "v_accvgpr_write_b32 a16, 0\n"
        "v_accvgpr_write_b32 a17, 0\n"
        "v_accvgpr_write_b32 a18, 0\n"
        "v_accvgpr_write_b32 a19, 0\n"
        "v_accvgpr_write_b32 a20, 0\n"
        "v_accvgpr_write_b32 a21, 0\n"
        "v_accvgpr_write_b32 a22, 0\n"
        "v_accvgpr_write_b32 a23, 0\n"
        "v_accvgpr_write_b32 a24, 0\n"
        "v_accvgpr_write_b32 a25, 0\n"
        "v_accvgpr_write_b32 a26, 0\n"
        "v_accvgpr_write_b32 a27, 0\n"
        "v_accvgpr_write_b32 a28, 0\n"
        "v_accvgpr_write_b32 a29, 0\n"
        "v_accvgpr_write_b32 a30, 0\n"
        "v_accvgpr_write_b32 a31, 0\n"
        "v_accvgpr_write_b32 a32, 0\n"
        "v_accvgpr_write_b32 a33, 0\n"
        "v_accvgpr_write_b32 a34, 0\n"
        "v_accvgpr_write_b32 a35, 0\n"
        "v_accvgpr_write_b32 a36, 0\n"
        "v_accvgpr_write_b32 a37, 0\n"
        "v_accvgpr_write_b32 a38, 0\n"
        "v_accvgpr_write_b32 a39, 0\n"
        "v_accvgpr_write_b32 a40, 0\n"
        "v_accvgpr_write_b32 a41, 0\n"
        "v_accvgpr_write_b32 a42, 0\n"
        "v_accvgpr_write_b32 a43, 0\n"
        "v_accvgpr_write_b32 a44, 0\n"
        "v_accvgpr_write_b32 a45, 0\n"
        "v_accvgpr_write_b32 a46, 0\n"
        "v_accvgpr_write_b32 a47, 0\n"
        "v_accvgpr_write_b32 a48, 0\n"
        "v_accvgpr_write_b32 a49, 0\n"
        "v_accvgpr_write_b32 a50, 0\n"
        "v_accvgpr_write_b32 a51, 0\n"
        "v_accvgpr_write_b32 a52, 0\n"
        "v_accvgpr_write_b32 a53, 0\n"
        "v_accvgpr_write_b32 a54, 0\n"
        "v_accvgpr_write_b32 a55, 0\n"
        "v_accvgpr_write_b32 a56, 0\n"
        "v_accvgpr_write_b32 a57, 0\n"
        "v_accvgpr_write_b32 a58, 0\n"
        "v_accvgpr_write_b32 a59, 0\n"
        "v_accvgpr_write_b32 a60, 0\n"
        "v_accvgpr_write_b32 a61, 0\n"
        "v_accvgpr_write_b32 a62, 0\n"
        "v_accvgpr_write_b32 a63, 0\n"
        "v_accvgpr_write_b32 a64, 0\n"
        "v_accvgpr_write_b32 a65, 0\n"
        "v_accvgpr_write_b32 a66, 0\n"
        "v_accvgpr_write_b32 a67, 0\n"
        "v_accvgpr_write_b32 a68, 0\n"
        "v_accvgpr_write_b32 a69, 0\n"
        "v_accvgpr_write_b32 a70, 0\n"
        "v_accvgpr_write_b32 a71, 0\n"
        "v_accvgpr_write_b32 a72, 0\n"
        "v_accvgpr_write_b32 a73, 0\n"
        "v_accvgpr_write_b32 a74, 0\n"
        "v_accvgpr_write_b32 a75, 0\n"
        "v_accvgpr_write_b32 a76, 0\n"
        "v_accvgpr_write_b32 a77, 0\n"
        "v_accvgpr_write_b32 a78, 0\n"
        "v_accvgpr_write_b32 a79, 0\n"
        "v_accvgpr_write_b32 a80, 0\n"
        "v_accvgpr_write_b32 a81, 0\n"
        "v_accvgpr_write_b32 a82, 0\n"
        "v_accvgpr_write_b32 a83, 0\n"
        "v_accvgpr_write_b32 a84, 0\n"
        "v_accvgpr_write_b32 a85, 0\n"
        "v_accvgpr_write_b32 a86, 0\n"
        "v_accvgpr_write_b32 a87, 0\n"
        "v_accvgpr_write_b32 a88, 0\n"
        "v_accvgpr_write_b32 a89, 0\n"
        "v_accvgpr_write_b32 a90, 0\n"
        "v_accvgpr_write_b32 a91, 0\n"
        "v_accvgpr_write_b32 a92, 0\n"
        "v_accvgpr_write_b32 a93, 0\n"
        "v_accvgpr_write_b32 a94, 0\n"
        "v_accvgpr_write_b32 a95, 0\n"
        "v_accvgpr_write_b32 a96, 0\n"
        "v_accvgpr_write_b32 a97, 0\n"
        "v_accvgpr_write_b32 a98, 0\n"
        "v_accvgpr_write_b32 a99, 0\n"
        "v_accvgpr_write_b32 a100, 0\n"
        "v_accvgpr_write_b32 a101, 0\n"
        "v_accvgpr_write_b32 a102, 0\n"
        "v_accvgpr_write_b32 a103, 0\n"
        "v_accvgpr_write_b32 a104, 0\n"
        "v_accvgpr_write_b32 a105, 0\n"
        "v_accvgpr_write_b32 a106, 0\n"
        "v_accvgpr_write_b32 a107, 0\n"
        "v_accvgpr_write_b32 a108, 0\n"
        "v_accvgpr_write_b32 a109, 0\n"
        "v_accvgpr_write_b32 a110, 0\n"
        "v_accvgpr_write_b32 a111, 0\n"
        "v_accvgpr_write_b32 a112, 0\n"
        "v_accvgpr_write_b32 a113, 0\n"
        "v_accvgpr_write_b32 a114, 0\n"
        "v_accvgpr_write_b32 a115, 0\n"
        "v_accvgpr_write_b32 a116, 0\n"
        "v_accvgpr_write_b32 a117, 0\n"
        "v_accvgpr_write_b32 a118, 0\n"
        "v_accvgpr_write_b32 a119, 0\n"
        "v_accvgpr_write_b32 a120, 0\n"
        "v_accvgpr_write_b32 a121, 0\n"
        "v_accvgpr_write_b32 a122, 0\n"
        "v_accvgpr_write_b32 a123, 0\n"
        "v_accvgpr_write_b32 a124, 0\n"
        "v_accvgpr_write_b32 a125, 0\n"
        "v_accvgpr_write_b32 a126, 0\n"
        "v_accvgpr_write_b32 a127, 0\n"
        "s_waitcnt vmcnt(0)\n"
        "v_readfirstlane_b32 s40, v243\n"
        "v_cvt_pk_f16_f32 v32, v0, v1\n"
        "v_cvt_pk_f16_f32 v33, v2, v3\n"
        "s_nop 1\n"
        "s_add_u32 s40, s40, 1\n"
        "s_and_b32 s44, s40, 1\n"
        "s_lshl_b32 s59, s44, 22\n"
        "v_add_u32 %[vpd], s59, %[vpd]\n"
        "buffer_store_dwordx2 v[32:33], %[vpd], %[rc], 0 offen sc1\n"
        "s_mov_b64 s[52:53], %[rx0]\n"
        "s_add_u32 s52, s52, s59\n"
        "s_addc_u32 s53, s53, 0\n"
        "s_mov_b32 s54, 0x400000\n"
        "s_mov_b32 s55, 0x20000\n"
        "v_mov_b32 %[vtk], s40\n"
        "v_accvgpr_write_b32 a128, 0\n"
        "v_accvgpr_write_b32 a129, 0\n"
        "v_accvgpr_write_b32 a130, 0\n"
        "v_accvgpr_write_b32 a131, 0\n"
        "v_accvgpr_write_b32 a132, 0\n"
        "v_accvgpr_write_b32 a133, 0\n"
        "v_accvgpr_write_b32 a134, 0\n"
        "v_accvgpr_write_b32 a135, 0\n"
        "v_accvgpr_write_b32 a136, 0\n"
        "v_accvgpr_write_b32 a137, 0\n"
        "v_accvgpr_write_b32 a138, 0\n"
        "v_accvgpr_write_b32 a139, 0\n"
        "v_accvgpr_write_b32 a140, 0\n"
        "v_accvgpr_write_b32 a141, 0\n"
        "v_accvgpr_write_b32 a142, 0\n"
        "v_accvgpr_write_b32 a143, 0\n"
        "v_accvgpr_write_b32 a144, 0\n"
        "v_accvgpr_write_b32 a145, 0\n"
        "v_accvgpr_write_b32 a146, 0\n"
        "v_accvgpr_write_b32 a147, 0\n"
        "v_accvgpr_write_b32 a148, 0\n"
        "v_accvgpr_write_b32 a149, 0\n"
        "v_accvgpr_write_b32 a150, 0\n"
        "v_accvgpr_write_b32 a151, 0\n"
        "v_accvgpr_write_b32 a152, 0\n"
        "v_accvgpr_write_b32 a153, 0\n"
        "v_accvgpr_write_b32 a154, 0\n"
        "v_accvgpr_write_b32 a155, 0\n"
        "v_accvgpr_write_b32 a156, 0\n"
        "v_accvgpr_write_b32 a157, 0\n"
        "v_accvgpr_write_b32 a158, 0\n"
        "v_accvgpr_write_b32 a159, 0\n"
        "v_accvgpr_write_b32 a160, 0\n"
        "v_accvgpr_write_b32 a161, 0\n"
        "v_accvgpr_write_b32 a162, 0\n"
        "v_accvgpr_write_b32 a163, 0\n"
        "v_accvgpr_write_b32 a164, 0\n"
        "v_accvgpr_write_b32 a165, 0\n"
        "v_accvgpr_write_b32 a166, 0\n"
        "v_accvgpr_write_b32 a167, 0\n"
        "v_accvgpr_write_b32 a168, 0\n"
        "v_accvgpr_write_b32 a169, 0\n"
        "v_accvgpr_write_b32 a170, 0\n"
        "v_accvgpr_write_b32 a171, 0\n"
        "v_accvgpr_write_b32 a172, 0\n"
        "v_accvgpr_write_b32 a173, 0\n"
        "v_accvgpr_write_b32 a174, 0\n"
        "v_accvgpr_write_b32 a175, 0\n"
        "v_accvgpr_write_b32 a176, 0\n"
        "v_accvgpr_write_b32 a177, 0\n"
        "v_accvgpr_write_b32 a178, 0\n"
        "v_accvgpr_write_b32 a179, 0\n"
        "v_accvgpr_write_b32 a180, 0\n"
        "v_accvgpr_write_b32 a181, 0\n"
        "v_accvgpr_write_b32 a182, 0\n"
        "v_accvgpr_write_b32 a183, 0\n"
        "v_accvgpr_write_b32 a184, 0\n"
        "v_accvgpr_write_b32 a185, 0\n"
        "v_accvgpr_write_b32 a186, 0\n"
        "v_accvgpr_write_b32 a187, 0\n"
        "v_accvgpr_write_b32 a188, 0\n"
        "v_accvgpr_write_b32 a189, 0\n"
        "v_accvgpr_write_b32 a190, 0\n"
        "v_accvgpr_write_b32 a191, 0\n"
        "v_accvgpr_write_b32 a192, 0\n"
        "v_accvgpr_write_b32 a193, 0\n"
        "v_accvgpr_write_b32 a194, 0\n"
        "v_accvgpr_write_b32 a195, 0\n"
        "v_accvgpr_write_b32 a196, 0\n"
        "v_accvgpr_write_b32 a197, 0\n"
        "v_accvgpr_write_b32 a198, 0\n"
        "v_accvgpr_write_b32 a199, 0\n"
        "v_accvgpr_write_b32 a200, 0\n"
        "v_accvgpr_write_b32 a201, 0\n"
        "v_accvgpr_write_b32 a202, 0\n"
        "v_accvgpr_write_b32 a203, 0\n"
        "v_accvgpr_write_b32 a204, 0\n"
        "v_accvgpr_write_b32 a205, 0\n"
        "v_accvgpr_write_b32 a206, 0\n"
        "v_accvgpr_write_b32 a207, 0\n"
        "v_accvgpr_write_b32 a208, 0\n"
        "v_accvgpr_write_b32 a209, 0\n"
        "v_accvgpr_write_b32 a210, 0\n"
        "v_accvgpr_write_b32 a211, 0\n"
        "v_accvgpr_write_b32 a212, 0\n"
        "v_accvgpr_write_b32 a213, 0\n"
        "v_accvgpr_write_b32 a214, 0\n"
        "v_accvgpr_write_b32 a215, 0\n"
        "v_accvgpr_write_b32 a216, 0\n"
        "v_accvgpr_write_b32 a217, 0\n"
        "v_accvgpr_write_b32 a218, 0\n"
        "v_accvgpr_write_b32 a219, 0\n"
        "v_accvgpr_write_b32 a220, 0\n"
        "v_accvgpr_write_b32 a221, 0\n"
        "v_accvgpr_write_b32 a222, 0\n"
        "v_accvgpr_write_b32 a223, 0\n"
        "v_accvgpr_write_b32 a224, 0\n"
        "v_accvgpr_write_b32 a225, 0\n"
        "v_accvgpr_write_b32 a226, 0\n"
        "v_accvgpr_write_b32 a227, 0\n"
        "v_accvgpr_write_b32 a228, 0\n"
        "v_accvgpr_write_b32 a229, 0\n"
        "v_accvgpr_write_b32 a230, 0\n"
        "v_accvgpr_write_b32 a231, 0\n"
        "v_accvgpr_write_b32 a232, 0\n"
        "v_accvgpr_write_b32 a233, 0\n"
        "v_accvgpr_write_b32 a234, 0\n"
        "v_accvgpr_write_b32 a235, 0\n"
        "v_accvgpr_write_b32 a236, 0\n"
        "v_accvgpr_write_b32 a237, 0\n"
        "v_accvgpr_write_b32 a238, 0\n"
        "v_accvgpr_write_b32 a239, 0\n"
        "v_accvgpr_write_b32 a240, 0\n"
        "v_accvgpr_write_b32 a241, 0\n"
        "v_accvgpr_write_b32 a242, 0\n"
        "v_accvgpr_write_b32 a243, 0\n"
        "v_accvgpr_write_b32 a244, 0\n"
        "v_accvgpr_write_b32 a245, 0\n"
        "v_accvgpr_write_b32 a246, 0\n"
        "v_accvgpr_write_b32 a247, 0\n"
        "v_accvgpr_write_b32 a248, 0\n"
        "v_accvgpr_write_b32 a249, 0\n"
        "v_accvgpr_write_b32 a250, 0\n"
        "v_accvgpr_write_b32 a251, 0\n"
        "v_accvgpr_write_b32 a252, 0\n"
        "v_accvgpr_write_b32 a253, 0\n"
        "v_accvgpr_write_b32 a254, 0\n"
        "v_accvgpr_write_b32 a255, 0\n"
        "s_waitcnt vmcnt(0)\n"
        "s_barrier\n"
        "s_cmp_lg_u32 %[swv], 0\n"
        "s_cbranch_scc1 L_nf_own_%=\n"
        "buffer_store_dword %[vtk], %[vpf], %[rc], 0 offen sc1\n"
        "L_nf_own_%=:\n"
        "s_mov_b32 s44, s43\n"
        "buffer_load_dwordx4 v[160:163], %[vwoff], %[rw], s44 offen nt\n"
        "s_add_u32 s44, s44, 0x20000\n"
        "buffer_load_dwordx4 v[164:167], %[vwoff], %[rw], s44 offen nt\n"
        "s_add_u32 s44, s44, 0x20000\n"
        "buffer_load_dwordx4 v[168:171], %[vwoff], %[rw], s44 offen nt\n"
        "s_add_u32 s44, s44, 0x20000\n"
        "buffer_load_dwordx4 v[172:175], %[vwoff], %[rw], s44 offen nt\n"
        "s_add_u32 s44, s44, 0x20000\n"
        "buffer_load_dwordx4 v[176:179], %[vwoff], %[rw], s44 offen nt\n"
        "s_add_u32 s44, s44, 0x20000\n"
        "buffer_load_dwordx4 v[180:183], %[vwoff], %[rw], s44 offen nt\n"
        "s_add_u32 s44, s44, 0x20000\n"
        "buffer_load_dwordx4 v[184:187], %[vwoff], %[rw], s44 offen nt\n"
        "s_add_u32 s44, s44, 0x20000\n"
        "buffer_load_dwordx4 v[188:191], %[vwoff], %[rw], s44 offen nt\n"
        "s_add_u32 s43, s43, 0x800000\n"
        "s_and_b32 s43, s43, 0x1fffffff\n"
        "s_lshl_b32 s44, %[sd0], 20\n"
        "buffer_load_dwordx4 v[4:7], %[vps], %[rs], s44 offen\n"
        "s_lshl_b32 s44, %[sd1], 20\n"
        "buffer_load_dwordx4 v[8:11], %[vps], %[rs], s44 offen\n"
        "s_lshl_b32 s44, %[sd2], 20\n"
        "buffer_load_dwordx4 v[12:15], %[vps], %[rs], s44 offen\n"
        "s_lshl_b32 s44, %[sd3], 20\n"
        "buffer_load_dwordx4 v[16:19], %[vps], %[rs], s44 offen\n"
        "s_lshl_b32 s44, %[sd4], 20\n"
        "buffer_load_dwordx4 v[20:23], %[vps], %[rs], s44 offen\n"
        "s_lshl_b32 s44, %[sd5], 20\n"
        "buffer_load_dwordx4 v[24:27], %[vps], %[rs], s44 offen\n"
        "s_lshl_b32 s44, %[sd6], 20\n"
        "buffer_load_dwordx4 v[28:31], %[vps], %[rs], s44 offen\n"
        "s_mov_b32 s44, s43\n"
        "buffer_load_dwordx4 v[192:195], %[vwoff], %[rw], s44 offen nt\n"
        "s_add_u32 s44, s44, 0x20000\n"
        "buffer_load_dwordx4 v[196:199], %[vwoff], %[rw], s44 offen nt\n"
        "s_add_u32 s44, s44, 0x20000\n"
        "buffer_load_dwordx4 v[200:203], %[vwoff], %[rw], s44 offen nt\n"
        "s_add_u32 s44, s44, 0x20000\n"
        "buffer_load_dwordx4 v[204:207], %[vwoff], %[rw], s44 offen nt\n"
        "s_add_u32 s44, s44, 0x20000\n"
        "buffer_load_dwordx4 v[208:211], %[vwoff], %[rw], s44 offen nt\n"
        "s_add_u32 s44, s44, 0x20000\n"
        "buffer_load_dwordx4 v[212:215], %[vwoff], %[rw], s44 offen nt\n"
        "s_add_u32 s44, s44, 0x20000\n"
        "buffer_load_dwordx4 v[216:219], %[vwoff], %[rw], s44 offen nt\n"
        "s_add_u32 s44, s44, 0x20000\n"
        "buffer_load_dwordx4 v[220:223], %[vwoff], %[rw], s44 offen nt\n"
        "s_add_u32 s43, s43, 0x800000\n"
        "s_and_b32 s43, s43, 0x1fffffff\n"
        "s_waitcnt vmcnt(8)\n"
        "v_cvt_pk_f16_f32 v34, v4, v5\n"
        "v_cvt_pk_f16_f32 v35, v6, v7\n"
        "v_cvt_pk_f16_f32 v36, v8, v9\n"
        "v_cvt_pk_f16_f32 v37, v10, v11\n"
        "v_cvt_pk_f16_f32 v38, v12, v13\n"
        "v_cvt_pk_f16_f32 v39, v14, v15\n"
        "v_cvt_pk_f16_f32 v40, v16, v17\n"
        "v_cvt_pk_f16_f32 v41, v18, v19\n"
        "v_cvt_pk_f16_f32 v42, v20, v21\n"
        "v_cvt_pk_f16_f32 v43, v22, v23\n"
        "v_cvt_pk_f16_f32 v44, v24, v25\n"
        "v_cvt_pk_f16_f32 v45, v26, v27\n"
        "v_cvt_pk_f16_f32 v46, v28, v29\n"
        "v_cvt_pk_f16_f32 v47, v30, v31\n"
        "ds_write_b64 %[vst], v[34:35] offset:32768\n"
        "ds_write_b64 %[vst], v[36:37] offset:34816\n"
        "ds_write_b64 %[vst], v[38:39] offset:36864\n"
        "ds_write_b64 %[vst], v[40:41] offset:38912\n"
        "ds_write_b64 %[vst], v[42:43] offset:40960\n"
        "ds_write_b64 %[vst], v[44:45] offset:43008\n"
        "ds_write_b64 %[vst], v[46:47] offset:45056\n"
        "v_cvt_pk_f16_f32 v224, v160, v164\n"
        "v_cvt_pk_f16_f32 v228, v161, v165\n"
        "v_cvt_pk_f16_f32 v232, v162, v166\n"
        "v_cvt_pk_f16_f32 v236, v163, v167\n"
        "v_cvt_pk_f16_f32 v225, v168, v172\n"
        "v_cvt_pk_f16_f32 v229, v169, v173\n"
        "v_cvt_pk_f16_f32 v233, v170, v174\n"
        "v_cvt_pk_f16_f32 v237, v171, v175\n"
        "v_cvt_pk_f16_f32 v226, v176, v180\n"
        "v_cvt_pk_f16_f32 v230, v177, v181\n"
        "v_cvt_pk_f16_f32 v234, v178, v182\n"
        "v_cvt_pk_f16_f32 v238, v179, v183\n"
        "v_cvt_pk_f16_f32 v227, v184, v188\n"
        "v_cvt_pk_f16_f32 v231, v185, v189\n"
        "v_cvt_pk_f16_f32 v235, v186, v190\n"
        "v_cvt_pk_f16_f32 v239, v187, v191\n"
        "ds_write_b128 %[vlw], v[224:227] offset:0\n"
        "ds_write_b128 %[vlw], v[228:231] offset:4096\n"
        "ds_write_b128 %[vlw], v[232:235] offset:8192\n"
        "ds_write_b128 %[vlw], v[236:239] offset:12288\n"
        "s_lshl_b32 s46, %[srank], 10\n"
        "s_cmp_lg_u32 s41, 0\n"
        "s_cbranch_scc1 L_ok_p_%=\n"
        "s_memrealtime s[48:49]\n"
        "L_spin_p_%=:\n"
        "buffer_load_dword v240, %[vfl], %[rc], s46 offen sc1\n"
        "buffer_load_dword v243, v241, %[rc], 0 offen sc1\n"
        "s_waitcnt vmcnt(0)\n"
        "v_cmp_gt_u32 vcc, s40, v240\n"
        "s_cbranch_vccz L_ok_p_%=\n"
        "v_readfirstlane_b32 s47, v243\n"
        "s_memrealtime s[50:51]\n"
        "s_waitcnt lgkmcnt(0)\n"
        "s_cmp_eq_u32 s47, s40\n"
        "s_cbranch_scc1 L_to_p_%=\n"
        "s_sub_u32 s50, s50, s48\n"
        "s_subb_u32 s51, s51, s49\n"
        "s_cmp_lg_u32 s51, 0\n"
        "s_cbranch_scc1 L_to_p_%=\n"
        "s_cmp_gt_u32 s50, 0x2625a00\n"
        "s_cbranch_scc1 L_to_p_%=\n"
        "s_sleep 1\n"
        "s_branch L_spin_p_%=\n"
        "L_to_p_%=:\n"
        "v_mov_b32 v243, s40\n"
        "buffer_store_dword v243, v241, %[rc], 0 offen sc1\n"
        "s_mov_b32 s41, 1\n"
        "s_waitcnt vmcnt(0)\n"
        "L_ok_p_%=:\n"
        "s_waitcnt lgkmcnt(0)\n"
        "s_nop 0\n"
        "s_waitcnt vmcnt(0)\n"
        "ds_read_b64 v[224:225], %[vst] offset:32768\n"
        "ds_read_b64 v[226:227], %[vst] offset:34816\n"
        "ds_read_b64 v[228:229], %[vst] offset:36864\n"
        "buffer_load_dwordx4 v[0:3], %[vxoff], s[52:55], s42 offen offset:0 sc1\n"
        "buffer_load_dwordx4 v[4:7], %[vxoff], s[52:55], s42 offen offset:1024 sc1\n"
        "buffer_load_dwordx4 v[8:11], %[vxoff], s[52:55], s42 offen offset:2048 sc1\n"
        "buffer_load_dwordx4 v[12:15], %[vxoff], s[52:55], s42 offen offset:3072 sc1\n"
        "buffer_load_dwordx4 v[16:19], %[vxoff], s[52:55], s88 offen offset:0 sc1\n"
        "buffer_load_dwordx4 v[20:23], %[vxoff], s[52:55], s88 offen offset:1024 sc1\n"
        "buffer_load_dwordx4 v[24:27], %[vxoff], s[52:55], s88 offen offset:2048 sc1\n"
        "buffer_load_dwordx4 v[28:31], %[vxoff], s[52:55], s88 offen offset:3072 sc1\n"
        "s_add_u32 s42, s42, 0x10000\n"
        "s_and_b32 s42, s42, 0x3fffff\n"
        "s_add_u32 s88, s42, 0x1000\n"
        "buffer_load_dwordx4 v[32:35], %[vxoff], s[52:55], s58 offen offset:0 sc1\n"
        "buffer_load_dwordx4 v[36:39], %[vxoff], s[52:55], s58 offen offset:1024 sc1\n"
        "buffer_load_dwordx4 v[40:43], %[vxoff], s[52:55], s58 offen offset:2048 sc1\n"
        "buffer_load_dwordx4 v[44:47], %[vxoff], s[52:55], s58 offen offset:3072 sc1\n"
        "buffer_load_dwordx4 v[48:51], %[vxoff], s[52:55], s89 offen offset:0 sc1\n"
        "buffer_load_dwordx4 v[52:55], %[vxoff], s[52:55], s89 offen offset:1024 sc1\n"
        "buffer_load_dwordx4 v[56:59], %[vxoff], s[52:55], s89 offen offset:2048 sc1\n"
        "buffer_load_dwordx4 v[60:63], %[vxoff], s[52:55], s89 offen offset:3072 sc1\n"
        "s_add_u32 s58, s58, 0x10000\n"
        "s_and_b32 s58, s58, 0x3fffff\n"
        "s_add_u32 s89, s58, 0x1000\n"
        "s_waitcnt lgkmcnt(0)\n"
        "global_store_dwordx2 %[vpd], v[224:225], %[pd0] sc0 sc1\n"
        "s_mov_b32 s44, s43\n"
        "buffer_load_dwordx4 v[160:163], %[vwoff], %[rw], s44 offen nt\n"
        "s_add_u32 s44, s44, 0x20000\n"
        "buffer_load_dwordx4 v[164:167], %[vwoff], %[rw], s44 offen nt\n"
        "s_add_u32 s44, s44, 0x20000\n"
        "buffer_load_dwordx4 v[168:171], %[vwoff], %[rw], s44 offen nt\n"
        "s_add_u32 s44, s44, 0x20000\n"
        "buffer_load_dwordx4 v[172:175], %[vwoff], %[rw], s44 offen nt\n"
        "s_add_u32 s44, s44, 0x20000\n"
        "buffer_load_dwordx4 v[176:179], %[vwoff], %[rw], s44 offen nt\n"
        "s_add_u32 s44, s44, 0x20000\n"
        "buffer_load_dwordx4 v[180:183], %[vwoff], %[rw], s44 offen nt\n"
        "s_add_u32 s44, s44, 0x20000\n"
        "buffer_load_dwordx4 v[184:187], %[vwoff], %[rw], s44 offen nt\n"
        "s_add_u32 s44, s44, 0x20000\n"
        "buffer_load_dwordx4 v[188:191], %[vwoff], %[rw], s44 offen nt\n"
        "s_add_u32 s43, s43, 0x800000\n"
        "s_and_b32 s43, s43, 0x1fffffff\n"
        "buffer_load_dwordx4 v[64:67], %[vxoff], s[52:55], s42 offen offset:0 sc1\n"
        "buffer_load_dwordx4 v[68:71], %[vxoff], s[52:55], s42 offen offset:1024 sc1\n"
        "buffer_load_dwordx4 v[72:75], %[vxoff], s[52:55], s42 offen offset:2048 sc1\n"
        "buffer_load_dwordx4 v[76:79], %[vxoff], s[52:55], s42 offen offset:3072 sc1\n"
        "buffer_load_dwordx4 v[80:83], %[vxoff], s[52:55], s88 offen offset:0 sc1\n"
        "buffer_load_dwordx4 v[84:87], %[vxoff], s[52:55], s88 offen offset:1024 sc1\n"
        "buffer_load_dwordx4 v[88:91], %[vxoff], s[52:55], s88 offen offset:2048 sc1\n"
        "buffer_load_dwordx4 v[92:95], %[vxoff], s[52:55], s88 offen offset:3072 sc1\n"
        "s_add_u32 s42, s42, 0x10000\n"
        "s_and_b32 s42, s42, 0x3fffff\n"
        "s_add_u32 s88, s42, 0x1000\n"
        "global_store_dwordx2 %[vpd], v[226:227], %[pd1] sc0 sc1\n"
        "buffer_load_dwordx4 v[96:99], %[vxoff], s[52:55], s58 offen offset:0 sc1\n"
        "buffer_load_dwordx4 v[100:103], %[vxoff], s[52:55], s58 offen offset:1024 sc1\n"
        "buffer_load_dwordx4 v[104:107], %[vxoff], s[52:55], s58 offen offset:2048 sc1\n"
        "buffer_load_dwordx4 v[108:111], %[vxoff], s[52:55], s58 offen offset:3072 sc1\n"
        "buffer_load_dwordx4 v[112:115], %[vxoff], s[52:55], s89 offen offset:0 sc1\n"
        "buffer_load_dwordx4 v[116:119], %[vxoff], s[52:55], s89 offen offset:1024 sc1\n"
        "global_store_dwordx2 %[vpd], v[228:229], %[pd2] sc0 sc1\n"
        "s_waitcnt lgkmcnt(0)\n"
        "s_barrier\n"
        "ds_read_b128 v[128:131], %[vlr0] offset:0\n"
        "ds_read_b128 v[132:135], %[vlr0] offset:2048\n"
        "ds_read_b128 v[136:139], %[vlr0] offset:4096\n"
        "ds_read_b128 v[140:143], %[vlr0] offset:6144\n"
        "s_mov_b32 s45, 0\n"
        "; ==== main\n"
        "s_and_b32 s47, s45, 3\n"
        "s_cmp_lg_u32 s47, 3\n"
        "s_cbranch_scc1 L_nc_i0_%=\n"
        "s_waitcnt vmcnt(48)\n"
        "v_cmp_gt_u32 vcc, s40, v240\n"
        "s_cbranch_vccnz L_slow_i0_%=\n"
        "s_nop 0\n"
        "s_branch L_nc_i0_%=\n"
        "L_slow_i0_%=:\n"
        "s_cmp_lg_u32 s41, 0\n"
        "s_cbranch_scc1 L_ok_i0_%=\n"
        "s_memrealtime s[48:49]\n"
        "L_spin_i0_%=:\n"
        "buffer_load_dword v240, %[vfl], %[rc], s46 offen sc0 sc1\n"
        "buffer_load_dword v243, v241, %[rc], 0 offen sc1\n"
        "s_waitcnt vmcnt(0)\n"
        "v_cmp_gt_u32 vcc, s40, v240\n"
        "s_cbranch_vccz L_ok_i0_%=\n"
        "v_readfirstlane_b32 s47, v243\n"
        "s_memrealtime s[50:51]\n"
        "s_waitcnt lgkmcnt(0)\n"
        "s_cmp_eq_u32 s47, s40\n"
        "s_cbranch_scc1 L_to_i0_%=\n"
        "s_sub_u32 s50, s50, s48\n"
        "s_subb_u32 s51, s51, s49\n"
        "s_cmp_lg_u32 s51, 0\n"
        "s_cbranch_scc1 L_to_i0_%=\n"
        "s_cmp_gt_u32 s50, 0x2625a00\n"
        "s_cbranch_scc1 L_to_i0_%=\n"
        "s_sleep 8\n"
        "s_branch L_spin_i0_%=\n"
        "L_to_i0_%=:\n"
        "v_mov_b32 v243, s40\n"
        "buffer_store_dword v243, v241, %[rc], 0 offen sc1\n"
        "s_mov_b32 s41, 1\n"
        "s_waitcnt vmcnt(0)\n"
        "L_ok_i0_%=:\n"
        "s_waitcnt lgkmcnt(0)\n"
        "s_nop 0\n"
        "L_nc_i0_%=:\n"
        "s_and_b32 s47, s45, 3\n"
        "s_cmp_lg_u32 s47, 2\n"
        "s_cbranch_scc1 L_np_i0_%=\n"
        "s_add_u32 s46, s46, 0x400\n"
        "s_and_b32 s46, s46, 0x1fff\n"
        "buffer_load_dword v240, %[vfl], %[rc], s46 offen sc0 sc1\n"
        "L_np_i0_%=:\n"
        "; ---- kstep\n"
        "s_waitcnt lgkmcnt(0)\n"
        "s_waitcnt vmcnt(39)\n"
        "v_mfma_f32_16x16x32_f16 a[0:3], v[0:3], v[128:131], a[0:3]\n"
        "ds_read_b128 v[144:147], %[vlr0] offset:8192\n"
        "v_mfma_f32_16x16x32_f16 a[4:7], v[0:3], v[132:135], a[4:7]\n"
        "ds_read_b128 v[148:151], %[vlr0] offset:10240\n"
        "v_mfma_f32_16x16x32_f16 a[8:11], v[0:3], v[136:139], a[8:11]\n"
        "ds_read_b128 v[152:155], %[vlr0] offset:12288\n"
        "v_mfma_f32_16x16x32_f16 a[12:15], v[0:3], v[140:143], a[12:15]\n"
        "ds_read_b128 v[156:159], %[vlr0] offset:14336\n"
        "v_mfma_f32_16x16x32_f16 a[32:35], v[4:7], v[128:131], a[32:35]\n"
        "buffer_load_dwordx4 v[120:123], %[vxoff], s[52:55], s89 offen offset:2048 sc1\n"
        "v_mfma_f32_16x16x32_f16 a[36:39], v[4:7], v[132:135], a[36:39]\n"
        "v_cvt_pk_f16_f32 v224, v192, v196\n"
        "v_mfma_f32_16x16x32_f16 a[40:43], v[4:7], v[136:139], a[40:43]\n"
        "v_cvt_pk_f16_f32 v228, v193, v197\n"
        "v_mfma_f32_16x16x32_f16 a[44:47], v[4:7], v[140:143], a[44:47]\n"
        "v_cvt_pk_f16_f32 v232, v194, v198\n"
        "s_waitcnt vmcnt(38)\n"
        "v_mfma_f32_16x16x32_f16 a[64:67], v[8:11], v[128:131], a[64:67]\n"
        "v_cvt_pk_f16_f32 v236, v195, v199\n"
        "v_mfma_f32_16x16x32_f16 a[68:71], v[8:11], v[132:135], a[68:71]\n"
        "buffer_load_dwordx4 v[124:127], %[vxoff], s[52:55], s89 offen offset:3072 sc1\n"
        "v_cvt_pk_f16_f32 v225, v200, v204\n"
        "v_mfma_f32_16x16x32_f16 a[72:75], v[8:11], v[136:139], a[72:75]\n"
        "s_add_u32 s58, s58, 0x10000\n"
        "s_and_b32 s58, s58, 0x3fffff\n"
        "s_add_u32 s89, s58, 0x1000\n"
        "v_cvt_pk_f16_f32 v229, v201, v205\n"
        "v_mfma_f32_16x16x32_f16 a[76:79], v[8:11], v[140:143], a[76:79]\n"
        "v_cvt_pk_f16_f32 v233, v202, v206\n"
        "v_mfma_f32_16x16x32_f16 a[96:99], v[12:15], v[128:131], a[96:99]\n"
        "v_cvt_pk_f16_f32 v237, v203, v207\n"
        "v_mfma_f32_16x16x32_f16 a[100:103], v[12:15], v[132:135], a[100:103]\n"
        "v_mfma_f32_16x16x32_f16 a[104:107], v[12:15], v[136:139], a[104:107]\n"
        "s_mov_b32 s44, s43\n"
        "buffer_load_dwordx4 v[192:195], %[vwoff], %[rw], s44 offen nt\n"
        "v_mfma_f32_16x16x32_f16 a[108:111], v[12:15], v[140:143], a[108:111]\n"
        "s_waitcnt vmcnt(38)\n"
        "v_mfma_f32_16x16x32_f16 a[128:131], v[16:19], v[128:131], a[128:131]\n"
        "v_mfma_f32_16x16x32_f16 a[132:135], v[16:19], v[132:135], a[132:135]\n"
        "v_mfma_f32_16x16x32_f16 a[136:139], v[16:19], v[136:139], a[136:139]\n"
        "v_mfma_f32_16x16x32_f16 a[140:143], v[16:19], v[140:143], a[140:143]\n"
        "s_add_u32 s44, s44, 0x20000\n"
        "buffer_load_dwordx4 v[196:199], %[vwoff], %[rw], s44 offen nt\n"
        "v_mfma_f32_16x16x32_f16 a[160:163], v[20:23], v[128:131], a[160:163]\n"
        "v_mfma_f32_16x16x32_f16 a[164:167], v[20:23], v[132:135], a[164:167]\n"
        "v_mfma_f32_16x16x32_f16 a[168:171], v[20:23], v[136:139], a[168:171]\n"
        "v_mfma_f32_16x16x32_f16 a[172:175], v[20:23], v[140:143], a[172:175]\n"
        "s_waitcnt vmcnt(37)\n"
        "v_mfma_f32_16x16x32_f16 a[192:195], v[24:27], v[128:131], a[192:195]\n"
        "s_add_u32 s44, s44, 0x20000\n"
        "buffer_load_dwordx4 v[200:203], %[vwoff], %[rw], s44 offen nt\n"
        "v_mfma_f32_16x16x32_f16 a[196:199], v[24:27], v[132:135], a[196:199]\n"
        "v_mfma_f32_16x16x32_f16 a[200:203], v[24:27], v[136:139], a[200:203]\n"
        "v_mfma_f32_16x16x32_f16 a[204:207], v[24:27], v[140:143], a[204:207]\n"
        "v_mfma_f32_16x16x32_f16 a[224:227], v[28:31], v[128:131], a[224:227]\n"
        "v_mfma_f32_16x16x32_f16 a[228:231], v[28:31], v[132:135], a[228:231]\n"
        "s_add_u32 s44, s44, 0x20000\n"
        "buffer_load_dwordx4 v[204:207], %[vwoff], %[rw], s44 offen nt\n"
        "v_mfma_f32_16x16x32_f16 a[232:235], v[28:31], v[136:139], a[232:235]\n"
        "v_mfma_f32_16x16x32_f16 a[236:239], v[28:31], v[140:143], a[236:239]\n"
        "s_waitcnt lgkmcnt(0)\n"
        "v_mfma_f32_16x16x32_f16 a[16:19], v[0:3], v[144:147], a[16:19]\n"
        "ds_read_b128 v[128:131], %[vlr1] offset:0\n"
        "v_mfma_f32_16x16x32_f16 a[20:23], v[0:3], v[148:151], a[20:23]\n"
        "ds_read_b128 v[132:135], %[vlr1] offset:2048\n"
        "v_mfma_f32_16x16x32_f16 a[24:27], v[0:3], v[152:155], a[24:27]\n"
        "ds_read_b128 v[136:139], %[vlr1] offset:4096\n"
        "v_mfma_f32_16x16x32_f16 a[28:31], v[0:3], v[156:159], a[28:31]\n"
        "ds_read_b128 v[140:143], %[vlr1] offset:6144\n"
        "v_mfma_f32_16x16x32_f16 a[48:51], v[4:7], v[144:147], a[48:51]\n"
        "v_cvt_pk_f16_f32 v226, v208, v212\n"
        "v_mfma_f32_16x16x32_f16 a[52:55], v[4:7], v[148:151], a[52:55]\n"
        "v_cvt_pk_f16_f32 v230, v209, v213\n"
        "buffer_load_dwordx4 v[0:3], %[vxoff], s[52:55], s42 offen offset:0 sc1\n"
        "v_mfma_f32_16x16x32_f16 a[56:59], v[4:7], v[152:155], a[56:59]\n"
        "v_cvt_pk_f16_f32 v234, v210, v214\n"
        "v_mfma_f32_16x16x32_f16 a[60:63], v[4:7], v[156:159], a[60:63]\n"
        "v_cvt_pk_f16_f32 v238, v211, v215\n"
        "v_mfma_f32_16x16x32_f16 a[80:83], v[8:11], v[144:147], a[80:83]\n"
        "v_cvt_pk_f16_f32 v227, v216, v220\n"
        "v_mfma_f32_16x16x32_f16 a[84:87], v[8:11], v[148:151], a[84:87]\n"
        "v_cvt_pk_f16_f32 v231, v217, v221\n"
        "v_mfma_f32_16x16x32_f16 a[88:91], v[8:11], v[152:155], a[88:91]\n"
        "v_cvt_pk_f16_f32 v235, v218, v222\n"
        "buffer_load_dwordx4 v[4:7], %[vxoff], s[52:55], s42 offen offset:1024 sc1\n"
        "v_mfma_f32_16x16x32_f16 a[92:95], v[8:11], v[156:159], a[92:95]\n"
        "v_cvt_pk_f16_f32 v239, v219, v223\n"
        "v_mfma_f32_16x16x32_f16 a[112:115], v[12:15], v[144:147], a[112:115]\n"
        "v_mfma_f32_16x16x32_f16 a[116:119], v[12:15], v[148:151], a[116:119]\n"
        "v_mfma_f32_16x16x32_f16 a[120:123], v[12:15], v[152:155], a[120:123]\n"
        "s_add_u32 s44, s44, 0x20000\n"
        "buffer_load_dwordx4 v[208:211], %[vwoff], %[rw], s44 offen nt\n"
        "v_mfma_f32_16x16x32_f16 a[124:127], v[12:15], v[156:159], a[124:127]\n"
        "v_mfma_f32_16x16x32_f16 a[144:147], v[16:19], v[144:147], a[144:147]\n"
        "v_mfma_f32_16x16x32_f16 a[148:151], v[16:19], v[148:151], a[148:151]\n"
        "v_mfma_f32_16x16x32_f16 a[152:155], v[16:19], v[152:155], a[152:155]\n"
        "v_mfma_f32_16x16x32_f16 a[156:159], v[16:19], v[156:159], a[156:159]\n"
        "s_add_u32 s44, s44, 0x20000\n"
        "buffer_load_dwordx4 v[212:215], %[vwoff], %[rw], s44 offen nt\n"
        "v_mfma_f32_16x16x32_f16 a[176:179], v[20:23], v[144:147], a[176:179]\n"
        "v_mfma_f32_16x16x32_f16 a[180:183], v[20:23], v[148:151], a[180:183]\n"
        "v_mfma_f32_16x16x32_f16 a[184:187], v[20:23], v[152:155], a[184:187]\n"
        "v_mfma_f32_16x16x32_f16 a[188:191], v[20:23], v[156:159], a[188:191]\n"
        "v_mfma_f32_16x16x32_f16 a[208:211], v[24:27], v[144:147], a[208:211]\n"
        "s_add_u32 s44, s44, 0x20000\n"
        "buffer_load_dwordx4 v[216:219], %[vwoff], %[rw], s44 offen nt\n"
        "v_mfma_f32_16x16x32_f16 a[212:215], v[24:27], v[148:151], a[212:215]\n"
        "v_mfma_f32_16x16x32_f16 a[216:219], v[24:27], v[152:155], a[216:219]\n"
        "v_mfma_f32_16x16x32_f16 a[220:223], v[24:27], v[156:159], a[220:223]\n"
        "v_mfma_f32_16x16x32_f16 a[240:243], v[28:31], v[144:147], a[240:243]\n"
        "v_mfma_f32_16x16x32_f16 a[244:247], v[28:31], v[148:151], a[244:247]\n"
        "s_add_u32 s44, s44, 0x20000\n"
        "buffer_load_dwordx4 v[220:223], %[vwoff], %[rw], s44 offen nt\n"
        "v_mfma_f32_16x16x32_f16 a[248:251], v[28:31], v[152:155], a[248:251]\n"
        "s_add_u32 s43, s43, 0x800000\n"
        "s_and_b32 s43, s43, 0x1fffffff\n"
        "v_mfma_f32_16x16x32_f16 a[252:255], v[28:31], v[156:159], a[252:255]\n"
        "s_waitcnt lgkmcnt(0)\n"
        "s_waitcnt vmcnt(43)\n"
        "v_mfma_f32_16x16x32_f16 a[0:3], v[32:35], v[128:131], a[0:3]\n"
        "ds_read_b128 v[144:147], %[vlr1] offset:8192\n"
        "v_mfma_f32_16x16x32_f16 a[4:7], v[32:35], v[132:135], a[4:7]\n"
        "ds_read_b128 v[148:151], %[vlr1] offset:10240\n"
        "v_mfma_f32_16x16x32_f16 a[8:11], v[32:35], v[136:139], a[8:11]\n"
        "ds_read_b128 v[152:155], %[vlr1] offset:12288\n"
        "v_mfma_f32_16x16x32_f16 a[12:15], v[32:35], v[140:143], a[12:15]\n"
        "ds_read_b128 v[156:159], %[vlr1] offset:14336\n"
        "buffer_load_dwordx4 v[8:11], %[vxoff], s[52:55], s42 offen offset:2048 sc1\n"
        "v_mfma_f32_16x16x32_f16 a[32:35], v[36:39], v[128:131], a[32:35]\n"
        "ds_write_b128 %[vlw], v[224:227] offset:16384\n"
        "v_mfma_f32_16x16x32_f16 a[36:39], v[36:39], v[132:135], a[36:39]\n"
        "v_mfma_f32_16x16x32_f16 a[40:43], v[36:39], v[136:139], a[40:43]\n"
        "ds_write_b128 %[vlw], v[228:231] offset:20480\n"
        "v_mfma_f32_16x16x32_f16 a[44:47], v[36:39], v[140:143], a[44:47]\n"
        "s_waitcnt vmcnt(42)\n"
        "v_mfma_f32_16x16x32_f16 a[64:67], v[40:43], v[128:131], a[64:67]\n"
        "ds_write_b128 %[vlw], v[232:235] offset:24576\n"
        "buffer_load_dwordx4 v[12:15], %[vxoff], s[52:55], s42 offen offset:3072 sc1\n"
        "v_mfma_f32_16x16x32_f16 a[68:71], v[40:43], v[132:135], a[68:71]\n"
        "v_mfma_f32_16x16x32_f16 a[72:75], v[40:43], v[136:139], a[72:75]\n"
        "ds_write_b128 %[vlw], v[236:239] offset:28672\n"
        "v_mfma_f32_16x16x32_f16 a[76:79], v[40:43], v[140:143], a[76:79]\n"
        "v_mfma_f32_16x16x32_f16 a[96:99], v[44:47], v[128:131], a[96:99]\n"
        "v_mfma_f32_16x16x32_f16 a[100:103], v[44:47], v[132:135], a[100:103]\n"
        "buffer_load_dwordx4 v[16:19], %[vxoff], s[52:55], s88 offen offset:0 sc1\n"
        "v_mfma_f32_16x16x32_f16 a[104:107], v[44:47], v[136:139], a[104:107]\n"
        "v_mfma_f32_16x16x32_f16 a[108:111], v[44:47], v[140:143], a[108:111]\n"
        "s_waitcnt vmcnt(42)\n"
        "v_mfma_f32_16x16x32_f16 a[128:131], v[48:51], v[128:131], a[128:131]\n"
        "v_mfma_f32_16x16x32_f16 a[132:135], v[48:51], v[132:135], a[132:135]\n"
        "v_mfma_f32_16x16x32_f16 a[136:139], v[48:51], v[136:139], a[136:139]\n"
        "buffer_load_dwordx4 v[20:23], %[vxoff], s[52:55], s88 offen offset:1024 sc1\n"
        "v_mfma_f32_16x16x32_f16 a[140:143], v[48:51], v[140:143], a[140:143]\n"
        "v_mfma_f32_16x16x32_f16 a[160:163], v[52:55], v[128:131], a[160:163]\n"
        "v_mfma_f32_16x16x32_f16 a[164:167], v[52:55], v[132:135], a[164:167]\n"
        "v_mfma_f32_16x16x32_f16 a[168:171], v[52:55], v[136:139], a[168:171]\n"
        "v_mfma_f32_16x16x32_f16 a[172:175], v[52:55], v[140:143], a[172:175]\n"
        "buffer_load_dwordx4 v[24:27], %[vxoff], s[52:55], s88 offen offset:2048 sc1\n"
        "s_waitcnt vmcnt(42)\n"
        "v_mfma_f32_16x16x32_f16 a[192:195], v[56:59], v[128:131], a[192:195]\n"
        "v_mfma_f32_16x16x32_f16 a[196:199], v[56:59], v[132:135], a[196:199]\n"
        "v_mfma_f32_16x16x32_f16 a[200:203], v[56:59], v[136:139], a[200:203]\n"
        "v_mfma_f32_16x16x32_f16 a[204:207], v[56:59], v[140:143], a[204:207]\n"
        "v_mfma_f32_16x16x32_f16 a[224:227], v[60:63], v[128:131], a[224:227]\n"
        "buffer_load_dwordx4 v[28:31], %[vxoff], s[52:55], s88 offen offset:3072 sc1\n"
        "v_mfma_f32_16x16x32_f16 a[228:231], v[60:63], v[132:135], a[228:231]\n"
        "s_add_u32 s42, s42, 0x10000\n"
        "s_and_b32 s42, s42, 0x3fffff\n"
        "s_add_u32 s88, s42, 0x1000\n"
        "v_mfma_f32_16x16x32_f16 a[232:235], v[60:63], v[136:139], a[232:235]\n"
        "v_mfma_f32_16x16x32_f16 a[236:239], v[60:63], v[140:143], a[236:239]\n"
        "s_waitcnt lgkmcnt(0)\n"
        "s_barrier\n"
        "v_mfma_f32_16x16x32_f16 a[16:19], v[32:35], v[144:147], a[16:19]\n"
        "ds_read_b128 v[128:131], %[vlr0] offset:16384\n"
        "v_mfma_f32_16x16x32_f16 a[20:23], v[32:35], v[148:151], a[20:23]\n"
        "ds_read_b128 v[132:135], %[vlr0] offset:18432\n"
        "v_mfma_f32_16x16x32_f16 a[24:27], v[32:35], v[152:155], a[24:27]\n"
        "ds_read_b128 v[136:139], %[vlr0] offset:20480\n"
        "v_mfma_f32_16x16x32_f16 a[28:31], v[32:35], v[156:159], a[28:31]\n"
        "ds_read_b128 v[140:143], %[vlr0] offset:22528\n"
        "v_mfma_f32_16x16x32_f16 a[48:51], v[36:39], v[144:147], a[48:51]\n"
        "buffer_load_dwordx4 v[32:35], %[vxoff], s[52:55], s58 offen offset:0 sc1\n"
        "v_mfma_f32_16x16x32_f16 a[52:55], v[36:39], v[148:151], a[52:55]\n"
        "ds_read_b64 v[224:225], %[vst] offset:38912\n"
        "v_mfma_f32_16x16x32_f16 a[56:59], v[36:39], v[152:155], a[56:59]\n"
        "v_mfma_f32_16x16x32_f16 a[60:63], v[36:39], v[156:159], a[60:63]\n"
        "v_mfma_f32_16x16x32_f16 a[80:83], v[40:43], v[144:147], a[80:83]\n"
        "v_mfma_f32_16x16x32_f16 a[84:87], v[40:43], v[148:151], a[84:87]\n"
        "buffer_load_dwordx4 v[36:39], %[vxoff], s[52:55], s58 offen offset:1024 sc1\n"
        "v_mfma_f32_16x16x32_f16 a[88:91], v[40:43], v[152:155], a[88:91]\n"
        "v_mfma_f32_16x16x32_f16 a[92:95], v[40:43], v[156:159], a[92:95]\n"
        "s_waitcnt lgkmcnt(0)\n"
        "v_mfma_f32_16x16x32_f16 a[112:115], v[44:47], v[144:147], a[112:115]\n"
        "global_store_dwordx2 %[vpd], v[224:225], %[pd3] sc0 sc1\n"
        "v_mfma_f32_16x16x32_f16 a[116:119], v[44:47], v[148:151], a[116:119]\n"
        "v_mfma_f32_16x16x32_f16 a[120:123], v[44:47], v[152:155], a[120:123]\n"
        "buffer_load_dwordx4 v[40:43], %[vxoff], s[52:55], s58 offen offset:2048 sc1\n"
        "v_mfma_f32_16x16x32_f16 a[124:127], v[44:47], v[156:159], a[124:127]\n"
        "v_mfma_f32_16x16x32_f16 a[144:147], v[48:51], v[144:147], a[144:147]\n"
        "v_mfma_f32_16x16x32_f16 a[148:151], v[48:51], v[148:151], a[148:151]\n"
        "v_mfma_f32_16x16x32_f16 a[152:155], v[48:51], v[152:155], a[152:155]\n"
        "v_mfma_f32_16x16x32_f16 a[156:159], v[48:51], v[156:159], a[156:159]\n"
        "buffer_load_dwordx4 v[44:47], %[vxoff], s[52:55], s58 offen offset:3072 sc1\n"
        "v_mfma_f32_16x16x32_f16 a[176:179], v[52:55], v[144:147], a[176:179]\n"
        "v_mfma_f32_16x16x32_f16 a[180:183], v[52:55], v[148:151], a[180:183]\n"
        "v_mfma_f32_16x16x32_f16 a[184:187], v[52:55], v[152:155], a[184:187]\n"
        "v_mfma_f32_16x16x32_f16 a[188:191], v[52:55], v[156:159], a[188:191]\n"
        "v_mfma_f32_16x16x32_f16 a[208:211], v[56:59], v[144:147], a[208:211]\n"
        "buffer_load_dwordx4 v[48:51], %[vxoff], s[52:55], s89 offen offset:0 sc1\n"
        "v_mfma_f32_16x16x32_f16 a[212:215], v[56:59], v[148:151], a[212:215]\n"
        "v_mfma_f32_16x16x32_f16 a[216:219], v[56:59], v[152:155], a[216:219]\n"
        "v_mfma_f32_16x16x32_f16 a[220:223], v[56:59], v[156:159], a[220:223]\n"
        "v_mfma_f32_16x16x32_f16 a[240:243], v[60:63], v[144:147], a[240:243]\n"
        "v_mfma_f32_16x16x32_f16 a[244:247], v[60:63], v[148:151], a[244:247]\n"
        "buffer_load_dwordx4 v[52:55], %[vxoff], s[52:55], s89 offen offset:1024 sc1\n"
        "v_mfma_f32_16x16x32_f16 a[248:251], v[60:63], v[152:155], a[248:251]\n"
        "v_mfma_f32_16x16x32_f16 a[252:255], v[60:63], v[156:159], a[252:255]\n"
        "; ---- kstep\n"
        "s_waitcnt vmcnt(39)\n"
        "v_mfma_f32_16x16x32_f16 a[0:3], v[64:67], v[128:131], a[0:3]\n"
        "ds_read_b128 v[144:147], %[vlr0] offset:24576\n"
        "v_mfma_f32_16x16x32_f16 a[4:7], v[64:67], v[132:135], a[4:7]\n"
        "ds_read_b128 v[148:151], %[vlr0] offset:26624\n"
        "v_mfma_f32_16x16x32_f16 a[8:11], v[64:67], v[136:139], a[8:11]\n"
        "ds_read_b128 v[152:155], %[vlr0] offset:28672\n"
        "v_mfma_f32_16x16x32_f16 a[12:15], v[64:67], v[140:143], a[12:15]\n"
        "ds_read_b128 v[156:159], %[vlr0] offset:30720\n"
        "v_mfma_f32_16x16x32_f16 a[32:35], v[68:71], v[128:131], a[32:35]\n"
        "buffer_load_dwordx4 v[56:59], %[vxoff], s[52:55], s89 offen offset:2048 sc1\n"
        "v_mfma_f32_16x16x32_f16 a[36:39], v[68:71], v[132:135], a[36:39]\n"
        "v_cvt_pk_f16_f32 v224, v160, v164\n"
        "v_mfma_f32_16x16x32_f16 a[40:43], v[68:71], v[136:139], a[40:43]\n"
        "v_cvt_pk_f16_f32 v228, v161, v165\n"
        "v_mfma_f32_16x16x32_f16 a[44:47], v[68:71], v[140:143], a[44:47]\n"
        "v_cvt_pk_f16_f32 v232, v162, v166\n"
        "s_waitcnt vmcnt(38)\n"
        "v_mfma_f32_16x16x32_f16 a[64:67], v[72:75], v[128:131], a[64:67]\n"
        "v_cvt_pk_f16_f32 v236, v163, v167\n"
        "v_mfma_f32_16x16x32_f16 a[68:71], v[72:75], v[132:135], a[68:71]\n"
        "buffer_load_dwordx4 v[60:63], %[vxoff], s[52:55], s89 offen offset:3072 sc1\n"
        "v_cvt_pk_f16_f32 v225, v168, v172\n"
        "v_mfma_f32_16x16x32_f16 a[72:75], v[72:75], v[136:139], a[72:75]\n"
        "s_add_u32 s58, s58, 0x10000\n"
        "s_and_b32 s58, s58, 0x3fffff\n"
        "s_add_u32 s89, s58, 0x1000\n"
        "v_cvt_pk_f16_f32 v229, v169, v173\n"
        "v_mfma_f32_16x16x32_f16 a[76:79], v[72:75], v[140:143], a[76:79]\n"
        "v_cvt_pk_f16_f32 v233, v170, v174\n"
        "v_mfma_f32_16x16x32_f16 a[96:99], v[76:79], v[128:131], a[96:99]\n"
        "v_cvt_pk_f16_f32 v237, v171, v175\n"
        "v_mfma_f32_16x16x32_f16 a[100:103], v[76:79], v[132:135], a[100:103]\n"
        "v_mfma_f32_16x16x32_f16 a[104:107], v[76:79], v[136:139], a[104:107]\n"
        "s_mov_b32 s44, s43\n"
        "buffer_load_dwordx4 v[160:163], %[vwoff], %[rw], s44 offen nt\n"
        "v_mfma_f32_16x16x32_f16 a[108:111], v[76:79], v[140:143], a[108:111]\n"
        "s_waitcnt vmcnt(38)\n"
        "v_mfma_f32_16x16x32_f16 a[128:131], v[80:83], v[128:131], a[128:131]\n"
        "v_mfma_f32_16x16x32_f16 a[132:135], v[80:83], v[132:135], a[132:135]\n"
        "v_mfma_f32_16x16x32_f16 a[136:139], v[80:83], v[136:139], a[136:139]\n"
        "v_mfma_f32_16x16x32_f16 a[140:143], v[80:83], v[140:143], a[140:143]\n"
        "s_add_u32 s44, s44, 0x20000\n"
        "buffer_load_dwordx4 v[164:167], %[vwoff], %[rw], s44 offen nt\n"
        "v_mfma_f32_16x16x32_f16 a[160:163], v[84:87], v[128:131], a[160:163]\n"
        "v_mfma_f32_16x16x32_f16 a[164:167], v[84:87], v[132:135], a[164:167]\n"
        "v_mfma_f32_16x16x32_f16 a[168:171], v[84:87], v[136:139], a[168:171]\n"
        "v_mfma_f32_16x16x32_f16 a[172:175], v[84:87], v[140:143], a[172:175]\n"
        "s_waitcnt vmcnt(37)\n"
        "v_mfma_f32_16x16x32_f16 a[192:195], v[88:91], v[128:131], a[192:195]\n"
        "s_add_u32 s44, s44, 0x20000\n"
        "buffer_load_dwordx4 v[168:171], %[vwoff], %[rw], s44 offen nt\n"
        "v_mfma_f32_16x16x32_f16 a[196:199], v[88:91], v[132:135], a[196:199]\n"
        "v_mfma_f32_16x16x32_f16 a[200:203], v[88:91], v[136:139], a[200:203]\n"
        "v_mfma_f32_16x16x32_f16 a[204:207], v[88:91], v[140:143], a[204:207]\n"
        "v_mfma_f32_16x16x32_f16 a[224:227], v[92:95], v[128:131], a[224:227]\n"
        "v_mfma_f32_16x16x32_f16 a[228:231], v[92:95], v[132:135], a[228:231]\n"
        "s_add_u32 s44, s44, 0x20000\n"
        "buffer_load_dwordx4 v[172:175], %[vwoff], %[rw], s44 offen nt\n"
        "v_mfma_f32_16x16x32_f16 a[232:235], v[92:95], v[136:139], a[232:235]\n"
        "v_mfma_f32_16x16x32_f16 a[236:239], v[92:95], v[140:143], a[236:239]\n"
        "s_waitcnt lgkmcnt(0)\n"
        "v_mfma_f32_16x16x32_f16 a[16:19], v[64:67], v[144:147], a[16:19]\n"
        "ds_read_b128 v[128:131], %[vlr1] offset:16384\n"
        "v_mfma_f32_16x16x32_f16 a[20:23], v[64:67], v[148:151], a[20:23]\n"
        "ds_read_b128 v[132:135], %[vlr1] offset:18432\n"
        "v_mfma_f32_16x16x32_f16 a[24:27], v[64:67], v[152:155], a[24:27]\n"
        "ds_read_b128 v[136:139], %[vlr1] offset:20480\n"
        "v_mfma_f32_16x16x32_f16 a[28:31], v[64:67], v[156:159], a[28:31]\n"
        "ds_read_b128 v[140:143], %[vlr1] offset:22528\n"
        "v_mfma_f32_16x16x32_f16 a[48:51], v[68:71], v[144:147], a[48:51]\n"
        "v_cvt_pk_f16_f32 v226, v176, v180\n"
        "v_mfma_f32_16x16x32_f16 a[52:55], v[68:71], v[148:151], a[52:55]\n"
        "v_cvt_pk_f16_f32 v230, v177, v181\n"
        "buffer_load_dwordx4 v[64:67], %[vxoff], s[52:55], s42 offen offset:0 sc1\n"
        "v_mfma_f32_16x16x32_f16 a[56:59], v[68:71], v[152:155], a[56:59]\n"
        "v_cvt_pk_f16_f32 v234, v178, v182\n"
        "v_mfma_f32_16x16x32_f16 a[60:63], v[68:71], v[156:159], a[60:63]\n"
        "v_cvt_pk_f16_f32 v238, v179, v183\n"
        "v_mfma_f32_16x16x32_f16 a[80:83], v[72:75], v[144:147], a[80:83]\n"
        "v_cvt_pk_f16_f32 v227, v184, v188\n"
        "v_mfma_f32_16x16x32_f16 a[84:87], v[72:75], v[148:151], a[84:87]\n"
        "v_cvt_pk_f16_f32 v231, v185, v189\n"
        "v_mfma_f32_16x16x32_f16 a[88:91], v[72:75], v[152:155], a[88:91]\n"
        "v_cvt_pk_f16_f32 v235, v186, v190\n"
        "buffer_load_dwordx4 v[68:71], %[vxoff], s[52:55], s42 offen offset:1024 sc1\n"
        "v_mfma_f32_16x16x32_f16 a[92:95], v[72:75], v[156:159], a[92:95]\n"
        "v_cvt_pk_f16_f32 v239, v187, v191\n"
        "v_mfma_f32_16x16x32_f16 a[112:115], v[76:79], v[144:147], a[112:115]\n"
        "v_mfma_f32_16x16x32_f16 a[116:119], v[76:79], v[148:151], a[116:119]\n"
        "v_mfma_f32_16x16x32_f16 a[120:123], v[76:79], v[152:155], a[120:123]\n"
        "s_add_u32 s44, s44, 0x20000\n"
        "buffer_load_dwordx4 v[176:179], %[vwoff], %[rw], s44 offen nt\n"
        "v_mfma_f32_16x16x32_f16 a[124:127], v[76:79], v[156:159], a[124:127]\n"
        "v_mfma_f32_16x16x32_f16 a[144:147], v[80:83], v[144:147], a[144:147]\n"
        "v_mfma_f32_16x16x32_f16 a[148:151], v[80:83], v[148:151], a[148:151]\n"
        "v_mfma_f32_16x16x32_f16 a[152:155], v[80:83], v[152:155], a[152:155]\n"
        "v_mfma_f32_16x16x32_f16 a[156:159], v[80:83], v[156:159], a[156:159]\n"
        "s_add_u32 s44, s44, 0x20000\n"
        "buffer_load_dwordx4 v[180:183], %[vwoff], %[rw], s44 offen nt\n"
        "v_mfma_f32_16x16x32_f16 a[176:179], v[84:87], v[144:147], a[176:179]\n"
        "v_mfma_f32_16x16x32_f16 a[180:183], v[84:87], v[148:151], a[180:183]\n"
        "v_mfma_f32_16x16x32_f16 a[184:187], v[84:87], v[152:155], a[184:187]\n"
        "v_mfma_f32_16x16x32_f16 a[188:191], v[84:87], v[156:159], a[188:191]\n"
        "v_mfma_f32_16x16x32_f16 a[208:211], v[88:91], v[144:147], a[208:211]\n"
        "s_add_u32 s44, s44, 0x20000\n"
        "buffer_load_dwordx4 v[184:187], %[vwoff], %[rw], s44 offen nt\n"
        "v_mfma_f32_16x16x32_f16 a[212:215], v[88:91], v[148:151], a[212:215]\n"
        "v_mfma_f32_16x16x32_f16 a[216:219], v[88:91], v[152:155], a[216:219]\n"
        "v_mfma_f32_16x16x32_f16 a[220:223], v[88:91], v[156:159], a[220:223]\n"
        "v_mfma_f32_16x16x32_f16 a[240:243], v[92:95], v[144:147], a[240:243]\n"
        "v_mfma_f32_16x16x32_f16 a[244:247], v[92:95], v[148:151], a[244:247]\n"
        "s_add_u32 s44, s44, 0x20000\n"
        "buffer_load_dwordx4 v[188:191], %[vwoff], %[rw], s44 offen nt\n"
        "v_mfma_f32_16x16x32_f16 a[248:251], v[92:95], v[152:155], a[248:251]\n"
        "s_add_u32 s43, s43, 0x800000\n"
        "s_and_b32 s43, s43, 0x1fffffff\n"
        "v_mfma_f32_16x16x32_f16 a[252:255], v[92:95], v[156:159], a[252:255]\n"
        "s_waitcnt lgkmcnt(0)\n"
        "s_waitcnt vmcnt(42)\n"
        "v_mfma_f32_16x16x32_f16 a[0:3], v[96:99], v[128:131], a[0:3]\n"
        "ds_read_b128 v[144:147], %[vlr1] offset:24576\n"
        "v_mfma_f32_16x16x32_f16 a[4:7], v[96:99], v[132:135], a[4:7]\n"
        "ds_read_b128 v[148:151], %[vlr1] offset:26624\n"
        "v_mfma_f32_16x16x32_f16 a[8:11], v[96:99], v[136:139], a[8:11]\n"
        "ds_read_b128 v[152:155], %[vlr1] offset:28672\n"
        "v_mfma_f32_16x16x32_f16 a[12:15], v[96:99], v[140:143], a[12:15]\n"
        "ds_read_b128 v[156:159], %[vlr1] offset:30720\n"
        "buffer_load_dwordx4 v[72:75], %[vxoff], s[52:55], s42 offen offset:2048 sc1\n"
        "v_mfma_f32_16x16x32_f16 a[32:35], v[100:103], v[128:131], a[32:35]\n"
        "ds_write_b128 %[vlw], v[224:227] offset:0\n"
        "v_mfma_f32_16x16x32_f16 a[36:39], v[100:103], v[132:135], a[36:39]\n"
        "v_mfma_f32_16x16x32_f16 a[40:43], v[100:103], v[136:139], a[40:43]\n"
        "ds_write_b128 %[vlw], v[228:231] offset:4096\n"
        "v_mfma_f32_16x16x32_f16 a[44:47], v[100:103], v[140:143], a[44:47]\n"
        "s_waitcnt vmcnt(41)\n"
        "v_mfma_f32_16x16x32_f16 a[64:67], v[104:107], v[128:131], a[64:67]\n"
        "ds_write_b128 %[vlw], v[232:235] offset:8192\n"
        "buffer_load_dwordx4 v[76:79], %[vxoff], s[52:55], s42 offen offset:3072 sc1\n"
        "v_mfma_f32_16x16x32_f16 a[68:71], v[104:107], v[132:135], a[68:71]\n"
        "v_mfma_f32_16x16x32_f16 a[72:75], v[104:107], v[136:139], a[72:75]\n"
        "ds_write_b128 %[vlw], v[236:239] offset:12288\n"
        "v_mfma_f32_16x16x32_f16 a[76:79], v[104:107], v[140:143], a[76:79]\n"
        "v_mfma_f32_16x16x32_f16 a[96:99], v[108:111], v[128:131], a[96:99]\n"
        "v_mfma_f32_16x16x32_f16 a[100:103], v[108:111], v[132:135], a[100:103]\n"
        "buffer_load_dwordx4 v[80:83], %[vxoff], s[52:55], s88 offen offset:0 sc1\n"
        "v_mfma_f32_16x16x32_f16 a[104:107], v[108:111], v[136:139], a[104:107]\n"
        "v_mfma_f32_16x16x32_f16 a[108:111], v[108:111], v[140:143], a[108:111]\n"
        "s_waitcnt vmcnt(41)\n"
        "v_mfma_f32_16x16x32_f16 a[128:131], v[112:115], v[128:131], a[128:131]\n"
        "v_mfma_f32_16x16x32_f16 a[132:135], v[112:115], v[132:135], a[132:135]\n"
        "v_mfma_f32_16x16x32_f16 a[136:139], v[112:115], v[136:139], a[136:139]\n"
        "buffer_load_dwordx4 v[84:87], %[vxoff], s[52:55], s88 offen offset:1024 sc1\n"
        "v_mfma_f32_16x16x32_f16 a[140:143], v[112:115], v[140:143], a[140:143]\n"
        "v_mfma_f32_16x16x32_f16 a[160:163], v[116:119], v[128:131], a[160:163]\n"
        "v_mfma_f32_16x16x32_f16 a[164:167], v[116:119], v[132:135], a[164:167]\n"
        "v_mfma_f32_16x16x32_f16 a[168:171], v[116:119], v[136:139], a[168:171]\n"
        "v_mfma_f32_16x16x32_f16 a[172:175], v[116:119], v[140:143], a[172:175]\n"
        "buffer_load_dwordx4 v[88:91], %[vxoff], s[52:55], s88 offen offset:2048 sc1\n"
        "s_waitcnt vmcnt(40)\n"
        "v_mfma_f32_16x16x32_f16 a[192:195], v[120:123], v[128:131], a[192:195]\n"
        "v_mfma_f32_16x16x32_f16 a[196:199], v[120:123], v[132:135], a[196:199]\n"
        "v_mfma_f32_16x16x32_f16 a[200:203], v[120:123], v[136:139], a[200:203]\n"
        "v_mfma_f32_16x16x32_f16 a[204:207], v[120:123], v[140:143], a[204:207]\n"
        "v_mfma_f32_16x16x32_f16 a[224:227], v[124:127], v[128:131], a[224:227]\n"
        "buffer_load_dwordx4 v[92:95], %[vxoff], s[52:55], s88 offen offset:3072 sc1\n"
        "v_mfma_f32_16x16x32_f16 a[228:231], v[124:127], v[132:135], a[228:231]\n"
        "s_add_u32 s42, s42, 0x10000\n"
        "s_and_b32 s42, s42, 0x3fffff\n"
        "s_add_u32 s88, s42, 0x1000\n"
        "v_mfma_f32_16x16x32_f16 a[232:235], v[124:127], v[136:139], a[232:235]\n"
        "v_mfma_f32_16x16x32_f16 a[236:239], v[124:127], v[140:143], a[236:239]\n"
        "s_waitcnt lgkmcnt(0)\n"
        "s_barrier\n"
        "v_mfma_f32_16x16x32_f16 a[16:19], v[96:99], v[144:147], a[16:19]\n"
        "ds_read_b128 v[128:131], %[vlr0] offset:0\n"
        "v_mfma_f32_16x16x32_f16 a[20:23], v[96:99], v[148:151], a[20:23]\n"
        "ds_read_b128 v[132:135], %[vlr0] offset:2048\n"
        "v_mfma_f32_16x16x32_f16 a[24:27], v[96:99], v[152:155], a[24:27]\n"
        "ds_read_b128 v[136:139], %[vlr0] offset:4096\n"
        "v_mfma_f32_16x16x32_f16 a[28:31], v[96:99], v[156:159], a[28:31]\n"
        "ds_read_b128 v[140:143], %[vlr0] offset:6144\n"
        "v_mfma_f32_16x16x32_f16 a[48:51], v[100:103], v[144:147], a[48:51]\n"
        "s_cmp_lg_u32 %[swv], 0\n"
        "s_cbranch_scc1 L_nf_0_%=\n"
        "global_store_dword %[vpf], %[vtk], %[pd0] sc0 sc1\n"
        "L_nf_0_%=:\n"
        "buffer_load_dwordx4 v[96:99], %[vxoff], s[52:55], s58 offen offset:0 sc1\n"
        "v_mfma_f32_16x16x32_f16 a[52:55], v[100:103], v[148:151], a[52:55]\n"
        "s_cmp_lg_u32 %[swv], 0\n"
        "s_cbranch_scc1 L_nf_1_%=\n"
        "global_store_dword %[vpf], %[vtk], %[pd1] sc0 sc1\n"
        "L_nf_1_%=:\n"
        "ds_read_b64 v[224:225], %[vst] offset:40960\n"
        "v_mfma_f32_16x16x32_f16 a[56:59], v[100:103], v[152:155], a[56:59]\n"
        "s_cmp_lg_u32 %[swv], 0\n"
        "s_cbranch_scc1 L_nf_2_%=\n"
        "global_store_dword %[vpf], %[vtk], %[pd2] sc0 sc1\n"
        "L_nf_2_%=:\n"
        "v_mfma_f32_16x16x32_f16 a[60:63], v[100:103], v[156:159], a[60:63]\n"
        "v_mfma_f32_16x16x32_f16 a[80:83], v[104:107], v[144:147], a[80:83]\n"
        "v_mfma_f32_16x16x32_f16 a[84:87], v[104:107], v[148:151], a[84:87]\n"
        "buffer_load_dwordx4 v[100:103], %[vxoff], s[52:55], s58 offen offset:1024 sc1\n"
        "v_mfma_f32_16x16x32_f16 a[88:91], v[104:107], v[152:155], a[88:91]\n"
        "v_mfma_f32_16x16x32_f16 a[92:95], v[104:107], v[156:159], a[92:95]\n"
        "s_waitcnt lgkmcnt(0)\n"
        "v_mfma_f32_16x16x32_f16 a[112:115], v[108:111], v[144:147], a[112:115]\n"
        "global_store_dwordx2 %[vpd], v[224:225], %[pd4] sc0 sc1\n"
        "v_mfma_f32_16x16x32_f16 a[116:119], v[108:111], v[148:151], a[116:119]\n"
        "v_mfma_f32_16x16x32_f16 a[120:123], v[108:111], v[152:155], a[120:123]\n"
        "buffer_load_dwordx4 v[104:107], %[vxoff], s[52:55], s58 offen offset:2048 sc1\n"
        "v_mfma_f32_16x16x32_f16 a[124:127], v[108:111], v[156:159], a[124:127]\n"
        "v_mfma_f32_16x16x32_f16 a[144:147], v[112:115], v[144:147], a[144:147]\n"
        "v_mfma_f32_16x16x32_f16 a[148:151], v[112:115], v[148:151], a[148:151]\n"
        "v_mfma_f32_16x16x32_f16 a[152:155], v[112:115], v[152:155], a[152:155]\n"
        "v_mfma_f32_16x16x32_f16 a[156:159], v[112:115], v[156:159], a[156:159]\n"
        "buffer_load_dwordx4 v[108:111], %[vxoff], s[52:55], s58 offen offset:3072 sc1\n"
        "v_mfma_f32_16x16x32_f16 a[176:179], v[116:119], v[144:147], a[176:179]\n"
        "v_mfma_f32_16x16x32_f16 a[180:183], v[116:119], v[148:151], a[180:183]\n"
        "v_mfma_f32_16x16x32_f16 a[184:187], v[116:119], v[152:155], a[184:187]\n"
        "v_mfma_f32_16x16x32_f16 a[188:191], v[116:119], v[156:159], a[188:191]\n"
        "v_mfma_f32_16x16x32_f16 a[208:211], v[120:123], v[144:147], a[208:211]\n"
        "buffer_load_dwordx4 v[112:115], %[vxoff], s[52:55], s89 offen offset:0 sc1\n"
        "v_mfma_f32_16x16x32_f16 a[212:215], v[120:123], v[148:151], a[212:215]\n"
        "v_mfma_f32_16x16x32_f16 a[216:219], v[120:123], v[152:155], a[216:219]\n"
        "v_mfma_f32_16x16x32_f16 a[220:223], v[120:123], v[156:159], a[220:223]\n"
        "v_mfma_f32_16x16x32_f16 a[240:243], v[124:127], v[144:147], a[240:243]\n"
        "v_mfma_f32_16x16x32_f16 a[244:247], v[124:127], v[148:151], a[244:247]\n"
        "buffer_load_dwordx4 v[116:119], %[vxoff], s[52:55], s89 offen offset:1024 sc1\n"
        "v_mfma_f32_16x16x32_f16 a[248:251], v[124:127], v[152:155], a[248:251]\n"
        "v_mfma_f32_16x16x32_f16 a[252:255], v[124:127], v[156:159], a[252:255]\n"
        "s_add_u32 s45, s45, 1\n"
        "s_and_b32 s47, s45, 3\n"
        "s_cmp_lg_u32 s47, 3\n"
        "s_cbranch_scc1 L_nc_i1_%=\n"
        "s_waitcnt vmcnt(48)\n"
        "v_cmp_gt_u32 vcc, s40, v240\n"
        "s_cbranch_vccnz L_slow_i1_%=\n"
        "s_nop 0\n"
        "s_branch L_nc_i1_%=\n"
        "L_slow_i1_%=:\n"
        "s_cmp_lg_u32 s41, 0\n"
        "s_cbranch_scc1 L_ok_i1_%=\n"
        "s_memrealtime s[48:49]\n"
        "L_spin_i1_%=:\n"
        "buffer_load_dword v240, %[vfl], %[rc], s46 offen sc0 sc1\n"
        "buffer_load_dword v243, v241, %[rc], 0 offen sc1\n"
        "s_waitcnt vmcnt(0)\n"
        "v_cmp_gt_u32 vcc, s40, v240\n"
        "s_cbranch_vccz L_ok_i1_%=\n"
        "v_readfirstlane_b32 s47, v243\n"
        "s_memrealtime s[50:51]\n"
        "s_waitcnt lgkmcnt(0)\n"
        "s_cmp_eq_u32 s47, s40\n"
        "s_cbranch_scc1 L_to_i1_%=\n"
        "s_sub_u32 s50, s50, s48\n"
        "s_subb_u32 s51, s51, s49\n"
        "s_cmp_lg_u32 s51, 0\n"
        "s_cbranch_scc1 L_to_i1_%=\n"
        "s_cmp_gt_u32 s50, 0x2625a00\n"
        "s_cbranch_scc1 L_to_i1_%=\n"
        "s_sleep 8\n"
        "s_branch L_spin_i1_%=\n"
        "L_to_i1_%=:\n"
        "v_mov_b32 v243, s40\n"
        "buffer_store_dword v243, v241, %[rc], 0 offen sc1\n"
        "s_mov_b32 s41, 1\n"
        "s_waitcnt vmcnt(0)\n"
        "L_ok_i1_%=:\n"
        "s_waitcnt lgkmcnt(0)\n"
        "s_nop 0\n"
        "L_nc_i1_%=:\n"
        "s_and_b32 s47, s45, 3\n"
        "s_cmp_lg_u32 s47, 2\n"
        "s_cbranch_scc1 L_np_i1_%=\n"
        "s_add_u32 s46, s46, 0x400\n"
        "s_and_b32 s46, s46, 0x1fff\n"
        "buffer_load_dword v240, %[vfl], %[rc], s46 offen sc0 sc1\n"
        "L_np_i1_%=:\n"
        "; ---- kstep\n"
        "s_waitcnt vmcnt(42)\n"
        "v_mfma_f32_16x16x32_f16 a[0:3], v[0:3], v[128:131], a[0:3]\n"
        "ds_read_b128 v[144:147], %[vlr0] offset:8192\n"
        "v_mfma_f32_16x16x32_f16 a[4:7], v[0:3], v[132:135], a[4:7]\n"
        "ds_read_b128 v[148:151], %[vlr0] offset:10240\n"
        "v_mfma_f32_16x16x32_f16 a[8:11], v[0:3], v[136:139], a[8:11]\n"
        "ds_read_b128 v[152:155], %[vlr0] offset:12288\n"
        "v_mfma_f32_16x16x32_f16 a[12:15], v[0:3], v[140:143], a[12:15]\n"
        "ds_read_b128 v[156:159], %[vlr0] offset:14336\n"
        "v_mfma_f32_16x16x32_f16 a[32:35], v[4:7], v[128:131], a[32:35]\n"
        "buffer_load_dwordx4 v[120:123], %[vxoff], s[52:55], s89 offen offset:2048 sc1\n"
        "v_mfma_f32_16x16x32_f16 a[36:39], v[4:7], v[132:135], a[36:39]\n"
        "v_cvt_pk_f16_f32 v224, v192, v196\n"
        "v_mfma_f32_16x16x32_f16 a[40:43], v[4:7], v[136:139], a[40:43]\n"
        "v_cvt_pk_f16_f32 v228, v193, v197\n"
        "v_mfma_f32_16x16x32_f16 a[44:47], v[4:7], v[140:143], a[44:47]\n"
        "v_cvt_pk_f16_f32 v232, v194, v198\n"
        "s_waitcnt vmcnt(37)\n"
        "v_mfma_f32_16x16x32_f16 a[64:67], v[8:11], v[128:131], a[64:67]\n"
        "v_cvt_pk_f16_f32 v236, v195, v199\n"
        "v_mfma_f32_16x16x32_f16 a[68:71], v[8:11], v[132:135], a[68:71]\n"
        "buffer_load_dwordx4 v[124:127], %[vxoff], s[52:55], s89 offen offset:3072 sc1\n"
        "v_cvt_pk_f16_f32 v225, v200, v204\n"
        "v_mfma_f32_16x16x32_f16 a[72:75], v[8:11], v[136:139], a[72:75]\n"
        "s_add_u32 s58, s58, 0x10000\n"
        "s_and_b32 s58, s58, 0x3fffff\n"
        "s_add_u32 s89, s58, 0x1000\n"
        "v_cvt_pk_f16_f32 v229, v201, v205\n"
        "v_mfma_f32_16x16x32_f16 a[76:79], v[8:11], v[140:143], a[76:79]\n"
        "v_cvt_pk_f16_f32 v233, v202, v206\n"
        "v_mfma_f32_16x16x32_f16 a[96:99], v[12:15], v[128:131], a[96:99]\n"
        "v_cvt_pk_f16_f32 v237, v203, v207\n"
        "v_mfma_f32_16x16x32_f16 a[100:103], v[12:15], v[132:135], a[100:103]\n"
        "v_mfma_f32_16x16x32_f16 a[104:107], v[12:15], v[136:139], a[104:107]\n"
        "s_mov_b32 s44, s43\n"
        "buffer_load_dwordx4 v[192:195], %[vwoff], %[rw], s44 offen nt\n"
        "v_mfma_f32_16x16x32_f16 a[108:111], v[12:15], v[140:143], a[108:111]\n"
        "s_waitcnt vmcnt(37)\n"
        "v_mfma_f32_16x16x32_f16 a[128:131], v[16:19], v[128:131], a[128:131]\n"
        "v_mfma_f32_16x16x32_f16 a[132:135], v[16:19], v[132:135], a[132:135]\n"
        "v_mfma_f32_16x16x32_f16 a[136:139], v[16:19], v[136:139], a[136:139]\n"
        "v_mfma_f32_16x16x32_f16 a[140:143], v[16:19], v[140:143], a[140:143]\n"
        "s_add_u32 s44, s44, 0x20000\n"
        "buffer_load_dwordx4 v[196:199], %[vwoff], %[rw], s44 offen nt\n"
        "v_mfma_f32_16x16x32_f16 a[160:163], v[20:23], v[128:131], a[160:163]\n"
        "v_mfma_f32_16x16x32_f16 a[164:167], v[20:23], v[132:135], a[164:167]\n"
        "v_mfma_f32_16x16x32_f16 a[168:171], v[20:23], v[136:139], a[168:171]\n"
        "v_mfma_f32_16x16x32_f16 a[172:175], v[20:23], v[140:143], a[172:175]\n"
        "s_waitcnt vmcnt(36)\n"
        "v_mfma_f32_16x16x32_f16 a[192:195], v[24:27], v[128:131], a[192:195]\n"
        "s_add_u32 s44, s44, 0x20000\n"
        "buffer_load_dwordx4 v[200:203], %[vwoff], %[rw], s44 offen nt\n"
        "v_mfma_f32_16x16x32_f16 a[196:199], v[24:27], v[132:135], a[196:199]\n"
        "v_mfma_f32_16x16x32_f16 a[200:203], v[24:27], v[136:139], a[200:203]\n"
        "v_mfma_f32_16x16x32_f16 a[204:207], v[24:27], v[140:143], a[204:207]\n"
        "v_mfma_f32_16x16x32_f16 a[224:227], v[28:31], v[128:131], a[224:227]\n"
        "v_mfma_f32_16x16x32_f16 a[228:231], v[28:31], v[132:135], a[228:231]\n"
        "s_add_u32 s44, s44, 0x20000\n"
        "buffer_load_dwordx4 v[204:207], %[vwoff], %[rw], s44 offen nt\n"
        "v_mfma_f32_16x16x32_f16 a[232:235], v[28:31], v[136:139], a[232:235]\n"
        "v_mfma_f32_16x16x32_f16 a[236:239], v[28:31], v[140:143], a[236:239]\n"
        "s_waitcnt lgkmcnt(0)\n"
        "v_mfma_f32_16x16x32_f16 a[16:19], v[0:3], v[144:147], a[16:19]\n"
        "ds_read_b128 v[128:131], %[vlr1] offset:0\n"
        "v_mfma_f32_16x16x32_f16 a[20:23], v[0:3], v[148:151], a[20:23]\n"
        "ds_read_b128 v[132:135], %[vlr1] offset:2048\n"
        "v_mfma_f32_16x16x32_f16 a[24:27], v[0:3], v[152:155], a[24:27]\n"
        "ds_read_b128 v[136:139], %[vlr1] offset:4096\n"
        "v_mfma_f32_16x16x32_f16 a[28:31], v[0:3], v[156:159], a[28:31]\n"
        "ds_read_b128 v[140:143], %[vlr1] offset:6144\n"
        "v_mfma_f32_16x16x32_f16 a[48:51], v[4:7], v[144:147], a[48:51]\n"
        "v_cvt_pk_f16_f32 v226, v208, v212\n"
        "v_mfma_f32_16x16x32_f16 a[52:55], v[4:7], v[148:151], a[52:55]\n"
        "v_cvt_pk_f16_f32 v230, v209, v213\n"
        "buffer_load_dwordx4 v[0:3], %[vxoff], s[52:55], s42 offen offset:0 sc1\n"
        "v_mfma_f32_16x16x32_f16 a[56:59], v[4:7], v[152:155], a[56:59]\n"
        "v_cvt_pk_f16_f32 v234, v210, v214\n"
        "v_mfma_f32_16x16x32_f16 a[60:63], v[4:7], v[156:159], a[60:63]\n"
        "v_cvt_pk_f16_f32 v238, v211, v215\n"
        "v_mfma_f32_16x16x32_f16 a[80:83], v[8:11], v[144:147], a[80:83]\n"
        "v_cvt_pk_f16_f32 v227, v216, v220\n"
        "v_mfma_f32_16x16x32_f16 a[84:87], v[8:11], v[148:151], a[84:87]\n"
        "v_cvt_pk_f16_f32 v231, v217, v221\n"
        "v_mfma_f32_16x16x32_f16 a[88:91], v[8:11], v[152:155], a[88:91]\n"
        "v_cvt_pk_f16_f32 v235, v218, v222\n"
        "buffer_load_dwordx4 v[4:7], %[vxoff], s[52:55], s42 offen offset:1024 sc1\n"
        "v_mfma_f32_16x16x32_f16 a[92:95], v[8:11], v[156:159], a[92:95]\n"
        "v_cvt_pk_f16_f32 v239, v219, v223\n"
        "v_mfma_f32_16x16x32_f16 a[112:115], v[12:15], v[144:147], a[112:115]\n"
        "v_mfma_f32_16x16x32_f16 a[116:119], v[12:15], v[148:151], a[116:119]\n"
        "v_mfma_f32_16x16x32_f16 a[120:123], v[12:15], v[152:155], a[120:123]\n"
        "s_add_u32 s44, s44, 0x20000\n"
        "buffer_load_dwordx4 v[208:211], %[vwoff], %[rw], s44 offen nt\n"
        "v_mfma_f32_16x16x32_f16 a[124:127], v[12:15], v[156:159], a[124:127]\n"
        "v_mfma_f32_16x16x32_f16 a[144:147], v[16:19], v[144:147], a[144:147]\n"
        "v_mfma_f32_16x16x32_f16 a[148:151], v[16:19], v[148:151], a[148:151]\n"
        "v_mfma_f32_16x16x32_f16 a[152:155], v[16:19], v[152:155], a[152:155]\n"
        "v_mfma_f32_16x16x32_f16 a[156:159], v[16:19], v[156:159], a[156:159]\n"
        "s_add_u32 s44, s44, 0x20000\n"
        "buffer_load_dwordx4 v[212:215], %[vwoff], %[rw], s44 offen nt\n"
        "v_mfma_f32_16x16x32_f16 a[176:179], v[20:23], v[144:147], a[176:179]\n"
        "v_mfma_f32_16x16x32_f16 a[180:183], v[20:23], v[148:151], a[180:183]\n"
        "v_mfma_f32_16x16x32_f16 a[184:187], v[20:23], v[152:155], a[184:187]\n"
        "v_mfma_f32_16x16x32_f16 a[188:191], v[20:23], v[156:159], a[188:191]\n"
        "v_mfma_f32_16x16x32_f16 a[208:211], v[24:27], v[144:147], a[208:211]\n"
        "s_add_u32 s44, s44, 0x20000\n"
        "buffer_load_dwordx4 v[216:219], %[vwoff], %[rw], s44 offen nt\n"
        "v_mfma_f32_16x16x32_f16 a[212:215], v[24:27], v[148:151], a[212:215]\n"
        "v_mfma_f32_16x16x32_f16 a[216:219], v[24:27], v[152:155], a[216:219]\n"
        "v_mfma_f32_16x16x32_f16 a[220:223], v[24:27], v[156:159], a[220:223]\n"
        "v_mfma_f32_16x16x32_f16 a[240:243], v[28:31], v[144:147], a[240:243]\n"
        "v_mfma_f32_16x16x32_f16 a[244:247], v[28:31], v[148:151], a[244:247]\n"
        "s_add_u32 s44, s44, 0x20000\n"
        "buffer_load_dwordx4 v[220:223], %[vwoff], %[rw], s44 offen nt\n"
        "v_mfma_f32_16x16x32_f16 a[248:251], v[28:31], v[152:155], a[248:251]\n"
        "s_add_u32 s43, s43, 0x800000\n"
        "s_and_b32 s43, s43, 0x1fffffff\n"
        "v_mfma_f32_16x16x32_f16 a[252:255], v[28:31], v[156:159], a[252:255]\n"
        "s_waitcnt lgkmcnt(0)\n"
        "s_waitcnt vmcnt(42)\n"
        "v_mfma_f32_16x16x32_f16 a[0:3], v[32:35], v[128:131], a[0:3]\n"
        "ds_read_b128 v[144:147], %[vlr1] offset:8192\n"
        "v_mfma_f32_16x16x32_f16 a[4:7], v[32:35], v[132:135], a[4:7]\n"
        "ds_read_b128 v[148:151], %[vlr1] offset:10240\n"
        "v_mfma_f32_16x16x32_f16 a[8:11], v[32:35], v[136:139], a[8:11]\n"
        "ds_read_b128 v[152:155], %[vlr1] offset:12288\n"
        "v_mfma_f32_16x16x32_f16 a[12:15], v[32:35], v[140:143], a[12:15]\n"
        "ds_read_b128 v[156:159], %[vlr1] offset:14336\n"
        "buffer_load_dwordx4 v[8:11], %[vxoff], s[52:55], s42 offen offset:2048 sc1\n"
        "v_mfma_f32_16x16x32_f16 a[32:35], v[36:39], v[128:131], a[32:35]\n"
        "ds_write_b128 %[vlw], v[224:227] offset:16384\n"
        "v_mfma_f32_16x16x32_f16 a[36:39], v[36:39], v[132:135], a[36:39]\n"
        "v_mfma_f32_16x16x32_f16 a[40:43], v[36:39], v[136:139], a[40:43]\n"
        "ds_write_b128 %[vlw], v[228:231] offset:20480\n"
        "v_mfma_f32_16x16x32_f16 a[44:47], v[36:39], v[140:143], a[44:47]\n"
        "s_waitcnt vmcnt(40)\n"
        "v_mfma_f32_16x16x32_f16 a[64:67], v[40:43], v[128:131], a[64:67]\n"
        "ds_write_b128 %[vlw], v[232:235] offset:24576\n"
        "buffer_load_dwordx4 v[12:15], %[vxoff], s[52:55], s42 offen offset:3072 sc1\n"
        "v_mfma_f32_16x16x32_f16 a[68:71], v[40:43], v[132:135], a[68:71]\n"
        "v_mfma_f32_16x16x32_f16 a[72:75], v[40:43], v[136:139], a[72:75]\n"
        "ds_write_b128 %[vlw], v[236:239] offset:28672\n"
        "v_mfma_f32_16x16x32_f16 a[76:79], v[40:43], v[140:143], a[76:79]\n"
        "v_mfma_f32_16x16x32_f16 a[96:99], v[44:47], v[128:131], a[96:99]\n"
        "v_mfma_f32_16x16x32_f16 a[100:103], v[44:47], v[132:135], a[100:103]\n"
        "buffer_load_dwordx4 v[16:19], %[vxoff], s[52:55], s88 offen offset:0 sc1\n"
        "v_mfma_f32_16x16x32_f16 a[104:107], v[44:47], v[136:139], a[104:107]\n"
        "v_mfma_f32_16x16x32_f16 a[108:111], v[44:47], v[140:143], a[108:111]\n"
        "s_waitcnt vmcnt(40)\n"
        "v_mfma_f32_16x16x32_f16 a[128:131], v[48:51], v[128:131], a[128:131]\n"
        "v_mfma_f32_16x16x32_f16 a[132:135], v[48:51], v[132:135], a[132:135]\n"
        "v_mfma_f32_16x16x32_f16 a[136:139], v[48:51], v[136:139], a[136:139]\n"
        "buffer_load_dwordx4 v[20:23], %[vxoff], s[52:55], s88 offen offset:1024 sc1\n"
        "v_mfma_f32_16x16x32_f16 a[140:143], v[48:51], v[140:143], a[140:143]\n"
        "v_mfma_f32_16x16x32_f16 a[160:163], v[52:55], v[128:131], a[160:163]\n"
        "v_mfma_f32_16x16x32_f16 a[164:167], v[52:55], v[132:135], a[164:167]\n"
        "v_mfma_f32_16x16x32_f16 a[168:171], v[52:55], v[136:139], a[168:171]\n"
        "v_mfma_f32_16x16x32_f16 a[172:175], v[52:55], v[140:143], a[172:175]\n"
        "buffer_load_dwordx4 v[24:27], %[vxoff], s[52:55], s88 offen offset:2048 sc1\n"
        "s_waitcnt vmcnt(40)\n"
        "v_mfma_f32_16x16x32_f16 a[192:195], v[56:59], v[128:131], a[192:195]\n"
        "v_mfma_f32_16x16x32_f16 a[196:199], v[56:59], v[132:135], a[196:199]\n"
        "v_mfma_f32_16x16x32_f16 a[200:203], v[56:59], v[136:139], a[200:203]\n"
        "v_mfma_f32_16x16x32_f16 a[204:207], v[56:59], v[140:143], a[204:207]\n"
        "v_mfma_f32_16x16x32_f16 a[224:227], v[60:63], v[128:131], a[224:227]\n"
        "buffer_load_dwordx4 v[28:31], %[vxoff], s[52:55], s88 offen offset:3072 sc1\n"
        "v_mfma_f32_16x16x32_f16 a[228:231], v[60:63], v[132:135], a[228:231]\n"
        "s_add_u32 s42, s42, 0x10000\n"
        "s_and_b32 s42, s42, 0x3fffff\n"
        "s_add_u32 s88, s42, 0x1000\n"
        "v_mfma_f32_16x16x32_f16 a[232:235], v[60:63], v[136:139], a[232:235]\n"
        "v_mfma_f32_16x16x32_f16 a[236:239], v[60:63], v[140:143], a[236:239]\n"
        "s_waitcnt lgkmcnt(0)\n"
        "s_barrier\n"
        "v_mfma_f32_16x16x32_f16 a[16:19], v[32:35], v[144:147], a[16:19]\n"
        "ds_read_b128 v[128:131], %[vlr0] offset:16384\n"
        "v_mfma_f32_16x16x32_f16 a[20:23], v[32:35], v[148:151], a[20:23]\n"
        "ds_read_b128 v[132:135], %[vlr0] offset:18432\n"
        "v_mfma_f32_16x16x32_f16 a[24:27], v[32:35], v[152:155], a[24:27]\n"
        "ds_read_b128 v[136:139], %[vlr0] offset:20480\n"
        "v_mfma_f32_16x16x32_f16 a[28:31], v[32:35], v[156:159], a[28:31]\n"
        "ds_read_b128 v[140:143], %[vlr0] offset:22528\n"
        "v_mfma_f32_16x16x32_f16 a[48:51], v[36:39], v[144:147], a[48:51]\n"
        "buffer_load_dwordx4 v[32:35], %[vxoff], s[52:55], s58 offen offset:0 sc1\n"
        "v_mfma_f32_16x16x32_f16 a[52:55], v[36:39], v[148:151], a[52:55]\n"
        "ds_read_b64 v[224:225], %[vst] offset:43008\n"
        "v_mfma_f32_16x16x32_f16 a[56:59], v[36:39], v[152:155], a[56:59]\n"
        "s_cmp_lg_u32 %[swv], 0\n"
        "s_cbranch_scc1 L_nf_3_%=\n"
        "global_store_dword %[vpf], %[vtk], %[pd3] sc0 sc1\n"
        "L_nf_3_%=:\n"
        "v_mfma_f32_16x16x32_f16 a[60:63], v[36:39], v[156:159], a[60:63]\n"
        "v_mfma_f32_16x16x32_f16 a[80:83], v[40:43], v[144:147], a[80:83]\n"
        "v_mfma_f32_16x16x32_f16 a[84:87], v[40:43], v[148:151], a[84:87]\n"
        "buffer_load_dwordx4 v[36:39], %[vxoff], s[52:55], s58 offen offset:1024 sc1\n"
        "v_mfma_f32_16x16x32_f16 a[88:91], v[40:43], v[152:155], a[88:91]\n"
        "v_mfma_f32_16x16x32_f16 a[92:95], v[40:43], v[156:159], a[92:95]\n"
        "s_waitcnt lgkmcnt(0)\n"
        "v_mfma_f32_16x16x32_f16 a[112:115], v[44:47], v[144:147], a[112:115]\n"
        "global_store_dwordx2 %[vpd], v[224:225], %[pd5] sc0 sc1\n"
        "v_mfma_f32_16x16x32_f16 a[116:119], v[44:47], v[148:151], a[116:119]\n"
        "v_mfma_f32_16x16x32_f16 a[120:123], v[44:47], v[152:155], a[120:123]\n"
        "buffer_load_dwordx4 v[40:43], %[vxoff], s[52:55], s58 offen offset:2048 sc1\n"
        "v_mfma_f32_16x16x32_f16 a[124:127], v[44:47], v[156:159], a[124:127]\n"
        "v_mfma_f32_16x16x32_f16 a[144:147], v[48:51], v[144:147], a[144:147]\n"
        "v_mfma_f32_16x16x32_f16 a[148:151], v[48:51], v[148:151], a[148:151]\n"
        "v_mfma_f32_16x16x32_f16 a[152:155], v[48:51], v[152:155], a[152:155]\n"
        "v_mfma_f32_16x16x32_f16 a[156:159], v[48:51], v[156:159], a[156:159]\n"
        "buffer_load_dwordx4 v[44:47], %[vxoff], s[52:55], s58 offen offset:3072 sc1\n"
        "v_mfma_f32_16x16x32_f16 a[176:179], v[52:55], v[144:147], a[176:179]\n"
        "v_mfma_f32_16x16x32_f16 a[180:183], v[52:55], v[148:151], a[180:183]\n"
        "v_mfma_f32_16x16x32_f16 a[184:187], v[52:55], v[152:155], a[184:187]\n"
        "v_mfma_f32_16x16x32_f16 a[188:191], v[52:55], v[156:159], a[188:191]\n"
        "v_mfma_f32_16x16x32_f16 a[208:211], v[56:59], v[144:147], a[208:211]\n"
        "buffer_load_dwordx4 v[48:51], %[vxoff], s[52:55], s89 offen offset:0 sc1\n"
        "v_mfma_f32_16x16x32_f16 a[212:215], v[56:59], v[148:151], a[212:215]\n"
        "v_mfma_f32_16x16x32_f16 a[216:219], v[56:59], v[152:155], a[216:219]\n"
        "v_mfma_f32_16x16x32_f16 a[220:223], v[56:59], v[156:159], a[220:223]\n"
        "v_mfma_f32_16x16x32_f16 a[240:243], v[60:63], v[144:147], a[240:243]\n"
        "v_mfma_f32_16x16x32_f16 a[244:247], v[60:63], v[148:151], a[244:247]\n"
        "buffer_load_dwordx4 v[52:55], %[vxoff], s[52:55], s89 offen offset:1024 sc1\n"
        "v_mfma_f32_16x16x32_f16 a[248:251], v[60:63], v[152:155], a[248:251]\n"
        "v_mfma_f32_16x16x32_f16 a[252:255], v[60:63], v[156:159], a[252:255]\n"
        "; ---- kstep\n"
        "s_waitcnt vmcnt(42)\n"
        "v_mfma_f32_16x16x32_f16 a[0:3], v[64:67], v[128:131], a[0:3]\n"
        "ds_read_b128 v[144:147], %[vlr0] offset:24576\n"
        "v_mfma_f32_16x16x32_f16 a[4:7], v[64:67], v[132:135], a[4:7]\n"
        "ds_read_b128 v[148:151], %[vlr0] offset:26624\n"
        "v_mfma_f32_16x16x32_f16 a[8:11], v[64:67], v[136:139], a[8:11]\n"
        "ds_read_b128 v[152:155], %[vlr0] offset:28672\n"
        "v_mfma_f32_16x16x32_f16 a[12:15], v[64:67], v[140:143], a[12:15]\n"
        "ds_read_b128 v[156:159], %[vlr0] offset:30720\n"
        "v_mfma_f32_16x16x32_f16 a[32:35], v[68:71], v[128:131], a[32:35]\n"
        "buffer_load_dwordx4 v[56:59], %[vxoff], s[52:55], s89 offen offset:2048 sc1\n"
        "v_mfma_f32_16x16x32_f16 a[36:39], v[68:71], v[132:135], a[36:39]\n"
        "v_cvt_pk_f16_f32 v224, v160, v164\n"
        "v_mfma_f32_16x16x32_f16 a[40:43], v[68:71], v[136:139], a[40:43]\n"
        "v_cvt_pk_f16_f32 v228, v161, v165\n"
        "v_mfma_f32_16x16x32_f16 a[44:47], v[68:71], v[140:143], a[44:47]\n"
        "v_cvt_pk_f16_f32 v232, v162, v166\n"
        "s_waitcnt vmcnt(37)\n"
        "v_mfma_f32_16x16x32_f16 a[64:67], v[72:75], v[128:131], a[64:67]\n"
        "v_cvt_pk_f16_f32 v236, v163, v167\n"
        "v_mfma_f32_16x16x32_f16 a[68:71], v[72:75], v[132:135], a[68:71]\n"
        "buffer_load_dwordx4 v[60:63], %[vxoff], s[52:55], s89 offen offset:3072 sc1\n"
        "v_cvt_pk_f16_f32 v225, v168, v172\n"
        "v_mfma_f32_16x16x32_f16 a[72:75], v[72:75], v[136:139], a[72:75]\n"
        "s_add_u32 s58, s58, 0x10000\n"
        "s_and_b32 s58, s58, 0x3fffff\n"
        "s_add_u32 s89, s58, 0x1000\n"
        "v_cvt_pk_f16_f32 v229, v169, v173\n"
        "v_mfma_f32_16x16x32_f16 a[76:79], v[72:75], v[140:143], a[76:79]\n"
        "v_cvt_pk_f16_f32 v233, v170, v174\n"
        "v_mfma_f32_16x16x32_f16 a[96:99], v[76:79], v[128:131], a[96:99]\n"
        "v_cvt_pk_f16_f32 v237, v171, v175\n"
        "v_mfma_f32_16x16x32_f16 a[100:103], v[76:79], v[132:135], a[100:103]\n"
        "v_mfma_f32_16x16x32_f16 a[104:107], v[76:79], v[136:139], a[104:107]\n"
        "s_mov_b32 s44, s43\n"
        "buffer_load_dwordx4 v[160:163], %[vwoff], %[rw], s44 offen nt\n"
        "v_mfma_f32_16x16x32_f16 a[108:111], v[76:79], v[140:143], a[108:111]\n"
        "s_waitcnt vmcnt(37)\n"
        "v_mfma_f32_16x16x32_f16 a[128:131], v[80:83], v[128:131], a[128:131]\n"
        "v_mfma_f32_16x16x32_f16 a[132:135], v[80:83], v[132:135], a[132:135]\n"
        "v_mfma_f32_16x16x32_f16 a[136:139], v[80:83], v[136:139], a[136:139]\n"
        "v_mfma_f32_16x16x32_f16 a[140:143], v[80:83], v[140:143], a[140:143]\n"
        "s_add_u32 s44, s44, 0x20000\n"
        "buffer_load_dwordx4 v[164:167], %[vwoff], %[rw], s44 offen nt\n"
        "v_mfma_f32_16x16x32_f16 a[160:163], v[84:87], v[128:131], a[160:163]\n"
        "v_mfma_f32_16x16x32_f16 a[164:167], v[84:87], v[132:135], a[164:167]\n"
        "v_mfma_f32_16x16x32_f16 a[168:171], v[84:87], v[136:139], a[168:171]\n"
        "v_mfma_f32_16x16x32_f16 a[172:175], v[84:87], v[140:143], a[172:175]\n"
        "s_waitcnt vmcnt(36)\n"
        "v_mfma_f32_16x16x32_f16 a[192:195], v[88:91], v[128:131], a[192:195]\n"
        "s_add_u32 s44, s44, 0x20000\n"
        "buffer_load_dwordx4 v[168:171], %[vwoff], %[rw], s44 offen nt\n"
        "v_mfma_f32_16x16x32_f16 a[196:199], v[88:91], v[132:135], a[196:199]\n"
        "v_mfma_f32_16x16x32_f16 a[200:203], v[88:91], v[136:139], a[200:203]\n"
        "v_mfma_f32_16x16x32_f16 a[204:207], v[88:91], v[140:143], a[204:207]\n"
        "v_mfma_f32_16x16x32_f16 a[224:227], v[92:95], v[128:131], a[224:227]\n"
        "v_mfma_f32_16x16x32_f16 a[228:231], v[92:95], v[132:135], a[228:231]\n"
        "s_add_u32 s44, s44, 0x20000\n"
        "buffer_load_dwordx4 v[172:175], %[vwoff], %[rw], s44 offen nt\n"
        "v_mfma_f32_16x16x32_f16 a[232:235], v[92:95], v[136:139], a[232:235]\n"
        "v_mfma_f32_16x16x32_f16 a[236:239], v[92:95], v[140:143], a[236:239]\n"
        "s_waitcnt lgkmcnt(0)\n"
        "v_mfma_f32_16x16x32_f16 a[16:19], v[64:67], v[144:147], a[16:19]\n"
        "ds_read_b128 v[128:131], %[vlr1] offset:16384\n"
        "v_mfma_f32_16x16x32_f16 a[20:23], v[64:67], v[148:151], a[20:23]\n"
        "ds_read_b128 v[132:135], %[vlr1] offset:18432\n"
        "v_mfma_f32_16x16x32_f16 a[24:27], v[64:67], v[152:155], a[24:27]\n"
        "ds_read_b128 v[136:139], %[vlr1] offset:20480\n"
        "v_mfma_f32_16x16x32_f16 a[28:31], v[64:67], v[156:159], a[28:31]\n"
        "ds_read_b128 v[140:143], %[vlr1] offset:22528\n"
        "v_mfma_f32_16x16x32_f16 a[48:51], v[68:71], v[144:147], a[48:51]\n"
        "v_cvt_pk_f16_f32 v226, v176, v180\n"
        "v_mfma_f32_16x16x32_f16 a[52:55], v[68:71], v[148:151], a[52:55]\n"
        "v_cvt_pk_f16_f32 v230, v177, v181\n"
        "buffer_load_dwordx4 v[64:67], %[vxoff], s[52:55], s42 offen offset:0 sc1\n"
        "v_mfma_f32_16x16x32_f16 a[56:59], v[68:71], v[152:155], a[56:59]\n"
        "v_cvt_pk_f16_f32 v234, v178, v182\n"
        "v_mfma_f32_16x16x32_f16 a[60:63], v[68:71], v[156:159], a[60:63]\n"
        "v_cvt_pk_f16_f32 v238, v179, v183\n"
        "v_mfma_f32_16x16x32_f16 a[80:83], v[72:75], v[144:147], a[80:83]\n"
        "v_cvt_pk_f16_f32 v227, v184, v188\n"
        "v_mfma_f32_16x16x32_f16 a[84:87], v[72:75], v[148:151], a[84:87]\n"
        "v_cvt_pk_f16_f32 v231, v185, v189\n"
        "v_mfma_f32_16x16x32_f16 a[88:91], v[72:75], v[152:155], a[88:91]\n"
        "v_cvt_pk_f16_f32 v235, v186, v190\n"
        "buffer_load_dwordx4 v[68:71], %[vxoff], s[52:55], s42 offen offset:1024 sc1\n"
        "v_mfma_f32_16x16x32_f16 a[92:95], v[72:75], v[156:159], a[92:95]\n"
        "v_cvt_pk_f16_f32 v239, v187, v191\n"
        "v_mfma_f32_16x16x32_f16 a[112:115], v[76:79], v[144:147], a[112:115]\n"
        "v_mfma_f32_16x16x32_f16 a[116:119], v[76:79], v[148:151], a[116:119]\n"
        "v_mfma_f32_16x16x32_f16 a[120:123], v[76:79], v[152:155], a[120:123]\n"
        "s_add_u32 s44, s44, 0x20000\n"
        "buffer_load_dwordx4 v[176:179], %[vwoff], %[rw], s44 offen nt\n"
        "v_mfma_f32_16x16x32_f16 a[124:127], v[76:79], v[156:159], a[124:127]\n"
        "v_mfma_f32_16x16x32_f16 a[144:147], v[80:83], v[144:147], a[144:147]\n"
        "v_mfma_f32_16x16x32_f16 a[148:151], v[80:83], v[148:151], a[148:151]\n"
        "v_mfma_f32_16x16x32_f16 a[152:155], v[80:83], v[152:155], a[152:155]\n"
        "v_mfma_f32_16x16x32_f16 a[156:159], v[80:83], v[156:159], a[156:159]\n"
        "s_add_u32 s44, s44, 0x20000\n"
        "buffer_load_dwordx4 v[180:183], %[vwoff], %[rw], s44 offen nt\n"
        "v_mfma_f32_16x16x32_f16 a[176:179], v[84:87], v[144:147], a[176:179]\n"
        "v_mfma_f32_16x16x32_f16 a[180:183], v[84:87], v[148:151], a[180:183]\n"
        "v_mfma_f32_16x16x32_f16 a[184:187], v[84:87], v[152:155], a[184:187]\n"
        "v_mfma_f32_16x16x32_f16 a[188:191], v[84:87], v[156:159], a[188:191]\n"
        "v_mfma_f32_16x16x32_f16 a[208:211], v[88:91], v[144:147], a[208:211]\n"
        "s_add_u32 s44, s44, 0x20000\n"
        "buffer_load_dwordx4 v[184:187], %[vwoff], %[rw], s44 offen nt\n"
        "v_mfma_f32_16x16x32_f16 a[212:215], v[88:91], v[148:151], a[212:215]\n"
        "v_mfma_f32_16x16x32_f16 a[216:219], v[88:91], v[152:155], a[216:219]\n"
        "v_mfma_f32_16x16x32_f16 a[220:223], v[88:91], v[156:159], a[220:223]\n"
        "v_mfma_f32_16x16x32_f16 a[240:243], v[92:95], v[144:147], a[240:243]\n"
        "v_mfma_f32_16x16x32_f16 a[244:247], v[92:95], v[148:151], a[244:247]\n"
        "s_add_u32 s44, s44, 0x20000\n"
        "buffer_load_dwordx4 v[188:191], %[vwoff], %[rw], s44 offen nt\n"
        "v_mfma_f32_16x16x32_f16 a[248:251], v[92:95], v[152:155], a[248:251]\n"
        "s_add_u32 s43, s43, 0x800000\n"
        "s_and_b32 s43, s43, 0x1fffffff\n"
        "v_mfma_f32_16x16x32_f16 a[252:255], v[92:95], v[156:159], a[252:255]\n"
        "s_waitcnt lgkmcnt(0)\n"
        "s_waitcnt vmcnt(42)\n"
        "v_mfma_f32_16x16x32_f16 a[0:3], v[96:99], v[128:131], a[0:3]\n"
        "ds_read_b128 v[144:147], %[vlr1] offset:24576\n"
        "v_mfma_f32_16x16x32_f16 a[4:7], v[96:99], v[132:135], a[4:7]\n"
        "ds_read_b128 v[148:151], %[vlr1] offset:26624\n"
        "v_mfma_f32_16x16x32_f16 a[8:11], v[96:99], v[136:139], a[8:11]\n"
        "ds_read_b128 v[152:155], %[vlr1] offset:28672\n"
        "v_mfma_f32_16x16x32_f16 a[12:15], v[96:99], v[140:143], a[12:15]\n"
        "ds_read_b128 v[156:159], %[vlr1] offset:30720\n"
        "buffer_load_dwordx4 v[72:75], %[vxoff], s[52:55], s42 offen offset:2048 sc1\n"
        "v_mfma_f32_16x16x32_f16 a[32:35], v[100:103], v[128:131], a[32:35]\n"
        "ds_write_b128 %[vlw], v[224:227] offset:0\n"
        "v_mfma_f32_16x16x32_f16 a[36:39], v[100:103], v[132:135], a[36:39]\n"
        "v_mfma_f32_16x16x32_f16 a[40:43], v[100:103], v[136:139], a[40:43]\n"
        "ds_write_b128 %[vlw], v[228:231] offset:4096\n"
        "v_mfma_f32_16x16x32_f16 a[44:47], v[100:103], v[140:143], a[44:47]\n"
        "s_waitcnt vmcnt(40)\n"
        "v_mfma_f32_16x16x32_f16 a[64:67], v[104:107], v[128:131], a[64:67]\n"
        "ds_write_b128 %[vlw], v[232:235] offset:8192\n"
        "buffer_load_dwordx4 v[76:79], %[vxoff], s[52:55], s42 offen offset:3072 sc1\n"
        "v_mfma_f32_16x16x32_f16 a[68:71], v[104:107], v[132:135], a[68:71]\n"
        "v_mfma_f32_16x16x32_f16 a[72:75], v[104:107], v[136:139], a[72:75]\n"
        "ds_write_b128 %[vlw], v[236:239] offset:12288\n"
        "v_mfma_f32_16x16x32_f16 a[76:79], v[104:107], v[140:143], a[76:79]\n"
        "v_mfma_f32_16x16x32_f16 a[96:99], v[108:111], v[128:131], a[96:99]\n"
        "v_mfma_f32_16x16x32_f16 a[100:103], v[108:111], v[132:135], a[100:103]\n"
        "buffer_load_dwordx4 v[80:83], %[vxoff], s[52:55], s88 offen offset:0 sc1\n"
        "v_mfma_f32_16x16x32_f16 a[104:107], v[108:111], v[136:139], a[104:107]\n"
        "v_mfma_f32_16x16x32_f16 a[108:111], v[108:111], v[140:143], a[108:111]\n"
        "s_waitcnt vmcnt(40)\n"
        "v_mfma_f32_16x16x32_f16 a[128:131], v[112:115], v[128:131], a[128:131]\n"
        "v_mfma_f32_16x16x32_f16 a[132:135], v[112:115], v[132:135], a[132:135]\n"
        "v_mfma_f32_16x16x32_f16 a[136:139], v[112:115], v[136:139], a[136:139]\n"
        "buffer_load_dwordx4 v[84:87], %[vxoff], s[52:55], s88 offen offset:1024 sc1\n"
        "v_mfma_f32_16x16x32_f16 a[140:143], v[112:115], v[140:143], a[140:143]\n"
        "v_mfma_f32_16x16x32_f16 a[160:163], v[116:119], v[128:131], a[160:163]\n"
        "v_mfma_f32_16x16x32_f16 a[164:167], v[116:119], v[132:135], a[164:167]\n"
        "v_mfma_f32_16x16x32_f16 a[168:171], v[116:119], v[136:139], a[168:171]\n"
        "v_mfma_f32_16x16x32_f16 a[172:175], v[116:119], v[140:143], a[172:175]\n"
        "buffer_load_dwordx4 v[88:91], %[vxoff], s[52:55], s88 offen offset:2048 sc1\n"
        "s_waitcnt vmcnt(40)\n"
        "v_mfma_f32_16x16x32_f16 a[192:195], v[120:123], v[128:131], a[192:195]\n"
        "v_mfma_f32_16x16x32_f16 a[196:199], v[120:123], v[132:135], a[196:199]\n"
        "v_mfma_f32_16x16x32_f16 a[200:203], v[120:123], v[136:139], a[200:203]\n"
        "v_mfma_f32_16x16x32_f16 a[204:207], v[120:123], v[140:143], a[204:207]\n"
        "v_mfma_f32_16x16x32_f16 a[224:227], v[124:127], v[128:131], a[224:227]\n"
        "buffer_load_dwordx4 v[92:95], %[vxoff], s[52:55], s88 offen offset:3072 sc1\n"
        "v_mfma_f32_16x16x32_f16 a[228:231], v[124:127], v[132:135], a[228:231]\n"
        "s_add_u32 s42, s42, 0x10000\n"
        "s_and_b32 s42, s42, 0x3fffff\n"
        "s_add_u32 s88, s42, 0x1000\n"
        "v_mfma_f32_16x16x32_f16 a[232:235], v[124:127], v[136:139], a[232:235]\n"
        "v_mfma_f32_16x16x32_f16 a[236:239], v[124:127], v[140:143], a[236:239]\n"
        "s_waitcnt lgkmcnt(0)\n"
        "s_barrier\n"
        "v_mfma_f32_16x16x32_f16 a[16:19], v[96:99], v[144:147], a[16:19]\n"
        "ds_read_b128 v[128:131], %[vlr0] offset:0\n"
        "v_mfma_f32_16x16x32_f16 a[20:23], v[96:99], v[148:151], a[20:23]\n"
        "ds_read_b128 v[132:135], %[vlr0] offset:2048\n"
        "v_mfma_f32_16x16x32_f16 a[24:27], v[96:99], v[152:155], a[24:27]\n"
        "ds_read_b128 v[136:139], %[vlr0] offset:4096\n"
        "v_mfma_f32_16x16x32_f16 a[28:31], v[96:99], v[156:159], a[28:31]\n"
        "ds_read_b128 v[140:143], %[vlr0] offset:6144\n"
        "v_mfma_f32_16x16x32_f16 a[48:51], v[100:103], v[144:147], a[48:51]\n"
        "buffer_load_dwordx4 v[96:99], %[vxoff], s[52:55], s58 offen offset:0 sc1\n"
        "v_mfma_f32_16x16x32_f16 a[52:55], v[100:103], v[148:151], a[52:55]\n"
        "ds_read_b64 v[224:225], %[vst] offset:45056\n"
        "v_mfma_f32_16x16x32_f16 a[56:59], v[100:103], v[152:155], a[56:59]\n"
        "s_cmp_lg_u32 %[swv], 0\n"
        "s_cbranch_scc1 L_nf_4_%=\n"
        "global_store_dword %[vpf], %[vtk], %[pd4] sc0 sc1\n"
        "L_nf_4_%=:\n"
        "v_mfma_f32_16x16x32_f16 a[60:63], v[100:103], v[156:159], a[60:63]\n"
        "v_mfma_f32_16x16x32_f16 a[80:83], v[104:107], v[144:147], a[80:83]\n"
        "v_mfma_f32_16x16x32_f16 a[84:87], v[104:107], v[148:151], a[84:87]\n"
        "buffer_load_dwordx4 v[100:103], %[vxoff], s[52:55], s58 offen offset:1024 sc1\n"
        "v_mfma_f32_16x16x32_f16 a[88:91], v[104:107], v[152:155], a[88:91]\n"
        "v_mfma_f32_16x16x32_f16 a[92:95], v[104:107], v[156:159], a[92:95]\n"
        "s_waitcnt lgkmcnt(0)\n"
        "v_mfma_f32_16x16x32_f16 a[112:115], v[108:111], v[144:147], a[112:115]\n"
        "global_store_dwordx2 %[vpd], v[224:225], %[pd6] sc0 sc1\n"
        "v_mfma_f32_16x16x32_f16 a[116:119], v[108:111], v[148:151], a[116:119]\n"
        "v_mfma_f32_16x16x32_f16 a[120:123], v[108:111], v[152:155], a[120:123]\n"
        "buffer_load_dwordx4 v[104:107], %[vxoff], s[52:55], s58 offen offset:2048 sc1\n"
        "v_mfma_f32_16x16x32_f16 a[124:127], v[108:111], v[156:159], a[124:127]\n"
        "v_mfma_f32_16x16x32_f16 a[144:147], v[112:115], v[144:147], a[144:147]\n"
        "v_mfma_f32_16x16x32_f16 a[148:151], v[112:115], v[148:151], a[148:151]\n"
        "v_mfma_f32_16x16x32_f16 a[152:155], v[112:115], v[152:155], a[152:155]\n"
        "v_mfma_f32_16x16x32_f16 a[156:159], v[112:115], v[156:159], a[156:159]\n"
        "buffer_load_dwordx4 v[108:111], %[vxoff], s[52:55], s58 offen offset:3072 sc1\n"
        "v_mfma_f32_16x16x32_f16 a[176:179], v[116:119], v[144:147], a[176:179]\n"
        "v_mfma_f32_16x16x32_f16 a[180:183], v[116:119], v[148:151], a[180:183]\n"
        "v_mfma_f32_16x16x32_f16 a[184:187], v[116:119], v[152:155], a[184:187]\n"
        "v_mfma_f32_16x16x32_f16 a[188:191], v[116:119], v[156:159], a[188:191]\n"
        "v_mfma_f32_16x16x32_f16 a[208:211], v[120:123], v[144:147], a[208:211]\n"
        "buffer_load_dwordx4 v[112:115], %[vxoff], s[52:55], s89 offen offset:0 sc1\n"
        "v_mfma_f32_16x16x32_f16 a[212:215], v[120:123], v[148:151], a[212:215]\n"
        "v_mfma_f32_16x16x32_f16 a[216:219], v[120:123], v[152:155], a[216:219]\n"
        "v_mfma_f32_16x16x32_f16 a[220:223], v[120:123], v[156:159], a[220:223]\n"
        "v_mfma_f32_16x16x32_f16 a[240:243], v[124:127], v[144:147], a[240:243]\n"
        "v_mfma_f32_16x16x32_f16 a[244:247], v[124:127], v[148:151], a[244:247]\n"
        "buffer_load_dwordx4 v[116:119], %[vxoff], s[52:55], s89 offen offset:1024 sc1\n"
        "v_mfma_f32_16x16x32_f16 a[248:251], v[124:127], v[152:155], a[248:251]\n"
        "v_mfma_f32_16x16x32_f16 a[252:255], v[124:127], v[156:159], a[252:255]\n"
        "s_add_u32 s45, s45, 1\n"
        "s_and_b32 s47, s45, 3\n"
        "s_cmp_lg_u32 s47, 3\n"
        "s_cbranch_scc1 L_nc_i2_%=\n"
        "s_waitcnt vmcnt(48)\n"
        "v_cmp_gt_u32 vcc, s40, v240\n"
        "s_cbranch_vccnz L_slow_i2_%=\n"
        "s_nop 0\n"
        "s_branch L_nc_i2_%=\n"
        "L_slow_i2_%=:\n"
        "s_cmp_lg_u32 s41, 0\n"
        "s_cbranch_scc1 L_ok_i2_%=\n"
        "s_memrealtime s[48:49]\n"
        "L_spin_i2_%=:\n"
        "buffer_load_dword v240, %[vfl], %[rc], s46 offen sc0 sc1\n"
        "buffer_load_dword v243, v241, %[rc], 0 offen sc1\n"
        "s_waitcnt vmcnt(0)\n"
        "v_cmp_gt_u32 vcc, s40, v240\n"
        "s_cbranch_vccz L_ok_i2_%=\n"
        "v_readfirstlane_b32 s47, v243\n"
        "s_memrealtime s[50:51]\n"
        "s_waitcnt lgkmcnt(0)\n"
        "s_cmp_eq_u32 s47, s40\n"
        "s_cbranch_scc1 L_to_i2_%=\n"
        "s_sub_u32 s50, s50, s48\n"
        "s_subb_u32 s51, s51, s49\n"
        "s_cmp_lg_u32 s51, 0\n"
        "s_cbranch_scc1 L_to_i2_%=\n"
        "s_cmp_gt_u32 s50, 0x2625a00\n"
        "s_cbranch_scc1 L_to_i2_%=\n"
        "s_sleep 8\n"
        "s_branch L_spin_i2_%=\n"
        "L_to_i2_%=:\n"
        "v_mov_b32 v243, s40\n"
        "buffer_store_dword v243, v241, %[rc], 0 offen sc1\n"
        "s_mov_b32 s41, 1\n"
        "s_waitcnt vmcnt(0)\n"
        "L_ok_i2_%=:\n"
        "s_waitcnt lgkmcnt(0)\n"
        "s_nop 0\n"
        "L_nc_i2_%=:\n"
        "s_and_b32 s47, s45, 3\n"
        "s_cmp_lg_u32 s47, 2\n"
        "s_cbranch_scc1 L_np_i2_%=\n"
        "s_add_u32 s46, s46, 0x400\n"
        "s_and_b32 s46, s46, 0x1fff\n"
        "buffer_load_dword v240, %[vfl], %[rc], s46 offen sc0 sc1\n"
        "L_np_i2_%=:\n"
        "; ---- kstep\n"
        "s_waitcnt vmcnt(42)\n"
        "v_mfma_f32_16x16x32_f16 a[0:3], v[0:3], v[128:131], a[0:3]\n"
        "ds_read_b128 v[144:147], %[vlr0] offset:8192\n"
        "v_mfma_f32_16x16x32_f16 a[4:7], v[0:3], v[132:135], a[4:7]\n"
        "ds_read_b128 v[148:151], %[vlr0] offset:10240\n"
        "v_mfma_f32_16x16x32_f16 a[8:11], v[0:3], v[136:139], a[8:11]\n"
        "ds_read_b128 v[152:155], %[vlr0] offset:12288\n"
        "v_mfma_f32_16x16x32_f16 a[12:15], v[0:3], v[140:143], a[12:15]\n"
        "ds_read_b128 v[156:159], %[vlr0] offset:14336\n"
        "v_mfma_f32_16x16x32_f16 a[32:35], v[4:7], v[128:131], a[32:35]\n"
        "buffer_load_dwordx4 v[120:123], %[vxoff], s[52:55], s89 offen offset:2048 sc1\n"
        "v_mfma_f32_16x16x32_f16 a[36:39], v[4:7], v[132:135], a[36:39]\n"
        "v_cvt_pk_f16_f32 v224, v192, v196\n"
        "v_mfma_f32_16x16x32_f16 a[40:43], v[4:7], v[136:139], a[40:43]\n"
        "v_cvt_pk_f16_f32 v228, v193, v197\n"
        "v_mfma_f32_16x16x32_f16 a[44:47], v[4:7], v[140:143], a[44:47]\n"
        "v_cvt_pk_f16_f32 v232, v194, v198\n"
        "s_waitcnt vmcnt(37)\n"
        "v_mfma_f32_16x16x32_f16 a[64:67], v[8:11], v[128:131], a[64:67]\n"
        "v_cvt_pk_f16_f32 v236, v195, v199\n"
        "v_mfma_f32_16x16x32_f16 a[68:71], v[8:11], v[132:135], a[68:71]\n"
        "buffer_load_dwordx4 v[124:127], %[vxoff], s[52:55], s89 offen offset:3072 sc1\n"
        "v_cvt_pk_f16_f32 v225, v200, v204\n"
        "v_mfma_f32_16x16x32_f16 a[72:75], v[8:11], v[136:139], a[72:75]\n"
        "s_add_u32 s58, s58, 0x10000\n"
        "s_and_b32 s58, s58, 0x3fffff\n"
        "s_add_u32 s89, s58, 0x1000\n"
        "v_cvt_pk_f16_f32 v229, v201, v205\n"
        "v_mfma_f32_16x16x32_f16 a[76:79], v[8:11], v[140:143], a[76:79]\n"
        "v_cvt_pk_f16_f32 v233, v202, v206\n"
        "v_mfma_f32_16x16x32_f16 a[96:99], v[12:15], v[128:131], a[96:99]\n"
        "v_cvt_pk_f16_f32 v237, v203, v207\n"
        "v_mfma_f32_16x16x32_f16 a[100:103], v[12:15], v[132:135], a[100:103]\n"
        "v_mfma_f32_16x16x32_f16 a[104:107], v[12:15], v[136:139], a[104:107]\n"
        "s_mov_b32 s44, s43\n"
        "buffer_load_dwordx4 v[192:195], %[vwoff], %[rw], s44 offen nt\n"
        "v_mfma_f32_16x16x32_f16 a[108:111], v[12:15], v[140:143], a[108:111]\n"
        "s_waitcnt vmcnt(37)\n"
        "v_mfma_f32_16x16x32_f16 a[128:131], v[16:19], v[128:131], a[128:131]\n"
        "v_mfma_f32_16x16x32_f16 a[132:135], v[16:19], v[132:135], a[132:135]\n"
        "v_mfma_f32_16x16x32_f16 a[136:139], v[16:19], v[136:139], a[136:139]\n"
        "v_mfma_f32_16x16x32_f16 a[140:143], v[16:19], v[140:143], a[140:143]\n"
        "s_add_u32 s44, s44, 0x20000\n"
        "buffer_load_dwordx4 v[196:199], %[vwoff], %[rw], s44 offen nt\n"
        "v_mfma_f32_16x16x32_f16 a[160:163], v[20:23], v[128:131], a[160:163]\n"
        "v_mfma_f32_16x16x32_f16 a[164:167], v[20:23], v[132:135], a[164:167]\n"
        "v_mfma_f32_16x16x32_f16 a[168:171], v[20:23], v[136:139], a[168:171]\n"
        "v_mfma_f32_16x16x32_f16 a[172:175], v[20:23], v[140:143], a[172:175]\n"
        "s_waitcnt vmcnt(36)\n"
        "v_mfma_f32_16x16x32_f16 a[192:195], v[24:27], v[128:131], a[192:195]\n"
        "s_add_u32 s44, s44, 0x20000\n"
        "buffer_load_dwordx4 v[200:203], %[vwoff], %[rw], s44 offen nt\n"
        "v_mfma_f32_16x16x32_f16 a[196:199], v[24:27], v[132:135], a[196:199]\n"
        "v_mfma_f32_16x16x32_f16 a[200:203], v[24:27], v[136:139], a[200:203]\n"
        "v_mfma_f32_16x16x32_f16 a[204:207], v[24:27], v[140:143], a[204:207]\n"
        "v_mfma_f32_16x16x32_f16 a[224:227], v[28:31], v[128:131], a[224:227]\n"
        "v_mfma_f32_16x16x32_f16 a[228:231], v[28:31], v[132:135], a[228:231]\n"
        "s_add_u32 s44, s44, 0x20000\n"
        "buffer_load_dwordx4 v[204:207], %[vwoff], %[rw], s44 offen nt\n"
        "v_mfma_f32_16x16x32_f16 a[232:235], v[28:31], v[136:139], a[232:235]\n"
        "v_mfma_f32_16x16x32_f16 a[236:239], v[28:31], v[140:143], a[236:239]\n"
        "s_waitcnt lgkmcnt(0)\n"
        "v_mfma_f32_16x16x32_f16 a[16:19], v[0:3], v[144:147], a[16:19]\n"
        "ds_read_b128 v[128:131], %[vlr1] offset:0\n"
        "v_mfma_f32_16x16x32_f16 a[20:23], v[0:3], v[148:151], a[20:23]\n"
        "ds_read_b128 v[132:135], %[vlr1] offset:2048\n"
        "v_mfma_f32_16x16x32_f16 a[24:27], v[0:3], v[152:155], a[24:27]\n"
        "ds_read_b128 v[136:139], %[vlr1] offset:4096\n"
        "v_mfma_f32_16x16x32_f16 a[28:31], v[0:3], v[156:159], a[28:31]\n"
        "ds_read_b128 v[140:143], %[vlr1] offset:6144\n"
        "v_mfma_f32_16x16x32_f16 a[48:51], v[4:7], v[144:147], a[48:51]\n"
        "v_cvt_pk_f16_f32 v226, v208, v212\n"
        "v_mfma_f32_16x16x32_f16 a[52:55], v[4:7], v[148:151], a[52:55]\n"
        "v_cvt_pk_f16_f32 v230, v209, v213\n"
        "buffer_load_dwordx4 v[0:3], %[vxoff], s[52:55], s42 offen offset:0 sc1\n"
        "v_mfma_f32_16x16x32_f16 a[56:59], v[4:7], v[152:155], a[56:59]\n"
        "v_cvt_pk_f16_f32 v234, v210, v214\n"
        "v_mfma_f32_16x16x32_f16 a[60:63], v[4:7], v[156:159], a[60:63]\n"
        "v_cvt_pk_f16_f32 v238, v211, v215\n"
        "v_mfma_f32_16x16x32_f16 a[80:83], v[8:11], v[144:147], a[80:83]\n"
        "v_cvt_pk_f16_f32 v227, v216, v220\n"
        "v_mfma_f32_16x16x32_f16 a[84:87], v[8:11], v[148:151], a[84:87]\n"
        "v_cvt_pk_f16_f32 v231, v217, v221\n"
        "v_mfma_f32_16x16x32_f16 a[88:91], v[8:11], v[152:155], a[88:91]\n"
        "v_cvt_pk_f16_f32 v235, v218, v222\n"
        "buffer_load_dwordx4 v[4:7], %[vxoff], s[52:55], s42 offen offset:1024 sc1\n"
        "v_mfma_f32_16x16x32_f16 a[92:95], v[8:11], v[156:159], a[92:95]\n"
        "v_cvt_pk_f16_f32 v239, v219, v223\n"
        "v_mfma_f32_16x16x32_f16 a[112:115], v[12:15], v[144:147], a[112:115]\n"
        "v_mfma_f32_16x16x32_f16 a[116:119], v[12:15], v[148:151], a[116:119]\n"
        "v_mfma_f32_16x16x32_f16 a[120:123], v[12:15], v[152:155], a[120:123]\n"
        "s_add_u32 s44, s44, 0x20000\n"
        "buffer_load_dwordx4 v[208:211], %[vwoff], %[rw], s44 offen nt\n"
        "v_mfma_f32_16x16x32_f16 a[124:127], v[12:15], v[156:159], a[124:127]\n"
        "v_mfma_f32_16x16x32_f16 a[144:147], v[16:19], v[144:147], a[144:147]\n"
        "v_mfma_f32_16x16x32_f16 a[148:151], v[16:19], v[148:151], a[148:151]\n"
        "v_mfma_f32_16x16x32_f16 a[152:155], v[16:19], v[152:155], a[152:155]\n"
        "v_mfma_f32_16x16x32_f16 a[156:159], v[16:19], v[156:159], a[156:159]\n"
        "s_add_u32 s44, s44, 0x20000\n"
        "buffer_load_dwordx4 v[212:215], %[vwoff], %[rw], s44 offen nt\n"
        "v_mfma_f32_16x16x32_f16 a[176:179], v[20:23], v[144:147], a[176:179]\n"
        "v_mfma_f32_16x16x32_f16 a[180:183], v[20:23], v[148:151], a[180:183]\n"
        "v_mfma_f32_16x16x32_f16 a[184:187], v[20:23], v[152:155], a[184:187]\n"
        "v_mfma_f32_16x16x32_f16 a[188:191], v[20:23], v[156:159], a[188:191]\n"
        "v_mfma_f32_16x16x32_f16 a[208:211], v[24:27], v[144:147], a[208:211]\n"
        "s_add_u32 s44, s44, 0x20000\n"
        "buffer_load_dwordx4 v[216:219], %[vwoff], %[rw], s44 offen nt\n"
        "v_mfma_f32_16x16x32_f16 a[212:215], v[24:27], v[148:151], a[212:215]\n"
        "v_mfma_f32_16x16x32_f16 a[216:219], v[24:27], v[152:155], a[216:219]\n"
        "v_mfma_f32_16x16x32_f16 a[220:223], v[24:27], v[156:159], a[220:223]\n"
        "v_mfma_f32_16x16x32_f16 a[240:243], v[28:31], v[144:147], a[240:243]\n"
        "v_mfma_f32_16x16x32_f16 a[244:247], v[28:31], v[148:151], a[244:247]\n"
        "s_add_u32 s44, s44, 0x20000\n"
        "buffer_load_dwordx4 v[220:223], %[vwoff], %[rw], s44 offen nt\n"
        "v_mfma_f32_16x16x32_f16 a[248:251], v[28:31], v[152:155], a[248:251]\n"
        "s_add_u32 s43, s43, 0x800000\n"
        "s_and_b32 s43, s43, 0x1fffffff\n"
        "v_mfma_f32_16x16x32_f16 a[252:255], v[28:31], v[156:159], a[252:255]\n"
        "s_waitcnt lgkmcnt(0)\n"
        "s_waitcnt vmcnt(42)\n"
        "v_mfma_f32_16x16x32_f16 a[0:3], v[32:35], v[128:131], a[0:3]\n"
        "ds_read_b128 v[144:147], %[vlr1] offset:8192\n"
        "v_mfma_f32_16x16x32_f16 a[4:7], v[32:35], v[132:135], a[4:7]\n"
        "ds_read_b128 v[148:151], %[vlr1] offset:10240\n"
        "v_mfma_f32_16x16x32_f16 a[8:11], v[32:35], v[136:139], a[8:11]\n"
        "ds_read_b128 v[152:155], %[vlr1] offset:12288\n"
        "v_mfma_f32_16x16x32_f16 a[12:15], v[32:35], v[140:143], a[12:15]\n"
        "ds_read_b128 v[156:159], %[vlr1] offset:14336\n"
        "buffer_load_dwordx4 v[8:11], %[vxoff], s[52:55], s42 offen offset:2048 sc1\n"
        "v_mfma_f32_16x16x32_f16 a[32:35], v[36:39], v[128:131], a[32:35]\n"
        "ds_write_b128 %[vlw], v[224:227] offset:16384\n"
        "v_mfma_f32_16x16x32_f16 a[36:39], v[36:39], v[132:135], a[36:39]\n"
        "v_mfma_f32_16x16x32_f16 a[40:43], v[36:39], v[136:139], a[40:43]\n"
        "ds_write_b128 %[vlw], v[228:231] offset:20480\n"
        "v_mfma_f32_16x16x32_f16 a[44:47], v[36:39], v[140:143], a[44:47]\n"
        "s_waitcnt vmcnt(40)\n"
        "v_mfma_f32_16x16x32_f16 a[64:67], v[40:43], v[128:131], a[64:67]\n"
        "ds_write_b128 %[vlw], v[232:235] offset:24576\n"
        "buffer_load_dwordx4 v[12:15], %[vxoff], s[52:55], s42 offen offset:3072 sc1\n"
        "v_mfma_f32_16x16x32_f16 a[68:71], v[40:43], v[132:135], a[68:71]\n"
        "v_mfma_f32_16x16x32_f16 a[72:75], v[40:43], v[136:139], a[72:75]\n"
        "ds_write_b128 %[vlw], v[236:239] offset:28672\n"
        "v_mfma_f32_16x16x32_f16 a[76:79], v[40:43], v[140:143], a[76:79]\n"
        "v_mfma_f32_16x16x32_f16 a[96:99], v[44:47], v[128:131], a[96:99]\n"
        "v_mfma_f32_16x16x32_f16 a[100:103], v[44:47], v[132:135], a[100:103]\n"
        "buffer_load_dwordx4 v[16:19], %[vxoff], s[52:55], s88 offen offset:0 sc1\n"
        "v_mfma_f32_16x16x32_f16 a[104:107], v[44:47], v[136:139], a[104:107]\n"
        "v_mfma_f32_16x16x32_f16 a[108:111], v[44:47], v[140:143], a[108:111]\n"
        "s_waitcnt vmcnt(40)\n"
        "v_mfma_f32_16x16x32_f16 a[128:131], v[48:51], v[128:131], a[128:131]\n"
        "v_mfma_f32_16x16x32_f16 a[132:135], v[48:51], v[132:135], a[132:135]\n"
        "v_mfma_f32_16x16x32_f16 a[136:139], v[48:51], v[136:139], a[136:139]\n"
        "buffer_load_dwordx4 v[20:23], %[vxoff], s[52:55], s88 offen offset:1024 sc1\n"
        "v_mfma_f32_16x16x32_f16 a[140:143], v[48:51], v[140:143], a[140:143]\n"
        "v_mfma_f32_16x16x32_f16 a[160:163], v[52:55], v[128:131], a[160:163]\n"
        "v_mfma_f32_16x16x32_f16 a[164:167], v[52:55], v[132:135], a[164:167]\n"
        "v_mfma_f32_16x16x32_f16 a[168:171], v[52:55], v[136:139], a[168:171]\n"
        "v_mfma_f32_16x16x32_f16 a[172:175], v[52:55], v[140:143], a[172:175]\n"
        "buffer_load_dwordx4 v[24:27], %[vxoff], s[52:55], s88 offen offset:2048 sc1\n"
        "s_waitcnt vmcnt(40)\n"
        "v_mfma_f32_16x16x32_f16 a[192:195], v[56:59], v[128:131], a[192:195]\n"
        "v_mfma_f32_16x16x32_f16 a[196:199], v[56:59], v[132:135], a[196:199]\n"
        "v_mfma_f32_16x16x32_f16 a[200:203], v[56:59], v[136:139], a[200:203]\n"
        "v_mfma_f32_16x16x32_f16 a[204:207], v[56:59], v[140:143], a[204:207]\n"
        "v_mfma_f32_16x16x32_f16 a[224:227], v[60:63], v[128:131], a[224:227]\n"
        "buffer_load_dwordx4 v[28:31], %[vxoff], s[52:55], s88 offen offset:3072 sc1\n"
        "v_mfma_f32_16x16x32_f16 a[228:231], v[60:63], v[132:135], a[228:231]\n"
        "s_add_u32 s42, s42, 0x10000\n"
        "s_and_b32 s42, s42, 0x3fffff\n"
        "s_add_u32 s88, s42, 0x1000\n"
        "v_mfma_f32_16x16x32_f16 a[232:235], v[60:63], v[136:139], a[232:235]\n"
        "v_mfma_f32_16x16x32_f16 a[236:239], v[60:63], v[140:143], a[236:239]\n"
        "s_waitcnt lgkmcnt(0)\n"
        "s_barrier\n"
        "v_mfma_f32_16x16x32_f16 a[16:19], v[32:35], v[144:147], a[16:19]\n"
        "ds_read_b128 v[128:131], %[vlr0] offset:16384\n"
        "v_mfma_f32_16x16x32_f16 a[20:23], v[32:35], v[148:151], a[20:23]\n"
        "ds_read_b128 v[132:135], %[vlr0] offset:18432\n"
        "v_mfma_f32_16x16x32_f16 a[24:27], v[32:35], v[152:155], a[24:27]\n"
        "ds_read_b128 v[136:139], %[vlr0] offset:20480\n"
        "v_mfma_f32_16x16x32_f16 a[28:31], v[32:35], v[156:159], a[28:31]\n"
        "ds_read_b128 v[140:143], %[vlr0] offset:22528\n"
        "v_mfma_f32_16x16x32_f16 a[48:51], v[36:39], v[144:147], a[48:51]\n"
        "buffer_load_dwordx4 v[32:35], %[vxoff], s[52:55], s58 offen offset:0 sc1\n"
        "v_mfma_f32_16x16x32_f16 a[52:55], v[36:39], v[148:151], a[52:55]\n"
        "v_mfma_f32_16x16x32_f16 a[56:59], v[36:39], v[152:155], a[56:59]\n"
        "s_cmp_lg_u32 %[swv], 0\n"
        "s_cbranch_scc1 L_nf_5_%=\n"
        "global_store_dword %[vpf], %[vtk], %[pd5] sc0 sc1\n"
        "L_nf_5_%=:\n"
        "v_mfma_f32_16x16x32_f16 a[60:63], v[36:39], v[156:159], a[60:63]\n"
        "v_mfma_f32_16x16x32_f16 a[80:83], v[40:43], v[144:147], a[80:83]\n"
        "v_mfma_f32_16x16x32_f16 a[84:87], v[40:43], v[148:151], a[84:87]\n"
        "buffer_load_dwordx4 v[36:39], %[vxoff], s[52:55], s58 offen offset:1024 sc1\n"
        "v_mfma_f32_16x16x32_f16 a[88:91], v[40:43], v[152:155], a[88:91]\n"
        "v_mfma_f32_16x16x32_f16 a[92:95], v[40:43], v[156:159], a[92:95]\n"
        "v_mfma_f32_16x16x32_f16 a[112:115], v[44:47], v[144:147], a[112:115]\n"
        "v_mfma_f32_16x16x32_f16 a[116:119], v[44:47], v[148:151], a[116:119]\n"
        "v_mfma_f32_16x16x32_f16 a[120:123], v[44:47], v[152:155], a[120:123]\n"
        "buffer_load_dwordx4 v[40:43], %[vxoff], s[52:55], s58 offen offset:2048 sc1\n"
        "v_mfma_f32_16x16x32_f16 a[124:127], v[44:47], v[156:159], a[124:127]\n"
        "v_mfma_f32_16x16x32_f16 a[144:147], v[48:51], v[144:147], a[144:147]\n"
        "v_mfma_f32_16x16x32_f16 a[148:151], v[48:51], v[148:151], a[148:151]\n"
        "v_mfma_f32_16x16x32_f16 a[152:155], v[48:51], v[152:155], a[152:155]\n"
        "v_mfma_f32_16x16x32_f16 a[156:159], v[48:51], v[156:159], a[156:159]\n"
        "buffer_load_dwordx4 v[44:47], %[vxoff], s[52:55], s58 offen offset:3072 sc1\n"
        "v_mfma_f32_16x16x32_f16 a[176:179], v[52:55], v[144:147], a[176:179]\n"
        "v_mfma_f32_16x16x32_f16 a[180:183], v[52:55], v[148:151], a[180:183]\n"
        "v_mfma_f32_16x16x32_f16 a[184:187], v[52:55], v[152:155], a[184:187]\n"
        "v_mfma_f32_16x16x32_f16 a[188:191], v[52:55], v[156:159], a[188:191]\n"
        "v_mfma_f32_16x16x32_f16 a[208:211], v[56:59], v[144:147], a[208:211]\n"
        "buffer_load_dwordx4 v[48:51], %[vxoff], s[52:55], s89 offen offset:0 sc1\n"
        "v_mfma_f32_16x16x32_f16 a[212:215], v[56:59], v[148:151], a[212:215]\n"
        "v_mfma_f32_16x16x32_f16 a[216:219], v[56:59], v[152:155], a[216:219]\n"
        "v_mfma_f32_16x16x32_f16 a[220:223], v[56:59], v[156:159], a[220:223]\n"
        "v_mfma_f32_16x16x32_f16 a[240:243], v[60:63], v[144:147], a[240:243]\n"
        "v_mfma_f32_16x16x32_f16 a[244:247], v[60:63], v[148:151], a[244:247]\n"
        "buffer_load_dwordx4 v[52:55], %[vxoff], s[52:55], s89 offen offset:1024 sc1\n"
        "v_mfma_f32_16x16x32_f16 a[248:251], v[60:63], v[152:155], a[248:251]\n"
        "v_mfma_f32_16x16x32_f16 a[252:255], v[60:63], v[156:159], a[252:255]\n"
        "; ---- kstep\n"
        "s_waitcnt lgkmcnt(0)\n"
        "s_waitcnt vmcnt(41)\n"
        "v_mfma_f32_16x16x32_f16 a[0:3], v[64:67], v[128:131], a[0:3]\n"
        "ds_read_b128 v[144:147], %[vlr0] offset:24576\n"
        "v_mfma_f32_16x16x32_f16 a[4:7], v[64:67], v[132:135], a[4:7]\n"
        "ds_read_b128 v[148:151], %[vlr0] offset:26624\n"
        "v_mfma_f32_16x16x32_f16 a[8:11], v[64:67], v[136:139], a[8:11]\n"
        "ds_read_b128 v[152:155], %[vlr0] offset:28672\n"
        "v_mfma_f32_16x16x32_f16 a[12:15], v[64:67], v[140:143], a[12:15]\n"
        "ds_read_b128 v[156:159], %[vlr0] offset:30720\n"
        "v_mfma_f32_16x16x32_f16 a[32:35], v[68:71], v[128:131], a[32:35]\n"
        "buffer_load_dwordx4 v[56:59], %[vxoff], s[52:55], s89 offen offset:2048 sc1\n"
        "v_mfma_f32_16x16x32_f16 a[36:39], v[68:71], v[132:135], a[36:39]\n"
        "v_cvt_pk_f16_f32 v224, v160, v164\n"
        "v_mfma_f32_16x16x32_f16 a[40:43], v[68:71], v[136:139], a[40:43]\n"
        "v_cvt_pk_f16_f32 v228, v161, v165\n"
        "v_mfma_f32_16x16x32_f16 a[44:47], v[68:71], v[140:143], a[44:47]\n"
        "v_cvt_pk_f16_f32 v232, v162, v166\n"
        "s_waitcnt vmcnt(36)\n"
        "v_mfma_f32_16x16x32_f16 a[64:67], v[72:75], v[128:131], a[64:67]\n"
        "v_cvt_pk_f16_f32 v236, v163, v167\n"
        "v_mfma_f32_16x16x32_f16 a[68:71], v[72:75], v[132:135], a[68:71]\n"
        "buffer_load_dwordx4 v[60:63], %[vxoff], s[52:55], s89 offen offset:3072 sc1\n"
        "v_cvt_pk_f16_f32 v225, v168, v172\n"
        "v_mfma_f32_16x16x32_f16 a[72:75], v[72:75], v[136:139], a[72:75]\n"
        "s_add_u32 s58, s58, 0x10000\n"
        "s_and_b32 s58, s58, 0x3fffff\n"
        "s_add_u32 s89, s58, 0x1000\n"
        "v_cvt_pk_f16_f32 v229, v169, v173\n"
        "v_mfma_f32_16x16x32_f16 a[76:79], v[72:75], v[140:143], a[76:79]\n"
        "v_cvt_pk_f16_f32 v233, v170, v174\n"
        "v_mfma_f32_16x16x32_f16 a[96:99], v[76:79], v[128:131], a[96:99]\n"
        "v_cvt_pk_f16_f32 v237, v171, v175\n"
        "v_mfma_f32_16x16x32_f16 a[100:103], v[76:79], v[132:135], a[100:103]\n"
        "v_mfma_f32_16x16x32_f16 a[104:107], v[76:79], v[136:139], a[104:107]\n"
        "s_mov_b32 s44, s43\n"
        "buffer_load_dwordx4 v[160:163], %[vwoff], %[rw], s44 offen nt\n"
        "v_mfma_f32_16x16x32_f16 a[108:111], v[76:79], v[140:143], a[108:111]\n"
        "s_waitcnt vmcnt(36)\n"
        "v_mfma_f32_16x16x32_f16 a[128:131], v[80:83], v[128:131], a[128:131]\n"
        "v_mfma_f32_16x16x32_f16 a[132:135], v[80:83], v[132:135], a[132:135]\n"
        "v_mfma_f32_16x16x32_f16 a[136:139], v[80:83], v[136:139], a[136:139]\n"
        "v_mfma_f32_16x16x32_f16 a[140:143], v[80:83], v[140:143], a[140:143]\n"
        "s_add_u32 s44, s44, 0x20000\n"
        "buffer_load_dwordx4 v[164:167], %[vwoff], %[rw], s44 offen nt\n"
        "v_mfma_f32_16x16x32_f16 a[160:163], v[84:87], v[128:131], a[160:163]\n"
        "v_mfma_f32_16x16x32_f16 a[164:167], v[84:87], v[132:135], a[164:167]\n"
        "v_mfma_f32_16x16x32_f16 a[168:171], v[84:87], v[136:139], a[168:171]\n"
        "v_mfma_f32_16x16x32_f16 a[172:175], v[84:87], v[140:143], a[172:175]\n"
        "s_waitcnt vmcnt(35)\n"
        "v_mfma_f32_16x16x32_f16 a[192:195], v[88:91], v[128:131], a[192:195]\n"
        "s_add_u32 s44, s44, 0x20000\n"
        "buffer_load_dwordx4 v[168:171], %[vwoff], %[rw], s44 offen nt\n"
        "v_mfma_f32_16x16x32_f16 a[196:199], v[88:91], v[132:135], a[196:199]\n"
        "v_mfma_f32_16x16x32_f16 a[200:203], v[88:91], v[136:139], a[200:203]\n"
        "v_mfma_f32_16x16x32_f16 a[204:207], v[88:91], v[140:143], a[204:207]\n"
        "v_mfma_f32_16x16x32_f16 a[224:227], v[92:95], v[128:131], a[224:227]\n"
        "v_mfma_f32_16x16x32_f16 a[228:231], v[92:95], v[132:135], a[228:231]\n"
        "s_add_u32 s44, s44, 0x20000\n"
        "buffer_load_dwordx4 v[172:175], %[vwoff], %[rw], s44 offen nt\n"
        "v_mfma_f32_16x16x32_f16 a[232:235], v[92:95], v[136:139], a[232:235]\n"
        "v_mfma_f32_16x16x32_f16 a[236:239], v[92:95], v[140:143], a[236:239]\n"
        "s_waitcnt lgkmcnt(0)\n"
        "v_mfma_f32_16x16x32_f16 a[16:19], v[64:67], v[144:147], a[16:19]\n"
        "ds_read_b128 v[128:131], %[vlr1] offset:16384\n"
        "v_mfma_f32_16x16x32_f16 a[20:23], v[64:67], v[148:151], a[20:23]\n"
        "ds_read_b128 v[132:135], %[vlr1] offset:18432\n"
        "v_mfma_f32_16x16x32_f16 a[24:27], v[64:67], v[152:155], a[24:27]\n"
        "ds_read_b128 v[136:139], %[vlr1] offset:20480\n"
        "v_mfma_f32_16x16x32_f16 a[28:31], v[64:67], v[156:159], a[28:31]\n"
        "ds_read_b128 v[140:143], %[vlr1] offset:22528\n"
        "v_mfma_f32_16x16x32_f16 a[48:51], v[68:71], v[144:147], a[48:51]\n"
        "v_cvt_pk_f16_f32 v226, v176, v180\n"
        "v_mfma_f32_16x16x32_f16 a[52:55], v[68:71], v[148:151], a[52:55]\n"
        "v_cvt_pk_f16_f32 v230, v177, v181\n"
        "buffer_load_dwordx4 v[64:67], %[vxoff], s[52:55], s42 offen offset:0 sc1\n"
        "v_mfma_f32_16x16x32_f16 a[56:59], v[68:71], v[152:155], a[56:59]\n"
        "v_cvt_pk_f16_f32 v234, v178, v182\n"
        "v_mfma_f32_16x16x32_f16 a[60:63], v[68:71], v[156:159], a[60:63]\n"
        "v_cvt_pk_f16_f32 v238, v179, v183\n"
        "v_mfma_f32_16x16x32_f16 a[80:83], v[72:75], v[144:147], a[80:83]\n"
        "v_cvt_pk_f16_f32 v227, v184, v188\n"
        "v_mfma_f32_16x16x32_f16 a[84:87], v[72:75], v[148:151], a[84:87]\n"
        "v_cvt_pk_f16_f32 v231, v185, v189\n"
        "v_mfma_f32_16x16x32_f16 a[88:91], v[72:75], v[152:155], a[88:91]\n"
        "v_cvt_pk_f16_f32 v235, v186, v190\n"
        "buffer_load_dwordx4 v[68:71], %[vxoff], s[52:55], s42 offen offset:1024 sc1\n"
        "v_mfma_f32_16x16x32_f16 a[92:95], v[72:75], v[156:159], a[92:95]\n"
        "v_cvt_pk_f16_f32 v239, v187, v191\n"
        "v_mfma_f32_16x16x32_f16 a[112:115], v[76:79], v[144:147], a[112:115]\n"
        "v_mfma_f32_16x16x32_f16 a[116:119], v[76:79], v[148:151], a[116:119]\n"
        "v_mfma_f32_16x16x32_f16 a[120:123], v[76:79], v[152:155], a[120:123]\n"
        "s_add_u32 s44, s44, 0x20000\n"
        "buffer_load_dwordx4 v[176:179], %[vwoff], %[rw], s44 offen nt\n"
        "v_mfma_f32_16x16x32_f16 a[124:127], v[76:79], v[156:159], a[124:127]\n"
        "v_mfma_f32_16x16x32_f16 a[144:147], v[80:83], v[144:147], a[144:147]\n"
        "v_mfma_f32_16x16x32_f16 a[148:151], v[80:83], v[148:151], a[148:151]\n"
        "v_mfma_f32_16x16x32_f16 a[152:155], v[80:83], v[152:155], a[152:155]\n"
        "v_mfma_f32_16x16x32_f16 a[156:159], v[80:83], v[156:159], a[156:159]\n"
        "s_add_u32 s44, s44, 0x20000\n"
        "buffer_load_dwordx4 v[180:183], %[vwoff], %[rw], s44 offen nt\n"
        "v_mfma_f32_16x16x32_f16 a[176:179], v[84:87], v[144:147], a[176:179]\n"
        "v_mfma_f32_16x16x32_f16 a[180:183], v[84:87], v[148:151], a[180:183]\n"
        "v_mfma_f32_16x16x32_f16 a[184:187], v[84:87], v[152:155], a[184:187]\n"
        "v_mfma_f32_16x16x32_f16 a[188:191], v[84:87], v[156:159], a[188:191]\n"
        "v_mfma_f32_16x16x32_f16 a[208:211], v[88:91], v[144:147], a[208:211]\n"
        "s_add_u32 s44, s44, 0x20000\n"
        "buffer_load_dwordx4 v[184:187], %[vwoff], %[rw], s44 offen nt\n"
        "v_mfma_f32_16x16x32_f16 a[212:215], v[88:91], v[148:151], a[212:215]\n"
        "v_mfma_f32_16x16x32_f16 a[216:219], v[88:91], v[152:155], a[216:219]\n"
        "v_mfma_f32_16x16x32_f16 a[220:223], v[88:91], v[156:159], a[220:223]\n"
        "v_mfma_f32_16x16x32_f16 a[240:243], v[92:95], v[144:147], a[240:243]\n"
        "v_mfma_f32_16x16x32_f16 a[244:247], v[92:95], v[148:151], a[244:247]\n"
        "s_add_u32 s44, s44, 0x20000\n"
        "buffer_load_dwordx4 v[188:191], %[vwoff], %[rw], s44 offen nt\n"
        "v_mfma_f32_16x16x32_f16 a[248:251], v[92:95], v[152:155], a[248:251]\n"
        "s_add_u32 s43, s43, 0x800000\n"
        "s_and_b32 s43, s43, 0x1fffffff\n"
        "v_mfma_f32_16x16x32_f16 a[252:255], v[92:95], v[156:159], a[252:255]\n"
        "s_waitcnt lgkmcnt(0)\n"
        "s_waitcnt vmcnt(41)\n"
        "v_mfma_f32_16x16x32_f16 a[0:3], v[96:99], v[128:131], a[0:3]\n"
        "ds_read_b128 v[144:147], %[vlr1] offset:24576\n"
        "v_mfma_f32_16x16x32_f16 a[4:7], v[96:99], v[132:135], a[4:7]\n"
        "ds_read_b128 v[148:151], %[vlr1] offset:26624\n"
        "v_mfma_f32_16x16x32_f16 a[8:11], v[96:99], v[136:139], a[8:11]\n"
        "ds_read_b128 v[152:155], %[vlr1] offset:28672\n"
        "v_mfma_f32_16x16x32_f16 a[12:15], v[96:99], v[140:143], a[12:15]\n"
        "ds_read_b128 v[156:159], %[vlr1] offset:30720\n"
        "buffer_load_dwordx4 v[72:75], %[vxoff], s[52:55], s42 offen offset:2048 sc1\n"
        "v_mfma_f32_16x16x32_f16 a[32:35], v[100:103], v[128:131], a[32:35]\n"
        "ds_write_b128 %[vlw], v[224:227] offset:0\n"
        "v_mfma_f32_16x16x32_f16 a[36:39], v[100:103], v[132:135], a[36:39]\n"
        "v_mfma_f32_16x16x32_f16 a[40:43], v[100:103], v[136:139], a[40:43]\n"
        "ds_write_b128 %[vlw], v[228:231] offset:4096\n"
        "v_mfma_f32_16x16x32_f16 a[44:47], v[100:103], v[140:143], a[44:47]\n"
        "s_waitcnt vmcnt(39)\n"
        "v_mfma_f32_16x16x32_f16 a[64:67], v[104:107], v[128:131], a[64:67]\n"
        "ds_write_b128 %[vlw], v[232:235] offset:8192\n"
        "buffer_load_dwordx4 v[76:79], %[vxoff], s[52:55], s42 offen offset:3072 sc1\n"
        "v_mfma_f32_16x16x32_f16 a[68:71], v[104:107], v[132:135], a[68:71]\n"
        "v_mfma_f32_16x16x32_f16 a[72:75], v[104:107], v[136:139], a[72:75]\n"
        "ds_write_b128 %[vlw], v[236:239] offset:12288\n"
        "v_mfma_f32_16x16x32_f16 a[76:79], v[104:107], v[140:143], a[76:79]\n"
        "v_mfma_f32_16x16x32_f16 a[96:99], v[108:111], v[128:131], a[96:99]\n"
        "v_mfma_f32_16x16x32_f16 a[100:103], v[108:111], v[132:135], a[100:103]\n"
        "buffer_load_dwordx4 v[80:83], %[vxoff], s[52:55], s88 offen offset:0 sc1\n"
        "v_mfma_f32_16x16x32_f16 a[104:107], v[108:111], v[136:139], a[104:107]\n"
        "v_mfma_f32_16x16x32_f16 a[108:111], v[108:111], v[140:143], a[108:111]\n"
        "s_waitcnt vmcnt(39)\n"
        "v_mfma_f32_16x16x32_f16 a[128:131], v[112:115], v[128:131], a[128:131]\n"
        "v_mfma_f32_16x16x32_f16 a[132:135], v[112:115], v[132:135], a[132:135]\n"
        "v_mfma_f32_16x16x32_f16 a[136:139], v[112:115], v[136:139], a[136:139]\n"
        "buffer_load_dwordx4 v[84:87], %[vxoff], s[52:55], s88 offen offset:1024 sc1\n"
        "v_mfma_f32_16x16x32_f16 a[140:143], v[112:115], v[140:143], a[140:143]\n"
        "v_mfma_f32_16x16x32_f16 a[160:163], v[116:119], v[128:131], a[160:163]\n"
        "v_mfma_f32_16x16x32_f16 a[164:167], v[116:119], v[132:135], a[164:167]\n"
        "v_mfma_f32_16x16x32_f16 a[168:171], v[116:119], v[136:139], a[168:171]\n"
        "v_mfma_f32_16x16x32_f16 a[172:175], v[116:119], v[140:143], a[172:175]\n"
        "buffer_load_dwordx4 v[88:91], %[vxoff], s[52:55], s88 offen offset:2048 sc1\n"
        "s_waitcnt vmcnt(39)\n"
        "v_mfma_f32_16x16x32_f16 a[192:195], v[120:123], v[128:131], a[192:195]\n"
        "v_mfma_f32_16x16x32_f16 a[196:199], v[120:123], v[132:135], a[196:199]\n"
        "v_mfma_f32_16x16x32_f16 a[200:203], v[120:123], v[136:139], a[200:203]\n"
        "v_mfma_f32_16x16x32_f16 a[204:207], v[120:123], v[140:143], a[204:207]\n"
        "v_mfma_f32_16x16x32_f16 a[224:227], v[124:127], v[128:131], a[224:227]\n"
        "buffer_load_dwordx4 v[92:95], %[vxoff], s[52:55], s88 offen offset:3072 sc1\n"
        "v_mfma_f32_16x16x32_f16 a[228:231], v[124:127], v[132:135], a[228:231]\n"
        "s_add_u32 s42, s42, 0x10000\n"
        "s_and_b32 s42, s42, 0x3fffff\n"
        "s_add_u32 s88, s42, 0x1000\n"
        "v_mfma_f32_16x16x32_f16 a[232:235], v[124:127], v[136:139], a[232:235]\n"
        "v_mfma_f32_16x16x32_f16 a[236:239], v[124:127], v[140:143], a[236:239]\n"
        "s_waitcnt lgkmcnt(0)\n"
        "s_barrier\n"
        "v_mfma_f32_16x16x32_f16 a[16:19], v[96:99], v[144:147], a[16:19]\n"
        "ds_read_b128 v[128:131], %[vlr0] offset:0\n"
        "v_mfma_f32_16x16x32_f16 a[20:23], v[96:99], v[148:151], a[20:23]\n"
        "ds_read_b128 v[132:135], %[vlr0] offset:2048\n"
        "v_mfma_f32_16x16x32_f16 a[24:27], v[96:99], v[152:155], a[24:27]\n"
        "ds_read_b128 v[136:139], %[vlr0] offset:4096\n"
        "v_mfma_f32_16x16x32_f16 a[28:31], v[96:99], v[156:159], a[28:31]\n"
        "ds_read_b128 v[140:143], %[vlr0] offset:6144\n"
        "v_mfma_f32_16x16x32_f16 a[48:51], v[100:103], v[144:147], a[48:51]\n"
        "buffer_load_dwordx4 v[96:99], %[vxoff], s[52:55], s58 offen offset:0 sc1\n"
        "v_mfma_f32_16x16x32_f16 a[52:55], v[100:103], v[148:151], a[52:55]\n"
        "v_mfma_f32_16x16x32_f16 a[56:59], v[100:103], v[152:155], a[56:59]\n"
        "s_cmp_lg_u32 %[swv], 0\n"
        "s_cbranch_scc1 L_nf_6_%=\n"
        "global_store_dword %[vpf], %[vtk], %[pd6] sc0 sc1\n"
        "L_nf_6_%=:\n"
        "v_mfma_f32_16x16x32_f16 a[60:63], v[100:103], v[156:159], a[60:63]\n"
        "v_mfma_f32_16x16x32_f16 a[80:83], v[104:107], v[144:147], a[80:83]\n"
        "v_mfma_f32_16x16x32_f16 a[84:87], v[104:107], v[148:151], a[84:87]\n"
        "buffer_load_dwordx4 v[100:103], %[vxoff], s[52:55], s58 offen offset:1024 sc1\n"
        "v_mfma_f32_16x16x32_f16 a[88:91], v[104:107], v[152:155], a[88:91]\n"
        "v_mfma_f32_16x16x32_f16 a[92:95], v[104:107], v[156:159], a[92:95]\n"
        "v_mfma_f32_16x16x32_f16 a[112:115], v[108:111], v[144:147], a[112:115]\n"
        "v_mfma_f32_16x16x32_f16 a[116:119], v[108:111], v[148:151], a[116:119]\n"
        "v_mfma_f32_16x16x32_f16 a[120:123], v[108:111], v[152:155], a[120:123]\n"
        "buffer_load_dwordx4 v[104:107], %[vxoff], s[52:55], s58 offen offset:2048 sc1\n"
        "v_mfma_f32_16x16x32_f16 a[124:127], v[108:111], v[156:159], a[124:127]\n"
        "v_mfma_f32_16x16x32_f16 a[144:147], v[112:115], v[144:147], a[144:147]\n"
        "v_mfma_f32_16x16x32_f16 a[148:151], v[112:115], v[148:151], a[148:151]\n"
        "v_mfma_f32_16x16x32_f16 a[152:155], v[112:115], v[152:155], a[152:155]\n"
        "v_mfma_f32_16x16x32_f16 a[156:159], v[112:115], v[156:159], a[156:159]\n"
        "buffer_load_dwordx4 v[108:111], %[vxoff], s[52:55], s58 offen offset:3072 sc1\n"
        "v_mfma_f32_16x16x32_f16 a[176:179], v[116:119], v[144:147], a[176:179]\n"
        "v_mfma_f32_16x16x32_f16 a[180:183], v[116:119], v[148:151], a[180:183]\n"
        "v_mfma_f32_16x16x32_f16 a[184:187], v[116:119], v[152:155], a[184:187]\n"
        "v_mfma_f32_16x16x32_f16 a[188:191], v[116:119], v[156:159], a[188:191]\n"
        "v_mfma_f32_16x16x32_f16 a[208:211], v[120:123], v[144:147], a[208:211]\n"
        "buffer_load_dwordx4 v[112:115], %[vxoff], s[52:55], s89 offen offset:0 sc1\n"
        "v_mfma_f32_16x16x32_f16 a[212:215], v[120:123], v[148:151], a[212:215]\n"
        "v_mfma_f32_16x16x32_f16 a[216:219], v[120:123], v[152:155], a[216:219]\n"
        "v_mfma_f32_16x16x32_f16 a[220:223], v[120:123], v[156:159], a[220:223]\n"
        "v_mfma_f32_16x16x32_f16 a[240:243], v[124:127], v[144:147], a[240:243]\n"
        "v_mfma_f32_16x16x32_f16 a[244:247], v[124:127], v[148:151], a[244:247]\n"
        "buffer_load_dwordx4 v[116:119], %[vxoff], s[52:55], s89 offen offset:1024 sc1\n"
        "v_mfma_f32_16x16x32_f16 a[248:251], v[124:127], v[152:155], a[248:251]\n"
        "v_mfma_f32_16x16x32_f16 a[252:255], v[124:127], v[156:159], a[252:255]\n"
        "s_add_u32 s45, s45, 1\n"
        "L_loop_%=:\n"
        "s_and_b32 s47, s45, 3\n"
        "s_cmp_lg_u32 s47, 3\n"
        "s_cbranch_scc1 L_nc_lp_%=\n"
        "s_waitcnt vmcnt(48)\n"
        "v_cmp_gt_u32 vcc, s40, v240\n"
        "s_cbranch_vccnz L_slow_lp_%=\n"
        "s_nop 0\n"
        "s_branch L_nc_lp_%=\n"
        "L_slow_lp_%=:\n"
        "s_cmp_lg_u32 s41, 0\n"
        "s_cbranch_scc1 L_ok_lp_%=\n"
        "s_memrealtime s[48:49]\n"
        "L_spin_lp_%=:\n"
        "buffer_load_dword v240, %[vfl], %[rc], s46 offen sc0 sc1\n"
        "buffer_load_dword v243, v241, %[rc], 0 offen sc1\n"
        "s_waitcnt vmcnt(0)\n"
        "v_cmp_gt_u32 vcc, s40, v240\n"
        "s_cbranch_vccz L_ok_lp_%=\n"
        "v_readfirstlane_b32 s47, v243\n"
        "s_memrealtime s[50:51]\n"
        "s_waitcnt lgkmcnt(0)\n"
        "s_cmp_eq_u32 s47, s40\n"
        "s_cbranch_scc1 L_to_lp_%=\n"
        "s_sub_u32 s50, s50, s48\n"
        "s_subb_u32 s51, s51, s49\n"
        "s_cmp_lg_u32 s51, 0\n"
        "s_cbranch_scc1 L_to_lp_%=\n"
        "s_cmp_gt_u32 s50, 0x2625a00\n"
        "s_cbranch_scc1 L_to_lp_%=\n"
        "s_sleep 8\n"
        "s_branch L_spin_lp_%=\n"
        "L_to_lp_%=:\n"
        "v_mov_b32 v243, s40\n"
        "buffer_store_dword v243, v241, %[rc], 0 offen sc1\n"
        "s_mov_b32 s41, 1\n"
        "s_waitcnt vmcnt(0)\n"
        "L_ok_lp_%=:\n"
        "s_waitcnt lgkmcnt(0)\n"
        "s_nop 0\n"
        "L_nc_lp_%=:\n"
        "s_and_b32 s47, s45, 3\n"
        "s_cmp_lg_u32 s47, 2\n"
        "s_cbranch_scc1 L_np_lp_%=\n"
        "s_add_u32 s46, s46, 0x400\n"
        "s_and_b32 s46, s46, 0x1fff\n"
        "buffer_load_dword v240, %[vfl], %[rc], s46 offen sc0 sc1\n"
        "L_np_lp_%=:\n"
        "; ---- kstep\n"
        "s_waitcnt lgkmcnt(0)\n"
        "s_waitcnt vmcnt(40)\n"
        "v_mfma_f32_16x16x32_f16 a[0:3], v[0:3], v[128:131], a[0:3]\n"
        "ds_read_b128 v[144:147], %[vlr0] offset:8192\n"
        "v_mfma_f32_16x16x32_f16 a[4:7], v[0:3], v[132:135], a[4:7]\n"
        "ds_read_b128 v[148:151], %[vlr0] offset:10240\n"
        "v_mfma_f32_16x16x32_f16 a[8:11], v[0:3], v[136:139], a[8:11]\n"
        "ds_read_b128 v[152:155], %[vlr0] offset:12288\n"
        "v_mfma_f32_16x16x32_f16 a[12:15], v[0:3], v[140:143], a[12:15]\n"
        "ds_read_b128 v[156:159], %[vlr0] offset:14336\n"
        "v_mfma_f32_16x16x32_f16 a[32:35], v[4:7], v[128:131], a[32:35]\n"
        "buffer_load_dwordx4 v[120:123], %[vxoff], s[52:55], s89 offen offset:2048 sc1\n"
        "v_mfma_f32_16x16x32_f16 a[36:39], v[4:7], v[132:135], a[36:39]\n"
        "v_cvt_pk_f16_f32 v224, v192, v196\n"
        "v_mfma_f32_16x16x32_f16 a[40:43], v[4:7], v[136:139], a[40:43]\n"
        "v_cvt_pk_f16_f32 v228, v193, v197\n"
        "v_mfma_f32_16x16x32_f16 a[44:47], v[4:7], v[140:143], a[44:47]\n"
        "v_cvt_pk_f16_f32 v232, v194, v198\n"
        "s_waitcnt vmcnt(35)\n"
        "v_mfma_f32_16x16x32_f16 a[64:67], v[8:11], v[128:131], a[64:67]\n"
        "v_cvt_pk_f16_f32 v236, v195, v199\n"
        "v_mfma_f32_16x16x32_f16 a[68:71], v[8:11], v[132:135], a[68:71]\n"
        "buffer_load_dwordx4 v[124:127], %[vxoff], s[52:55], s89 offen offset:3072 sc1\n"
        "v_cvt_pk_f16_f32 v225, v200, v204\n"
        "v_mfma_f32_16x16x32_f16 a[72:75], v[8:11], v[136:139], a[72:75]\n"
        "s_add_u32 s58, s58, 0x10000\n"
        "s_and_b32 s58, s58, 0x3fffff\n"
        "s_add_u32 s89, s58, 0x1000\n"
        "v_cvt_pk_f16_f32 v229, v201, v205\n"
        "v_mfma_f32_16x16x32_f16 a[76:79], v[8:11], v[140:143], a[76:79]\n"
        "v_cvt_pk_f16_f32 v233, v202, v206\n"
        "v_mfma_f32_16x16x32_f16 a[96:99], v[12:15], v[128:131], a[96:99]\n"
        "v_cvt_pk_f16_f32 v237, v203, v207\n"
        "v_mfma_f32_16x16x32_f16 a[100:103], v[12:15], v[132:135], a[100:103]\n"
        "v_mfma_f32_16x16x32_f16 a[104:107], v[12:15], v[136:139], a[104:107]\n"
        "s_mov_b32 s44, s43\n"
        "buffer_load_dwordx4 v[192:195], %[vwoff], %[rw], s44 offen nt\n"
        "v_mfma_f32_16x16x32_f16 a[108:111], v[12:15], v[140:143], a[108:111]\n"
        "s_waitcnt vmcnt(35)\n"
        "v_mfma_f32_16x16x32_f16 a[128:131], v[16:19], v[128:131], a[128:131]\n"
        "v_mfma_f32_16x16x32_f16 a[132:135], v[16:19], v[132:135], a[132:135]\n"
        "v_mfma_f32_16x16x32_f16 a[136:139], v[16:19], v[136:139], a[136:139]\n"
        "v_mfma_f32_16x16x32_f16 a[140:143], v[16:19], v[140:143], a[140:143]\n"
        "s_add_u32 s44, s44, 0x20000\n"
        "buffer_load_dwordx4 v[196:199], %[vwoff], %[rw], s44 offen nt\n"
        "v_mfma_f32_16x16x32_f16 a[160:163], v[20:23], v[128:131], a[160:163]\n"
        "v_mfma_f32_16x16x32_f16 a[164:167], v[20:23], v[132:135], a[164:167]\n"
        "v_mfma_f32_16x16x32_f16 a[168:171], v[20:23], v[136:139], a[168:171]\n"
        "v_mfma_f32_16x16x32_f16 a[172:175], v[20:23], v[140:143], a[172:175]\n"
        "s_waitcnt vmcnt(34)\n"
        "v_mfma_f32_16x16x32_f16 a[192:195], v[24:27], v[128:131], a[192:195]\n"
        "s_add_u32 s44, s44, 0x20000\n"
        "buffer_load_dwordx4 v[200:203], %[vwoff], %[rw], s44 offen nt\n"
        "v_mfma_f32_16x16x32_f16 a[196:199], v[24:27], v[132:135], a[196:199]\n"
        "v_mfma_f32_16x16x32_f16 a[200:203], v[24:27], v[136:139], a[200:203]\n"
        "v_mfma_f32_16x16x32_f16 a[204:207], v[24:27], v[140:143], a[204:207]\n"
        "v_mfma_f32_16x16x32_f16 a[224:227], v[28:31], v[128:131], a[224:227]\n"
        "v_mfma_f32_16x16x32_f16 a[228:231], v[28:31], v[132:135], a[228:231]\n"
        "s_add_u32 s44, s44, 0x20000\n"
        "buffer_load_dwordx4 v[204:207], %[vwoff], %[rw], s44 offen nt\n"
        "v_mfma_f32_16x16x32_f16 a[232:235], v[28:31], v[136:139], a[232:235]\n"
        "v_mfma_f32_16x16x32_f16 a[236:239], v[28:31], v[140:143], a[236:239]\n"
        "s_waitcnt lgkmcnt(0)\n"
        "v_mfma_f32_16x16x32_f16 a[16:19], v[0:3], v[144:147], a[16:19]\n"
        "ds_read_b128 v[128:131], %[vlr1] offset:0\n"
        "v_mfma_f32_16x16x32_f16 a[20:23], v[0:3], v[148:151], a[20:23]\n"
        "ds_read_b128 v[132:135], %[vlr1] offset:2048\n"
        "v_mfma_f32_16x16x32_f16 a[24:27], v[0:3], v[152:155], a[24:27]\n"
        "ds_read_b128 v[136:139], %[vlr1] offset:4096\n"
        "v_mfma_f32_16x16x32_f16 a[28:31], v[0:3], v[156:159], a[28:31]\n"
        "ds_read_b128 v[140:143], %[vlr1] offset:6144\n"
        "v_mfma_f32_16x16x32_f16 a[48:51], v[4:7], v[144:147], a[48:51]\n"
        "v_cvt_pk_f16_f32 v226, v208, v212\n"
        "v_mfma_f32_16x16x32_f16 a[52:55], v[4:7], v[148:151], a[52:55]\n"
        "v_cvt_pk_f16_f32 v230, v209, v213\n"
        "buffer_load_dwordx4 v[0:3], %[vxoff], s[52:55], s42 offen offset:0 sc1\n"
        "v_mfma_f32_16x16x32_f16 a[56:59], v[4:7], v[152:155], a[56:59]\n"
        "v_cvt_pk_f16_f32 v234, v210, v214\n"
        "v_mfma_f32_16x16x32_f16 a[60:63], v[4:7], v[156:159], a[60:63]\n"
        "v_cvt_pk_f16_f32 v238, v211, v215\n"
        "v_mfma_f32_16x16x32_f16 a[80:83], v[8:11], v[144:147], a[80:83]\n"
        "v_cvt_pk_f16_f32 v227, v216, v220\n"
        "v_mfma_f32_16x16x32_f16 a[84:87], v[8:11], v[148:151], a[84:87]\n"
        "v_cvt_pk_f16_f32 v231, v217, v221\n"
        "v_mfma_f32_16x16x32_f16 a[88:91], v[8:11], v[152:155], a[88:91]\n"
        "v_cvt_pk_f16_f32 v235, v218, v222\n"
        "buffer_load_dwordx4 v[4:7], %[vxoff], s[52:55], s42 offen offset:1024 sc1\n"
        "v_mfma_f32_16x16x32_f16 a[92:95], v[8:11], v[156:159], a[92:95]\n"
        "v_cvt_pk_f16_f32 v239, v219, v223\n"
        "v_mfma_f32_16x16x32_f16 a[112:115], v[12:15], v[144:147], a[112:115]\n"
        "v_mfma_f32_16x16x32_f16 a[116:119], v[12:15], v[148:151], a[116:119]\n"
        "v_mfma_f32_16x16x32_f16 a[120:123], v[12:15], v[152:155], a[120:123]\n"
        "s_add_u32 s44, s44, 0x20000\n"
        "buffer_load_dwordx4 v[208:211], %[vwoff], %[rw], s44 offen nt\n"
        "v_mfma_f32_16x16x32_f16 a[124:127], v[12:15], v[156:159], a[124:127]\n"
        "v_mfma_f32_16x16x32_f16 a[144:147], v[16:19], v[144:147], a[144:147]\n"
        "v_mfma_f32_16x16x32_f16 a[148:151], v[16:19], v[148:151], a[148:151]\n"
        "v_mfma_f32_16x16x32_f16 a[152:155], v[16:19], v[152:155], a[152:155]\n"
        "v_mfma_f32_16x16x32_f16 a[156:159], v[16:19], v[156:159], a[156:159]\n"
        "s_add_u32 s44, s44, 0x20000\n"
        "buffer_load_dwordx4 v[212:215], %[vwoff], %[rw], s44 offen nt\n"
        "v_mfma_f32_16x16x32_f16 a[176:179], v[20:23], v[144:147], a[176:179]\n"
        "v_mfma_f32_16x16x32_f16 a[180:183], v[20:23], v[148:151], a[180:183]\n"
        "v_mfma_f32_16x16x32_f16 a[184:187], v[20:23], v[152:155], a[184:187]\n"
        "v_mfma_f32_16x16x32_f16 a[188:191], v[20:23], v[156:159], a[188:191]\n"
        "v_mfma_f32_16x16x32_f16 a[208:211], v[24:27], v[144:147], a[208:211]\n"
        "s_add_u32 s44, s44, 0x20000\n"
        "buffer_load_dwordx4 v[216:219], %[vwoff], %[rw], s44 offen nt\n"
        "v_mfma_f32_16x16x32_f16 a[212:215], v[24:27], v[148:151], a[212:215]\n"
        "v_mfma_f32_16x16x32_f16 a[216:219], v[24:27], v[152:155], a[216:219]\n"
        "v_mfma_f32_16x16x32_f16 a[220:223], v[24:27], v[156:159], a[220:223]\n"
        "v_mfma_f32_16x16x32_f16 a[240:243], v[28:31], v[144:147], a[240:243]\n"
        "v_mfma_f32_16x16x32_f16 a[244:247], v[28:31], v[148:151], a[244:247]\n"
        "s_add_u32 s44, s44, 0x20000\n"
        "buffer_load_dwordx4 v[220:223], %[vwoff], %[rw], s44 offen nt\n"
        "v_mfma_f32_16x16x32_f16 a[248:251], v[28:31], v[152:155], a[248:251]\n"
        "s_add_u32 s43, s43, 0x800000\n"
        "s_and_b32 s43, s43, 0x1fffffff\n"
        "v_mfma_f32_16x16x32_f16 a[252:255], v[28:31], v[156:159], a[252:255]\n"
        "s_waitcnt lgkmcnt(0)\n"
        "s_waitcnt vmcnt(40)\n"
        "v_mfma_f32_16x16x32_f16 a[0:3], v[32:35], v[128:131], a[0:3]\n"
        "ds_read_b128 v[144:147], %[vlr1] offset:8192\n"
        "v_mfma_f32_16x16x32_f16 a[4:7], v[32:35], v[132:135], a[4:7]\n"
        "ds_read_b128 v[148:151], %[vlr1] offset:10240\n"
        "v_mfma_f32_16x16x32_f16 a[8:11], v[32:35], v[136:139], a[8:11]\n"
        "ds_read_b128 v[152:155], %[vlr1] offset:12288\n"
        "v_mfma_f32_16x16x32_f16 a[12:15], v[32:35], v[140:143], a[12:15]\n"
        "ds_read_b128 v[156:159], %[vlr1] offset:14336\n"
        "buffer_load_dwordx4 v[8:11], %[vxoff], s[52:55], s42 offen offset:2048 sc1\n"
        "v_mfma_f32_16x16x32_f16 a[32:35], v[36:39], v[128:131], a[32:35]\n"
        "ds_write_b128 %[vlw], v[224:227] offset:16384\n"
        "v_mfma_f32_16x16x32_f16 a[36:39], v[36:39], v[132:135], a[36:39]\n"
        "v_mfma_f32_16x16x32_f16 a[40:43], v[36:39], v[136:139], a[40:43]\n"
        "ds_write_b128 %[vlw], v[228:231] offset:20480\n"
        "v_mfma_f32_16x16x32_f16 a[44:47], v[36:39], v[140:143], a[44:47]\n"
        "s_waitcnt vmcnt(39)\n"
        "v_mfma_f32_16x16x32_f16 a[64:67], v[40:43], v[128:131], a[64:67]\n"
        "ds_write_b128 %[vlw], v[232:235] offset:24576\n"
        "buffer_load_dwordx4 v[12:15], %[vxoff], s[52:55], s42 offen offset:3072 sc1\n"
        "v_mfma_f32_16x16x32_f16 a[68:71], v[40:43], v[132:135], a[68:71]\n"
        "v_mfma_f32_16x16x32_f16 a[72:75], v[40:43], v[136:139], a[72:75]\n"
        "ds_write_b128 %[vlw], v[236:239] offset:28672\n"
        "v_mfma_f32_16x16x32_f16 a[76:79], v[40:43], v[140:143], a[76:79]\n"
        "v_mfma_f32_16x16x32_f16 a[96:99], v[44:47], v[128:131], a[96:99]\n"
        "v_mfma_f32_16x16x32_f16 a[100:103], v[44:47], v[132:135], a[100:103]\n"
        "buffer_load_dwordx4 v[16:19], %[vxoff], s[52:55], s88 offen offset:0 sc1\n"
        "v_mfma_f32_16x16x32_f16 a[104:107], v[44:47], v[136:139], a[104:107]\n"
        "v_mfma_f32_16x16x32_f16 a[108:111], v[44:47], v[140:143], a[108:111]\n"
        "s_waitcnt vmcnt(39)\n"
        "v_mfma_f32_16x16x32_f16 a[128:131], v[48:51], v[128:131], a[128:131]\n"
        "v_mfma_f32_16x16x32_f16 a[132:135], v[48:51], v[132:135], a[132:135]\n"
        "v_mfma_f32_16x16x32_f16 a[136:139], v[48:51], v[136:139], a[136:139]\n"
        "buffer_load_dwordx4 v[20:23], %[vxoff], s[52:55], s88 offen offset:1024 sc1\n"
        "v_mfma_f32_16x16x32_f16 a[140:143], v[48:51], v[140:143], a[140:143]\n"
        "v_mfma_f32_16x16x32_f16 a[160:163], v[52:55], v[128:131], a[160:163]\n"
        "v_mfma_f32_16x16x32_f16 a[164:167], v[52:55], v[132:135], a[164:167]\n"
        "v_mfma_f32_16x16x32_f16 a[168:171], v[52:55], v[136:139], a[168:171]\n"
        "v_mfma_f32_16x16x32_f16 a[172:175], v[52:55], v[140:143], a[172:175]\n"
        "buffer_load_dwordx4 v[24:27], %[vxoff], s[52:55], s88 offen offset:2048 sc1\n"
        "s_waitcnt vmcnt(39)\n"
        "v_mfma_f32_16x16x32_f16 a[192:195], v[56:59], v[128:131], a[192:195]\n"
        "v_mfma_f32_16x16x32_f16 a[196:199], v[56:59], v[132:135], a[196:199]\n"
        "v_mfma_f32_16x16x32_f16 a[200:203], v[56:59], v[136:139], a[200:203]\n"
        "v_mfma_f32_16x16x32_f16 a[204:207], v[56:59], v[140:143], a[204:207]\n"
        "v_mfma_f32_16x16x32_f16 a[224:227], v[60:63], v[128:131], a[224:227]\n"
        "buffer_load_dwordx4 v[28:31], %[vxoff], s[52:55], s88 offen offset:3072 sc1\n"
        "v_mfma_f32_16x16x32_f16 a[228:231], v[60:63], v[132:135], a[228:231]\n"
        "s_add_u32 s42, s42, 0x10000\n"
        "s_and_b32 s42, s42, 0x3fffff\n"
        "s_add_u32 s88, s42, 0x1000\n"
        "v_mfma_f32_16x16x32_f16 a[232:235], v[60:63], v[136:139], a[232:235]\n"
        "v_mfma_f32_16x16x32_f16 a[236:239], v[60:63], v[140:143], a[236:239]\n"
        "s_waitcnt lgkmcnt(0)\n"
        "s_barrier\n"
        "v_mfma_f32_16x16x32_f16 a[16:19], v[32:35], v[144:147], a[16:19]\n"
        "ds_read_b128 v[128:131], %[vlr0] offset:16384\n"
        "v_mfma_f32_16x16x32_f16 a[20:23], v[32:35], v[148:151], a[20:23]\n"
        "ds_read_b128 v[132:135], %[vlr0] offset:18432\n"
        "v_mfma_f32_16x16x32_f16 a[24:27], v[32:35], v[152:155], a[24:27]\n"
        "ds_read_b128 v[136:139], %[vlr0] offset:20480\n"
        "v_mfma_f32_16x16x32_f16 a[28:31], v[32:35], v[156:159], a[28:31]\n"
        "ds_read_b128 v[140:143], %[vlr0] offset:22528\n"
        "v_mfma_f32_16x16x32_f16 a[48:51], v[36:39], v[144:147], a[48:51]\n"
        "buffer_load_dwordx4 v[32:35], %[vxoff], s[52:55], s58 offen offset:0 sc1\n"
        "v_mfma_f32_16x16x32_f16 a[52:55], v[36:39], v[148:151], a[52:55]\n"
        "v_mfma_f32_16x16x32_f16 a[56:59], v[36:39], v[152:155], a[56:59]\n"
        "v_mfma_f32_16x16x32_f16 a[60:63], v[36:39], v[156:159], a[60:63]\n"
        "v_mfma_f32_16x16x32_f16 a[80:83], v[40:43], v[144:147], a[80:83]\n"
        "v_mfma_f32_16x16x32_f16 a[84:87], v[40:43], v[148:151], a[84:87]\n"
        "buffer_load_dwordx4 v[36:39], %[vxoff], s[52:55], s58 offen offset:1024 sc1\n"
        "v_mfma_f32_16x16x32_f16 a[88:91], v[40:43], v[152:155], a[88:91]\n"
        "v_mfma_f32_16x16x32_f16 a[92:95], v[40:43], v[156:159], a[92:95]\n"
        "v_mfma_f32_16x16x32_f16 a[112:115], v[44:47], v[144:147], a[112:115]\n"
        "v_mfma_f32_16x16x32_f16 a[116:119], v[44:47], v[148:151], a[116:119]\n"
        "v_mfma_f32_16x16x32_f16 a[120:123], v[44:47], v[152:155], a[120:123]\n"
        "buffer_load_dwordx4 v[40:43], %[vxoff], s[52:55], s58 offen offset:2048 sc1\n"
        "v_mfma_f32_16x16x32_f16 a[124:127], v[44:47], v[156:159], a[124:127]\n"
        "v_mfma_f32_16x16x32_f16 a[144:147], v[48:51], v[144:147], a[144:147]\n"
        "v_mfma_f32_16x16x32_f16 a[148:151], v[48:51], v[148:151], a[148:151]\n"
        "v_mfma_f32_16x16x32_f16 a[152:155], v[48:51], v[152:155], a[152:155]\n"
        "v_mfma_f32_16x16x32_f16 a[156:159], v[48:51], v[156:159], a[156:159]\n"
        "buffer_load_dwordx4 v[44:47], %[vxoff], s[52:55], s58 offen offset:3072 sc1\n"
        "v_mfma_f32_16x16x32_f16 a[176:179], v[52:55], v[144:147], a[176:179]\n"
        "v_mfma_f32_16x16x32_f16 a[180:183], v[52:55], v[148:151], a[180:183]\n"
        "v_mfma_f32_16x16x32_f16 a[184:187], v[52:55], v[152:155], a[184:187]\n"
        "v_mfma_f32_16x16x32_f16 a[188:191], v[52:55], v[156:159], a[188:191]\n"
        "v_mfma_f32_16x16x32_f16 a[208:211], v[56:59], v[144:147], a[208:211]\n"
        "buffer_load_dwordx4 v[48:51], %[vxoff], s[52:55], s89 offen offset:0 sc1\n"
        "v_mfma_f32_16x16x32_f16 a[212:215], v[56:59], v[148:151], a[212:215]\n"
        "v_mfma_f32_16x16x32_f16 a[216:219], v[56:59], v[152:155], a[216:219]\n"
        "v_mfma_f32_16x16x32_f16 a[220:223], v[56:59], v[156:159], a[220:223]\n"
        "v_mfma_f32_16x16x32_f16 a[240:243], v[60:63], v[144:147], a[240:243]\n"
        "v_mfma_f32_16x16x32_f16 a[244:247], v[60:63], v[148:151], a[244:247]\n"
        "buffer_load_dwordx4 v[52:55], %[vxoff], s[52:55], s89 offen offset:1024 sc1\n"
        "v_mfma_f32_16x16x32_f16 a[248:251], v[60:63], v[152:155], a[248:251]\n"
        "v_mfma_f32_16x16x32_f16 a[252:255], v[60:63], v[156:159], a[252:255]\n"
        "; ---- kstep\n"
        "s_waitcnt lgkmcnt(0)\n"
        "s_waitcnt vmcnt(40)\n"
        "v_mfma_f32_16x16x32_f16 a[0:3], v[64:67], v[128:131], a[0:3]\n"
        "ds_read_b128 v[144:147], %[vlr0] offset:24576\n"
        "v_mfma_f32_16x16x32_f16 a[4:7], v[64:67], v[132:135], a[4:7]\n"
        "ds_read_b128 v[148:151], %[vlr0] offset:26624\n"
        "v_mfma_f32_16x16x32_f16 a[8:11], v[64:67], v[136:139], a[8:11]\n"
        "ds_read_b128 v[152:155], %[vlr0] offset:28672\n"
        "v_mfma_f32_16x16x32_f16 a[12:15], v[64:67], v[140:143], a[12:15]\n"
        "ds_read_b128 v[156:159], %[vlr0] offset:30720\n"
        "v_mfma_f32_16x16x32_f16 a[32:35], v[68:71], v[128:131], a[32:35]\n"
        "buffer_load_dwordx4 v[56:59], %[vxoff], s[52:55], s89 offen offset:2048 sc1\n"
        "v_mfma_f32_16x16x32_f16 a[36:39], v[68:71], v[132:135], a[36:39]\n"
        "v_cvt_pk_f16_f32 v224, v160, v164\n"
        "v_mfma_f32_16x16x32_f16 a[40:43], v[68:71], v[136:139], a[40:43]\n"
        "v_cvt_pk_f16_f32 v228, v161, v165\n"
        "v_mfma_f32_16x16x32_f16 a[44:47], v[68:71], v[140:143], a[44:47]\n"
        "v_cvt_pk_f16_f32 v232, v162, v166\n"
        "s_waitcnt vmcnt(35)\n"
        "v_mfma_f32_16x16x32_f16 a[64:67], v[72:75], v[128:131], a[64:67]\n"
        "v_cvt_pk_f16_f32 v236, v163, v167\n"
        "v_mfma_f32_16x16x32_f16 a[68:71], v[72:75], v[132:135], a[68:71]\n"
        "buffer_load_dwordx4 v[60:63], %[vxoff], s[52:55], s89 offen offset:3072 sc1\n"
        "v_cvt_pk_f16_f32 v225, v168, v172\n"
        "v_mfma_f32_16x16x32_f16 a[72:75], v[72:75], v[136:139], a[72:75]\n"
        "s_add_u32 s58, s58, 0x10000\n"
        "s_and_b32 s58, s58, 0x3fffff\n"
        "s_add_u32 s89, s58, 0x1000\n"
        "v_cvt_pk_f16_f32 v229, v169, v173\n"
        "v_mfma_f32_16x16x32_f16 a[76:79], v[72:75], v[140:143], a[76:79]\n"
        "v_cvt_pk_f16_f32 v233, v170, v174\n"
        "v_mfma_f32_16x16x32_f16 a[96:99], v[76:79], v[128:131], a[96:99]\n"
        "v_cvt_pk_f16_f32 v237, v171, v175\n"
        "v_mfma_f32_16x16x32_f16 a[100:103], v[76:79], v[132:135], a[100:103]\n"
        "v_mfma_f32_16x16x32_f16 a[104:107], v[76:79], v[136:139], a[104:107]\n"
        "s_mov_b32 s44, s43\n"
        "buffer_load_dwordx4 v[160:163], %[vwoff], %[rw], s44 offen nt\n"
        "v_mfma_f32_16x16x32_f16 a[108:111], v[76:79], v[140:143], a[108:111]\n"
        "s_waitcnt vmcnt(35)\n"
        "v_mfma_f32_16x16x32_f16 a[128:131], v[80:83], v[128:131], a[128:131]\n"
        "v_mfma_f32_16x16x32_f16 a[132:135], v[80:83], v[132:135], a[132:135]\n"
        "v_mfma_f32_16x16x32_f16 a[136:139], v[80:83], v[136:139], a[136:139]\n"
        "v_mfma_f32_16x16x32_f16 a[140:143], v[80:83], v[140:143], a[140:143]\n"
        "s_add_u32 s44, s44, 0x20000\n"
        "buffer_load_dwordx4 v[164:167], %[vwoff], %[rw], s44 offen nt\n"
        "v_mfma_f32_16x16x32_f16 a[160:163], v[84:87], v[128:131], a[160:163]\n"
        "v_mfma_f32_16x16x32_f16 a[164:167], v[84:87], v[132:135], a[164:167]\n"
        "v_mfma_f32_16x16x32_f16 a[168:171], v[84:87], v[136:139], a[168:171]\n"
        "v_mfma_f32_16x16x32_f16 a[172:175], v[84:87], v[140:143], a[172:175]\n"
        "s_waitcnt vmcnt(34)\n"
        "v_mfma_f32_16x16x32_f16 a[192:195], v[88:91], v[128:131], a[192:195]\n"
        "s_add_u32 s44, s44, 0x20000\n"
        "buffer_load_dwordx4 v[168:171], %[vwoff], %[rw], s44 offen nt\n"
        "v_mfma_f32_16x16x32_f16 a[196:199], v[88:91], v[132:135], a[196:199]\n"
        "v_mfma_f32_16x16x32_f16 a[200:203], v[88:91], v[136:139], a[200:203]\n"
        "v_mfma_f32_16x16x32_f16 a[204:207], v[88:91], v[140:143], a[204:207]\n"
        "v_mfma_f32_16x16x32_f16 a[224:227], v[92:95], v[128:131], a[224:227]\n"
        "v_mfma_f32_16x16x32_f16 a[228:231], v[92:95], v[132:135], a[228:231]\n"
        "s_add_u32 s44, s44, 0x20000\n"
        "buffer_load_dwordx4 v[172:175], %[vwoff], %[rw], s44 offen nt\n"
        "v_mfma_f32_16x16x32_f16 a[232:235], v[92:95], v[136:139], a[232:235]\n"
        "v_mfma_f32_16x16x32_f16 a[236:239], v[92:95], v[140:143], a[236:239]\n"
        "s_waitcnt lgkmcnt(0)\n"
        "v_mfma_f32_16x16x32_f16 a[16:19], v[64:67], v[144:147], a[16:19]\n"
        "ds_read_b128 v[128:131], %[vlr1] offset:16384\n"
        "v_mfma_f32_16x16x32_f16 a[20:23], v[64:67], v[148:151], a[20:23]\n"
        "ds_read_b128 v[132:135], %[vlr1] offset:18432\n"
        "v_mfma_f32_16x16x32_f16 a[24:27], v[64:67], v[152:155], a[24:27]\n"
        "ds_read_b128 v[136:139], %[vlr1] offset:20480\n"
        "v_mfma_f32_16x16x32_f16 a[28:31], v[64:67], v[156:159], a[28:31]\n"
        "ds_read_b128 v[140:143], %[vlr1] offset:22528\n"
        "v_mfma_f32_16x16x32_f16 a[48:51], v[68:71], v[144:147], a[48:51]\n"
        "v_cvt_pk_f16_f32 v226, v176, v180\n"
        "v_mfma_f32_16x16x32_f16 a[52:55], v[68:71], v[148:151], a[52:55]\n"
        "v_cvt_pk_f16_f32 v230, v177, v181\n"
        "buffer_load_dwordx4 v[64:67], %[vxoff], s[52:55], s42 offen offset:0 sc1\n"
        "v_mfma_f32_16x16x32_f16 a[56:59], v[68:71], v[152:155], a[56:59]\n"
        "v_cvt_pk_f16_f32 v234, v178, v182\n"
        "v_mfma_f32_16x16x32_f16 a[60:63], v[68:71], v[156:159], a[60:63]\n"
        "v_cvt_pk_f16_f32 v238, v179, v183\n"
        "v_mfma_f32_16x16x32_f16 a[80:83], v[72:75], v[144:147], a[80:83]\n"
        "v_cvt_pk_f16_f32 v227, v184, v188\n"
        "v_mfma_f32_16x16x32_f16 a[84:87], v[72:75], v[148:151], a[84:87]\n"
        "v_cvt_pk_f16_f32 v231, v185, v189\n"
        "v_mfma_f32_16x16x32_f16 a[88:91], v[72:75], v[152:155], a[88:91]\n"
        "v_cvt_pk_f16_f32 v235, v186, v190\n"
        "buffer_load_dwordx4 v[68:71], %[vxoff], s[52:55], s42 offen offset:1024 sc1\n"
        "v_mfma_f32_16x16x32_f16 a[92:95], v[72:75], v[156:159], a[92:95]\n"
        "v_cvt_pk_f16_f32 v239, v187, v191\n"
        "v_mfma_f32_16x16x32_f16 a[112:115], v[76:79], v[144:147], a[112:115]\n"
        "v_mfma_f32_16x16x32_f16 a[116:119], v[76:79], v[148:151], a[116:119]\n"
        "v_mfma_f32_16x16x32_f16 a[120:123], v[76:79], v[152:155], a[120:123]\n"
        "s_add_u32 s44, s44, 0x20000\n"
        "buffer_load_dwordx4 v[176:179], %[vwoff], %[rw], s44 offen nt\n"
        "v_mfma_f32_16x16x32_f16 a[124:127], v[76:79], v[156:159], a[124:127]\n"
        "v_mfma_f32_16x16x32_f16 a[144:147], v[80:83], v[144:147], a[144:147]\n"
        "v_mfma_f32_16x16x32_f16 a[148:151], v[80:83], v[148:151], a[148:151]\n"
        "v_mfma_f32_16x16x32_f16 a[152:155], v[80:83], v[152:155], a[152:155]\n"
        "v_mfma_f32_16x16x32_f16 a[156:159], v[80:83], v[156:159], a[156:159]\n"
        "s_add_u32 s44, s44, 0x20000\n"
        "buffer_load_dwordx4 v[180:183], %[vwoff], %[rw], s44 offen nt\n"
        "v_mfma_f32_16x16x32_f16 a[176:179], v[84:87], v[144:147], a[176:179]\n"
        "v_mfma_f32_16x16x32_f16 a[180:183], v[84:87], v[148:151], a[180:183]\n"
        "v_mfma_f32_16x16x32_f16 a[184:187], v[84:87], v[152:155], a[184:187]\n"
        "v_mfma_f32_16x16x32_f16 a[188:191], v[84:87], v[156:159], a[188:191]\n"
        "v_mfma_f32_16x16x32_f16 a[208:211], v[88:91], v[144:147], a[208:211]\n"
        "s_add_u32 s44, s44, 0x20000\n"
        "buffer_load_dwordx4 v[184:187], %[vwoff], %[rw], s44 offen nt\n"
        "v_mfma_f32_16x16x32_f16 a[212:215], v[88:91], v[148:151], a[212:215]\n"
        "v_mfma_f32_16x16x32_f16 a[216:219], v[88:91], v[152:155], a[216:219]\n"
        "v_mfma_f32_16x16x32_f16 a[220:223], v[88:91], v[156:159], a[220:223]\n"
        "v_mfma_f32_16x16x32_f16 a[240:243], v[92:95], v[144:147], a[240:243]\n"
        "v_mfma_f32_16x16x32_f16 a[244:247], v[92:95], v[148:151], a[244:247]\n"
        "s_add_u32 s44, s44, 0x20000\n"
        "buffer_load_dwordx4 v[188:191], %[vwoff], %[rw], s44 offen nt\n"
        "v_mfma_f32_16x16x32_f16 a[248:251], v[92:95], v[152:155], a[248:251]\n"
        "s_add_u32 s43, s43, 0x800000\n"
        "s_and_b32 s43, s43, 0x1fffffff\n"
        "v_mfma_f32_16x16x32_f16 a[252:255], v[92:95], v[156:159], a[252:255]\n"
        "s_waitcnt lgkmcnt(0)\n"
        "s_waitcnt vmcnt(40)\n"
        "v_mfma_f32_16x16x32_f16 a[0:3], v[96:99], v[128:131], a[0:3]\n"
        "ds_read_b128 v[144:147], %[vlr1] offset:24576\n"
        "v_mfma_f32_16x16x32_f16 a[4:7], v[96:99], v[132:135], a[4:7]\n"
        "ds_read_b128 v[148:151], %[vlr1] offset:26624\n"
        "v_mfma_f32_16x16x32_f16 a[8:11], v[96:99], v[136:139], a[8:11]\n"
        "ds_read_b128 v[152:155], %[vlr1] offset:28672\n"
        "v_mfma_f32_16x16x32_f16 a[12:15], v[96:99], v[140:143], a[12:15]\n"
        "ds_read_b128 v[156:159], %[vlr1] offset:30720\n"
        "buffer_load_dwordx4 v[72:75], %[vxoff], s[52:55], s42 offen offset:2048 sc1\n"
        "v_mfma_f32_16x16x32_f16 a[32:35], v[100:103], v[128:131], a[32:35]\n"
        "ds_write_b128 %[vlw], v[224:227] offset:0\n"
        "v_mfma_f32_16x16x32_f16 a[36:39], v[100:103], v[132:135], a[36:39]\n"
        "v_mfma_f32_16x16x32_f16 a[40:43], v[100:103], v[136:139], a[40:43]\n"
        "ds_write_b128 %[vlw], v[228:231] offset:4096\n"
        "v_mfma_f32_16x16x32_f16 a[44:47], v[100:103], v[140:143], a[44:47]\n"
        "s_waitcnt vmcnt(39)\n"
        "v_mfma_f32_16x16x32_f16 a[64:67], v[104:107], v[128:131], a[64:67]\n"
        "ds_write_b128 %[vlw], v[232:235] offset:8192\n"
        "buffer_load_dwordx4 v[76:79], %[vxoff], s[52:55], s42 offen offset:3072 sc1\n"
        "v_mfma_f32_16x16x32_f16 a[68:71], v[104:107], v[132:135], a[68:71]\n"
        "v_mfma_f32_16x16x32_f16 a[72:75], v[104:107], v[136:139], a[72:75]\n"
        "ds_write_b128 %[vlw], v[236:239] offset:12288\n"
        "v_mfma_f32_16x16x32_f16 a[76:79], v[104:107], v[140:143], a[76:79]\n"
        "v_mfma_f32_16x16x32_f16 a[96:99], v[108:111], v[128:131], a[96:99]\n"
        "v_mfma_f32_16x16x32_f16 a[100:103], v[108:111], v[132:135], a[100:103]\n"
        "buffer_load_dwordx4 v[80:83], %[vxoff], s[52:55], s88 offen offset:0 sc1\n"
        "v_mfma_f32_16x16x32_f16 a[104:107], v[108:111], v[136:139], a[104:107]\n"
        "v_mfma_f32_16x16x32_f16 a[108:111], v[108:111], v[140:143], a[108:111]\n"
        "s_waitcnt vmcnt(39)\n"
        "v_mfma_f32_16x16x32_f16 a[128:131], v[112:115], v[128:131], a[128:131]\n"
        "v_mfma_f32_16x16x32_f16 a[132:135], v[112:115], v[132:135], a[132:135]\n"
        "v_mfma_f32_16x16x32_f16 a[136:139], v[112:115], v[136:139], a[136:139]\n"
        "buffer_load_dwordx4 v[84:87], %[vxoff], s[52:55], s88 offen offset:1024 sc1\n"
        "v_mfma_f32_16x16x32_f16 a[140:143], v[112:115], v[140:143], a[140:143]\n"
        "v_mfma_f32_16x16x32_f16 a[160:163], v[116:119], v[128:131], a[160:163]\n"
        "v_mfma_f32_16x16x32_f16 a[164:167], v[116:119], v[132:135], a[164:167]\n"
        "v_mfma_f32_16x16x32_f16 a[168:171], v[116:119], v[136:139], a[168:171]\n"
        "v_mfma_f32_16x16x32_f16 a[172:175], v[116:119], v[140:143], a[172:175]\n"
        "buffer_load_dwordx4 v[88:91], %[vxoff], s[52:55], s88 offen offset:2048 sc1\n"
        "s_waitcnt vmcnt(39)\n"
        "v_mfma_f32_16x16x32_f16 a[192:195], v[120:123], v[128:131], a[192:195]\n"
        "v_mfma_f32_16x16x32_f16 a[196:199], v[120:123], v[132:135], a[196:199]\n"
        "v_mfma_f32_16x16x32_f16 a[200:203], v[120:123], v[136:139], a[200:203]\n"
        "v_mfma_f32_16x16x32_f16 a[204:207], v[120:123], v[140:143], a[204:207]\n"
        "v_mfma_f32_16x16x32_f16 a[224:227], v[124:127], v[128:131], a[224:227]\n"
        "buffer_load_dwordx4 v[92:95], %[vxoff], s[52:55], s88 offen offset:3072 sc1\n"
        "v_mfma_f32_16x16x32_f16 a[228:231], v[124:127], v[132:135], a[228:231]\n"
        "s_add_u32 s42, s42, 0x10000\n"
        "s_and_b32 s42, s42, 0x3fffff\n"
        "s_add_u32 s88, s42, 0x1000\n"
        "v_mfma_f32_16x16x32_f16 a[232:235], v[124:127], v[136:139], a[232:235]\n"
        "v_mfma_f32_16x16x32_f16 a[236:239], v[124:127], v[140:143], a[236:239]\n"
        "s_waitcnt lgkmcnt(0)\n"
        "s_barrier\n"
        "v_mfma_f32_16x16x32_f16 a[16:19], v[96:99], v[144:147], a[16:19]\n"
        "ds_read_b128 v[128:131], %[vlr0] offset:0\n"
        "v_mfma_f32_16x16x32_f16 a[20:23], v[96:99], v[148:151], a[20:23]\n"
        "ds_read_b128 v[132:135], %[vlr0] offset:2048\n"
        "v_mfma_f32_16x16x32_f16 a[24:27], v[96:99], v[152:155], a[24:27]\n"
        "ds_read_b128 v[136:139], %[vlr0] offset:4096\n"
        "v_mfma_f32_16x16x32_f16 a[28:31], v[96:99], v[156:159], a[28:31]\n"
        "ds_read_b128 v[140:143], %[vlr0] offset:6144\n"
        "v_mfma_f32_16x16x32_f16 a[48:51], v[100:103], v[144:147], a[48:51]\n"
        "buffer_load_dwordx4 v[96:99], %[vxoff], s[52:55], s58 offen offset:0 sc1\n"
        "v_mfma_f32_16x16x32_f16 a[52:55], v[100:103], v[148:151], a[52:55]\n"
        "v_mfma_f32_16x16x32_f16 a[56:59], v[100:103], v[152:155], a[56:59]\n"
        "v_mfma_f32_16x16x32_f16 a[60:63], v[100:103], v[156:159], a[60:63]\n"
        "v_mfma_f32_16x16x32_f16 a[80:83], v[104:107], v[144:147], a[80:83]\n"
        "v_mfma_f32_16x16x32_f16 a[84:87], v[104:107], v[148:151], a[84:87]\n"
        "buffer_load_dwordx4 v[100:103], %[vxoff], s[52:55], s58 offen offset:1024 sc1\n"
        "v_mfma_f32_16x16x32_f16 a[88:91], v[104:107], v[152:155], a[88:91]\n"
        "v_mfma_f32_16x16x32_f16 a[92:95], v[104:107], v[156:159], a[92:95]\n"
        "v_mfma_f32_16x16x32_f16 a[112:115], v[108:111], v[144:147], a[112:115]\n"
        "v_mfma_f32_16x16x32_f16 a[116:119], v[108:111], v[148:151], a[116:119]\n"
        "v_mfma_f32_16x16x32_f16 a[120:123], v[108:111], v[152:155], a[120:123]\n"
        "buffer_load_dwordx4 v[104:107], %[vxoff], s[52:55], s58 offen offset:2048 sc1\n"
        "v_mfma_f32_16x16x32_f16 a[124:127], v[108:111], v[156:159], a[124:127]\n"
        "v_mfma_f32_16x16x32_f16 a[144:147], v[112:115], v[144:147], a[144:147]\n"
        "v_mfma_f32_16x16x32_f16 a[148:151], v[112:115], v[148:151], a[148:151]\n"
        "v_mfma_f32_16x16x32_f16 a[152:155], v[112:115], v[152:155], a[152:155]\n"
        "v_mfma_f32_16x16x32_f16 a[156:159], v[112:115], v[156:159], a[156:159]\n"
        "buffer_load_dwordx4 v[108:111], %[vxoff], s[52:55], s58 offen offset:3072 sc1\n"
        "v_mfma_f32_16x16x32_f16 a[176:179], v[116:119], v[144:147], a[176:179]\n"
        "v_mfma_f32_16x16x32_f16 a[180:183], v[116:119], v[148:151], a[180:183]\n"
        "v_mfma_f32_16x16x32_f16 a[184:187], v[116:119], v[152:155], a[184:187]\n"
        "v_mfma_f32_16x16x32_f16 a[188:191], v[116:119], v[156:159], a[188:191]\n"
        "v_mfma_f32_16x16x32_f16 a[208:211], v[120:123], v[144:147], a[208:211]\n"
        "buffer_load_dwordx4 v[112:115], %[vxoff], s[52:55], s89 offen offset:0 sc1\n"
        "v_mfma_f32_16x16x32_f16 a[212:215], v[120:123], v[148:151], a[212:215]\n"
        "v_mfma_f32_16x16x32_f16 a[216:219], v[120:123], v[152:155], a[216:219]\n"
        "v_mfma_f32_16x16x32_f16 a[220:223], v[120:123], v[156:159], a[220:223]\n"
        "v_mfma_f32_16x16x32_f16 a[240:243], v[124:127], v[144:147], a[240:243]\n"
        "v_mfma_f32_16x16x32_f16 a[244:247], v[124:127], v[148:151], a[244:247]\n"
        "buffer_load_dwordx4 v[116:119], %[vxoff], s[52:55], s89 offen offset:1024 sc1\n"
        "v_mfma_f32_16x16x32_f16 a[248:251], v[124:127], v[152:155], a[248:251]\n"
        "v_mfma_f32_16x16x32_f16 a[252:255], v[124:127], v[156:159], a[252:255]\n"
        "s_add_u32 s45, s45, 1\n"
        "s_cmp_le_u32 s45, 29\n"
        "s_cbranch_scc1 L_loop_%=\n"
        "; ---- kstep\n"
        "s_waitcnt lgkmcnt(0)\n"
        "s_waitcnt vmcnt(40)\n"
        "v_mfma_f32_16x16x32_f16 a[0:3], v[0:3], v[128:131], a[0:3]\n"
        "ds_read_b128 v[144:147], %[vlr0] offset:8192\n"
        "v_mfma_f32_16x16x32_f16 a[4:7], v[0:3], v[132:135], a[4:7]\n"
        "ds_read_b128 v[148:151], %[vlr0] offset:10240\n"
        "v_mfma_f32_16x16x32_f16 a[8:11], v[0:3], v[136:139], a[8:11]\n"
        "ds_read_b128 v[152:155], %[vlr0] offset:12288\n"
        "v_mfma_f32_16x16x32_f16 a[12:15], v[0:3], v[140:143], a[12:15]\n"
        "ds_read_b128 v[156:159], %[vlr0] offset:14336\n"
        "v_mfma_f32_16x16x32_f16 a[32:35], v[4:7], v[128:131], a[32:35]\n"
        "buffer_load_dwordx4 v[120:123], %[vxoff], s[52:55], s89 offen offset:2048 sc1\n"
        "v_mfma_f32_16x16x32_f16 a[36:39], v[4:7], v[132:135], a[36:39]\n"
        "v_cvt_pk_f16_f32 v224, v192, v196\n"
        "v_mfma_f32_16x16x32_f16 a[40:43], v[4:7], v[136:139], a[40:43]\n"
        "v_cvt_pk_f16_f32 v228, v193, v197\n"
        "v_mfma_f32_16x16x32_f16 a[44:47], v[4:7], v[140:143], a[44:47]\n"
        "v_cvt_pk_f16_f32 v232, v194, v198\n"
        "s_waitcnt vmcnt(35)\n"
        "v_mfma_f32_16x16x32_f16 a[64:67], v[8:11], v[128:131], a[64:67]\n"
        "v_cvt_pk_f16_f32 v236, v195, v199\n"
        "v_mfma_f32_16x16x32_f16 a[68:71], v[8:11], v[132:135], a[68:71]\n"
        "buffer_load_dwordx4 v[124:127], %[vxoff], s[52:55], s89 offen offset:3072 sc1\n"
        "v_cvt_pk_f16_f32 v225, v200, v204\n"
        "v_mfma_f32_16x16x32_f16 a[72:75], v[8:11], v[136:139], a[72:75]\n"
        "s_add_u32 s58, s58, 0x10000\n"
        "s_and_b32 s58, s58, 0x3fffff\n"
        "s_add_u32 s89, s58, 0x1000\n"
        "v_cvt_pk_f16_f32 v229, v201, v205\n"
        "v_mfma_f32_16x16x32_f16 a[76:79], v[8:11], v[140:143], a[76:79]\n"
        "v_cvt_pk_f16_f32 v233, v202, v206\n"
        "v_mfma_f32_16x16x32_f16 a[96:99], v[12:15], v[128:131], a[96:99]\n"
        "v_cvt_pk_f16_f32 v237, v203, v207\n"
        "v_mfma_f32_16x16x32_f16 a[100:103], v[12:15], v[132:135], a[100:103]\n"
        "v_mfma_f32_16x16x32_f16 a[104:107], v[12:15], v[136:139], a[104:107]\n"
        "s_mov_b32 s44, s43\n"
        "buffer_load_dwordx4 v[192:195], %[vwoff], %[rw], s44 offen nt\n"
        "v_mfma_f32_16x16x32_f16 a[108:111], v[12:15], v[140:143], a[108:111]\n"
        "s_waitcnt vmcnt(35)\n"
        "v_mfma_f32_16x16x32_f16 a[128:131], v[16:19], v[128:131], a[128:131]\n"
        "v_mfma_f32_16x16x32_f16 a[132:135], v[16:19], v[132:135], a[132:135]\n"
        "v_mfma_f32_16x16x32_f16 a[136:139], v[16:19], v[136:139], a[136:139]\n"
        "v_mfma_f32_16x16x32_f16 a[140:143], v[16:19], v[140:143], a[140:143]\n"
        "s_add_u32 s44, s44, 0x20000\n"
        "buffer_load_dwordx4 v[196:199], %[vwoff], %[rw], s44 offen nt\n"
        "v_mfma_f32_16x16x32_f16 a[160:163], v[20:23], v[128:131], a[160:163]\n"
        "v_mfma_f32_16x16x32_f16 a[164:167], v[20:23], v[132:135], a[164:167]\n"
        "v_mfma_f32_16x16x32_f16 a[168:171], v[20:23], v[136:139], a[168:171]\n"
        "v_mfma_f32_16x16x32_f16 a[172:175], v[20:23], v[140:143], a[172:175]\n"
        "s_waitcnt vmcnt(34)\n"
        "v_mfma_f32_16x16x32_f16 a[192:195], v[24:27], v[128:131], a[192:195]\n"
        "s_add_u32 s44, s44, 0x20000\n"
        "buffer_load_dwordx4 v[200:203], %[vwoff], %[rw], s44 offen nt\n"
        "v_mfma_f32_16x16x32_f16 a[196:199], v[24:27], v[132:135], a[196:199]\n"
        "v_mfma_f32_16x16x32_f16 a[200:203], v[24:27], v[136:139], a[200:203]\n"
        "v_mfma_f32_16x16x32_f16 a[204:207], v[24:27], v[140:143], a[204:207]\n"
        "v_mfma_f32_16x16x32_f16 a[224:227], v[28:31], v[128:131], a[224:227]\n"
        "v_mfma_f32_16x16x32_f16 a[228:231], v[28:31], v[132:135], a[228:231]\n"
        "s_add_u32 s44, s44, 0x20000\n"
        "buffer_load_dwordx4 v[204:207], %[vwoff], %[rw], s44 offen nt\n"
        "v_mfma_f32_16x16x32_f16 a[232:235], v[28:31], v[136:139], a[232:235]\n"
        "v_mfma_f32_16x16x32_f16 a[236:239], v[28:31], v[140:143], a[236:239]\n"
        "s_waitcnt lgkmcnt(0)\n"
        "v_mfma_f32_16x16x32_f16 a[16:19], v[0:3], v[144:147], a[16:19]\n"
        "ds_read_b128 v[128:131], %[vlr1] offset:0\n"
        "v_mfma_f32_16x16x32_f16 a[20:23], v[0:3], v[148:151], a[20:23]\n"
        "ds_read_b128 v[132:135], %[vlr1] offset:2048\n"
        "v_mfma_f32_16x16x32_f16 a[24:27], v[0:3], v[152:155], a[24:27]\n"
        "ds_read_b128 v[136:139], %[vlr1] offset:4096\n"
        "v_mfma_f32_16x16x32_f16 a[28:31], v[0:3], v[156:159], a[28:31]\n"
        "ds_read_b128 v[140:143], %[vlr1] offset:6144\n"
        "v_mfma_f32_16x16x32_f16 a[48:51], v[4:7], v[144:147], a[48:51]\n"
        "v_cvt_pk_f16_f32 v226, v208, v212\n"
        "v_mfma_f32_16x16x32_f16 a[52:55], v[4:7], v[148:151], a[52:55]\n"
        "v_cvt_pk_f16_f32 v230, v209, v213\n"
        "buffer_load_dwordx4 v[0:3], %[vxoff], s[52:55], s42 offen offset:0 sc1\n"
        "v_mfma_f32_16x16x32_f16 a[56:59], v[4:7], v[152:155], a[56:59]\n"
        "v_cvt_pk_f16_f32 v234, v210, v214\n"
        "v_mfma_f32_16x16x32_f16 a[60:63], v[4:7], v[156:159], a[60:63]\n"
        "v_cvt_pk_f16_f32 v238, v211, v215\n"
        "v_mfma_f32_16x16x32_f16 a[80:83], v[8:11], v[144:147], a[80:83]\n"
        "v_cvt_pk_f16_f32 v227, v216, v220\n"
        "v_mfma_f32_16x16x32_f16 a[84:87], v[8:11], v[148:151], a[84:87]\n"
        "v_cvt_pk_f16_f32 v231, v217, v221\n"
        "v_mfma_f32_16x16x32_f16 a[88:91], v[8:11], v[152:155], a[88:91]\n"
        "v_cvt_pk_f16_f32 v235, v218, v222\n"
        "buffer_load_dwordx4 v[4:7], %[vxoff], s[52:55], s42 offen offset:1024 sc1\n"
        "v_mfma_f32_16x16x32_f16 a[92:95], v[8:11], v[156:159], a[92:95]\n"
        "v_cvt_pk_f16_f32 v239, v219, v223\n"
        "v_mfma_f32_16x16x32_f16 a[112:115], v[12:15], v[144:147], a[112:115]\n"
        "v_mfma_f32_16x16x32_f16 a[116:119], v[12:15], v[148:151], a[116:119]\n"
        "v_mfma_f32_16x16x32_f16 a[120:123], v[12:15], v[152:155], a[120:123]\n"
        "s_add_u32 s44, s44, 0x20000\n"
        "buffer_load_dwordx4 v[208:211], %[vwoff], %[rw], s44 offen nt\n"
        "v_mfma_f32_16x16x32_f16 a[124:127], v[12:15], v[156:159], a[124:127]\n"
        "v_mfma_f32_16x16x32_f16 a[144:147], v[16:19], v[144:147], a[144:147]\n"
        "v_mfma_f32_16x16x32_f16 a[148:151], v[16:19], v[148:151], a[148:151]\n"
        "v_mfma_f32_16x16x32_f16 a[152:155], v[16:19], v[152:155], a[152:155]\n"
        "v_mfma_f32_16x16x32_f16 a[156:159], v[16:19], v[156:159], a[156:159]\n"
        "s_add_u32 s44, s44, 0x20000\n"
        "buffer_load_dwordx4 v[212:215], %[vwoff], %[rw], s44 offen nt\n"
        "v_mfma_f32_16x16x32_f16 a[176:179], v[20:23], v[144:147], a[176:179]\n"
        "v_mfma_f32_16x16x32_f16 a[180:183], v[20:23], v[148:151], a[180:183]\n"
        "v_mfma_f32_16x16x32_f16 a[184:187], v[20:23], v[152:155], a[184:187]\n"
        "v_mfma_f32_16x16x32_f16 a[188:191], v[20:23], v[156:159], a[188:191]\n"
        "v_mfma_f32_16x16x32_f16 a[208:211], v[24:27], v[144:147], a[208:211]\n"
        "s_add_u32 s44, s44, 0x20000\n"
        "buffer_load_dwordx4 v[216:219], %[vwoff], %[rw], s44 offen nt\n"
        "v_mfma_f32_16x16x32_f16 a[212:215], v[24:27], v[148:151], a[212:215]\n"
        "v_mfma_f32_16x16x32_f16 a[216:219], v[24:27], v[152:155], a[216:219]\n"
        "v_mfma_f32_16x16x32_f16 a[220:223], v[24:27], v[156:159], a[220:223]\n"
        "v_mfma_f32_16x16x32_f16 a[240:243], v[28:31], v[144:147], a[240:243]\n"
        "v_mfma_f32_16x16x32_f16 a[244:247], v[28:31], v[148:151], a[244:247]\n"
        "s_add_u32 s44, s44, 0x20000\n"
        "buffer_load_dwordx4 v[220:223], %[vwoff], %[rw], s44 offen nt\n"
        "v_mfma_f32_16x16x32_f16 a[248:251], v[28:31], v[152:155], a[248:251]\n"
        "s_add_u32 s43, s43, 0x800000\n"
        "s_and_b32 s43, s43, 0x1fffffff\n"
        "v_mfma_f32_16x16x32_f16 a[252:255], v[28:31], v[156:159], a[252:255]\n"
        "s_waitcnt lgkmcnt(0)\n"
        "s_waitcnt vmcnt(40)\n"
        "v_mfma_f32_16x16x32_f16 a[0:3], v[32:35], v[128:131], a[0:3]\n"
        "ds_read_b128 v[144:147], %[vlr1] offset:8192\n"
        "v_mfma_f32_16x16x32_f16 a[4:7], v[32:35], v[132:135], a[4:7]\n"
        "ds_read_b128 v[148:151], %[vlr1] offset:10240\n"
        "v_mfma_f32_16x16x32_f16 a[8:11], v[32:35], v[136:139], a[8:11]\n"
        "ds_read_b128 v[152:155], %[vlr1] offset:12288\n"
        "v_mfma_f32_16x16x32_f16 a[12:15], v[32:35], v[140:143], a[12:15]\n"
        "ds_read_b128 v[156:159], %[vlr1] offset:14336\n"
        "buffer_load_dwordx4 v[8:11], %[vxoff], s[52:55], s42 offen offset:2048 sc1\n"
        "v_mfma_f32_16x16x32_f16 a[32:35], v[36:39], v[128:131], a[32:35]\n"
        "ds_write_b128 %[vlw], v[224:227] offset:16384\n"
        "v_mfma_f32_16x16x32_f16 a[36:39], v[36:39], v[132:135], a[36:39]\n"
        "v_mfma_f32_16x16x32_f16 a[40:43], v[36:39], v[136:139], a[40:43]\n"
        "ds_write_b128 %[vlw], v[228:231] offset:20480\n"
        "v_mfma_f32_16x16x32_f16 a[44:47], v[36:39], v[140:143], a[44:47]\n"
        "s_waitcnt vmcnt(39)\n"
        "v_mfma_f32_16x16x32_f16 a[64:67], v[40:43], v[128:131], a[64:67]\n"
        "ds_write_b128 %[vlw], v[232:235] offset:24576\n"
        "buffer_load_dwordx4 v[12:15], %[vxoff], s[52:55], s42 offen offset:3072 sc1\n"
        "v_mfma_f32_16x16x32_f16 a[68:71], v[40:43], v[132:135], a[68:71]\n"
        "v_mfma_f32_16x16x32_f16 a[72:75], v[40:43], v[136:139], a[72:75]\n"
        "ds_write_b128 %[vlw], v[236:239] offset:28672\n"
        "v_mfma_f32_16x16x32_f16 a[76:79], v[40:43], v[140:143], a[76:79]\n"
        "v_mfma_f32_16x16x32_f16 a[96:99], v[44:47], v[128:131], a[96:99]\n"
        "v_mfma_f32_16x16x32_f16 a[100:103], v[44:47], v[132:135], a[100:103]\n"
        "buffer_load_dwordx4 v[16:19], %[vxoff], s[52:55], s88 offen offset:0 sc1\n"
        "v_mfma_f32_16x16x32_f16 a[104:107], v[44:47], v[136:139], a[104:107]\n"
        "v_mfma_f32_16x16x32_f16 a[108:111], v[44:47], v[140:143], a[108:111]\n"
        "s_waitcnt vmcnt(39)\n"
        "v_mfma_f32_16x16x32_f16 a[128:131], v[48:51], v[128:131], a[128:131]\n"
        "v_mfma_f32_16x16x32_f16 a[132:135], v[48:51], v[132:135], a[132:135]\n"
        "v_mfma_f32_16x16x32_f16 a[136:139], v[48:51], v[136:139], a[136:139]\n"
        "buffer_load_dwordx4 v[20:23], %[vxoff], s[52:55], s88 offen offset:1024 sc1\n"
        "v_mfma_f32_16x16x32_f16 a[140:143], v[48:51], v[140:143], a[140:143]\n"
        "v_mfma_f32_16x16x32_f16 a[160:163], v[52:55], v[128:131], a[160:163]\n"
        "v_mfma_f32_16x16x32_f16 a[164:167], v[52:55], v[132:135], a[164:167]\n"
        "v_mfma_f32_16x16x32_f16 a[168:171], v[52:55], v[136:139], a[168:171]\n"
        "v_mfma_f32_16x16x32_f16 a[172:175], v[52:55], v[140:143], a[172:175]\n"
        "buffer_load_dwordx4 v[24:27], %[vxoff], s[52:55], s88 offen offset:2048 sc1\n"
        "s_waitcnt vmcnt(39)\n"
        "v_mfma_f32_16x16x32_f16 a[192:195], v[56:59], v[128:131], a[192:195]\n"
        "v_mfma_f32_16x16x32_f16 a[196:199], v[56:59], v[132:135], a[196:199]\n"
        "v_mfma_f32_16x16x32_f16 a[200:203], v[56:59], v[136:139], a[200:203]\n"
        "v_mfma_f32_16x16x32_f16 a[204:207], v[56:59], v[140:143], a[204:207]\n"
        "v_mfma_f32_16x16x32_f16 a[224:227], v[60:63], v[128:131], a[224:227]\n"
        "buffer_load_dwordx4 v[28:31], %[vxoff], s[52:55], s88 offen offset:3072 sc1\n"
        "v_mfma_f32_16x16x32_f16 a[228:231], v[60:63], v[132:135], a[228:231]\n"
        "s_add_u32 s42, s42, 0x10000\n"
        "s_and_b32 s42, s42, 0x3fffff\n"
        "s_add_u32 s88, s42, 0x1000\n"
        "v_mfma_f32_16x16x32_f16 a[232:235], v[60:63], v[136:139], a[232:235]\n"
        "v_mfma_f32_16x16x32_f16 a[236:239], v[60:63], v[140:143], a[236:239]\n"
        "s_waitcnt lgkmcnt(0)\n"
        "s_barrier\n"
        "v_mfma_f32_16x16x32_f16 a[16:19], v[32:35], v[144:147], a[16:19]\n"
        "ds_read_b128 v[128:131], %[vlr0] offset:16384\n"
        "v_mfma_f32_16x16x32_f16 a[20:23], v[32:35], v[148:151], a[20:23]\n"
        "ds_read_b128 v[132:135], %[vlr0] offset:18432\n"
        "v_mfma_f32_16x16x32_f16 a[24:27], v[32:35], v[152:155], a[24:27]\n"
        "ds_read_b128 v[136:139], %[vlr0] offset:20480\n"
        "v_mfma_f32_16x16x32_f16 a[28:31], v[32:35], v[156:159], a[28:31]\n"
        "ds_read_b128 v[140:143], %[vlr0] offset:22528\n"
        "v_mfma_f32_16x16x32_f16 a[48:51], v[36:39], v[144:147], a[48:51]\n"
        "buffer_load_dwordx4 v[32:35], %[vxoff], s[52:55], s58 offen offset:0 sc1\n"
        "v_mfma_f32_16x16x32_f16 a[52:55], v[36:39], v[148:151], a[52:55]\n"
        "v_mfma_f32_16x16x32_f16 a[56:59], v[36:39], v[152:155], a[56:59]\n"
        "v_mfma_f32_16x16x32_f16 a[60:63], v[36:39], v[156:159], a[60:63]\n"
        "v_mfma_f32_16x16x32_f16 a[80:83], v[40:43], v[144:147], a[80:83]\n"
        "v_mfma_f32_16x16x32_f16 a[84:87], v[40:43], v[148:151], a[84:87]\n"
        "buffer_load_dwordx4 v[36:39], %[vxoff], s[52:55], s58 offen offset:1024 sc1\n"
        "v_mfma_f32_16x16x32_f16 a[88:91], v[40:43], v[152:155], a[88:91]\n"
        "v_mfma_f32_16x16x32_f16 a[92:95], v[40:43], v[156:159], a[92:95]\n"
        "v_mfma_f32_16x16x32_f16 a[112:115], v[44:47], v[144:147], a[112:115]\n"
        "v_mfma_f32_16x16x32_f16 a[116:119], v[44:47], v[148:151], a[116:119]\n"
        "v_mfma_f32_16x16x32_f16 a[120:123], v[44:47], v[152:155], a[120:123]\n"
        "buffer_load_dwordx4 v[40:43], %[vxoff], s[52:55], s58 offen offset:2048 sc1\n"
        "v_mfma_f32_16x16x32_f16 a[124:127], v[44:47], v[156:159], a[124:127]\n"
        "v_mfma_f32_16x16x32_f16 a[144:147], v[48:51], v[144:147], a[144:147]\n"
        "v_mfma_f32_16x16x32_f16 a[148:151], v[48:51], v[148:151], a[148:151]\n"
        "v_mfma_f32_16x16x32_f16 a[152:155], v[48:51], v[152:155], a[152:155]\n"
        "v_mfma_f32_16x16x32_f16 a[156:159], v[48:51], v[156:159], a[156:159]\n"
        "buffer_load_dwordx4 v[44:47], %[vxoff], s[52:55], s58 offen offset:3072 sc1\n"
        "v_mfma_f32_16x16x32_f16 a[176:179], v[52:55], v[144:147], a[176:179]\n"
        "v_mfma_f32_16x16x32_f16 a[180:183], v[52:55], v[148:151], a[180:183]\n"
        "v_mfma_f32_16x16x32_f16 a[184:187], v[52:55], v[152:155], a[184:187]\n"
        "v_mfma_f32_16x16x32_f16 a[188:191], v[52:55], v[156:159], a[188:191]\n"
        "v_mfma_f32_16x16x32_f16 a[208:211], v[56:59], v[144:147], a[208:211]\n"
        "buffer_load_dwordx4 v[48:51], %[vxoff], s[52:55], s89 offen offset:0 sc1\n"
        "v_mfma_f32_16x16x32_f16 a[212:215], v[56:59], v[148:151], a[212:215]\n"
        "v_mfma_f32_16x16x32_f16 a[216:219], v[56:59], v[152:155], a[216:219]\n"
        "v_mfma_f32_16x16x32_f16 a[220:223], v[56:59], v[156:159], a[220:223]\n"
        "v_mfma_f32_16x16x32_f16 a[240:243], v[60:63], v[144:147], a[240:243]\n"
        "v_mfma_f32_16x16x32_f16 a[244:247], v[60:63], v[148:151], a[244:247]\n"
        "buffer_load_dwordx4 v[52:55], %[vxoff], s[52:55], s89 offen offset:1024 sc1\n"
        "v_mfma_f32_16x16x32_f16 a[248:251], v[60:63], v[152:155], a[248:251]\n"
        "v_mfma_f32_16x16x32_f16 a[252:255], v[60:63], v[156:159], a[252:255]\n"
        "; ---- kstep\n"
        "s_waitcnt lgkmcnt(0)\n"
        "s_waitcnt vmcnt(40)\n"
        "v_mfma_f32_16x16x32_f16 a[0:3], v[64:67], v[128:131], a[0:3]\n"
        "ds_read_b128 v[144:147], %[vlr0] offset:24576\n"
        "v_mfma_f32_16x16x32_f16 a[4:7], v[64:67], v[132:135], a[4:7]\n"
        "ds_read_b128 v[148:151], %[vlr0] offset:26624\n"
        "v_mfma_f32_16x16x32_f16 a[8:11], v[64:67], v[136:139], a[8:11]\n"
        "ds_read_b128 v[152:155], %[vlr0] offset:28672\n"
        "v_mfma_f32_16x16x32_f16 a[12:15], v[64:67], v[140:143], a[12:15]\n"
        "ds_read_b128 v[156:159], %[vlr0] offset:30720\n"
        "v_mfma_f32_16x16x32_f16 a[32:35], v[68:71], v[128:131], a[32:35]\n"
        "buffer_load_dwordx4 v[56:59], %[vxoff], s[52:55], s89 offen offset:2048 sc1\n"
        "v_mfma_f32_16x16x32_f16 a[36:39], v[68:71], v[132:135], a[36:39]\n"
        "v_cvt_pk_f16_f32 v224, v160, v164\n"
        "v_mfma_f32_16x16x32_f16 a[40:43], v[68:71], v[136:139], a[40:43]\n"
        "v_cvt_pk_f16_f32 v228, v161, v165\n"
        "v_mfma_f32_16x16x32_f16 a[44:47], v[68:71], v[140:143], a[44:47]\n"
        "v_cvt_pk_f16_f32 v232, v162, v166\n"
        "s_waitcnt vmcnt(35)\n"
        "v_mfma_f32_16x16x32_f16 a[64:67], v[72:75], v[128:131], a[64:67]\n"
        "v_cvt_pk_f16_f32 v236, v163, v167\n"
        "v_mfma_f32_16x16x32_f16 a[68:71], v[72:75], v[132:135], a[68:71]\n"
        "buffer_load_dwordx4 v[60:63], %[vxoff], s[52:55], s89 offen offset:3072 sc1\n"
        "v_cvt_pk_f16_f32 v225, v168, v172\n"
        "v_mfma_f32_16x16x32_f16 a[72:75], v[72:75], v[136:139], a[72:75]\n"
        "s_add_u32 s58, s58, 0x10000\n"
        "s_and_b32 s58, s58, 0x3fffff\n"
        "s_add_u32 s89, s58, 0x1000\n"
        "v_cvt_pk_f16_f32 v229, v169, v173\n"
        "v_mfma_f32_16x16x32_f16 a[76:79], v[72:75], v[140:143], a[76:79]\n"
        "v_cvt_pk_f16_f32 v233, v170, v174\n"
        "v_mfma_f32_16x16x32_f16 a[96:99], v[76:79], v[128:131], a[96:99]\n"
        "v_cvt_pk_f16_f32 v237, v171, v175\n"
        "v_mfma_f32_16x16x32_f16 a[100:103], v[76:79], v[132:135], a[100:103]\n"
        "v_mfma_f32_16x16x32_f16 a[104:107], v[76:79], v[136:139], a[104:107]\n"
        "v_mfma_f32_16x16x32_f16 a[108:111], v[76:79], v[140:143], a[108:111]\n"
        "s_waitcnt vmcnt(34)\n"
        "v_mfma_f32_16x16x32_f16 a[128:131], v[80:83], v[128:131], a[128:131]\n"
        "v_mfma_f32_16x16x32_f16 a[132:135], v[80:83], v[132:135], a[132:135]\n"
        "v_mfma_f32_16x16x32_f16 a[136:139], v[80:83], v[136:139], a[136:139]\n"
        "v_mfma_f32_16x16x32_f16 a[140:143], v[80:83], v[140:143], a[140:143]\n"
        "v_mfma_f32_16x16x32_f16 a[160:163], v[84:87], v[128:131], a[160:163]\n"
        "v_mfma_f32_16x16x32_f16 a[164:167], v[84:87], v[132:135], a[164:167]\n"
        "v_mfma_f32_16x16x32_f16 a[168:171], v[84:87], v[136:139], a[168:171]\n"
        "v_mfma_f32_16x16x32_f16 a[172:175], v[84:87], v[140:143], a[172:175]\n"
        "s_waitcnt vmcnt(32)\n"
        "v_mfma_f32_16x16x32_f16 a[192:195], v[88:91], v[128:131], a[192:195]\n"
        "v_mfma_f32_16x16x32_f16 a[196:199], v[88:91], v[132:135], a[196:199]\n"
        "v_mfma_f32_16x16x32_f16 a[200:203], v[88:91], v[136:139], a[200:203]\n"
        "v_mfma_f32_16x16x32_f16 a[204:207], v[88:91], v[140:143], a[204:207]\n"
        "v_mfma_f32_16x16x32_f16 a[224:227], v[92:95], v[128:131], a[224:227]\n"
        "v_mfma_f32_16x16x32_f16 a[228:231], v[92:95], v[132:135], a[228:231]\n"
        "v_mfma_f32_16x16x32_f16 a[232:235], v[92:95], v[136:139], a[232:235]\n"
        "v_mfma_f32_16x16x32_f16 a[236:239], v[92:95], v[140:143], a[236:239]\n"
        "s_waitcnt lgkmcnt(0)\n"
        "v_mfma_f32_16x16x32_f16 a[16:19], v[64:67], v[144:147], a[16:19]\n"
        "ds_read_b128 v[128:131], %[vlr1] offset:16384\n"
        "v_mfma_f32_16x16x32_f16 a[20:23], v[64:67], v[148:151], a[20:23]\n"
        "ds_read_b128 v[132:135], %[vlr1] offset:18432\n"
        "v_mfma_f32_16x16x32_f16 a[24:27], v[64:67], v[152:155], a[24:27]\n"
        "ds_read_b128 v[136:139], %[vlr1] offset:20480\n"
        "v_mfma_f32_16x16x32_f16 a[28:31], v[64:67], v[156:159], a[28:31]\n"
        "ds_read_b128 v[140:143], %[vlr1] offset:22528\n"
        "v_mfma_f32_16x16x32_f16 a[48:51], v[68:71], v[144:147], a[48:51]\n"
        "v_cvt_pk_f16_f32 v226, v176, v180\n"
        "v_mfma_f32_16x16x32_f16 a[52:55], v[68:71], v[148:151], a[52:55]\n"
        "v_cvt_pk_f16_f32 v230, v177, v181\n"
        "buffer_load_dwordx4 v[64:67], %[vxoff], s[52:55], s42 offen offset:0 sc1\n"
        "v_mfma_f32_16x16x32_f16 a[56:59], v[68:71], v[152:155], a[56:59]\n"
        "v_cvt_pk_f16_f32 v234, v178, v182\n"
        "v_mfma_f32_16x16x32_f16 a[60:63], v[68:71], v[156:159], a[60:63]\n"
        "v_cvt_pk_f16_f32 v238, v179, v183\n"
        "v_mfma_f32_16x16x32_f16 a[80:83], v[72:75], v[144:147], a[80:83]\n"
        "v_cvt_pk_f16_f32 v227, v184, v188\n"
        "v_mfma_f32_16x16x32_f16 a[84:87], v[72:75], v[148:151], a[84:87]\n"
        "v_cvt_pk_f16_f32 v231, v185, v189\n"
        "v_mfma_f32_16x16x32_f16 a[88:91], v[72:75], v[152:155], a[88:91]\n"
        "v_cvt_pk_f16_f32 v235, v186, v190\n"
        "buffer_load_dwordx4 v[68:71], %[vxoff], s[52:55], s42 offen offset:1024 sc1\n"
        "v_mfma_f32_16x16x32_f16 a[92:95], v[72:75], v[156:159], a[92:95]\n"
        "v_cvt_pk_f16_f32 v239, v187, v191\n"
        "v_mfma_f32_16x16x32_f16 a[112:115], v[76:79], v[144:147], a[112:115]\n"
        "v_mfma_f32_16x16x32_f16 a[116:119], v[76:79], v[148:151], a[116:119]\n"
        "v_mfma_f32_16x16x32_f16 a[120:123], v[76:79], v[152:155], a[120:123]\n"
        "v_mfma_f32_16x16x32_f16 a[124:127], v[76:79], v[156:159], a[124:127]\n"
        "v_mfma_f32_16x16x32_f16 a[144:147], v[80:83], v[144:147], a[144:147]\n"
        "v_mfma_f32_16x16x32_f16 a[148:151], v[80:83], v[148:151], a[148:151]\n"
        "v_mfma_f32_16x16x32_f16 a[152:155], v[80:83], v[152:155], a[152:155]\n"
        "v_mfma_f32_16x16x32_f16 a[156:159], v[80:83], v[156:159], a[156:159]\n"
        "v_mfma_f32_16x16x32_f16 a[176:179], v[84:87], v[144:147], a[176:179]\n"
        "v_mfma_f32_16x16x32_f16 a[180:183], v[84:87], v[148:151], a[180:183]\n"
        "v_mfma_f32_16x16x32_f16 a[184:187], v[84:87], v[152:155], a[184:187]\n"
        "v_mfma_f32_16x16x32_f16 a[188:191], v[84:87], v[156:159], a[188:191]\n"
        "v_mfma_f32_16x16x32_f16 a[208:211], v[88:91], v[144:147], a[208:211]\n"
        "v_mfma_f32_16x16x32_f16 a[212:215], v[88:91], v[148:151], a[212:215]\n"
        "v_mfma_f32_16x16x32_f16 a[216:219], v[88:91], v[152:155], a[216:219]\n"
        "v_mfma_f32_16x16x32_f16 a[220:223], v[88:91], v[156:159], a[220:223]\n"
        "v_mfma_f32_16x16x32_f16 a[240:243], v[92:95], v[144:147], a[240:243]\n"
        "v_mfma_f32_16x16x32_f16 a[244:247], v[92:95], v[148:151], a[244:247]\n"
        "v_mfma_f32_16x16x32_f16 a[248:251], v[92:95], v[152:155], a[248:251]\n"
        "v_mfma_f32_16x16x32_f16 a[252:255], v[92:95], v[156:159], a[252:255]\n"
        "s_waitcnt lgkmcnt(0)\n"
        "s_waitcnt vmcnt(32)\n"
        "v_mfma_f32_16x16x32_f16 a[0:3], v[96:99], v[128:131], a[0:3]\n"
        "ds_read_b128 v[144:147], %[vlr1] offset:24576\n"
        "v_mfma_f32_16x16x32_f16 a[4:7], v[96:99], v[132:135], a[4:7]\n"
        "ds_read_b128 v[148:151], %[vlr1] offset:26624\n"
        "v_mfma_f32_16x16x32_f16 a[8:11], v[96:99], v[136:139], a[8:11]\n"
        "ds_read_b128 v[152:155], %[vlr1] offset:28672\n"
        "v_mfma_f32_16x16x32_f16 a[12:15], v[96:99], v[140:143], a[12:15]\n"
        "ds_read_b128 v[156:159], %[vlr1] offset:30720\n"
        "buffer_load_dwordx4 v[72:75], %[vxoff], s[52:55], s42 offen offset:2048 sc1\n"
        "v_mfma_f32_16x16x32_f16 a[32:35], v[100:103], v[128:131], a[32:35]\n"
        "ds_write_b128 %[vlw], v[224:227] offset:0\n"
        "v_mfma_f32_16x16x32_f16 a[36:39], v[100:103], v[132:135], a[36:39]\n"
        "v_mfma_f32_16x16x32_f16 a[40:43], v[100:103], v[136:139], a[40:43]\n"
        "ds_write_b128 %[vlw], v[228:231] offset:4096\n"
        "v_mfma_f32_16x16x32_f16 a[44:47], v[100:103], v[140:143], a[44:47]\n"
        "s_waitcnt vmcnt(31)\n"
        "v_mfma_f32_16x16x32_f16 a[64:67], v[104:107], v[128:131], a[64:67]\n"
        "ds_write_b128 %[vlw], v[232:235] offset:8192\n"
        "buffer_load_dwordx4 v[76:79], %[vxoff], s[52:55], s42 offen offset:3072 sc1\n"
        "v_mfma_f32_16x16x32_f16 a[68:71], v[104:107], v[132:135], a[68:71]\n"
        "v_mfma_f32_16x16x32_f16 a[72:75], v[104:107], v[136:139], a[72:75]\n"
        "ds_write_b128 %[vlw], v[236:239] offset:12288\n"
        "v_mfma_f32_16x16x32_f16 a[76:79], v[104:107], v[140:143], a[76:79]\n"
        "v_mfma_f32_16x16x32_f16 a[96:99], v[108:111], v[128:131], a[96:99]\n"
        "v_mfma_f32_16x16x32_f16 a[100:103], v[108:111], v[132:135], a[100:103]\n"
        "buffer_load_dwordx4 v[80:83], %[vxoff], s[52:55], s88 offen offset:0 sc1\n"
        "v_mfma_f32_16x16x32_f16 a[104:107], v[108:111], v[136:139], a[104:107]\n"
        "v_mfma_f32_16x16x32_f16 a[108:111], v[108:111], v[140:143], a[108:111]\n"
        "s_waitcnt vmcnt(31)\n"
        "v_mfma_f32_16x16x32_f16 a[128:131], v[112:115], v[128:131], a[128:131]\n"
        "v_mfma_f32_16x16x32_f16 a[132:135], v[112:115], v[132:135], a[132:135]\n"
        "v_mfma_f32_16x16x32_f16 a[136:139], v[112:115], v[136:139], a[136:139]\n"
        "buffer_load_dwordx4 v[84:87], %[vxoff], s[52:55], s88 offen offset:1024 sc1\n"
        "v_mfma_f32_16x16x32_f16 a[140:143], v[112:115], v[140:143], a[140:143]\n"
        "v_mfma_f32_16x16x32_f16 a[160:163], v[116:119], v[128:131], a[160:163]\n"
        "v_mfma_f32_16x16x32_f16 a[164:167], v[116:119], v[132:135], a[164:167]\n"
        "v_mfma_f32_16x16x32_f16 a[168:171], v[116:119], v[136:139], a[168:171]\n"
        "v_mfma_f32_16x16x32_f16 a[172:175], v[116:119], v[140:143], a[172:175]\n"
        "buffer_load_dwordx4 v[88:91], %[vxoff], s[52:55], s88 offen offset:2048 sc1\n"
        "s_waitcnt vmcnt(31)\n"
        "v_mfma_f32_16x16x32_f16 a[192:195], v[120:123], v[128:131], a[192:195]\n"
        "v_mfma_f32_16x16x32_f16 a[196:199], v[120:123], v[132:135], a[196:199]\n"
        "v_mfma_f32_16x16x32_f16 a[200:203], v[120:123], v[136:139], a[200:203]\n"
        "v_mfma_f32_16x16x32_f16 a[204:207], v[120:123], v[140:143], a[204:207]\n"
        "v_mfma_f32_16x16x32_f16 a[224:227], v[124:127], v[128:131], a[224:227]\n"
        "buffer_load_dwordx4 v[92:95], %[vxoff], s[52:55], s88 offen offset:3072 sc1\n"
        "v_mfma_f32_16x16x32_f16 a[228:231], v[124:127], v[132:135], a[228:231]\n"
        "s_add_u32 s42, s42, 0x10000\n"
        "s_and_b32 s42, s42, 0x3fffff\n"
        "s_add_u32 s88, s42, 0x1000\n"
        "v_mfma_f32_16x16x32_f16 a[232:235], v[124:127], v[136:139], a[232:235]\n"
        "v_mfma_f32_16x16x32_f16 a[236:239], v[124:127], v[140:143], a[236:239]\n"
        "s_waitcnt lgkmcnt(0)\n"
        "s_barrier\n"
        "v_mfma_f32_16x16x32_f16 a[16:19], v[96:99], v[144:147], a[16:19]\n"
        "ds_read_b128 v[128:131], %[vlr0] offset:0\n"
        "v_mfma_f32_16x16x32_f16 a[20:23], v[96:99], v[148:151], a[20:23]\n"
        "ds_read_b128 v[132:135], %[vlr0] offset:2048\n"
        "v_mfma_f32_16x16x32_f16 a[24:27], v[96:99], v[152:155], a[24:27]\n"
        "ds_read_b128 v[136:139], %[vlr0] offset:4096\n"
        "v_mfma_f32_16x16x32_f16 a[28:31], v[96:99], v[156:159], a[28:31]\n"
        "ds_read_b128 v[140:143], %[vlr0] offset:6144\n"
        "v_mfma_f32_16x16x32_f16 a[48:51], v[100:103], v[144:147], a[48:51]\n"
        "buffer_load_dwordx4 v[96:99], %[vxoff], s[52:55], s58 offen offset:0 sc1\n"
        "v_mfma_f32_16x16x32_f16 a[52:55], v[100:103], v[148:151], a[52:55]\n"
        "v_mfma_f32_16x16x32_f16 a[56:59], v[100:103], v[152:155], a[56:59]\n"
        "v_mfma_f32_16x16x32_f16 a[60:63], v[100:103], v[156:159], a[60:63]\n"
        "v_mfma_f32_16x16x32_f16 a[80:83], v[104:107], v[144:147], a[80:83]\n"
        "v_mfma_f32_16x16x32_f16 a[84:87], v[104:107], v[148:151], a[84:87]\n"
        "buffer_load_dwordx4 v[100:103], %[vxoff], s[52:55], s58 offen offset:1024 sc1\n"
        "v_mfma_f32_16x16x32_f16 a[88:91], v[104:107], v[152:155], a[88:91]\n"
        "buffer_load_dword v243, v241, %[rc], 0 offen sc1\n"
        "v_mfma_f32_16x16x32_f16 a[92:95], v[104:107], v[156:159], a[92:95]\n"
        "v_mfma_f32_16x16x32_f16 a[112:115], v[108:111], v[144:147], a[112:115]\n"
        "v_mfma_f32_16x16x32_f16 a[116:119], v[108:111], v[148:151], a[116:119]\n"
        "v_mfma_f32_16x16x32_f16 a[120:123], v[108:111], v[152:155], a[120:123]\n"
        "buffer_load_dwordx4 v[104:107], %[vxoff], s[52:55], s58 offen offset:2048 sc1\n"
        "v_mfma_f32_16x16x32_f16 a[124:127], v[108:111], v[156:159], a[124:127]\n"
        "v_mfma_f32_16x16x32_f16 a[144:147], v[112:115], v[144:147], a[144:147]\n"
        "v_mfma_f32_16x16x32_f16 a[148:151], v[112:115], v[148:151], a[148:151]\n"
        "v_mfma_f32_16x16x32_f16 a[152:155], v[112:115], v[152:155], a[152:155]\n"
        "v_mfma_f32_16x16x32_f16 a[156:159], v[112:115], v[156:159], a[156:159]\n"
        "buffer_load_dwordx4 v[108:111], %[vxoff], s[52:55], s58 offen offset:3072 sc1\n"
        "v_mfma_f32_16x16x32_f16 a[176:179], v[116:119], v[144:147], a[176:179]\n"
        "v_mfma_f32_16x16x32_f16 a[180:183], v[116:119], v[148:151], a[180:183]\n"
        "v_mfma_f32_16x16x32_f16 a[184:187], v[116:119], v[152:155], a[184:187]\n"
        "v_mfma_f32_16x16x32_f16 a[188:191], v[116:119], v[156:159], a[188:191]\n"
        "v_mfma_f32_16x16x32_f16 a[208:211], v[120:123], v[144:147], a[208:211]\n"
        "buffer_load_dwordx4 v[112:115], %[vxoff], s[52:55], s89 offen offset:0 sc1\n"
        "v_mfma_f32_16x16x32_f16 a[212:215], v[120:123], v[148:151], a[212:215]\n"
        "v_mfma_f32_16x16x32_f16 a[216:219], v[120:123], v[152:155], a[216:219]\n"
        "v_mfma_f32_16x16x32_f16 a[220:223], v[120:123], v[156:159], a[220:223]\n"
        "v_mfma_f32_16x16x32_f16 a[240:243], v[124:127], v[144:147], a[240:243]\n"
        "v_mfma_f32_16x16x32_f16 a[244:247], v[124:127], v[148:151], a[244:247]\n"
        "buffer_load_dwordx4 v[116:119], %[vxoff], s[52:55], s89 offen offset:1024 sc1\n"
        "v_mfma_f32_16x16x32_f16 a[248:251], v[124:127], v[152:155], a[248:251]\n"
        "v_mfma_f32_16x16x32_f16 a[252:255], v[124:127], v[156:159], a[252:255]\n"
        "s_add_u32 s45, s45, 1\n"
        "; ---- kstep\n"
        "s_waitcnt lgkmcnt(0)\n"
        "s_waitcnt vmcnt(33)\n"
        "v_mfma_f32_16x16x32_f16 a[0:3], v[0:3], v[128:131], a[0:3]\n"
        "ds_read_b128 v[144:147], %[vlr0] offset:8192\n"
        "v_mfma_f32_16x16x32_f16 a[4:7], v[0:3], v[132:135], a[4:7]\n"
        "ds_read_b128 v[148:151], %[vlr0] offset:10240\n"
        "v_mfma_f32_16x16x32_f16 a[8:11], v[0:3], v[136:139], a[8:11]\n"
        "ds_read_b128 v[152:155], %[vlr0] offset:12288\n"
        "v_mfma_f32_16x16x32_f16 a[12:15], v[0:3], v[140:143], a[12:15]\n"
        "ds_read_b128 v[156:159], %[vlr0] offset:14336\n"
        "v_mfma_f32_16x16x32_f16 a[32:35], v[4:7], v[128:131], a[32:35]\n"
        "buffer_load_dwordx4 v[120:123], %[vxoff], s[52:55], s89 offen offset:2048 sc1\n"
        "v_mfma_f32_16x16x32_f16 a[36:39], v[4:7], v[132:135], a[36:39]\n"
        "v_cvt_pk_f16_f32 v224, v192, v196\n"
        "v_mfma_f32_16x16x32_f16 a[40:43], v[4:7], v[136:139], a[40:43]\n"
        "v_cvt_pk_f16_f32 v228, v193, v197\n"
        "v_mfma_f32_16x16x32_f16 a[44:47], v[4:7], v[140:143], a[44:47]\n"
        "v_cvt_pk_f16_f32 v232, v194, v198\n"
        "s_waitcnt vmcnt(28)\n"
        "v_mfma_f32_16x16x32_f16 a[64:67], v[8:11], v[128:131], a[64:67]\n"
        "v_cvt_pk_f16_f32 v236, v195, v199\n"
        "v_mfma_f32_16x16x32_f16 a[68:71], v[8:11], v[132:135], a[68:71]\n"
        "buffer_load_dwordx4 v[124:127], %[vxoff], s[52:55], s89 offen offset:3072 sc1\n"
        "v_cvt_pk_f16_f32 v225, v200, v204\n"
        "v_mfma_f32_16x16x32_f16 a[72:75], v[8:11], v[136:139], a[72:75]\n"
        "s_add_u32 s58, s58, 0x10000\n"
        "s_and_b32 s58, s58, 0x3fffff\n"
        "s_add_u32 s89, s58, 0x1000\n"
        "v_cvt_pk_f16_f32 v229, v201, v205\n"
        "v_mfma_f32_16x16x32_f16 a[76:79], v[8:11], v[140:143], a[76:79]\n"
        "v_cvt_pk_f16_f32 v233, v202, v206\n"
        "v_mfma_f32_16x16x32_f16 a[96:99], v[12:15], v[128:131], a[96:99]\n"
        "v_cvt_pk_f16_f32 v237, v203, v207\n"
        "v_mfma_f32_16x16x32_f16 a[100:103], v[12:15], v[132:135], a[100:103]\n"
        "v_mfma_f32_16x16x32_f16 a[104:107], v[12:15], v[136:139], a[104:107]\n"
        "v_mfma_f32_16x16x32_f16 a[108:111], v[12:15], v[140:143], a[108:111]\n"
        "s_waitcnt vmcnt(27)\n"
        "v_mfma_f32_16x16x32_f16 a[128:131], v[16:19], v[128:131], a[128:131]\n"
        "v_mfma_f32_16x16x32_f16 a[132:135], v[16:19], v[132:135], a[132:135]\n"
        "v_mfma_f32_16x16x32_f16 a[136:139], v[16:19], v[136:139], a[136:139]\n"
        "v_mfma_f32_16x16x32_f16 a[140:143], v[16:19], v[140:143], a[140:143]\n"
        "v_mfma_f32_16x16x32_f16 a[160:163], v[20:23], v[128:131], a[160:163]\n"
        "v_mfma_f32_16x16x32_f16 a[164:167], v[20:23], v[132:135], a[164:167]\n"
        "v_mfma_f32_16x16x32_f16 a[168:171], v[20:23], v[136:139], a[168:171]\n"
        "v_mfma_f32_16x16x32_f16 a[172:175], v[20:23], v[140:143], a[172:175]\n"
        "s_waitcnt vmcnt(25)\n"
        "v_mfma_f32_16x16x32_f16 a[192:195], v[24:27], v[128:131], a[192:195]\n"
        "v_mfma_f32_16x16x32_f16 a[196:199], v[24:27], v[132:135], a[196:199]\n"
        "v_mfma_f32_16x16x32_f16 a[200:203], v[24:27], v[136:139], a[200:203]\n"
        "v_mfma_f32_16x16x32_f16 a[204:207], v[24:27], v[140:143], a[204:207]\n"
        "v_mfma_f32_16x16x32_f16 a[224:227], v[28:31], v[128:131], a[224:227]\n"
        "v_mfma_f32_16x16x32_f16 a[228:231], v[28:31], v[132:135], a[228:231]\n"
        "v_mfma_f32_16x16x32_f16 a[232:235], v[28:31], v[136:139], a[232:235]\n"
        "v_mfma_f32_16x16x32_f16 a[236:239], v[28:31], v[140:143], a[236:239]\n"
        "s_waitcnt lgkmcnt(0)\n"
        "v_mfma_f32_16x16x32_f16 a[16:19], v[0:3], v[144:147], a[16:19]\n"
        "ds_read_b128 v[128:131], %[vlr1] offset:0\n"
        "v_mfma_f32_16x16x32_f16 a[20:23], v[0:3], v[148:151], a[20:23]\n"
        "ds_read_b128 v[132:135], %[vlr1] offset:2048\n"
        "v_mfma_f32_16x16x32_f16 a[24:27], v[0:3], v[152:155], a[24:27]\n"
        "ds_read_b128 v[136:139], %[vlr1] offset:4096\n"
        "v_mfma_f32_16x16x32_f16 a[28:31], v[0:3], v[156:159], a[28:31]\n"
        "ds_read_b128 v[140:143], %[vlr1] offset:6144\n"
        "v_mfma_f32_16x16x32_f16 a[48:51], v[4:7], v[144:147], a[48:51]\n"
        "v_cvt_pk_f16_f32 v226, v208, v212\n"
        "v_mfma_f32_16x16x32_f16 a[52:55], v[4:7], v[148:151], a[52:55]\n"
        "v_cvt_pk_f16_f32 v230, v209, v213\n"
        "v_mfma_f32_16x16x32_f16 a[56:59], v[4:7], v[152:155], a[56:59]\n"
        "v_cvt_pk_f16_f32 v234, v210, v214\n"
        "v_mfma_f32_16x16x32_f16 a[60:63], v[4:7], v[156:159], a[60:63]\n"
        "v_cvt_pk_f16_f32 v238, v211, v215\n"
        "v_mfma_f32_16x16x32_f16 a[80:83], v[8:11], v[144:147], a[80:83]\n"
        "v_cvt_pk_f16_f32 v227, v216, v220\n"
        "v_mfma_f32_16x16x32_f16 a[84:87], v[8:11], v[148:151], a[84:87]\n"
        "v_cvt_pk_f16_f32 v231, v217, v221\n"
        "v_mfma_f32_16x16x32_f16 a[88:91], v[8:11], v[152:155], a[88:91]\n"
        "v_cvt_pk_f16_f32 v235, v218, v222\n"
        "v_mfma_f32_16x16x32_f16 a[92:95], v[8:11], v[156:159], a[92:95]\n"
        "v_cvt_pk_f16_f32 v239, v219, v223\n"
        "v_mfma_f32_16x16x32_f16 a[112:115], v[12:15], v[144:147], a[112:115]\n"
        "v_mfma_f32_16x16x32_f16 a[116:119], v[12:15], v[148:151], a[116:119]\n"
        "v_mfma_f32_16x16x32_f16 a[120:123], v[12:15], v[152:155], a[120:123]\n"
        "v_mfma_f32_16x16x32_f16 a[124:127], v[12:15], v[156:159], a[124:127]\n"
        "v_mfma_f32_16x16x32_f16 a[144:147], v[16:19], v[144:147], a[144:147]\n"
        "v_mfma_f32_16x16x32_f16 a[148:151], v[16:19], v[148:151], a[148:151]\n"
        "v_mfma_f32_16x16x32_f16 a[152:155], v[16:19], v[152:155], a[152:155]\n"
        "v_mfma_f32_16x16x32_f16 a[156:159], v[16:19], v[156:159], a[156:159]\n"
        "v_mfma_f32_16x16x32_f16 a[176:179], v[20:23], v[144:147], a[176:179]\n"
        "v_mfma_f32_16x16x32_f16 a[180:183], v[20:23], v[148:151], a[180:183]\n"
        "v_mfma_f32_16x16x32_f16 a[184:187], v[20:23], v[152:155], a[184:187]\n"
        "v_mfma_f32_16x16x32_f16 a[188:191], v[20:23], v[156:159], a[188:191]\n"
        "v_mfma_f32_16x16x32_f16 a[208:211], v[24:27], v[144:147], a[208:211]\n"
        "v_mfma_f32_16x16x32_f16 a[212:215], v[24:27], v[148:151], a[212:215]\n"
        "v_mfma_f32_16x16x32_f16 a[216:219], v[24:27], v[152:155], a[216:219]\n"
        "v_mfma_f32_16x16x32_f16 a[220:223], v[24:27], v[156:159], a[220:223]\n"
        "v_mfma_f32_16x16x32_f16 a[240:243], v[28:31], v[144:147], a[240:243]\n"
        "v_mfma_f32_16x16x32_f16 a[244:247], v[28:31], v[148:151], a[244:247]\n"
        "v_mfma_f32_16x16x32_f16 a[248:251], v[28:31], v[152:155], a[248:251]\n"
        "v_mfma_f32_16x16x32_f16 a[252:255], v[28:31], v[156:159], a[252:255]\n"
        "s_waitcnt lgkmcnt(0)\n"
        "s_waitcnt vmcnt(23)\n"
        "v_mfma_f32_16x16x32_f16 a[0:3], v[32:35], v[128:131], a[0:3]\n"
        "ds_read_b128 v[144:147], %[vlr1] offset:8192\n"
        "v_mfma_f32_16x16x32_f16 a[4:7], v[32:35], v[132:135], a[4:7]\n"
        "ds_read_b128 v[148:151], %[vlr1] offset:10240\n"
        "v_mfma_f32_16x16x32_f16 a[8:11], v[32:35], v[136:139], a[8:11]\n"
        "ds_read_b128 v[152:155], %[vlr1] offset:12288\n"
        "v_mfma_f32_16x16x32_f16 a[12:15], v[32:35], v[140:143], a[12:15]\n"
        "ds_read_b128 v[156:159], %[vlr1] offset:14336\n"
        "v_mfma_f32_16x16x32_f16 a[32:35], v[36:39], v[128:131], a[32:35]\n"
        "ds_write_b128 %[vlw], v[224:227] offset:16384\n"
        "v_mfma_f32_16x16x32_f16 a[36:39], v[36:39], v[132:135], a[36:39]\n"
        "v_mfma_f32_16x16x32_f16 a[40:43], v[36:39], v[136:139], a[40:43]\n"
        "ds_write_b128 %[vlw], v[228:231] offset:20480\n"
        "v_mfma_f32_16x16x32_f16 a[44:47], v[36:39], v[140:143], a[44:47]\n"
        "s_waitcnt vmcnt(21)\n"
        "v_mfma_f32_16x16x32_f16 a[64:67], v[40:43], v[128:131], a[64:67]\n"
        "ds_write_b128 %[vlw], v[232:235] offset:24576\n"
        "v_mfma_f32_16x16x32_f16 a[68:71], v[40:43], v[132:135], a[68:71]\n"
        "v_mfma_f32_16x16x32_f16 a[72:75], v[40:43], v[136:139], a[72:75]\n"
        "ds_write_b128 %[vlw], v[236:239] offset:28672\n"
        "v_mfma_f32_16x16x32_f16 a[76:79], v[40:43], v[140:143], a[76:79]\n"
        "v_mfma_f32_16x16x32_f16 a[96:99], v[44:47], v[128:131], a[96:99]\n"
        "v_mfma_f32_16x16x32_f16 a[100:103], v[44:47], v[132:135], a[100:103]\n"
        "v_mfma_f32_16x16x32_f16 a[104:107], v[44:47], v[136:139], a[104:107]\n"
        "v_mfma_f32_16x16x32_f16 a[108:111], v[44:47], v[140:143], a[108:111]\n"
        "s_waitcnt vmcnt(19)\n"
        "v_mfma_f32_16x16x32_f16 a[128:131], v[48:51], v[128:131], a[128:131]\n"
        "v_mfma_f32_16x16x32_f16 a[132:135], v[48:51], v[132:135], a[132:135]\n"
        "v_mfma_f32_16x16x32_f16 a[136:139], v[48:51], v[136:139], a[136:139]\n"
        "v_mfma_f32_16x16x32_f16 a[140:143], v[48:51], v[140:143], a[140:143]\n"
        "v_mfma_f32_16x16x32_f16 a[160:163], v[52:55], v[128:131], a[160:163]\n"
        "v_mfma_f32_16x16x32_f16 a[164:167], v[52:55], v[132:135], a[164:167]\n"
        "v_mfma_f32_16x16x32_f16 a[168:171], v[52:55], v[136:139], a[168:171]\n"
        "v_mfma_f32_16x16x32_f16 a[172:175], v[52:55], v[140:143], a[172:175]\n"
        "s_waitcnt vmcnt(17)\n"
        "v_mfma_f32_16x16x32_f16 a[192:195], v[56:59], v[128:131], a[192:195]\n"
        "v_mfma_f32_16x16x32_f16 a[196:199], v[56:59], v[132:135], a[196:199]\n"
        "v_mfma_f32_16x16x32_f16 a[200:203], v[56:59], v[136:139], a[200:203]\n"
        "v_mfma_f32_16x16x32_f16 a[204:207], v[56:59], v[140:143], a[204:207]\n"
        "v_mfma_f32_16x16x32_f16 a[224:227], v[60:63], v[128:131], a[224:227]\n"
        "v_mfma_f32_16x16x32_f16 a[228:231], v[60:63], v[132:135], a[228:231]\n"
        "v_mfma_f32_16x16x32_f16 a[232:235], v[60:63], v[136:139], a[232:235]\n"
        "v_mfma_f32_16x16x32_f16 a[236:239], v[60:63], v[140:143], a[236:239]\n"
        "s_waitcnt lgkmcnt(0)\n"
        "s_barrier\n"
        "v_mfma_f32_16x16x32_f16 a[16:19], v[32:35], v[144:147], a[16:19]\n"
        "ds_read_b128 v[128:131], %[vlr0] offset:16384\n"
        "v_mfma_f32_16x16x32_f16 a[20:23], v[32:35], v[148:151], a[20:23]\n"
        "ds_read_b128 v[132:135], %[vlr0] offset:18432\n"
        "v_mfma_f32_16x16x32_f16 a[24:27], v[32:35], v[152:155], a[24:27]\n"
        "ds_read_b128 v[136:139], %[vlr0] offset:20480\n"
        "v_mfma_f32_16x16x32_f16 a[28:31], v[32:35], v[156:159], a[28:31]\n"
        "ds_read_b128 v[140:143], %[vlr0] offset:22528\n"
        "v_mfma_f32_16x16x32_f16 a[48:51], v[36:39], v[144:147], a[48:51]\n"
        "v_mfma_f32_16x16x32_f16 a[52:55], v[36:39], v[148:151], a[52:55]\n"
        "v_mfma_f32_16x16x32_f16 a[56:59], v[36:39], v[152:155], a[56:59]\n"
        "v_mfma_f32_16x16x32_f16 a[60:63], v[36:39], v[156:159], a[60:63]\n"
        "v_mfma_f32_16x16x32_f16 a[80:83], v[40:43], v[144:147], a[80:83]\n"
        "v_mfma_f32_16x16x32_f16 a[84:87], v[40:43], v[148:151], a[84:87]\n"
        "v_mfma_f32_16x16x32_f16 a[88:91], v[40:43], v[152:155], a[88:91]\n"
        "v_mfma_f32_16x16x32_f16 a[92:95], v[40:43], v[156:159], a[92:95]\n"
        "v_mfma_f32_16x16x32_f16 a[112:115], v[44:47], v[144:147], a[112:115]\n"
        "v_mfma_f32_16x16x32_f16 a[116:119], v[44:47], v[148:151], a[116:119]\n"
        "v_mfma_f32_16x16x32_f16 a[120:123], v[44:47], v[152:155], a[120:123]\n"
        "v_mfma_f32_16x16x32_f16 a[124:127], v[44:47], v[156:159], a[124:127]\n"
        "v_mfma_f32_16x16x32_f16 a[144:147], v[48:51], v[144:147], a[144:147]\n"
        "v_mfma_f32_16x16x32_f16 a[148:151], v[48:51], v[148:151], a[148:151]\n"
        "v_mfma_f32_16x16x32_f16 a[152:155], v[48:51], v[152:155], a[152:155]\n"
        "v_mfma_f32_16x16x32_f16 a[156:159], v[48:51], v[156:159], a[156:159]\n"
        "v_mfma_f32_16x16x32_f16 a[176:179], v[52:55], v[144:147], a[176:179]\n"
        "v_mfma_f32_16x16x32_f16 a[180:183], v[52:55], v[148:151], a[180:183]\n"
        "v_mfma_f32_16x16x32_f16 a[184:187], v[52:55], v[152:155], a[184:187]\n"
        "v_mfma_f32_16x16x32_f16 a[188:191], v[52:55], v[156:159], a[188:191]\n"
        "v_mfma_f32_16x16x32_f16 a[208:211], v[56:59], v[144:147], a[208:211]\n"
        "v_mfma_f32_16x16x32_f16 a[212:215], v[56:59], v[148:151], a[212:215]\n"
        "v_mfma_f32_16x16x32_f16 a[216:219], v[56:59], v[152:155], a[216:219]\n"
        "v_mfma_f32_16x16x32_f16 a[220:223], v[56:59], v[156:159], a[220:223]\n"
        "v_mfma_f32_16x16x32_f16 a[240:243], v[60:63], v[144:147], a[240:243]\n"
        "v_mfma_f32_16x16x32_f16 a[244:247], v[60:63], v[148:151], a[244:247]\n"
        "v_mfma_f32_16x16x32_f16 a[248:251], v[60:63], v[152:155], a[248:251]\n"
        "v_mfma_f32_16x16x32_f16 a[252:255], v[60:63], v[156:159], a[252:255]\n"
        "; ---- kstep\n"
        "s_waitcnt lgkmcnt(0)\n"
        "s_waitcnt vmcnt(15)\n"
        "v_mfma_f32_16x16x32_f16 a[0:3], v[64:67], v[128:131], a[0:3]\n"
        "ds_read_b128 v[144:147], %[vlr0] offset:24576\n"
        "v_mfma_f32_16x16x32_f16 a[4:7], v[64:67], v[132:135], a[4:7]\n"
        "ds_read_b128 v[148:151], %[vlr0] offset:26624\n"
        "v_mfma_f32_16x16x32_f16 a[8:11], v[64:67], v[136:139], a[8:11]\n"
        "ds_read_b128 v[152:155], %[vlr0] offset:28672\n"
        "v_mfma_f32_16x16x32_f16 a[12:15], v[64:67], v[140:143], a[12:15]\n"
        "ds_read_b128 v[156:159], %[vlr0] offset:30720\n"
        "v_mfma_f32_16x16x32_f16 a[32:35], v[68:71], v[128:131], a[32:35]\n"
        "v_mfma_f32_16x16x32_f16 a[36:39], v[68:71], v[132:135], a[36:39]\n"
        "v_mfma_f32_16x16x32_f16 a[40:43], v[68:71], v[136:139], a[40:43]\n"
        "v_mfma_f32_16x16x32_f16 a[44:47], v[68:71], v[140:143], a[44:47]\n"
        "s_waitcnt vmcnt(13)\n"
        "v_mfma_f32_16x16x32_f16 a[64:67], v[72:75], v[128:131], a[64:67]\n"
        "v_mfma_f32_16x16x32_f16 a[68:71], v[72:75], v[132:135], a[68:71]\n"
        "v_mfma_f32_16x16x32_f16 a[72:75], v[72:75], v[136:139], a[72:75]\n"
        "v_mfma_f32_16x16x32_f16 a[76:79], v[72:75], v[140:143], a[76:79]\n"
        "v_mfma_f32_16x16x32_f16 a[96:99], v[76:79], v[128:131], a[96:99]\n"
        "v_mfma_f32_16x16x32_f16 a[100:103], v[76:79], v[132:135], a[100:103]\n"
        "v_mfma_f32_16x16x32_f16 a[104:107], v[76:79], v[136:139], a[104:107]\n"
        "v_mfma_f32_16x16x32_f16 a[108:111], v[76:79], v[140:143], a[108:111]\n"
        "s_waitcnt vmcnt(11)\n"
        "v_mfma_f32_16x16x32_f16 a[128:131], v[80:83], v[128:131], a[128:131]\n"
        "v_mfma_f32_16x16x32_f16 a[132:135], v[80:83], v[132:135], a[132:135]\n"
        "v_mfma_f32_16x16x32_f16 a[136:139], v[80:83], v[136:139], a[136:139]\n"
        "v_mfma_f32_16x16x32_f16 a[140:143], v[80:83], v[140:143], a[140:143]\n"
        "v_mfma_f32_16x16x32_f16 a[160:163], v[84:87], v[128:131], a[160:163]\n"
        "v_mfma_f32_16x16x32_f16 a[164:167], v[84:87], v[132:135], a[164:167]\n"
        "v_mfma_f32_16x16x32_f16 a[168:171], v[84:87], v[136:139], a[168:171]\n"
        "v_mfma_f32_16x16x32_f16 a[172:175], v[84:87], v[140:143], a[172:175]\n"
        "s_waitcnt vmcnt(9)\n"
        "v_mfma_f32_16x16x32_f16 a[192:195], v[88:91], v[128:131], a[192:195]\n"
        "v_mfma_f32_16x16x32_f16 a[196:199], v[88:91], v[132:135], a[196:199]\n"
        "v_mfma_f32_16x16x32_f16 a[200:203], v[88:91], v[136:139], a[200:203]\n"
        "v_mfma_f32_16x16x32_f16 a[204:207], v[88:91], v[140:143], a[204:207]\n"
        "v_mfma_f32_16x16x32_f16 a[224:227], v[92:95], v[128:131], a[224:227]\n"
        "v_mfma_f32_16x16x32_f16 a[228:231], v[92:95], v[132:135], a[228:231]\n"
        "v_mfma_f32_16x16x32_f16 a[232:235], v[92:95], v[136:139], a[232:235]\n"
        "v_mfma_f32_16x16x32_f16 a[236:239], v[92:95], v[140:143], a[236:239]\n"
        "s_waitcnt lgkmcnt(0)\n"
        "v_mfma_f32_16x16x32_f16 a[16:19], v[64:67], v[144:147], a[16:19]\n"
        "ds_read_b128 v[128:131], %[vlr1] offset:16384\n"
        "v_mfma_f32_16x16x32_f16 a[20:23], v[64:67], v[148:151], a[20:23]\n"
        "ds_read_b128 v[132:135], %[vlr1] offset:18432\n"
        "v_mfma_f32_16x16x32_f16 a[24:27], v[64:67], v[152:155], a[24:27]\n"
        "ds_read_b128 v[136:139], %[vlr1] offset:20480\n"
        "v_mfma_f32_16x16x32_f16 a[28:31], v[64:67], v[156:159], a[28:31]\n"
        "ds_read_b128 v[140:143], %[vlr1] offset:22528\n"
        "v_mfma_f32_16x16x32_f16 a[48:51], v[68:71], v[144:147], a[48:51]\n"
        "v_mfma_f32_16x16x32_f16 a[52:55], v[68:71], v[148:151], a[52:55]\n"
        "v_mfma_f32_16x16x32_f16 a[56:59], v[68:71], v[152:155], a[56:59]\n"
        "v_mfma_f32_16x16x32_f16 a[60:63], v[68:71], v[156:159], a[60:63]\n"
        "v_mfma_f32_16x16x32_f16 a[80:83], v[72:75], v[144:147], a[80:83]\n"
        "v_mfma_f32_16x16x32_f16 a[84:87], v[72:75], v[148:151], a[84:87]\n"
        "v_mfma_f32_16x16x32_f16 a[88:91], v[72:75], v[152:155], a[88:91]\n"
        "v_mfma_f32_16x16x32_f16 a[92:95], v[72:75], v[156:159], a[92:95]\n"
        "v_mfma_f32_16x16x32_f16 a[112:115], v[76:79], v[144:147], a[112:115]\n"
        "v_mfma_f32_16x16x32_f16 a[116:119], v[76:79], v[148:151], a[116:119]\n"
        "v_mfma_f32_16x16x32_f16 a[120:123], v[76:79], v[152:155], a[120:123]\n"
        "v_mfma_f32_16x16x32_f16 a[124:127], v[76:79], v[156:159], a[124:127]\n"
        "v_mfma_f32_16x16x32_f16 a[144:147], v[80:83], v[144:147], a[144:147]\n"
        "v_mfma_f32_16x16x32_f16 a[148:151], v[80:83], v[148:151], a[148:151]\n"
        "v_mfma_f32_16x16x32_f16 a[152:155], v[80:83], v[152:155], a[152:155]\n"
        "v_mfma_f32_16x16x32_f16 a[156:159], v[80:83], v[156:159], a[156:159]\n"
        "v_mfma_f32_16x16x32_f16 a[176:179], v[84:87], v[144:147], a[176:179]\n"
        "v_mfma_f32_16x16x32_f16 a[180:183], v[84:87], v[148:151], a[180:183]\n"
        "v_mfma_f32_16x16x32_f16 a[184:187], v[84:87], v[152:155], a[184:187]\n"
        "v_mfma_f32_16x16x32_f16 a[188:191], v[84:87], v[156:159], a[188:191]\n"
        "v_mfma_f32_16x16x32_f16 a[208:211], v[88:91], v[144:147], a[208:211]\n"
        "v_mfma_f32_16x16x32_f16 a[212:215], v[88:91], v[148:151], a[212:215]\n"
        "v_mfma_f32_16x16x32_f16 a[216:219], v[88:91], v[152:155], a[216:219]\n"
        "v_mfma_f32_16x16x32_f16 a[220:223], v[88:91], v[156:159], a[220:223]\n"
        "v_mfma_f32_16x16x32_f16 a[240:243], v[92:95], v[144:147], a[240:243]\n"
        "v_mfma_f32_16x16x32_f16 a[244:247], v[92:95], v[148:151], a[244:247]\n"
        "v_mfma_f32_16x16x32_f16 a[248:251], v[92:95], v[152:155], a[248:251]\n"
        "v_mfma_f32_16x16x32_f16 a[252:255], v[92:95], v[156:159], a[252:255]\n"
        "s_waitcnt lgkmcnt(0)\n"
        "s_waitcnt vmcnt(7)\n"
        "v_mfma_f32_16x16x32_f16 a[0:3], v[96:99], v[128:131], a[0:3]\n"
        "ds_read_b128 v[144:147], %[vlr1] offset:24576\n"
        "v_mfma_f32_16x16x32_f16 a[4:7], v[96:99], v[132:135], a[4:7]\n"
        "ds_read_b128 v[148:151], %[vlr1] offset:26624\n"
        "v_mfma_f32_16x16x32_f16 a[8:11], v[96:99], v[136:139], a[8:11]\n"
        "ds_read_b128 v[152:155], %[vlr1] offset:28672\n"
        "v_mfma_f32_16x16x32_f16 a[12:15], v[96:99], v[140:143], a[12:15]\n"
        "ds_read_b128 v[156:159], %[vlr1] offset:30720\n"
        "v_mfma_f32_16x16x32_f16 a[32:35], v[100:103], v[128:131], a[32:35]\n"
        "v_mfma_f32_16x16x32_f16 a[36:39], v[100:103], v[132:135], a[36:39]\n"
        "v_mfma_f32_16x16x32_f16 a[40:43], v[100:103], v[136:139], a[40:43]\n"
        "v_mfma_f32_16x16x32_f16 a[44:47], v[100:103], v[140:143], a[44:47]\n"
        "s_waitcnt vmcnt(4)\n"
        "v_mfma_f32_16x16x32_f16 a[64:67], v[104:107], v[128:131], a[64:67]\n"
        "v_mfma_f32_16x16x32_f16 a[68:71], v[104:107], v[132:135], a[68:71]\n"
        "v_mfma_f32_16x16x32_f16 a[72:75], v[104:107], v[136:139], a[72:75]\n"
        "v_mfma_f32_16x16x32_f16 a[76:79], v[104:107], v[140:143], a[76:79]\n"
        "v_mfma_f32_16x16x32_f16 a[96:99], v[108:111], v[128:131], a[96:99]\n"
        "v_mfma_f32_16x16x32_f16 a[100:103], v[108:111], v[132:135], a[100:103]\n"
        "v_mfma_f32_16x16x32_f16 a[104:107], v[108:111], v[136:139], a[104:107]\n"
        "v_mfma_f32_16x16x32_f16 a[108:111], v[108:111], v[140:143], a[108:111]\n"
        "s_waitcnt vmcnt(2)\n"
        "v_mfma_f32_16x16x32_f16 a[128:131], v[112:115], v[128:131], a[128:131]\n"
        "v_mfma_f32_16x16x32_f16 a[132:135], v[112:115], v[132:135], a[132:135]\n"
        "v_mfma_f32_16x16x32_f16 a[136:139], v[112:115], v[136:139], a[136:139]\n"
        "v_mfma_f32_16x16x32_f16 a[140:143], v[112:115], v[140:143], a[140:143]\n"
        "v_mfma_f32_16x16x32_f16 a[160:163], v[116:119], v[128:131], a[160:163]\n"
        "v_mfma_f32_16x16x32_f16 a[164:167], v[116:119], v[132:135], a[164:167]\n"
        "v_mfma_f32_16x16x32_f16 a[168:171], v[116:119], v[136:139], a[168:171]\n"
        "v_mfma_f32_16x16x32_f16 a[172:175], v[116:119], v[140:143], a[172:175]\n"
        "s_waitcnt vmcnt(0)\n"
        "v_mfma_f32_16x16x32_f16 a[192:195], v[120:123], v[128:131], a[192:195]\n"
        "v_mfma_f32_16x16x32_f16 a[196:199], v[120:123], v[132:135], a[196:199]\n"
        "v_mfma_f32_16x16x32_f16 a[200:203], v[120:123], v[136:139], a[200:203]\n"
        "v_mfma_f32_16x16x32_f16 a[204:207], v[120:123], v[140:143], a[204:207]\n"
        "v_mfma_f32_16x16x32_f16 a[224:227], v[124:127], v[128:131], a[224:227]\n"
        "v_mfma_f32_16x16x32_f16 a[228:231], v[124:127], v[132:135], a[228:231]\n"
        "v_mfma_f32_16x16x32_f16 a[232:235], v[124:127], v[136:139], a[232:235]\n"
        "v_mfma_f32_16x16x32_f16 a[236:239], v[124:127], v[140:143], a[236:239]\n"
        "s_waitcnt lgkmcnt(0)\n"
        "s_barrier\n"
        "v_mfma_f32_16x16x32_f16 a[16:19], v[96:99], v[144:147], a[16:19]\n"
        "v_mfma_f32_16x16x32_f16 a[20:23], v[96:99], v[148:151], a[20:23]\n"
        "v_mfma_f32_16x16x32_f16 a[24:27], v[96:99], v[152:155], a[24:27]\n"
        "v_mfma_f32_16x16x32_f16 a[28:31], v[96:99], v[156:159], a[28:31]\n"
        "v_mfma_f32_16x16x32_f16 a[48:51], v[100:103], v[144:147], a[48:51]\n"
        "v_mfma_f32_16x16x32_f16 a[52:55], v[100:103], v[148:151], a[52:55]\n"
        "v_mfma_f32_16x16x32_f16 a[56:59], v[100:103], v[152:155], a[56:59]\n"
        "v_mfma_f32_16x16x32_f16 a[60:63], v[100:103], v[156:159], a[60:63]\n"
        "v_mfma_f32_16x16x32_f16 a[80:83], v[104:107], v[144:147], a[80:83]\n"
        "v_mfma_f32_16x16x32_f16 a[84:87], v[104:107], v[148:151], a[84:87]\n"
        "v_mfma_f32_16x16x32_f16 a[88:91], v[104:107], v[152:155], a[88:91]\n"
        "v_mfma_f32_16x16x32_f16 a[92:95], v[104:107], v[156:159], a[92:95]\n"
        "v_mfma_f32_16x16x32_f16 a[112:115], v[108:111], v[144:147], a[112:115]\n"
        "v_mfma_f32_16x16x32_f16 a[116:119], v[108:111], v[148:151], a[116:119]\n"
        "v_mfma_f32_16x16x32_f16 a[120:123], v[108:111], v[152:155], a[120:123]\n"
        "v_mfma_f32_16x16x32_f16 a[124:127], v[108:111], v[156:159], a[124:127]\n"
        "v_mfma_f32_16x16x32_f16 a[144:147], v[112:115], v[144:147], a[144:147]\n"
        "v_mfma_f32_16x16x32_f16 a[148:151], v[112:115], v[148:151], a[148:151]\n"
        "v_mfma_f32_16x16x32_f16 a[152:155], v[112:115], v[152:155], a[152:155]\n"
        "v_mfma_f32_16x16x32_f16 a[156:159], v[112:115], v[156:159], a[156:159]\n"
        "v_mfma_f32_16x16x32_f16 a[176:179], v[116:119], v[144:147], a[176:179]\n"
        "v_mfma_f32_16x16x32_f16 a[180:183], v[116:119], v[148:151], a[180:183]\n"
        "v_mfma_f32_16x16x32_f16 a[184:187], v[116:119], v[152:155], a[184:187]\n"
        "v_mfma_f32_16x16x32_f16 a[188:191], v[116:119], v[156:159], a[188:191]\n"
        "v_mfma_f32_16x16x32_f16 a[208:211], v[120:123], v[144:147], a[208:211]\n"
        "v_mfma_f32_16x16x32_f16 a[212:215], v[120:123], v[148:151], a[212:215]\n"
        "v_mfma_f32_16x16x32_f16 a[216:219], v[120:123], v[152:155], a[216:219]\n"
        "v_mfma_f32_16x16x32_f16 a[220:223], v[120:123], v[156:159], a[220:223]\n"
        "v_mfma_f32_16x16x32_f16 a[240:243], v[124:127], v[144:147], a[240:243]\n"
        "v_mfma_f32_16x16x32_f16 a[244:247], v[124:127], v[148:151], a[244:247]\n"
        "v_mfma_f32_16x16x32_f16 a[248:251], v[124:127], v[152:155], a[248:251]\n"
        "v_mfma_f32_16x16x32_f16 a[252:255], v[124:127], v[156:159], a[252:255]\n"
        "s_add_u32 s45, s45, 1\n"
        "s_nop 15\n"
        "s_waitcnt vmcnt(0)\n"
        "v_readfirstlane_b32 s47, v243\n"
        "s_nop 3\n"
        "s_cmp_eq_u32 s47, s40\n"
        "s_cselect_b32 s41, 1, s41\n"
        "s_cmp_lg_u32 s41, 0\n"
        "s_cbranch_scc1 L_bad_%=\n"
        "v_mov_b32 v128, 0xc0135761\n"
        "v_mov_b32 v129, 0xc0135761\n"
        "v_mov_b32 v130, 0xbdd2d3e8\n"
        "v_mov_b32 v131, 0xbdd2d3e8\n"
        "v_mov_b32 v132, 1.0\n"
        "v_mov_b32 v133, 1.0\n"
        "v_accvgpr_read_b32 v0, a0\n"
        "v_accvgpr_read_b32 v1, a8\n"
        "v_accvgpr_read_b32 v2, a16\n"
        "v_accvgpr_read_b32 v3, a24\n"
        "v_pk_mul_f32 v[4:5], v[0:1], v[0:1]\n"
        "v_pk_mul_f32 v[6:7], v[2:3], v[2:3]\n"
        "v_pk_fma_f32 v[4:5], v[4:5], v[130:131], v[128:129]\n"
        "v_pk_fma_f32 v[6:7], v[6:7], v[130:131], v[128:129]\n"
        "v_pk_mul_f32 v[4:5], v[4:5], v[0:1]\n"
        "v_pk_mul_f32 v[6:7], v[6:7], v[2:3]\n"
        "v_exp_f32 v8, v4\n"
        "v_exp_f32 v9, v5\n"
        "v_exp_f32 v10, v6\n"
        "v_exp_f32 v11, v7\n"
        "v_pk_add_f32 v[8:9], v[8:9], v[132:133]\n"
        "v_pk_add_f32 v[10:11], v[10:11], v[132:133]\n"
        "v_rcp_f32 v8, v8\n"
        "v_rcp_f32 v9, v9\n"
        "v_rcp_f32 v10, v10\n"
        "v_rcp_f32 v11, v11\n"
        "v_pk_mul_f32 v[4:5], v[8:9], v[0:1]\n"
        "v_pk_mul_f32 v[6:7], v[10:11], v[2:3]\n"
        "s_mov_b32 s44, 0x0\n"
        "buffer_store_dwordx4 v[4:7], %[vout0], %[ro], s44 offen\n"
        "v_accvgpr_read_b32 v16, a4\n"
        "v_accvgpr_read_b32 v17, a12\n"
        "v_accvgpr_read_b32 v18, a20\n"
        "v_accvgpr_read_b32 v19, a28\n"
        "v_pk_mul_f32 v[20:21], v[16:17], v[16:17]\n"
        "v_pk_mul_f32 v[22:23], v[18:19], v[18:19]\n"
        "v_pk_fma_f32 v[20:21], v[20:21], v[130:131], v[128:129]\n"
        "v_pk_fma_f32 v[22:23], v[22:23], v[130:131], v[128:129]\n"
        "v_pk_mul_f32 v[20:21], v[20:21], v[16:17]\n"
        "v_pk_mul_f32 v[22:23], v[22:23], v[18:19]\n"
        "v_exp_f32 v24, v20\n"
        "v_exp_f32 v25, v21\n"
        "v_exp_f32 v26, v22\n"
        "v_exp_f32 v27, v23\n"
        "v_pk_add_f32 v[24:25], v[24:25], v[132:133]\n"
        "v_pk_add_f32 v[26:27], v[26:27], v[132:133]\n"
        "v_rcp_f32 v24, v24\n"
        "v_rcp_f32 v25, v25\n"
        "v_rcp_f32 v26, v26\n"
        "v_rcp_f32 v27, v27\n"
        "v_pk_mul_f32 v[20:21], v[24:25], v[16:17]\n"
        "v_pk_mul_f32 v[22:23], v[26:27], v[18:19]\n"
        "s_mov_b32 s44, 0x0\n"
        "buffer_store_dwordx4 v[20:23], %[vout1], %[ro], s44 offen\n"
        "v_accvgpr_read_b32 v32, a1\n"
        "v_accvgpr_read_b32 v33, a9\n"
        "v_accvgpr_read_b32 v34, a17\n"
        "v_accvgpr_read_b32 v35, a25\n"
        "v_pk_mul_f32 v[36:37], v[32:33], v[32:33]\n"
        "v_pk_mul_f32 v[38:39], v[34:35], v[34:35]\n"
        "v_pk_fma_f32 v[36:37], v[36:37], v[130:131], v[128:129]\n"
        "v_pk_fma_f32 v[38:39], v[38:39], v[130:131], v[128:129]\n"
        "v_pk_mul_f32 v[36:37], v[36:37], v[32:33]\n"
        "v_pk_mul_f32 v[38:39], v[38:39], v[34:35]\n"
        "v_exp_f32 v40, v36\n"
        "v_exp_f32 v41, v37\n"
        "v_exp_f32 v42, v38\n"
        "v_exp_f32 v43, v39\n"
        "v_pk_add_f32 v[40:41], v[40:41], v[132:133]\n"
        "v_pk_add_f32 v[42:43], v[42:43], v[132:133]\n"
        "v_rcp_f32 v40, v40\n"
        "v_rcp_f32 v41, v41\n"
        "v_rcp_f32 v42, v42\n"
        "v_rcp_f32 v43, v43\n"
        "v_pk_mul_f32 v[36:37], v[40:41], v[32:33]\n"
        "v_pk_mul_f32 v[38:39], v[42:43], v[34:35]\n"
        "s_mov_b32 s44, 0x20000\n"
        "buffer_store_dwordx4 v[36:39], %[vout0], %[ro], s44 offen\n"
        "v_accvgpr_read_b32 v48, a5\n"
        "v_accvgpr_read_b32 v49, a13\n"
        "v_accvgpr_read_b32 v50, a21\n"
        "v_accvgpr_read_b32 v51, a29\n"
        "v_pk_mul_f32 v[52:53], v[48:49], v[48:49]\n"
        "v_pk_mul_f32 v[54:55], v[50:51], v[50:51]\n"
        "v_pk_fma_f32 v[52:53], v[52:53], v[130:131], v[128:129]\n"
        "v_pk_fma_f32 v[54:55], v[54:55], v[130:131], v[128:129]\n"
        "v_pk_mul_f32 v[52:53], v[52:53], v[48:49]\n"
        "v_pk_mul_f32 v[54:55], v[54:55], v[50:51]\n"
        "v_exp_f32 v56, v52\n"
        "v_exp_f32 v57, v53\n"
        "v_exp_f32 v58, v54\n"
        "v_exp_f32 v59, v55\n"
        "v_pk_add_f32 v[56:57], v[56:57], v[132:133]\n"
        "v_pk_add_f32 v[58:59], v[58:59], v[132:133]\n"
        "v_rcp_f32 v56, v56\n"
        "v_rcp_f32 v57, v57\n"
        "v_rcp_f32 v58, v58\n"
        "v_rcp_f32 v59, v59\n"
        "v_pk_mul_f32 v[52:53], v[56:57], v[48:49]\n"
        "v_pk_mul_f32 v[54:55], v[58:59], v[50:51]\n"
        "s_mov_b32 s44, 0x20000\n"
        "buffer_store_dwordx4 v[52:55], %[vout1], %[ro], s44 offen\n"
        "v_accvgpr_read_b32 v64, a2\n"
        "v_accvgpr_read_b32 v65, a10\n"
        "v_accvgpr_read_b32 v66, a18\n"
        "v_accvgpr_read_b32 v67, a26\n"
        "v_pk_mul_f32 v[68:69], v[64:65], v[64:65]\n"
        "v_pk_mul_f32 v[70:71], v[66:67], v[66:67]\n"
        "v_pk_fma_f32 v[68:69], v[68:69], v[130:131], v[128:129]\n"
        "v_pk_fma_f32 v[70:71], v[70:71], v[130:131], v[128:129]\n"
        "v_pk_mul_f32 v[68:69], v[68:69], v[64:65]\n"
        "v_pk_mul_f32 v[70:71], v[70:71], v[66:67]\n"
        "v_exp_f32 v72, v68\n"
        "v_exp_f32 v73, v69\n"
        "v_exp_f32 v74, v70\n"
        "v_exp_f32 v75, v71\n"
        "v_pk_add_f32 v[72:73], v[72:73], v[132:133]\n"
        "v_pk_add_f32 v[74:75], v[74:75], v[132:133]\n"
        "v_rcp_f32 v72, v72\n"
        "v_rcp_f32 v73, v73\n"
        "v_rcp_f32 v74, v74\n"
        "v_rcp_f32 v75, v75\n"
        "v_pk_mul_f32 v[68:69], v[72:73], v[64:65]\n"
        "v_pk_mul_f32 v[70:71], v[74:75], v[66:67]\n"
        "s_mov_b32 s44, 0x40000\n"
        "buffer_store_dwordx4 v[68:71], %[vout0], %[ro], s44 offen\n"
        "v_accvgpr_read_b32 v80, a6\n"
        "v_accvgpr_read_b32 v81, a14\n"
        "v_accvgpr_read_b32 v82, a22\n"
        "v_accvgpr_read_b32 v83, a30\n"
        "v_pk_mul_f32 v[84:85], v[80:81], v[80:81]\n"
        "v_pk_mul_f32 v[86:87], v[82:83], v[82:83]\n"
        "v_pk_fma_f32 v[84:85], v[84:85], v[130:131], v[128:129]\n"
        "v_pk_fma_f32 v[86:87], v[86:87], v[130:131], v[128:129]\n"
        "v_pk_mul_f32 v[84:85], v[84:85], v[80:81]\n"
        "v_pk_mul_f32 v[86:87], v[86:87], v[82:83]\n"
        "v_exp_f32 v88, v84\n"
        "v_exp_f32 v89, v85\n"
        "v_exp_f32 v90, v86\n"
        "v_exp_f32 v91, v87\n"
        "v_pk_add_f32 v[88:89], v[88:89], v[132:133]\n"
        "v_pk_add_f32 v[90:91], v[90:91], v[132:133]\n"
        "v_rcp_f32 v88, v88\n"
        "v_rcp_f32 v89, v89\n"
        "v_rcp_f32 v90, v90\n"
        "v_rcp_f32 v91, v91\n"
        "v_pk_mul_f32 v[84:85], v[88:89], v[80:81]\n"
        "v_pk_mul_f32 v[86:87], v[90:91], v[82:83]\n"
        "s_mov_b32 s44, 0x40000\n"
        "buffer_store_dwordx4 v[84:87], %[vout1], %[ro], s44 offen\n"
        "v_accvgpr_read_b32 v96, a3\n"
        "v_accvgpr_read_b32 v97, a11\n"
        "v_accvgpr_read_b32 v98, a19\n"
        "v_accvgpr_read_b32 v99, a27\n"
        "v_pk_mul_f32 v[100:101], v[96:97], v[96:97]\n"
        "v_pk_mul_f32 v[102:103], v[98:99], v[98:99]\n"
        "v_pk_fma_f32 v[100:101], v[100:101], v[130:131], v[128:129]\n"
        "v_pk_fma_f32 v[102:103], v[102:103], v[130:131], v[128:129]\n"
        "v_pk_mul_f32 v[100:101], v[100:101], v[96:97]\n"
        "v_pk_mul_f32 v[102:103], v[102:103], v[98:99]\n"
        "v_exp_f32 v104, v100\n"
        "v_exp_f32 v105, v101\n"
        "v_exp_f32 v106, v102\n"
        "v_exp_f32 v107, v103\n"
        "v_pk_add_f32 v[104:105], v[104:105], v[132:133]\n"
        "v_pk_add_f32 v[106:107], v[106:107], v[132:133]\n"
        "v_rcp_f32 v104, v104\n"
        "v_rcp_f32 v105, v105\n"
        "v_rcp_f32 v106, v106\n"
        "v_rcp_f32 v107, v107\n"
        "v_pk_mul_f32 v[100:101], v[104:105], v[96:97]\n"
        "v_pk_mul_f32 v[102:103], v[106:107], v[98:99]\n"
        "s_mov_b32 s44, 0x60000\n"
        "buffer_store_dwordx4 v[100:103], %[vout0], %[ro], s44 offen\n"
        "v_accvgpr_read_b32 v112, a7\n"
        "v_accvgpr_read_b32 v113, a15\n"
        "v_accvgpr_read_b32 v114, a23\n"
        "v_accvgpr_read_b32 v115, a31\n"
        "v_pk_mul_f32 v[116:117], v[112:113], v[112:113]\n"
        "v_pk_mul_f32 v[118:119], v[114:115], v[114:115]\n"
        "v_pk_fma_f32 v[116:117], v[116:117], v[130:131], v[128:129]\n"
        "v_pk_fma_f32 v[118:119], v[118:119], v[130:131], v[128:129]\n"
        "v_pk_mul_f32 v[116:117], v[116:117], v[112:113]\n"
        "v_pk_mul_f32 v[118:119], v[118:119], v[114:115]\n"
        "v_exp_f32 v120, v116\n"
        "v_exp_f32 v121, v117\n"
        "v_exp_f32 v122, v118\n"
        "v_exp_f32 v123, v119\n"
        "v_pk_add_f32 v[120:121], v[120:121], v[132:133]\n"
        "v_pk_add_f32 v[122:123], v[122:123], v[132:133]\n"
        "v_rcp_f32 v120, v120\n"
        "v_rcp_f32 v121, v121\n"
        "v_rcp_f32 v122, v122\n"
        "v_rcp_f32 v123, v123\n"
        "v_pk_mul_f32 v[116:117], v[120:121], v[112:113]\n"
        "v_pk_mul_f32 v[118:119], v[122:123], v[114:115]\n"
        "s_mov_b32 s44, 0x60000\n"
        "buffer_store_dwordx4 v[116:119], %[vout1], %[ro], s44 offen\n"
        "v_accvgpr_read_b32 v0, a32\n"
        "v_accvgpr_read_b32 v1, a40\n"
        "v_accvgpr_read_b32 v2, a48\n"
        "v_accvgpr_read_b32 v3, a56\n"
        "v_pk_mul_f32 v[4:5], v[0:1], v[0:1]\n"
        "v_pk_mul_f32 v[6:7], v[2:3], v[2:3]\n"
        "v_pk_fma_f32 v[4:5], v[4:5], v[130:131], v[128:129]\n"
        "v_pk_fma_f32 v[6:7], v[6:7], v[130:131], v[128:129]\n"
        "v_pk_mul_f32 v[4:5], v[4:5], v[0:1]\n"
        "v_pk_mul_f32 v[6:7], v[6:7], v[2:3]\n"
        "v_exp_f32 v8, v4\n"
        "v_exp_f32 v9, v5\n"
        "v_exp_f32 v10, v6\n"
        "v_exp_f32 v11, v7\n"
        "v_pk_add_f32 v[8:9], v[8:9], v[132:133]\n"
        "v_pk_add_f32 v[10:11], v[10:11], v[132:133]\n"
        "v_rcp_f32 v8, v8\n"
        "v_rcp_f32 v9, v9\n"
        "v_rcp_f32 v10, v10\n"
        "v_rcp_f32 v11, v11\n"
        "v_pk_mul_f32 v[4:5], v[8:9], v[0:1]\n"
        "v_pk_mul_f32 v[6:7], v[10:11], v[2:3]\n"
        "s_mov_b32 s44, 0x200000\n"
        "buffer_store_dwordx4 v[4:7], %[vout0], %[ro], s44 offen\n"
        "v_accvgpr_read_b32 v16, a36\n"
        "v_accvgpr_read_b32 v17, a44\n"
        "v_accvgpr_read_b32 v18, a52\n"
        "v_accvgpr_read_b32 v19, a60\n"
        "v_pk_mul_f32 v[20:21], v[16:17], v[16:17]\n"
        "v_pk_mul_f32 v[22:23], v[18:19], v[18:19]\n"
        "v_pk_fma_f32 v[20:21], v[20:21], v[130:131], v[128:129]\n"
        "v_pk_fma_f32 v[22:23], v[22:23], v[130:131], v[128:129]\n"
        "v_pk_mul_f32 v[20:21], v[20:21], v[16:17]\n"
        "v_pk_mul_f32 v[22:23], v[22:23], v[18:19]\n"
        "v_exp_f32 v24, v20\n"
        "v_exp_f32 v25, v21\n"
        "v_exp_f32 v26, v22\n"
        "v_exp_f32 v27, v23\n"
        "v_pk_add_f32 v[24:25], v[24:25], v[132:133]\n"
        "v_pk_add_f32 v[26:27], v[26:27], v[132:133]\n"
        "v_rcp_f32 v24, v24\n"
        "v_rcp_f32 v25, v25\n"
        "v_rcp_f32 v26, v26\n"
        "v_rcp_f32 v27, v27\n"
        "v_pk_mul_f32 v[20:21], v[24:25], v[16:17]\n"
        "v_pk_mul_f32 v[22:23], v[26:27], v[18:19]\n"
        "s_mov_b32 s44, 0x200000\n"
        "buffer_store_dwordx4 v[20:23], %[vout1], %[ro], s44 offen\n"
        "v_accvgpr_read_b32 v32, a33\n"
        "v_accvgpr_read_b32 v33, a41\n"
        "v_accvgpr_read_b32 v34, a49\n"
        "v_accvgpr_read_b32 v35, a57\n"
        "v_pk_mul_f32 v[36:37], v[32:33], v[32:33]\n"
        "v_pk_mul_f32 v[38:39], v[34:35], v[34:35]\n"
        "v_pk_fma_f32 v[36:37], v[36:37], v[130:131], v[128:129]\n"
        "v_pk_fma_f32 v[38:39], v[38:39], v[130:131], v[128:129]\n"
        "v_pk_mul_f32 v[36:37], v[36:37], v[32:33]\n"
        "v_pk_mul_f32 v[38:39], v[38:39], v[34:35]\n"
        "v_exp_f32 v40, v36\n"
        "v_exp_f32 v41, v37\n"
        "v_exp_f32 v42, v38\n"
        "v_exp_f32 v43, v39\n"
        "v_pk_add_f32 v[40:41], v[40:41], v[132:133]\n"
        "v_pk_add_f32 v[42:43], v[42:43], v[132:133]\n"
        "v_rcp_f32 v40, v40\n"
        "v_rcp_f32 v41, v41\n"
        "v_rcp_f32 v42, v42\n"
        "v_rcp_f32 v43, v43\n"
        "v_pk_mul_f32 v[36:37], v[40:41], v[32:33]\n"
        "v_pk_mul_f32 v[38:39], v[42:43], v[34:35]\n"
        "s_mov_b32 s44, 0x220000\n"
        "buffer_store_dwordx4 v[36:39], %[vout0], %[ro], s44 offen\n"
        "v_accvgpr_read_b32 v48, a37\n"
        "v_accvgpr_read_b32 v49, a45\n"
        "v_accvgpr_read_b32 v50, a53\n"
        "v_accvgpr_read_b32 v51, a61\n"
        "v_pk_mul_f32 v[52:53], v[48:49], v[48:49]\n"
        "v_pk_mul_f32 v[54:55], v[50:51], v[50:51]\n"
        "v_pk_fma_f32 v[52:53], v[52:53], v[130:131], v[128:129]\n"
        "v_pk_fma_f32 v[54:55], v[54:55], v[130:131], v[128:129]\n"
        "v_pk_mul_f32 v[52:53], v[52:53], v[48:49]\n"
        "v_pk_mul_f32 v[54:55], v[54:55], v[50:51]\n"
        "v_exp_f32 v56, v52\n"
        "v_exp_f32 v57, v53\n"
        "v_exp_f32 v58, v54\n"
        "v_exp_f32 v59, v55\n"
        "v_pk_add_f32 v[56:57], v[56:57], v[132:133]\n"
        "v_pk_add_f32 v[58:59], v[58:59], v[132:133]\n"
        "v_rcp_f32 v56, v56\n"
        "v_rcp_f32 v57, v57\n"
        "v_rcp_f32 v58, v58\n"
        "v_rcp_f32 v59, v59\n"
        "v_pk_mul_f32 v[52:53], v[56:57], v[48:49]\n"
        "v_pk_mul_f32 v[54:55], v[58:59], v[50:51]\n"
        "s_mov_b32 s44, 0x220000\n"
        "buffer_store_dwordx4 v[52:55], %[vout1], %[ro], s44 offen\n"
        "v_accvgpr_read_b32 v64, a34\n"
        "v_accvgpr_read_b32 v65, a42\n"
        "v_accvgpr_read_b32 v66, a50\n"
        "v_accvgpr_read_b32 v67, a58\n"
        "v_pk_mul_f32 v[68:69], v[64:65], v[64:65]\n"
        "v_pk_mul_f32 v[70:71], v[66:67], v[66:67]\n"
        "v_pk_fma_f32 v[68:69], v[68:69], v[130:131], v[128:129]\n"
        "v_pk_fma_f32 v[70:71], v[70:71], v[130:131], v[128:129]\n"
        "v_pk_mul_f32 v[68:69], v[68:69], v[64:65]\n"
        "v_pk_mul_f32 v[70:71], v[70:71], v[66:67]\n"
        "v_exp_f32 v72, v68\n"
        "v_exp_f32 v73, v69\n"
        "v_exp_f32 v74, v70\n"
        "v_exp_f32 v75, v71\n"
        "v_pk_add_f32 v[72:73], v[72:73], v[132:133]\n"
        "v_pk_add_f32 v[74:75], v[74:75], v[132:133]\n"
        "v_rcp_f32 v72, v72\n"
        "v_rcp_f32 v73, v73\n"
        "v_rcp_f32 v74, v74\n"
        "v_rcp_f32 v75, v75\n"
        "v_pk_mul_f32 v[68:69], v[72:73], v[64:65]\n"
        "v_pk_mul_f32 v[70:71], v[74:75], v[66:67]\n"
        "s_mov_b32 s44, 0x240000\n"
        "buffer_store_dwordx4 v[68:71], %[vout0], %[ro], s44 offen\n"
        "v_accvgpr_read_b32 v80, a38\n"
        "v_accvgpr_read_b32 v81, a46\n"
        "v_accvgpr_read_b32 v82, a54\n"
        "v_accvgpr_read_b32 v83, a62\n"
        "v_pk_mul_f32 v[84:85], v[80:81], v[80:81]\n"
        "v_pk_mul_f32 v[86:87], v[82:83], v[82:83]\n"
        "v_pk_fma_f32 v[84:85], v[84:85], v[130:131], v[128:129]\n"
        "v_pk_fma_f32 v[86:87], v[86:87], v[130:131], v[128:129]\n"
        "v_pk_mul_f32 v[84:85], v[84:85], v[80:81]\n"
        "v_pk_mul_f32 v[86:87], v[86:87], v[82:83]\n"
        "v_exp_f32 v88, v84\n"
        "v_exp_f32 v89, v85\n"
        "v_exp_f32 v90, v86\n"
        "v_exp_f32 v91, v87\n"
        "v_pk_add_f32 v[88:89], v[88:89], v[132:133]\n"
        "v_pk_add_f32 v[90:91], v[90:91], v[132:133]\n"
        "v_rcp_f32 v88, v88\n"
        "v_rcp_f32 v89, v89\n"
        "v_rcp_f32 v90, v90\n"
        "v_rcp_f32 v91, v91\n"
        "v_pk_mul_f32 v[84:85], v[88:89], v[80:81]\n"
        "v_pk_mul_f32 v[86:87], v[90:91], v[82:83]\n"
        "s_mov_b32 s44, 0x240000\n"
        "buffer_store_dwordx4 v[84:87], %[vout1], %[ro], s44 offen\n"
        "v_accvgpr_read_b32 v96, a35\n"
        "v_accvgpr_read_b32 v97, a43\n"
        "v_accvgpr_read_b32 v98, a51\n"
        "v_accvgpr_read_b32 v99, a59\n"
        "v_pk_mul_f32 v[100:101], v[96:97], v[96:97]\n"
        "v_pk_mul_f32 v[102:103], v[98:99], v[98:99]\n"
        "v_pk_fma_f32 v[100:101], v[100:101], v[130:131], v[128:129]\n"
        "v_pk_fma_f32 v[102:103], v[102:103], v[130:131], v[128:129]\n"
        "v_pk_mul_f32 v[100:101], v[100:101], v[96:97]\n"
        "v_pk_mul_f32 v[102:103], v[102:103], v[98:99]\n"
        "v_exp_f32 v104, v100\n"
        "v_exp_f32 v105, v101\n"
        "v_exp_f32 v106, v102\n"
        "v_exp_f32 v107, v103\n"
        "v_pk_add_f32 v[104:105], v[104:105], v[132:133]\n"
        "v_pk_add_f32 v[106:107], v[106:107], v[132:133]\n"
        "v_rcp_f32 v104, v104\n"
        "v_rcp_f32 v105, v105\n"
        "v_rcp_f32 v106, v106\n"
        "v_rcp_f32 v107, v107\n"
        "v_pk_mul_f32 v[100:101], v[104:105], v[96:97]\n"
        "v_pk_mul_f32 v[102:103], v[106:107], v[98:99]\n"
        "s_mov_b32 s44, 0x260000\n"
        "buffer_store_dwordx4 v[100:103], %[vout0], %[ro], s44 offen\n"
        "v_accvgpr_read_b32 v112, a39\n"
        "v_accvgpr_read_b32 v113, a47\n"
        "v_accvgpr_read_b32 v114, a55\n"
        "v_accvgpr_read_b32 v115, a63\n"
        "v_pk_mul_f32 v[116:117], v[112:113], v[112:113]\n"
        "v_pk_mul_f32 v[118:119], v[114:115], v[114:115]\n"
        "v_pk_fma_f32 v[116:117], v[116:117], v[130:131], v[128:129]\n"
        "v_pk_fma_f32 v[118:119], v[118:119], v[130:131], v[128:129]\n"
        "v_pk_mul_f32 v[116:117], v[116:117], v[112:113]\n"
        "v_pk_mul_f32 v[118:119], v[118:119], v[114:115]\n"
        "v_exp_f32 v120, v116\n"
        "v_exp_f32 v121, v117\n"
        "v_exp_f32 v122, v118\n"
        "v_exp_f32 v123, v119\n"
        "v_pk_add_f32 v[120:121], v[120:121], v[132:133]\n"
        "v_pk_add_f32 v[122:123], v[122:123], v[132:133]\n"
        "v_rcp_f32 v120, v120\n"
        "v_rcp_f32 v121, v121\n"
        "v_rcp_f32 v122, v122\n"
        "v_rcp_f32 v123, v123\n"
        "v_pk_mul_f32 v[116:117], v[120:121], v[112:113]\n"
        "v_pk_mul_f32 v[118:119], v[122:123], v[114:115]\n"
        "s_mov_b32 s44, 0x260000\n"
        "buffer_store_dwordx4 v[116:119], %[vout1], %[ro], s44 offen\n"
        "v_accvgpr_read_b32 v0, a64\n"
        "v_accvgpr_read_b32 v1, a72\n"
        "v_accvgpr_read_b32 v2, a80\n"
        "v_accvgpr_read_b32 v3, a88\n"
        "v_pk_mul_f32 v[4:5], v[0:1], v[0:1]\n"
        "v_pk_mul_f32 v[6:7], v[2:3], v[2:3]\n"
        "v_pk_fma_f32 v[4:5], v[4:5], v[130:131], v[128:129]\n"
        "v_pk_fma_f32 v[6:7], v[6:7], v[130:131], v[128:129]\n"
        "v_pk_mul_f32 v[4:5], v[4:5], v[0:1]\n"
        "v_pk_mul_f32 v[6:7], v[6:7], v[2:3]\n"
        "v_exp_f32 v8, v4\n"
        "v_exp_f32 v9, v5\n"
        "v_exp_f32 v10, v6\n"
        "v_exp_f32 v11, v7\n"
        "v_pk_add_f32 v[8:9], v[8:9], v[132:133]\n"
        "v_pk_add_f32 v[10:11], v[10:11], v[132:133]\n"
        "v_rcp_f32 v8, v8\n"
        "v_rcp_f32 v9, v9\n"
        "v_rcp_f32 v10, v10\n"
        "v_rcp_f32 v11, v11\n"
        "v_pk_mul_f32 v[4:5], v[8:9], v[0:1]\n"
        "v_pk_mul_f32 v[6:7], v[10:11], v[2:3]\n"
        "s_mov_b32 s44, 0x400000\n"
        "buffer_store_dwordx4 v[4:7], %[vout0], %[ro], s44 offen\n"
        "v_accvgpr_read_b32 v16, a68\n"
        "v_accvgpr_read_b32 v17, a76\n"
        "v_accvgpr_read_b32 v18, a84\n"
        "v_accvgpr_read_b32 v19, a92\n"
        "v_pk_mul_f32 v[20:21], v[16:17], v[16:17]\n"
        "v_pk_mul_f32 v[22:23], v[18:19], v[18:19]\n"
        "v_pk_fma_f32 v[20:21], v[20:21], v[130:131], v[128:129]\n"
        "v_pk_fma_f32 v[22:23], v[22:23], v[130:131], v[128:129]\n"
        "v_pk_mul_f32 v[20:21], v[20:21], v[16:17]\n"
        "v_pk_mul_f32 v[22:23], v[22:23], v[18:19]\n"
        "v_exp_f32 v24, v20\n"
        "v_exp_f32 v25, v21\n"
        "v_exp_f32 v26, v22\n"
        "v_exp_f32 v27, v23\n"
        "v_pk_add_f32 v[24:25], v[24:25], v[132:133]\n"
        "v_pk_add_f32 v[26:27], v[26:27], v[132:133]\n"
        "v_rcp_f32 v24, v24\n"
        "v_rcp_f32 v25, v25\n"
        "v_rcp_f32 v26, v26\n"
        "v_rcp_f32 v27, v27\n"
        "v_pk_mul_f32 v[20:21], v[24:25], v[16:17]\n"
        "v_pk_mul_f32 v[22:23], v[26:27], v[18:19]\n"
        "s_mov_b32 s44, 0x400000\n"
        "buffer_store_dwordx4 v[20:23], %[vout1], %[ro], s44 offen\n"
        "v_accvgpr_read_b32 v32, a65\n"
        "v_accvgpr_read_b32 v33, a73\n"
        "v_accvgpr_read_b32 v34, a81\n"
        "v_accvgpr_read_b32 v35, a89\n"
        "v_pk_mul_f32 v[36:37], v[32:33], v[32:33]\n"
        "v_pk_mul_f32 v[38:39], v[34:35], v[34:35]\n"
        "v_pk_fma_f32 v[36:37], v[36:37], v[130:131], v[128:129]\n"
        "v_pk_fma_f32 v[38:39], v[38:39], v[130:131], v[128:129]\n"
        "v_pk_mul_f32 v[36:37], v[36:37], v[32:33]\n"
        "v_pk_mul_f32 v[38:39], v[38:39], v[34:35]\n"
        "v_exp_f32 v40, v36\n"
        "v_exp_f32 v41, v37\n"
        "v_exp_f32 v42, v38\n"
        "v_exp_f32 v43, v39\n"
        "v_pk_add_f32 v[40:41], v[40:41], v[132:133]\n"
        "v_pk_add_f32 v[42:43], v[42:43], v[132:133]\n"
        "v_rcp_f32 v40, v40\n"
        "v_rcp_f32 v41, v41\n"
        "v_rcp_f32 v42, v42\n"
        "v_rcp_f32 v43, v43\n"
        "v_pk_mul_f32 v[36:37], v[40:41], v[32:33]\n"
        "v_pk_mul_f32 v[38:39], v[42:43], v[34:35]\n"
        "s_mov_b32 s44, 0x420000\n"
        "buffer_store_dwordx4 v[36:39], %[vout0], %[ro], s44 offen\n"
        "v_accvgpr_read_b32 v48, a69\n"
        "v_accvgpr_read_b32 v49, a77\n"
        "v_accvgpr_read_b32 v50, a85\n"
        "v_accvgpr_read_b32 v51, a93\n"
        "v_pk_mul_f32 v[52:53], v[48:49], v[48:49]\n"
        "v_pk_mul_f32 v[54:55], v[50:51], v[50:51]\n"
        "v_pk_fma_f32 v[52:53], v[52:53], v[130:131], v[128:129]\n"
        "v_pk_fma_f32 v[54:55], v[54:55], v[130:131], v[128:129]\n"
        "v_pk_mul_f32 v[52:53], v[52:53], v[48:49]\n"
        "v_pk_mul_f32 v[54:55], v[54:55], v[50:51]\n"
        "v_exp_f32 v56, v52\n"
        "v_exp_f32 v57, v53\n"
        "v_exp_f32 v58, v54\n"
        "v_exp_f32 v59, v55\n"
        "v_pk_add_f32 v[56:57], v[56:57], v[132:133]\n"
        "v_pk_add_f32 v[58:59], v[58:59], v[132:133]\n"
        "v_rcp_f32 v56, v56\n"
        "v_rcp_f32 v57, v57\n"
        "v_rcp_f32 v58, v58\n"
        "v_rcp_f32 v59, v59\n"
        "v_pk_mul_f32 v[52:53], v[56:57], v[48:49]\n"
        "v_pk_mul_f32 v[54:55], v[58:59], v[50:51]\n"
        "s_mov_b32 s44, 0x420000\n"
        "buffer_store_dwordx4 v[52:55], %[vout1], %[ro], s44 offen\n"
        "v_accvgpr_read_b32 v64, a66\n"
        "v_accvgpr_read_b32 v65, a74\n"
        "v_accvgpr_read_b32 v66, a82\n"
        "v_accvgpr_read_b32 v67, a90\n"
        "v_pk_mul_f32 v[68:69], v[64:65], v[64:65]\n"
        "v_pk_mul_f32 v[70:71], v[66:67], v[66:67]\n"
        "v_pk_fma_f32 v[68:69], v[68:69], v[130:131], v[128:129]\n"
        "v_pk_fma_f32 v[70:71], v[70:71], v[130:131], v[128:129]\n"
        "v_pk_mul_f32 v[68:69], v[68:69], v[64:65]\n"
        "v_pk_mul_f32 v[70:71], v[70:71], v[66:67]\n"
        "v_exp_f32 v72, v68\n"
        "v_exp_f32 v73, v69\n"
        "v_exp_f32 v74, v70\n"
        "v_exp_f32 v75, v71\n"
        "v_pk_add_f32 v[72:73], v[72:73], v[132:133]\n"
        "v_pk_add_f32 v[74:75], v[74:75], v[132:133]\n"
        "v_rcp_f32 v72, v72\n"
        "v_rcp_f32 v73, v73\n"
        "v_rcp_f32 v74, v74\n"
        "v_rcp_f32 v75, v75\n"
        "v_pk_mul_f32 v[68:69], v[72:73], v[64:65]\n"
        "v_pk_mul_f32 v[70:71], v[74:75], v[66:67]\n"
        "s_mov_b32 s44, 0x440000\n"
        "buffer_store_dwordx4 v[68:71], %[vout0], %[ro], s44 offen\n"
        "v_accvgpr_read_b32 v80, a70\n"
        "v_accvgpr_read_b32 v81, a78\n"
        "v_accvgpr_read_b32 v82, a86\n"
        "v_accvgpr_read_b32 v83, a94\n"
        "v_pk_mul_f32 v[84:85], v[80:81], v[80:81]\n"
        "v_pk_mul_f32 v[86:87], v[82:83], v[82:83]\n"
        "v_pk_fma_f32 v[84:85], v[84:85], v[130:131], v[128:129]\n"
        "v_pk_fma_f32 v[86:87], v[86:87], v[130:131], v[128:129]\n"
        "v_pk_mul_f32 v[84:85], v[84:85], v[80:81]\n"
        "v_pk_mul_f32 v[86:87], v[86:87], v[82:83]\n"
        "v_exp_f32 v88, v84\n"
        "v_exp_f32 v89, v85\n"
        "v_exp_f32 v90, v86\n"
        "v_exp_f32 v91, v87\n"
        "v_pk_add_f32 v[88:89], v[88:89], v[132:133]\n"
        "v_pk_add_f32 v[90:91], v[90:91], v[132:133]\n"
        "v_rcp_f32 v88, v88\n"
        "v_rcp_f32 v89, v89\n"
        "v_rcp_f32 v90, v90\n"
        "v_rcp_f32 v91, v91\n"
        "v_pk_mul_f32 v[84:85], v[88:89], v[80:81]\n"
        "v_pk_mul_f32 v[86:87], v[90:91], v[82:83]\n"
        "s_mov_b32 s44, 0x440000\n"
        "buffer_store_dwordx4 v[84:87], %[vout1], %[ro], s44 offen\n"
        "v_accvgpr_read_b32 v96, a67\n"
        "v_accvgpr_read_b32 v97, a75\n"
        "v_accvgpr_read_b32 v98, a83\n"
        "v_accvgpr_read_b32 v99, a91\n"
        "v_pk_mul_f32 v[100:101], v[96:97], v[96:97]\n"
        "v_pk_mul_f32 v[102:103], v[98:99], v[98:99]\n"
        "v_pk_fma_f32 v[100:101], v[100:101], v[130:131], v[128:129]\n"
        "v_pk_fma_f32 v[102:103], v[102:103], v[130:131], v[128:129]\n"
        "v_pk_mul_f32 v[100:101], v[100:101], v[96:97]\n"
        "v_pk_mul_f32 v[102:103], v[102:103], v[98:99]\n"
        "v_exp_f32 v104, v100\n"
        "v_exp_f32 v105, v101\n"
        "v_exp_f32 v106, v102\n"
        "v_exp_f32 v107, v103\n"
        "v_pk_add_f32 v[104:105], v[104:105], v[132:133]\n"
        "v_pk_add_f32 v[106:107], v[106:107], v[132:133]\n"
        "v_rcp_f32 v104, v104\n"
        "v_rcp_f32 v105, v105\n"
        "v_rcp_f32 v106, v106\n"
        "v_rcp_f32 v107, v107\n"
        "v_pk_mul_f32 v[100:101], v[104:105], v[96:97]\n"
        "v_pk_mul_f32 v[102:103], v[106:107], v[98:99]\n"
        "s_mov_b32 s44, 0x460000\n"
        "buffer_store_dwordx4 v[100:103], %[vout0], %[ro], s44 offen\n"
        "v_accvgpr_read_b32 v112, a71\n"
        "v_accvgpr_read_b32 v113, a79\n"
        "v_accvgpr_read_b32 v114, a87\n"
        "v_accvgpr_read_b32 v115, a95\n"
        "v_pk_mul_f32 v[116:117], v[112:113], v[112:113]\n"
        "v_pk_mul_f32 v[118:119], v[114:115], v[114:115]\n"
        "v_pk_fma_f32 v[116:117], v[116:117], v[130:131], v[128:129]\n"
        "v_pk_fma_f32 v[118:119], v[118:119], v[130:131], v[128:129]\n"
        "v_pk_mul_f32 v[116:117], v[116:117], v[112:113]\n"
        "v_pk_mul_f32 v[118:119], v[118:119], v[114:115]\n"
        "v_exp_f32 v120, v116\n"
        "v_exp_f32 v121, v117\n"
        "v_exp_f32 v122, v118\n"
        "v_exp_f32 v123, v119\n"
        "v_pk_add_f32 v[120:121], v[120:121], v[132:133]\n"
        "v_pk_add_f32 v[122:123], v[122:123], v[132:133]\n"
        "v_rcp_f32 v120, v120\n"
        "v_rcp_f32 v121, v121\n"
        "v_rcp_f32 v122, v122\n"
        "v_rcp_f32 v123, v123\n"
        "v_pk_mul_f32 v[116:117], v[120:121], v[112:113]\n"
        "v_pk_mul_f32 v[118:119], v[122:123], v[114:115]\n"
        "s_mov_b32 s44, 0x460000\n"
        "buffer_store_dwordx4 v[116:119], %[vout1], %[ro], s44 offen\n"
        "v_accvgpr_read_b32 v0, a96\n"
        "v_accvgpr_read_b32 v1, a104\n"
        "v_accvgpr_read_b32 v2, a112\n"
        "v_accvgpr_read_b32 v3, a120\n"
        "v_pk_mul_f32 v[4:5], v[0:1], v[0:1]\n"
        "v_pk_mul_f32 v[6:7], v[2:3], v[2:3]\n"
        "v_pk_fma_f32 v[4:5], v[4:5], v[130:131], v[128:129]\n"
        "v_pk_fma_f32 v[6:7], v[6:7], v[130:131], v[128:129]\n"
        "v_pk_mul_f32 v[4:5], v[4:5], v[0:1]\n"
        "v_pk_mul_f32 v[6:7], v[6:7], v[2:3]\n"
        "v_exp_f32 v8, v4\n"
        "v_exp_f32 v9, v5\n"
        "v_exp_f32 v10, v6\n"
        "v_exp_f32 v11, v7\n"
        "v_pk_add_f32 v[8:9], v[8:9], v[132:133]\n"
        "v_pk_add_f32 v[10:11], v[10:11], v[132:133]\n"
        "v_rcp_f32 v8, v8\n"
        "v_rcp_f32 v9, v9\n"
        "v_rcp_f32 v10, v10\n"
        "v_rcp_f32 v11, v11\n"
        "v_pk_mul_f32 v[4:5], v[8:9], v[0:1]\n"
        "v_pk_mul_f32 v[6:7], v[10:11], v[2:3]\n"
        "s_mov_b32 s44, 0x600000\n"
        "buffer_store_dwordx4 v[4:7], %[vout0], %[ro], s44 offen\n"
        "v_accvgpr_read_b32 v16, a100\n"
        "v_accvgpr_read_b32 v17, a108\n"
        "v_accvgpr_read_b32 v18, a116\n"
        "v_accvgpr_read_b32 v19, a124\n"
        "v_pk_mul_f32 v[20:21], v[16:17], v[16:17]\n"
        "v_pk_mul_f32 v[22:23], v[18:19], v[18:19]\n"
        "v_pk_fma_f32 v[20:21], v[20:21], v[130:131], v[128:129]\n"
        "v_pk_fma_f32 v[22:23], v[22:23], v[130:131], v[128:129]\n"
        "v_pk_mul_f32 v[20:21], v[20:21], v[16:17]\n"
        "v_pk_mul_f32 v[22:23], v[22:23], v[18:19]\n"
        "v_exp_f32 v24, v20\n"
        "v_exp_f32 v25, v21\n"
        "v_exp_f32 v26, v22\n"
        "v_exp_f32 v27, v23\n"
        "v_pk_add_f32 v[24:25], v[24:25], v[132:133]\n"
        "v_pk_add_f32 v[26:27], v[26:27], v[132:133]\n"
        "v_rcp_f32 v24, v24\n"
        "v_rcp_f32 v25, v25\n"
        "v_rcp_f32 v26, v26\n"
        "v_rcp_f32 v27, v27\n"
        "v_pk_mul_f32 v[20:21], v[24:25], v[16:17]\n"
        "v_pk_mul_f32 v[22:23], v[26:27], v[18:19]\n"
        "s_mov_b32 s44, 0x600000\n"
        "buffer_store_dwordx4 v[20:23], %[vout1], %[ro], s44 offen\n"
        "v_accvgpr_read_b32 v32, a97\n"
        "v_accvgpr_read_b32 v33, a105\n"
        "v_accvgpr_read_b32 v34, a113\n"
        "v_accvgpr_read_b32 v35, a121\n"
        "v_pk_mul_f32 v[36:37], v[32:33], v[32:33]\n"
        "v_pk_mul_f32 v[38:39], v[34:35], v[34:35]\n"
        "v_pk_fma_f32 v[36:37], v[36:37], v[130:131], v[128:129]\n"
        "v_pk_fma_f32 v[38:39], v[38:39], v[130:131], v[128:129]\n"
        "v_pk_mul_f32 v[36:37], v[36:37], v[32:33]\n"
        "v_pk_mul_f32 v[38:39], v[38:39], v[34:35]\n"
        "v_exp_f32 v40, v36\n"
        "v_exp_f32 v41, v37\n"
        "v_exp_f32 v42, v38\n"
        "v_exp_f32 v43, v39\n"
        "v_pk_add_f32 v[40:41], v[40:41], v[132:133]\n"
        "v_pk_add_f32 v[42:43], v[42:43], v[132:133]\n"
        "v_rcp_f32 v40, v40\n"
        "v_rcp_f32 v41, v41\n"
        "v_rcp_f32 v42, v42\n"
        "v_rcp_f32 v43, v43\n"
        "v_pk_mul_f32 v[36:37], v[40:41], v[32:33]\n"
        "v_pk_mul_f32 v[38:39], v[42:43], v[34:35]\n"
        "s_mov_b32 s44, 0x620000\n"
        "buffer_store_dwordx4 v[36:39], %[vout0], %[ro], s44 offen\n"
        "v_accvgpr_read_b32 v48, a101\n"
        "v_accvgpr_read_b32 v49, a109\n"
        "v_accvgpr_read_b32 v50, a117\n"
        "v_accvgpr_read_b32 v51, a125\n"
        "v_pk_mul_f32 v[52:53], v[48:49], v[48:49]\n"
        "v_pk_mul_f32 v[54:55], v[50:51], v[50:51]\n"
        "v_pk_fma_f32 v[52:53], v[52:53], v[130:131], v[128:129]\n"
        "v_pk_fma_f32 v[54:55], v[54:55], v[130:131], v[128:129]\n"
        "v_pk_mul_f32 v[52:53], v[52:53], v[48:49]\n"
        "v_pk_mul_f32 v[54:55], v[54:55], v[50:51]\n"
        "v_exp_f32 v56, v52\n"
        "v_exp_f32 v57, v53\n"
        "v_exp_f32 v58, v54\n"
        "v_exp_f32 v59, v55\n"
        "v_pk_add_f32 v[56:57], v[56:57], v[132:133]\n"
        "v_pk_add_f32 v[58:59], v[58:59], v[132:133]\n"
        "v_rcp_f32 v56, v56\n"
        "v_rcp_f32 v57, v57\n"
        "v_rcp_f32 v58, v58\n"
        "v_rcp_f32 v59, v59\n"
        "v_pk_mul_f32 v[52:53], v[56:57], v[48:49]\n"
        "v_pk_mul_f32 v[54:55], v[58:59], v[50:51]\n"
        "s_mov_b32 s44, 0x620000\n"
        "buffer_store_dwordx4 v[52:55], %[vout1], %[ro], s44 offen\n"
        "v_accvgpr_read_b32 v64, a98\n"
        "v_accvgpr_read_b32 v65, a106\n"
        "v_accvgpr_read_b32 v66, a114\n"
        "v_accvgpr_read_b32 v67, a122\n"
        "v_pk_mul_f32 v[68:69], v[64:65], v[64:65]\n"
        "v_pk_mul_f32 v[70:71], v[66:67], v[66:67]\n"
        "v_pk_fma_f32 v[68:69], v[68:69], v[130:131], v[128:129]\n"
        "v_pk_fma_f32 v[70:71], v[70:71], v[130:131], v[128:129]\n"
        "v_pk_mul_f32 v[68:69], v[68:69], v[64:65]\n"
        "v_pk_mul_f32 v[70:71], v[70:71], v[66:67]\n"
        "v_exp_f32 v72, v68\n"
        "v_exp_f32 v73, v69\n"
        "v_exp_f32 v74, v70\n"
        "v_exp_f32 v75, v71\n"
        "v_pk_add_f32 v[72:73], v[72:73], v[132:133]\n"
        "v_pk_add_f32 v[74:75], v[74:75], v[132:133]\n"
        "v_rcp_f32 v72, v72\n"
        "v_rcp_f32 v73, v73\n"
        "v_rcp_f32 v74, v74\n"
        "v_rcp_f32 v75, v75\n"
        "v_pk_mul_f32 v[68:69], v[72:73], v[64:65]\n"
        "v_pk_mul_f32 v[70:71], v[74:75], v[66:67]\n"
        "s_mov_b32 s44, 0x640000\n"
        "buffer_store_dwordx4 v[68:71], %[vout0], %[ro], s44 offen\n"
        "v_accvgpr_read_b32 v80, a102\n"
        "v_accvgpr_read_b32 v81, a110\n"
        "v_accvgpr_read_b32 v82, a118\n"
        "v_accvgpr_read_b32 v83, a126\n"
        "v_pk_mul_f32 v[84:85], v[80:81], v[80:81]\n"
        "v_pk_mul_f32 v[86:87], v[82:83], v[82:83]\n"
        "v_pk_fma_f32 v[84:85], v[84:85], v[130:131], v[128:129]\n"
        "v_pk_fma_f32 v[86:87], v[86:87], v[130:131], v[128:129]\n"
        "v_pk_mul_f32 v[84:85], v[84:85], v[80:81]\n"
        "v_pk_mul_f32 v[86:87], v[86:87], v[82:83]\n"
        "v_exp_f32 v88, v84\n"
        "v_exp_f32 v89, v85\n"
        "v_exp_f32 v90, v86\n"
        "v_exp_f32 v91, v87\n"
        "v_pk_add_f32 v[88:89], v[88:89], v[132:133]\n"
        "v_pk_add_f32 v[90:91], v[90:91], v[132:133]\n"
        "v_rcp_f32 v88, v88\n"
        "v_rcp_f32 v89, v89\n"
        "v_rcp_f32 v90, v90\n"
        "v_rcp_f32 v91, v91\n"
        "v_pk_mul_f32 v[84:85], v[88:89], v[80:81]\n"
        "v_pk_mul_f32 v[86:87], v[90:91], v[82:83]\n"
        "s_mov_b32 s44, 0x640000\n"
        "buffer_store_dwordx4 v[84:87], %[vout1], %[ro], s44 offen\n"
        "v_accvgpr_read_b32 v96, a99\n"
        "v_accvgpr_read_b32 v97, a107\n"
        "v_accvgpr_read_b32 v98, a115\n"
        "v_accvgpr_read_b32 v99, a123\n"
        "v_pk_mul_f32 v[100:101], v[96:97], v[96:97]\n"
        "v_pk_mul_f32 v[102:103], v[98:99], v[98:99]\n"
        "v_pk_fma_f32 v[100:101], v[100:101], v[130:131], v[128:129]\n"
        "v_pk_fma_f32 v[102:103], v[102:103], v[130:131], v[128:129]\n"
        "v_pk_mul_f32 v[100:101], v[100:101], v[96:97]\n"
        "v_pk_mul_f32 v[102:103], v[102:103], v[98:99]\n"
        "v_exp_f32 v104, v100\n"
        "v_exp_f32 v105, v101\n"
        "v_exp_f32 v106, v102\n"
        "v_exp_f32 v107, v103\n"
        "v_pk_add_f32 v[104:105], v[104:105], v[132:133]\n"
        "v_pk_add_f32 v[106:107], v[106:107], v[132:133]\n"
        "v_rcp_f32 v104, v104\n"
        "v_rcp_f32 v105, v105\n"
        "v_rcp_f32 v106, v106\n"
        "v_rcp_f32 v107, v107\n"
        "v_pk_mul_f32 v[100:101], v[104:105], v[96:97]\n"
        "v_pk_mul_f32 v[102:103], v[106:107], v[98:99]\n"
        "s_mov_b32 s44, 0x660000\n"
        "buffer_store_dwordx4 v[100:103], %[vout0], %[ro], s44 offen\n"
        "v_accvgpr_read_b32 v112, a103\n"
        "v_accvgpr_read_b32 v113, a111\n"
        "v_accvgpr_read_b32 v114, a119\n"
        "v_accvgpr_read_b32 v115, a127\n"
        "v_pk_mul_f32 v[116:117], v[112:113], v[112:113]\n"
        "v_pk_mul_f32 v[118:119], v[114:115], v[114:115]\n"
        "v_pk_fma_f32 v[116:117], v[116:117], v[130:131], v[128:129]\n"
        "v_pk_fma_f32 v[118:119], v[118:119], v[130:131], v[128:129]\n"
        "v_pk_mul_f32 v[116:117], v[116:117], v[112:113]\n"
        "v_pk_mul_f32 v[118:119], v[118:119], v[114:115]\n"
        "v_exp_f32 v120, v116\n"
        "v_exp_f32 v121, v117\n"
        "v_exp_f32 v122, v118\n"
        "v_exp_f32 v123, v119\n"
        "v_pk_add_f32 v[120:121], v[120:121], v[132:133]\n"
        "v_pk_add_f32 v[122:123], v[122:123], v[132:133]\n"
        "v_rcp_f32 v120, v120\n"
        "v_rcp_f32 v121, v121\n"
        "v_rcp_f32 v122, v122\n"
        "v_rcp_f32 v123, v123\n"
        "v_pk_mul_f32 v[116:117], v[120:121], v[112:113]\n"
        "v_pk_mul_f32 v[118:119], v[122:123], v[114:115]\n"
        "s_mov_b32 s44, 0x660000\n"
        "buffer_store_dwordx4 v[116:119], %[vout1], %[ro], s44 offen\n"
        "v_accvgpr_read_b32 v0, a128\n"
        "v_accvgpr_read_b32 v1, a136\n"
        "v_accvgpr_read_b32 v2, a144\n"
        "v_accvgpr_read_b32 v3, a152\n"
        "v_pk_mul_f32 v[4:5], v[0:1], v[0:1]\n"
        "v_pk_mul_f32 v[6:7], v[2:3], v[2:3]\n"
        "v_pk_fma_f32 v[4:5], v[4:5], v[130:131], v[128:129]\n"
        "v_pk_fma_f32 v[6:7], v[6:7], v[130:131], v[128:129]\n"
        "v_pk_mul_f32 v[4:5], v[4:5], v[0:1]\n"
        "v_pk_mul_f32 v[6:7], v[6:7], v[2:3]\n"
        "v_exp_f32 v8, v4\n"
        "v_exp_f32 v9, v5\n"
        "v_exp_f32 v10, v6\n"
        "v_exp_f32 v11, v7\n"
        "v_pk_add_f32 v[8:9], v[8:9], v[132:133]\n"
        "v_pk_add_f32 v[10:11], v[10:11], v[132:133]\n"
        "v_rcp_f32 v8, v8\n"
        "v_rcp_f32 v9, v9\n"
        "v_rcp_f32 v10, v10\n"
        "v_rcp_f32 v11, v11\n"
        "v_pk_mul_f32 v[4:5], v[8:9], v[0:1]\n"
        "v_pk_mul_f32 v[6:7], v[10:11], v[2:3]\n"
        "s_mov_b32 s44, 0x800000\n"
        "buffer_store_dwordx4 v[4:7], %[vout0], %[ro], s44 offen\n"
        "v_accvgpr_read_b32 v16, a132\n"
        "v_accvgpr_read_b32 v17, a140\n"
        "v_accvgpr_read_b32 v18, a148\n"
        "v_accvgpr_read_b32 v19, a156\n"
        "v_pk_mul_f32 v[20:21], v[16:17], v[16:17]\n"
        "v_pk_mul_f32 v[22:23], v[18:19], v[18:19]\n"
        "v_pk_fma_f32 v[20:21], v[20:21], v[130:131], v[128:129]\n"
        "v_pk_fma_f32 v[22:23], v[22:23], v[130:131], v[128:129]\n"
        "v_pk_mul_f32 v[20:21], v[20:21], v[16:17]\n"
        "v_pk_mul_f32 v[22:23], v[22:23], v[18:19]\n"
        "v_exp_f32 v24, v20\n"
        "v_exp_f32 v25, v21\n"
        "v_exp_f32 v26, v22\n"
        "v_exp_f32 v27, v23\n"
        "v_pk_add_f32 v[24:25], v[24:25], v[132:133]\n"
        "v_pk_add_f32 v[26:27], v[26:27], v[132:133]\n"
        "v_rcp_f32 v24, v24\n"
        "v_rcp_f32 v25, v25\n"
        "v_rcp_f32 v26, v26\n"
        "v_rcp_f32 v27, v27\n"
        "v_pk_mul_f32 v[20:21], v[24:25], v[16:17]\n"
        "v_pk_mul_f32 v[22:23], v[26:27], v[18:19]\n"
        "s_mov_b32 s44, 0x800000\n"
        "buffer_store_dwordx4 v[20:23], %[vout1], %[ro], s44 offen\n"
        "v_accvgpr_read_b32 v32, a129\n"
        "v_accvgpr_read_b32 v33, a137\n"
        "v_accvgpr_read_b32 v34, a145\n"
        "v_accvgpr_read_b32 v35, a153\n"
        "v_pk_mul_f32 v[36:37], v[32:33], v[32:33]\n"
        "v_pk_mul_f32 v[38:39], v[34:35], v[34:35]\n"
        "v_pk_fma_f32 v[36:37], v[36:37], v[130:131], v[128:129]\n"
        "v_pk_fma_f32 v[38:39], v[38:39], v[130:131], v[128:129]\n"
        "v_pk_mul_f32 v[36:37], v[36:37], v[32:33]\n"
        "v_pk_mul_f32 v[38:39], v[38:39], v[34:35]\n"
        "v_exp_f32 v40, v36\n"
        "v_exp_f32 v41, v37\n"
        "v_exp_f32 v42, v38\n"
        "v_exp_f32 v43, v39\n"
        "v_pk_add_f32 v[40:41], v[40:41], v[132:133]\n"
        "v_pk_add_f32 v[42:43], v[42:43], v[132:133]\n"
        "v_rcp_f32 v40, v40\n"
        "v_rcp_f32 v41, v41\n"
        "v_rcp_f32 v42, v42\n"
        "v_rcp_f32 v43, v43\n"
        "v_pk_mul_f32 v[36:37], v[40:41], v[32:33]\n"
        "v_pk_mul_f32 v[38:39], v[42:43], v[34:35]\n"
        "s_mov_b32 s44, 0x820000\n"
        "buffer_store_dwordx4 v[36:39], %[vout0], %[ro], s44 offen\n"
        "v_accvgpr_read_b32 v48, a133\n"
        "v_accvgpr_read_b32 v49, a141\n"
        "v_accvgpr_read_b32 v50, a149\n"
        "v_accvgpr_read_b32 v51, a157\n"
        "v_pk_mul_f32 v[52:53], v[48:49], v[48:49]\n"
        "v_pk_mul_f32 v[54:55], v[50:51], v[50:51]\n"
        "v_pk_fma_f32 v[52:53], v[52:53], v[130:131], v[128:129]\n"
        "v_pk_fma_f32 v[54:55], v[54:55], v[130:131], v[128:129]\n"
        "v_pk_mul_f32 v[52:53], v[52:53], v[48:49]\n"
        "v_pk_mul_f32 v[54:55], v[54:55], v[50:51]\n"
        "v_exp_f32 v56, v52\n"
        "v_exp_f32 v57, v53\n"
        "v_exp_f32 v58, v54\n"
        "v_exp_f32 v59, v55\n"
        "v_pk_add_f32 v[56:57], v[56:57], v[132:133]\n"
        "v_pk_add_f32 v[58:59], v[58:59], v[132:133]\n"
        "v_rcp_f32 v56, v56\n"
        "v_rcp_f32 v57, v57\n"
        "v_rcp_f32 v58, v58\n"
        "v_rcp_f32 v59, v59\n"
        "v_pk_mul_f32 v[52:53], v[56:57], v[48:49]\n"
        "v_pk_mul_f32 v[54:55], v[58:59], v[50:51]\n"
        "s_mov_b32 s44, 0x820000\n"
        "buffer_store_dwordx4 v[52:55], %[vout1], %[ro], s44 offen\n"
        "v_accvgpr_read_b32 v64, a130\n"
        "v_accvgpr_read_b32 v65, a138\n"
        "v_accvgpr_read_b32 v66, a146\n"
        "v_accvgpr_read_b32 v67, a154\n"
        "v_pk_mul_f32 v[68:69], v[64:65], v[64:65]\n"
        "v_pk_mul_f32 v[70:71], v[66:67], v[66:67]\n"
        "v_pk_fma_f32 v[68:69], v[68:69], v[130:131], v[128:129]\n"
        "v_pk_fma_f32 v[70:71], v[70:71], v[130:131], v[128:129]\n"
        "v_pk_mul_f32 v[68:69], v[68:69], v[64:65]\n"
        "v_pk_mul_f32 v[70:71], v[70:71], v[66:67]\n"
        "v_exp_f32 v72, v68\n"
        "v_exp_f32 v73, v69\n"
        "v_exp_f32 v74, v70\n"
        "v_exp_f32 v75, v71\n"
        "v_pk_add_f32 v[72:73], v[72:73], v[132:133]\n"
        "v_pk_add_f32 v[74:75], v[74:75], v[132:133]\n"
        "v_rcp_f32 v72, v72\n"
        "v_rcp_f32 v73, v73\n"
        "v_rcp_f32 v74, v74\n"
        "v_rcp_f32 v75, v75\n"
        "v_pk_mul_f32 v[68:69], v[72:73], v[64:65]\n"
        "v_pk_mul_f32 v[70:71], v[74:75], v[66:67]\n"
        "s_mov_b32 s44, 0x840000\n"
        "buffer_store_dwordx4 v[68:71], %[vout0], %[ro], s44 offen\n"
        "v_accvgpr_read_b32 v80, a134\n"
        "v_accvgpr_read_b32 v81, a142\n"
        "v_accvgpr_read_b32 v82, a150\n"
        "v_accvgpr_read_b32 v83, a158\n"
        "v_pk_mul_f32 v[84:85], v[80:81], v[80:81]\n"
        "v_pk_mul_f32 v[86:87], v[82:83], v[82:83]\n"
        "v_pk_fma_f32 v[84:85], v[84:85], v[130:131], v[128:129]\n"
        "v_pk_fma_f32 v[86:87], v[86:87], v[130:131], v[128:129]\n"
        "v_pk_mul_f32 v[84:85], v[84:85], v[80:81]\n"
        "v_pk_mul_f32 v[86:87], v[86:87], v[82:83]\n"
        "v_exp_f32 v88, v84\n"
        "v_exp_f32 v89, v85\n"
        "v_exp_f32 v90, v86\n"
        "v_exp_f32 v91, v87\n"
        "v_pk_add_f32 v[88:89], v[88:89], v[132:133]\n"
        "v_pk_add_f32 v[90:91], v[90:91], v[132:133]\n"
        "v_rcp_f32 v88, v88\n"
        "v_rcp_f32 v89, v89\n"
        "v_rcp_f32 v90, v90\n"
        "v_rcp_f32 v91, v91\n"
        "v_pk_mul_f32 v[84:85], v[88:89], v[80:81]\n"
        "v_pk_mul_f32 v[86:87], v[90:91], v[82:83]\n"
        "s_mov_b32 s44, 0x840000\n"
        "buffer_store_dwordx4 v[84:87], %[vout1], %[ro], s44 offen\n"
        "v_accvgpr_read_b32 v96, a131\n"
        "v_accvgpr_read_b32 v97, a139\n"
        "v_accvgpr_read_b32 v98, a147\n"
        "v_accvgpr_read_b32 v99, a155\n"
        "v_pk_mul_f32 v[100:101], v[96:97], v[96:97]\n"
        "v_pk_mul_f32 v[102:103], v[98:99], v[98:99]\n"
        "v_pk_fma_f32 v[100:101], v[100:101], v[130:131], v[128:129]\n"
        "v_pk_fma_f32 v[102:103], v[102:103], v[130:131], v[128:129]\n"
        "v_pk_mul_f32 v[100:101], v[100:101], v[96:97]\n"
        "v_pk_mul_f32 v[102:103], v[102:103], v[98:99]\n"
        "v_exp_f32 v104, v100\n"
        "v_exp_f32 v105, v101\n"
        "v_exp_f32 v106, v102\n"
        "v_exp_f32 v107, v103\n"
        "v_pk_add_f32 v[104:105], v[104:105], v[132:133]\n"
        "v_pk_add_f32 v[106:107], v[106:107], v[132:133]\n"
        "v_rcp_f32 v104, v104\n"
        "v_rcp_f32 v105, v105\n"
        "v_rcp_f32 v106, v106\n"
        "v_rcp_f32 v107, v107\n"
        "v_pk_mul_f32 v[100:101], v[104:105], v[96:97]\n"
        "v_pk_mul_f32 v[102:103], v[106:107], v[98:99]\n"
        "s_mov_b32 s44, 0x860000\n"
        "buffer_store_dwordx4 v[100:103], %[vout0], %[ro], s44 offen\n"
        "v_accvgpr_read_b32 v112, a135\n"
        "v_accvgpr_read_b32 v113, a143\n"
        "v_accvgpr_read_b32 v114, a151\n"
        "v_accvgpr_read_b32 v115, a159\n"
        "v_pk_mul_f32 v[116:117], v[112:113], v[112:113]\n"
        "v_pk_mul_f32 v[118:119], v[114:115], v[114:115]\n"
        "v_pk_fma_f32 v[116:117], v[116:117], v[130:131], v[128:129]\n"
        "v_pk_fma_f32 v[118:119], v[118:119], v[130:131], v[128:129]\n"
        "v_pk_mul_f32 v[116:117], v[116:117], v[112:113]\n"
        "v_pk_mul_f32 v[118:119], v[118:119], v[114:115]\n"
        "v_exp_f32 v120, v116\n"
        "v_exp_f32 v121, v117\n"
        "v_exp_f32 v122, v118\n"
        "v_exp_f32 v123, v119\n"
        "v_pk_add_f32 v[120:121], v[120:121], v[132:133]\n"
        "v_pk_add_f32 v[122:123], v[122:123], v[132:133]\n"
        "v_rcp_f32 v120, v120\n"
        "v_rcp_f32 v121, v121\n"
        "v_rcp_f32 v122, v122\n"
        "v_rcp_f32 v123, v123\n"
        "v_pk_mul_f32 v[116:117], v[120:121], v[112:113]\n"
        "v_pk_mul_f32 v[118:119], v[122:123], v[114:115]\n"
        "s_mov_b32 s44, 0x860000\n"
        "buffer_store_dwordx4 v[116:119], %[vout1], %[ro], s44 offen\n"
        "v_accvgpr_read_b32 v0, a160\n"
        "v_accvgpr_read_b32 v1, a168\n"
        "v_accvgpr_read_b32 v2, a176\n"
        "v_accvgpr_read_b32 v3, a184\n"
        "v_pk_mul_f32 v[4:5], v[0:1], v[0:1]\n"
        "v_pk_mul_f32 v[6:7], v[2:3], v[2:3]\n"
        "v_pk_fma_f32 v[4:5], v[4:5], v[130:131], v[128:129]\n"
        "v_pk_fma_f32 v[6:7], v[6:7], v[130:131], v[128:129]\n"
        "v_pk_mul_f32 v[4:5], v[4:5], v[0:1]\n"
        "v_pk_mul_f32 v[6:7], v[6:7], v[2:3]\n"
        "v_exp_f32 v8, v4\n"
        "v_exp_f32 v9, v5\n"
        "v_exp_f32 v10, v6\n"
        "v_exp_f32 v11, v7\n"
        "v_pk_add_f32 v[8:9], v[8:9], v[132:133]\n"
        "v_pk_add_f32 v[10:11], v[10:11], v[132:133]\n"
        "v_rcp_f32 v8, v8\n"
        "v_rcp_f32 v9, v9\n"
        "v_rcp_f32 v10, v10\n"
        "v_rcp_f32 v11, v11\n"
        "v_pk_mul_f32 v[4:5], v[8:9], v[0:1]\n"
        "v_pk_mul_f32 v[6:7], v[10:11], v[2:3]\n"
        "s_mov_b32 s44, 0xa00000\n"
        "buffer_store_dwordx4 v[4:7], %[vout0], %[ro], s44 offen\n"
        "v_accvgpr_read_b32 v16, a164\n"
        "v_accvgpr_read_b32 v17, a172\n"
        "v_accvgpr_read_b32 v18, a180\n"
        "v_accvgpr_read_b32 v19, a188\n"
        "v_pk_mul_f32 v[20:21], v[16:17], v[16:17]\n"
        "v_pk_mul_f32 v[22:23], v[18:19], v[18:19]\n"
        "v_pk_fma_f32 v[20:21], v[20:21], v[130:131], v[128:129]\n"
        "v_pk_fma_f32 v[22:23], v[22:23], v[130:131], v[128:129]\n"
        "v_pk_mul_f32 v[20:21], v[20:21], v[16:17]\n"
        "v_pk_mul_f32 v[22:23], v[22:23], v[18:19]\n"
        "v_exp_f32 v24, v20\n"
        "v_exp_f32 v25, v21\n"
        "v_exp_f32 v26, v22\n"
        "v_exp_f32 v27, v23\n"
        "v_pk_add_f32 v[24:25], v[24:25], v[132:133]\n"
        "v_pk_add_f32 v[26:27], v[26:27], v[132:133]\n"
        "v_rcp_f32 v24, v24\n"
        "v_rcp_f32 v25, v25\n"
        "v_rcp_f32 v26, v26\n"
        "v_rcp_f32 v27, v27\n"
        "v_pk_mul_f32 v[20:21], v[24:25], v[16:17]\n"
        "v_pk_mul_f32 v[22:23], v[26:27], v[18:19]\n"
        "s_mov_b32 s44, 0xa00000\n"
        "buffer_store_dwordx4 v[20:23], %[vout1], %[ro], s44 offen\n"
        "v_accvgpr_read_b32 v32, a161\n"
        "v_accvgpr_read_b32 v33, a169\n"
        "v_accvgpr_read_b32 v34, a177\n"
        "v_accvgpr_read_b32 v35, a185\n"
        "v_pk_mul_f32 v[36:37], v[32:33], v[32:33]\n"
        "v_pk_mul_f32 v[38:39], v[34:35], v[34:35]\n"
        "v_pk_fma_f32 v[36:37], v[36:37], v[130:131], v[128:129]\n"
        "v_pk_fma_f32 v[38:39], v[38:39], v[130:131], v[128:129]\n"
        "v_pk_mul_f32 v[36:37], v[36:37], v[32:33]\n"
        "v_pk_mul_f32 v[38:39], v[38:39], v[34:35]\n"
        "v_exp_f32 v40, v36\n"
        "v_exp_f32 v41, v37\n"
        "v_exp_f32 v42, v38\n"
        "v_exp_f32 v43, v39\n"
        "v_pk_add_f32 v[40:41], v[40:41], v[132:133]\n"
        "v_pk_add_f32 v[42:43], v[42:43], v[132:133]\n"
        "v_rcp_f32 v40, v40\n"
        "v_rcp_f32 v41, v41\n"
        "v_rcp_f32 v42, v42\n"
        "v_rcp_f32 v43, v43\n"
        "v_pk_mul_f32 v[36:37], v[40:41], v[32:33]\n"
        "v_pk_mul_f32 v[38:39], v[42:43], v[34:35]\n"
        "s_mov_b32 s44, 0xa20000\n"
        "buffer_store_dwordx4 v[36:39], %[vout0], %[ro], s44 offen\n"
        "v_accvgpr_read_b32 v48, a165\n"
        "v_accvgpr_read_b32 v49, a173\n"
        "v_accvgpr_read_b32 v50, a181\n"
        "v_accvgpr_read_b32 v51, a189\n"
        "v_pk_mul_f32 v[52:53], v[48:49], v[48:49]\n"
        "v_pk_mul_f32 v[54:55], v[50:51], v[50:51]\n"
        "v_pk_fma_f32 v[52:53], v[52:53], v[130:131], v[128:129]\n"
        "v_pk_fma_f32 v[54:55], v[54:55], v[130:131], v[128:129]\n"
        "v_pk_mul_f32 v[52:53], v[52:53], v[48:49]\n"
        "v_pk_mul_f32 v[54:55], v[54:55], v[50:51]\n"
        "v_exp_f32 v56, v52\n"
        "v_exp_f32 v57, v53\n"
        "v_exp_f32 v58, v54\n"
        "v_exp_f32 v59, v55\n"
        "v_pk_add_f32 v[56:57], v[56:57], v[132:133]\n"
        "v_pk_add_f32 v[58:59], v[58:59], v[132:133]\n"
        "v_rcp_f32 v56, v56\n"
        "v_rcp_f32 v57, v57\n"
        "v_rcp_f32 v58, v58\n"
        "v_rcp_f32 v59, v59\n"
        "v_pk_mul_f32 v[52:53], v[56:57], v[48:49]\n"
        "v_pk_mul_f32 v[54:55], v[58:59], v[50:51]\n"
        "s_mov_b32 s44, 0xa20000\n"
        "buffer_store_dwordx4 v[52:55], %[vout1], %[ro], s44 offen\n"
        "v_accvgpr_read_b32 v64, a162\n"
        "v_accvgpr_read_b32 v65, a170\n"
        "v_accvgpr_read_b32 v66, a178\n"
        "v_accvgpr_read_b32 v67, a186\n"
        "v_pk_mul_f32 v[68:69], v[64:65], v[64:65]\n"
        "v_pk_mul_f32 v[70:71], v[66:67], v[66:67]\n"
        "v_pk_fma_f32 v[68:69], v[68:69], v[130:131], v[128:129]\n"
        "v_pk_fma_f32 v[70:71], v[70:71], v[130:131], v[128:129]\n"
        "v_pk_mul_f32 v[68:69], v[68:69], v[64:65]\n"
        "v_pk_mul_f32 v[70:71], v[70:71], v[66:67]\n"
        "v_exp_f32 v72, v68\n"
        "v_exp_f32 v73, v69\n"
        "v_exp_f32 v74, v70\n"
        "v_exp_f32 v75, v71\n"
        "v_pk_add_f32 v[72:73], v[72:73], v[132:133]\n"
        "v_pk_add_f32 v[74:75], v[74:75], v[132:133]\n"
        "v_rcp_f32 v72, v72\n"
        "v_rcp_f32 v73, v73\n"
        "v_rcp_f32 v74, v74\n"
        "v_rcp_f32 v75, v75\n"
        "v_pk_mul_f32 v[68:69], v[72:73], v[64:65]\n"
        "v_pk_mul_f32 v[70:71], v[74:75], v[66:67]\n"
        "s_mov_b32 s44, 0xa40000\n"
        "buffer_store_dwordx4 v[68:71], %[vout0], %[ro], s44 offen\n"
        "v_accvgpr_read_b32 v80, a166\n"
        "v_accvgpr_read_b32 v81, a174\n"
        "v_accvgpr_read_b32 v82, a182\n"
        "v_accvgpr_read_b32 v83, a190\n"
        "v_pk_mul_f32 v[84:85], v[80:81], v[80:81]\n"
        "v_pk_mul_f32 v[86:87], v[82:83], v[82:83]\n"
        "v_pk_fma_f32 v[84:85], v[84:85], v[130:131], v[128:129]\n"
        "v_pk_fma_f32 v[86:87], v[86:87], v[130:131], v[128:129]\n"
        "v_pk_mul_f32 v[84:85], v[84:85], v[80:81]\n"
        "v_pk_mul_f32 v[86:87], v[86:87], v[82:83]\n"
        "v_exp_f32 v88, v84\n"
        "v_exp_f32 v89, v85\n"
        "v_exp_f32 v90, v86\n"
        "v_exp_f32 v91, v87\n"
        "v_pk_add_f32 v[88:89], v[88:89], v[132:133]\n"
        "v_pk_add_f32 v[90:91], v[90:91], v[132:133]\n"
        "v_rcp_f32 v88, v88\n"
        "v_rcp_f32 v89, v89\n"
        "v_rcp_f32 v90, v90\n"
        "v_rcp_f32 v91, v91\n"
        "v_pk_mul_f32 v[84:85], v[88:89], v[80:81]\n"
        "v_pk_mul_f32 v[86:87], v[90:91], v[82:83]\n"
        "s_mov_b32 s44, 0xa40000\n"
        "buffer_store_dwordx4 v[84:87], %[vout1], %[ro], s44 offen\n"
        "v_accvgpr_read_b32 v96, a163\n"
        "v_accvgpr_read_b32 v97, a171\n"
        "v_accvgpr_read_b32 v98, a179\n"
        "v_accvgpr_read_b32 v99, a187\n"
        "v_pk_mul_f32 v[100:101], v[96:97], v[96:97]\n"
        "v_pk_mul_f32 v[102:103], v[98:99], v[98:99]\n"
        "v_pk_fma_f32 v[100:101], v[100:101], v[130:131], v[128:129]\n"
        "v_pk_fma_f32 v[102:103], v[102:103], v[130:131], v[128:129]\n"
        "v_pk_mul_f32 v[100:101], v[100:101], v[96:97]\n"
        "v_pk_mul_f32 v[102:103], v[102:103], v[98:99]\n"
        "v_exp_f32 v104, v100\n"
        "v_exp_f32 v105, v101\n"
        "v_exp_f32 v106, v102\n"
        "v_exp_f32 v107, v103\n"
        "v_pk_add_f32 v[104:105], v[104:105], v[132:133]\n"
        "v_pk_add_f32 v[106:107], v[106:107], v[132:133]\n"
        "v_rcp_f32 v104, v104\n"
        "v_rcp_f32 v105, v105\n"
        "v_rcp_f32 v106, v106\n"
        "v_rcp_f32 v107, v107\n"
        "v_pk_mul_f32 v[100:101], v[104:105], v[96:97]\n"
        "v_pk_mul_f32 v[102:103], v[106:107], v[98:99]\n"
        "s_mov_b32 s44, 0xa60000\n"
        "buffer_store_dwordx4 v[100:103], %[vout0], %[ro], s44 offen\n"
        "v_accvgpr_read_b32 v112, a167\n"
        "v_accvgpr_read_b32 v113, a175\n"
        "v_accvgpr_read_b32 v114, a183\n"
        "v_accvgpr_read_b32 v115, a191\n"
        "v_pk_mul_f32 v[116:117], v[112:113], v[112:113]\n"
        "v_pk_mul_f32 v[118:119], v[114:115], v[114:115]\n"
        "v_pk_fma_f32 v[116:117], v[116:117], v[130:131], v[128:129]\n"
        "v_pk_fma_f32 v[118:119], v[118:119], v[130:131], v[128:129]\n"
        "v_pk_mul_f32 v[116:117], v[116:117], v[112:113]\n"
        "v_pk_mul_f32 v[118:119], v[118:119], v[114:115]\n"
        "v_exp_f32 v120, v116\n"
        "v_exp_f32 v121, v117\n"
        "v_exp_f32 v122, v118\n"
        "v_exp_f32 v123, v119\n"
        "v_pk_add_f32 v[120:121], v[120:121], v[132:133]\n"
        "v_pk_add_f32 v[122:123], v[122:123], v[132:133]\n"
        "v_rcp_f32 v120, v120\n"
        "v_rcp_f32 v121, v121\n"
        "v_rcp_f32 v122, v122\n"
        "v_rcp_f32 v123, v123\n"
        "v_pk_mul_f32 v[116:117], v[120:121], v[112:113]\n"
        "v_pk_mul_f32 v[118:119], v[122:123], v[114:115]\n"
        "s_mov_b32 s44, 0xa60000\n"
        "buffer_store_dwordx4 v[116:119], %[vout1], %[ro], s44 offen\n"
        "v_accvgpr_read_b32 v0, a192\n"
        "v_accvgpr_read_b32 v1, a200\n"
        "v_accvgpr_read_b32 v2, a208\n"
        "v_accvgpr_read_b32 v3, a216\n"
        "v_pk_mul_f32 v[4:5], v[0:1], v[0:1]\n"
        "v_pk_mul_f32 v[6:7], v[2:3], v[2:3]\n"
        "v_pk_fma_f32 v[4:5], v[4:5], v[130:131], v[128:129]\n"
        "v_pk_fma_f32 v[6:7], v[6:7], v[130:131], v[128:129]\n"
        "v_pk_mul_f32 v[4:5], v[4:5], v[0:1]\n"
        "v_pk_mul_f32 v[6:7], v[6:7], v[2:3]\n"
        "v_exp_f32 v8, v4\n"
        "v_exp_f32 v9, v5\n"
        "v_exp_f32 v10, v6\n"
        "v_exp_f32 v11, v7\n"
        "v_pk_add_f32 v[8:9], v[8:9], v[132:133]\n"
        "v_pk_add_f32 v[10:11], v[10:11], v[132:133]\n"
        "v_rcp_f32 v8, v8\n"
        "v_rcp_f32 v9, v9\n"
        "v_rcp_f32 v10, v10\n"
        "v_rcp_f32 v11, v11\n"
        "v_pk_mul_f32 v[4:5], v[8:9], v[0:1]\n"
        "v_pk_mul_f32 v[6:7], v[10:11], v[2:3]\n"
        "s_mov_b32 s44, 0xc00000\n"
        "buffer_store_dwordx4 v[4:7], %[vout0], %[ro], s44 offen\n"
        "v_accvgpr_read_b32 v16, a196\n"
        "v_accvgpr_read_b32 v17, a204\n"
        "v_accvgpr_read_b32 v18, a212\n"
        "v_accvgpr_read_b32 v19, a220\n"
        "v_pk_mul_f32 v[20:21], v[16:17], v[16:17]\n"
        "v_pk_mul_f32 v[22:23], v[18:19], v[18:19]\n"
        "v_pk_fma_f32 v[20:21], v[20:21], v[130:131], v[128:129]\n"
        "v_pk_fma_f32 v[22:23], v[22:23], v[130:131], v[128:129]\n"
        "v_pk_mul_f32 v[20:21], v[20:21], v[16:17]\n"
        "v_pk_mul_f32 v[22:23], v[22:23], v[18:19]\n"
        "v_exp_f32 v24, v20\n"
        "v_exp_f32 v25, v21\n"
        "v_exp_f32 v26, v22\n"
        "v_exp_f32 v27, v23\n"
        "v_pk_add_f32 v[24:25], v[24:25], v[132:133]\n"
        "v_pk_add_f32 v[26:27], v[26:27], v[132:133]\n"
        "v_rcp_f32 v24, v24\n"
        "v_rcp_f32 v25, v25\n"
        "v_rcp_f32 v26, v26\n"
        "v_rcp_f32 v27, v27\n"
        "v_pk_mul_f32 v[20:21], v[24:25], v[16:17]\n"
        "v_pk_mul_f32 v[22:23], v[26:27], v[18:19]\n"
        "s_mov_b32 s44, 0xc00000\n"
        "buffer_store_dwordx4 v[20:23], %[vout1], %[ro], s44 offen\n"
        "v_accvgpr_read_b32 v32, a193\n"
        "v_accvgpr_read_b32 v33, a201\n"
        "v_accvgpr_read_b32 v34, a209\n"
        "v_accvgpr_read_b32 v35, a217\n"
        "v_pk_mul_f32 v[36:37], v[32:33], v[32:33]\n"
        "v_pk_mul_f32 v[38:39], v[34:35], v[34:35]\n"
        "v_pk_fma_f32 v[36:37], v[36:37], v[130:131], v[128:129]\n"
        "v_pk_fma_f32 v[38:39], v[38:39], v[130:131], v[128:129]\n"
        "v_pk_mul_f32 v[36:37], v[36:37], v[32:33]\n"
        "v_pk_mul_f32 v[38:39], v[38:39], v[34:35]\n"
        "v_exp_f32 v40, v36\n"
        "v_exp_f32 v41, v37\n"
        "v_exp_f32 v42, v38\n"
        "v_exp_f32 v43, v39\n"
        "v_pk_add_f32 v[40:41], v[40:41], v[132:133]\n"
        "v_pk_add_f32 v[42:43], v[42:43], v[132:133]\n"
        "v_rcp_f32 v40, v40\n"
        "v_rcp_f32 v41, v41\n"
        "v_rcp_f32 v42, v42\n"
        "v_rcp_f32 v43, v43\n"
        "v_pk_mul_f32 v[36:37], v[40:41], v[32:33]\n"
        "v_pk_mul_f32 v[38:39], v[42:43], v[34:35]\n"
        "s_mov_b32 s44, 0xc20000\n"
        "buffer_store_dwordx4 v[36:39], %[vout0], %[ro], s44 offen\n"
        "v_accvgpr_read_b32 v48, a197\n"
        "v_accvgpr_read_b32 v49, a205\n"
        "v_accvgpr_read_b32 v50, a213\n"
        "v_accvgpr_read_b32 v51, a221\n"
        "v_pk_mul_f32 v[52:53], v[48:49], v[48:49]\n"
        "v_pk_mul_f32 v[54:55], v[50:51], v[50:51]\n"
        "v_pk_fma_f32 v[52:53], v[52:53], v[130:131], v[128:129]\n"
        "v_pk_fma_f32 v[54:55], v[54:55], v[130:131], v[128:129]\n"
        "v_pk_mul_f32 v[52:53], v[52:53], v[48:49]\n"
        "v_pk_mul_f32 v[54:55], v[54:55], v[50:51]\n"
        "v_exp_f32 v56, v52\n"
        "v_exp_f32 v57, v53\n"
        "v_exp_f32 v58, v54\n"
        "v_exp_f32 v59, v55\n"
        "v_pk_add_f32 v[56:57], v[56:57], v[132:133]\n"
        "v_pk_add_f32 v[58:59], v[58:59], v[132:133]\n"
        "v_rcp_f32 v56, v56\n"
        "v_rcp_f32 v57, v57\n"
        "v_rcp_f32 v58, v58\n"
        "v_rcp_f32 v59, v59\n"
        "v_pk_mul_f32 v[52:53], v[56:57], v[48:49]\n"
        "v_pk_mul_f32 v[54:55], v[58:59], v[50:51]\n"
        "s_mov_b32 s44, 0xc20000\n"
        "buffer_store_dwordx4 v[52:55], %[vout1], %[ro], s44 offen\n"
        "v_accvgpr_read_b32 v64, a194\n"
        "v_accvgpr_read_b32 v65, a202\n"
        "v_accvgpr_read_b32 v66, a210\n"
        "v_accvgpr_read_b32 v67, a218\n"
        "v_pk_mul_f32 v[68:69], v[64:65], v[64:65]\n"
        "v_pk_mul_f32 v[70:71], v[66:67], v[66:67]\n"
        "v_pk_fma_f32 v[68:69], v[68:69], v[130:131], v[128:129]\n"
        "v_pk_fma_f32 v[70:71], v[70:71], v[130:131], v[128:129]\n"
        "v_pk_mul_f32 v[68:69], v[68:69], v[64:65]\n"
        "v_pk_mul_f32 v[70:71], v[70:71], v[66:67]\n"
        "v_exp_f32 v72, v68\n"
        "v_exp_f32 v73, v69\n"
        "v_exp_f32 v74, v70\n"
        "v_exp_f32 v75, v71\n"
        "v_pk_add_f32 v[72:73], v[72:73], v[132:133]\n"
        "v_pk_add_f32 v[74:75], v[74:75], v[132:133]\n"
        "v_rcp_f32 v72, v72\n"
        "v_rcp_f32 v73, v73\n"
        "v_rcp_f32 v74, v74\n"
        "v_rcp_f32 v75, v75\n"
        "v_pk_mul_f32 v[68:69], v[72:73], v[64:65]\n"
        "v_pk_mul_f32 v[70:71], v[74:75], v[66:67]\n"
        "s_mov_b32 s44, 0xc40000\n"
        "buffer_store_dwordx4 v[68:71], %[vout0], %[ro], s44 offen\n"
        "v_accvgpr_read_b32 v80, a198\n"
        "v_accvgpr_read_b32 v81, a206\n"
        "v_accvgpr_read_b32 v82, a214\n"
        "v_accvgpr_read_b32 v83, a222\n"
        "v_pk_mul_f32 v[84:85], v[80:81], v[80:81]\n"
        "v_pk_mul_f32 v[86:87], v[82:83], v[82:83]\n"
        "v_pk_fma_f32 v[84:85], v[84:85], v[130:131], v[128:129]\n"
        "v_pk_fma_f32 v[86:87], v[86:87], v[130:131], v[128:129]\n"
        "v_pk_mul_f32 v[84:85], v[84:85], v[80:81]\n"
        "v_pk_mul_f32 v[86:87], v[86:87], v[82:83]\n"
        "v_exp_f32 v88, v84\n"
        "v_exp_f32 v89, v85\n"
        "v_exp_f32 v90, v86\n"
        "v_exp_f32 v91, v87\n"
        "v_pk_add_f32 v[88:89], v[88:89], v[132:133]\n"
        "v_pk_add_f32 v[90:91], v[90:91], v[132:133]\n"
        "v_rcp_f32 v88, v88\n"
        "v_rcp_f32 v89, v89\n"
        "v_rcp_f32 v90, v90\n"
        "v_rcp_f32 v91, v91\n"
        "v_pk_mul_f32 v[84:85], v[88:89], v[80:81]\n"
        "v_pk_mul_f32 v[86:87], v[90:91], v[82:83]\n"
        "s_mov_b32 s44, 0xc40000\n"
        "buffer_store_dwordx4 v[84:87], %[vout1], %[ro], s44 offen\n"
        "v_accvgpr_read_b32 v96, a195\n"
        "v_accvgpr_read_b32 v97, a203\n"
        "v_accvgpr_read_b32 v98, a211\n"
        "v_accvgpr_read_b32 v99, a219\n"
        "v_pk_mul_f32 v[100:101], v[96:97], v[96:97]\n"
        "v_pk_mul_f32 v[102:103], v[98:99], v[98:99]\n"
        "v_pk_fma_f32 v[100:101], v[100:101], v[130:131], v[128:129]\n"
        "v_pk_fma_f32 v[102:103], v[102:103], v[130:131], v[128:129]\n"
        "v_pk_mul_f32 v[100:101], v[100:101], v[96:97]\n"
        "v_pk_mul_f32 v[102:103], v[102:103], v[98:99]\n"
        "v_exp_f32 v104, v100\n"
        "v_exp_f32 v105, v101\n"
        "v_exp_f32 v106, v102\n"
        "v_exp_f32 v107, v103\n"
        "v_pk_add_f32 v[104:105], v[104:105], v[132:133]\n"
        "v_pk_add_f32 v[106:107], v[106:107], v[132:133]\n"
        "v_rcp_f32 v104, v104\n"
        "v_rcp_f32 v105, v105\n"
        "v_rcp_f32 v106, v106\n"
        "v_rcp_f32 v107, v107\n"
        "v_pk_mul_f32 v[100:101], v[104:105], v[96:97]\n"
        "v_pk_mul_f32 v[102:103], v[106:107], v[98:99]\n"
        "s_mov_b32 s44, 0xc60000\n"
        "buffer_store_dwordx4 v[100:103], %[vout0], %[ro], s44 offen\n"
        "v_accvgpr_read_b32 v112, a199\n"
        "v_accvgpr_read_b32 v113, a207\n"
        "v_accvgpr_read_b32 v114, a215\n"
        "v_accvgpr_read_b32 v115, a223\n"
        "v_pk_mul_f32 v[116:117], v[112:113], v[112:113]\n"
        "v_pk_mul_f32 v[118:119], v[114:115], v[114:115]\n"
        "v_pk_fma_f32 v[116:117], v[116:117], v[130:131], v[128:129]\n"
        "v_pk_fma_f32 v[118:119], v[118:119], v[130:131], v[128:129]\n"
        "v_pk_mul_f32 v[116:117], v[116:117], v[112:113]\n"
        "v_pk_mul_f32 v[118:119], v[118:119], v[114:115]\n"
        "v_exp_f32 v120, v116\n"
        "v_exp_f32 v121, v117\n"
        "v_exp_f32 v122, v118\n"
        "v_exp_f32 v123, v119\n"
        "v_pk_add_f32 v[120:121], v[120:121], v[132:133]\n"
        "v_pk_add_f32 v[122:123], v[122:123], v[132:133]\n"
        "v_rcp_f32 v120, v120\n"
        "v_rcp_f32 v121, v121\n"
        "v_rcp_f32 v122, v122\n"
        "v_rcp_f32 v123, v123\n"
        "v_pk_mul_f32 v[116:117], v[120:121], v[112:113]\n"
        "v_pk_mul_f32 v[118:119], v[122:123], v[114:115]\n"
        "s_mov_b32 s44, 0xc60000\n"
        "buffer_store_dwordx4 v[116:119], %[vout1], %[ro], s44 offen\n"
        "v_accvgpr_read_b32 v0, a224\n"
        "v_accvgpr_read_b32 v1, a232\n"
        "v_accvgpr_read_b32 v2, a240\n"
        "v_accvgpr_read_b32 v3, a248\n"
        "v_pk_mul_f32 v[4:5], v[0:1], v[0:1]\n"
        "v_pk_mul_f32 v[6:7], v[2:3], v[2:3]\n"
        "v_pk_fma_f32 v[4:5], v[4:5], v[130:131], v[128:129]\n"
        "v_pk_fma_f32 v[6:7], v[6:7], v[130:131], v[128:129]\n"
        "v_pk_mul_f32 v[4:5], v[4:5], v[0:1]\n"
        "v_pk_mul_f32 v[6:7], v[6:7], v[2:3]\n"
        "v_exp_f32 v8, v4\n"
        "v_exp_f32 v9, v5\n"
        "v_exp_f32 v10, v6\n"
        "v_exp_f32 v11, v7\n"
        "v_pk_add_f32 v[8:9], v[8:9], v[132:133]\n"
        "v_pk_add_f32 v[10:11], v[10:11], v[132:133]\n"
        "v_rcp_f32 v8, v8\n"
        "v_rcp_f32 v9, v9\n"
        "v_rcp_f32 v10, v10\n"
        "v_rcp_f32 v11, v11\n"
        "v_pk_mul_f32 v[4:5], v[8:9], v[0:1]\n"
        "v_pk_mul_f32 v[6:7], v[10:11], v[2:3]\n"
        "s_mov_b32 s44, 0xe00000\n"
        "buffer_store_dwordx4 v[4:7], %[vout0], %[ro], s44 offen\n"
        "v_accvgpr_read_b32 v16, a228\n"
        "v_accvgpr_read_b32 v17, a236\n"
        "v_accvgpr_read_b32 v18, a244\n"
        "v_accvgpr_read_b32 v19, a252\n"
        "v_pk_mul_f32 v[20:21], v[16:17], v[16:17]\n"
        "v_pk_mul_f32 v[22:23], v[18:19], v[18:19]\n"
        "v_pk_fma_f32 v[20:21], v[20:21], v[130:131], v[128:129]\n"
        "v_pk_fma_f32 v[22:23], v[22:23], v[130:131], v[128:129]\n"
        "v_pk_mul_f32 v[20:21], v[20:21], v[16:17]\n"
        "v_pk_mul_f32 v[22:23], v[22:23], v[18:19]\n"
        "v_exp_f32 v24, v20\n"
        "v_exp_f32 v25, v21\n"
        "v_exp_f32 v26, v22\n"
        "v_exp_f32 v27, v23\n"
        "v_pk_add_f32 v[24:25], v[24:25], v[132:133]\n"
        "v_pk_add_f32 v[26:27], v[26:27], v[132:133]\n"
        "v_rcp_f32 v24, v24\n"
        "v_rcp_f32 v25, v25\n"
        "v_rcp_f32 v26, v26\n"
        "v_rcp_f32 v27, v27\n"
        "v_pk_mul_f32 v[20:21], v[24:25], v[16:17]\n"
        "v_pk_mul_f32 v[22:23], v[26:27], v[18:19]\n"
        "s_mov_b32 s44, 0xe00000\n"
        "buffer_store_dwordx4 v[20:23], %[vout1], %[ro], s44 offen\n"
        "v_accvgpr_read_b32 v32, a225\n"
        "v_accvgpr_read_b32 v33, a233\n"
        "v_accvgpr_read_b32 v34, a241\n"
        "v_accvgpr_read_b32 v35, a249\n"
        "v_pk_mul_f32 v[36:37], v[32:33], v[32:33]\n"
        "v_pk_mul_f32 v[38:39], v[34:35], v[34:35]\n"
        "v_pk_fma_f32 v[36:37], v[36:37], v[130:131], v[128:129]\n"
        "v_pk_fma_f32 v[38:39], v[38:39], v[130:131], v[128:129]\n"
        "v_pk_mul_f32 v[36:37], v[36:37], v[32:33]\n"
        "v_pk_mul_f32 v[38:39], v[38:39], v[34:35]\n"
        "v_exp_f32 v40, v36\n"
        "v_exp_f32 v41, v37\n"
        "v_exp_f32 v42, v38\n"
        "v_exp_f32 v43, v39\n"
        "v_pk_add_f32 v[40:41], v[40:41], v[132:133]\n"
        "v_pk_add_f32 v[42:43], v[42:43], v[132:133]\n"
        "v_rcp_f32 v40, v40\n"
        "v_rcp_f32 v41, v41\n"
        "v_rcp_f32 v42, v42\n"
        "v_rcp_f32 v43, v43\n"
        "v_pk_mul_f32 v[36:37], v[40:41], v[32:33]\n"
        "v_pk_mul_f32 v[38:39], v[42:43], v[34:35]\n"
        "s_mov_b32 s44, 0xe20000\n"
        "buffer_store_dwordx4 v[36:39], %[vout0], %[ro], s44 offen\n"
        "v_accvgpr_read_b32 v48, a229\n"
        "v_accvgpr_read_b32 v49, a237\n"
        "v_accvgpr_read_b32 v50, a245\n"
        "v_accvgpr_read_b32 v51, a253\n"
        "v_pk_mul_f32 v[52:53], v[48:49], v[48:49]\n"
        "v_pk_mul_f32 v[54:55], v[50:51], v[50:51]\n"
        "v_pk_fma_f32 v[52:53], v[52:53], v[130:131], v[128:129]\n"
        "v_pk_fma_f32 v[54:55], v[54:55], v[130:131], v[128:129]\n"
        "v_pk_mul_f32 v[52:53], v[52:53], v[48:49]\n"
        "v_pk_mul_f32 v[54:55], v[54:55], v[50:51]\n"
        "v_exp_f32 v56, v52\n"
        "v_exp_f32 v57, v53\n"
        "v_exp_f32 v58, v54\n"
        "v_exp_f32 v59, v55\n"
        "v_pk_add_f32 v[56:57], v[56:57], v[132:133]\n"
        "v_pk_add_f32 v[58:59], v[58:59], v[132:133]\n"
        "v_rcp_f32 v56, v56\n"
        "v_rcp_f32 v57, v57\n"
        "v_rcp_f32 v58, v58\n"
        "v_rcp_f32 v59, v59\n"
        "v_pk_mul_f32 v[52:53], v[56:57], v[48:49]\n"
        "v_pk_mul_f32 v[54:55], v[58:59], v[50:51]\n"
        "s_mov_b32 s44, 0xe20000\n"
        "buffer_store_dwordx4 v[52:55], %[vout1], %[ro], s44 offen\n"
        "v_accvgpr_read_b32 v64, a226\n"
        "v_accvgpr_read_b32 v65, a234\n"
        "v_accvgpr_read_b32 v66, a242\n"
        "v_accvgpr_read_b32 v67, a250\n"
        "v_pk_mul_f32 v[68:69], v[64:65], v[64:65]\n"
        "v_pk_mul_f32 v[70:71], v[66:67], v[66:67]\n"
        "v_pk_fma_f32 v[68:69], v[68:69], v[130:131], v[128:129]\n"
        "v_pk_fma_f32 v[70:71], v[70:71], v[130:131], v[128:129]\n"
        "v_pk_mul_f32 v[68:69], v[68:69], v[64:65]\n"
        "v_pk_mul_f32 v[70:71], v[70:71], v[66:67]\n"
        "v_exp_f32 v72, v68\n"
        "v_exp_f32 v73, v69\n"
        "v_exp_f32 v74, v70\n"
        "v_exp_f32 v75, v71\n"
        "v_pk_add_f32 v[72:73], v[72:73], v[132:133]\n"
        "v_pk_add_f32 v[74:75], v[74:75], v[132:133]\n"
        "v_rcp_f32 v72, v72\n"
        "v_rcp_f32 v73, v73\n"
        "v_rcp_f32 v74, v74\n"
        "v_rcp_f32 v75, v75\n"
        "v_pk_mul_f32 v[68:69], v[72:73], v[64:65]\n"
        "v_pk_mul_f32 v[70:71], v[74:75], v[66:67]\n"
        "s_mov_b32 s44, 0xe40000\n"
        "buffer_store_dwordx4 v[68:71], %[vout0], %[ro], s44 offen\n"
        "v_accvgpr_read_b32 v80, a230\n"
        "v_accvgpr_read_b32 v81, a238\n"
        "v_accvgpr_read_b32 v82, a246\n"
        "v_accvgpr_read_b32 v83, a254\n"
        "v_pk_mul_f32 v[84:85], v[80:81], v[80:81]\n"
        "v_pk_mul_f32 v[86:87], v[82:83], v[82:83]\n"
        "v_pk_fma_f32 v[84:85], v[84:85], v[130:131], v[128:129]\n"
        "v_pk_fma_f32 v[86:87], v[86:87], v[130:131], v[128:129]\n"
        "v_pk_mul_f32 v[84:85], v[84:85], v[80:81]\n"
        "v_pk_mul_f32 v[86:87], v[86:87], v[82:83]\n"
        "v_exp_f32 v88, v84\n"
        "v_exp_f32 v89, v85\n"
        "v_exp_f32 v90, v86\n"
        "v_exp_f32 v91, v87\n"
        "v_pk_add_f32 v[88:89], v[88:89], v[132:133]\n"
        "v_pk_add_f32 v[90:91], v[90:91], v[132:133]\n"
        "v_rcp_f32 v88, v88\n"
        "v_rcp_f32 v89, v89\n"
        "v_rcp_f32 v90, v90\n"
        "v_rcp_f32 v91, v91\n"
        "v_pk_mul_f32 v[84:85], v[88:89], v[80:81]\n"
        "v_pk_mul_f32 v[86:87], v[90:91], v[82:83]\n"
        "s_mov_b32 s44, 0xe40000\n"
        "buffer_store_dwordx4 v[84:87], %[vout1], %[ro], s44 offen\n"
        "v_accvgpr_read_b32 v96, a227\n"
        "v_accvgpr_read_b32 v97, a235\n"
        "v_accvgpr_read_b32 v98, a243\n"
        "v_accvgpr_read_b32 v99, a251\n"
        "v_pk_mul_f32 v[100:101], v[96:97], v[96:97]\n"
        "v_pk_mul_f32 v[102:103], v[98:99], v[98:99]\n"
        "v_pk_fma_f32 v[100:101], v[100:101], v[130:131], v[128:129]\n"
        "v_pk_fma_f32 v[102:103], v[102:103], v[130:131], v[128:129]\n"
        "v_pk_mul_f32 v[100:101], v[100:101], v[96:97]\n"
        "v_pk_mul_f32 v[102:103], v[102:103], v[98:99]\n"
        "v_exp_f32 v104, v100\n"
        "v_exp_f32 v105, v101\n"
        "v_exp_f32 v106, v102\n"
        "v_exp_f32 v107, v103\n"
        "v_pk_add_f32 v[104:105], v[104:105], v[132:133]\n"
        "v_pk_add_f32 v[106:107], v[106:107], v[132:133]\n"
        "v_rcp_f32 v104, v104\n"
        "v_rcp_f32 v105, v105\n"
        "v_rcp_f32 v106, v106\n"
        "v_rcp_f32 v107, v107\n"
        "v_pk_mul_f32 v[100:101], v[104:105], v[96:97]\n"
        "v_pk_mul_f32 v[102:103], v[106:107], v[98:99]\n"
        "s_mov_b32 s44, 0xe60000\n"
        "buffer_store_dwordx4 v[100:103], %[vout0], %[ro], s44 offen\n"
        "v_accvgpr_read_b32 v112, a231\n"
        "v_accvgpr_read_b32 v113, a239\n"
        "v_accvgpr_read_b32 v114, a247\n"
        "v_accvgpr_read_b32 v115, a255\n"
        "v_pk_mul_f32 v[116:117], v[112:113], v[112:113]\n"
        "v_pk_mul_f32 v[118:119], v[114:115], v[114:115]\n"
        "v_pk_fma_f32 v[116:117], v[116:117], v[130:131], v[128:129]\n"
        "v_pk_fma_f32 v[118:119], v[118:119], v[130:131], v[128:129]\n"
        "v_pk_mul_f32 v[116:117], v[116:117], v[112:113]\n"
        "v_pk_mul_f32 v[118:119], v[118:119], v[114:115]\n"
        "v_exp_f32 v120, v116\n"
        "v_exp_f32 v121, v117\n"
        "v_exp_f32 v122, v118\n"
        "v_exp_f32 v123, v119\n"
        "v_pk_add_f32 v[120:121], v[120:121], v[132:133]\n"
        "v_pk_add_f32 v[122:123], v[122:123], v[132:133]\n"
        "v_rcp_f32 v120, v120\n"
        "v_rcp_f32 v121, v121\n"
        "v_rcp_f32 v122, v122\n"
        "v_rcp_f32 v123, v123\n"
        "v_pk_mul_f32 v[116:117], v[120:121], v[112:113]\n"
        "v_pk_mul_f32 v[118:119], v[122:123], v[114:115]\n"
        "s_mov_b32 s44, 0xe60000\n"
        "buffer_store_dwordx4 v[116:119], %[vout1], %[ro], s44 offen\n"
        "s_branch L_done_%=\n"
        "L_bad_%=:\n"
        "v_mov_b32 v0, 0x7fc00000\n"
        "v_mov_b32 v1, 0x7fc00000\n"
        "v_mov_b32 v2, 0x7fc00000\n"
        "v_mov_b32 v3, 0x7fc00000\n"
        "s_mov_b32 s44, 0x0\n"
        "buffer_store_dwordx4 v[0:3], %[vout0], %[ro], s44 offen\n"
        "buffer_store_dwordx4 v[0:3], %[vout1], %[ro], s44 offen\n"
        "s_mov_b32 s44, 0x20000\n"
        "buffer_store_dwordx4 v[0:3], %[vout0], %[ro], s44 offen\n"
        "buffer_store_dwordx4 v[0:3], %[vout1], %[ro], s44 offen\n"
        "s_mov_b32 s44, 0x40000\n"
        "buffer_store_dwordx4 v[0:3], %[vout0], %[ro], s44 offen\n"
        "buffer_store_dwordx4 v[0:3], %[vout1], %[ro], s44 offen\n"
        "s_mov_b32 s44, 0x60000\n"
        "buffer_store_dwordx4 v[0:3], %[vout0], %[ro], s44 offen\n"
        "buffer_store_dwordx4 v[0:3], %[vout1], %[ro], s44 offen\n"
        "s_mov_b32 s44, 0x200000\n"
        "buffer_store_dwordx4 v[0:3], %[vout0], %[ro], s44 offen\n"
        "buffer_store_dwordx4 v[0:3], %[vout1], %[ro], s44 offen\n"
        "s_mov_b32 s44, 0x220000\n"
        "buffer_store_dwordx4 v[0:3], %[vout0], %[ro], s44 offen\n"
        "buffer_store_dwordx4 v[0:3], %[vout1], %[ro], s44 offen\n"
        "s_mov_b32 s44, 0x240000\n"
        "buffer_store_dwordx4 v[0:3], %[vout0], %[ro], s44 offen\n"
        "buffer_store_dwordx4 v[0:3], %[vout1], %[ro], s44 offen\n"
        "s_mov_b32 s44, 0x260000\n"
        "buffer_store_dwordx4 v[0:3], %[vout0], %[ro], s44 offen\n"
        "buffer_store_dwordx4 v[0:3], %[vout1], %[ro], s44 offen\n"
        "s_mov_b32 s44, 0x400000\n"
        "buffer_store_dwordx4 v[0:3], %[vout0], %[ro], s44 offen\n"
        "buffer_store_dwordx4 v[0:3], %[vout1], %[ro], s44 offen\n"
        "s_mov_b32 s44, 0x420000\n"
        "buffer_store_dwordx4 v[0:3], %[vout0], %[ro], s44 offen\n"
        "buffer_store_dwordx4 v[0:3], %[vout1], %[ro], s44 offen\n"
        "s_mov_b32 s44, 0x440000\n"
        "buffer_store_dwordx4 v[0:3], %[vout0], %[ro], s44 offen\n"
        "buffer_store_dwordx4 v[0:3], %[vout1], %[ro], s44 offen\n"
        "s_mov_b32 s44, 0x460000\n"
        "buffer_store_dwordx4 v[0:3], %[vout0], %[ro], s44 offen\n"
        "buffer_store_dwordx4 v[0:3], %[vout1], %[ro], s44 offen\n"
        "s_mov_b32 s44, 0x600000\n"
        "buffer_store_dwordx4 v[0:3], %[vout0], %[ro], s44 offen\n"
        "buffer_store_dwordx4 v[0:3], %[vout1], %[ro], s44 offen\n"
        "s_mov_b32 s44, 0x620000\n"
        "buffer_store_dwordx4 v[0:3], %[vout0], %[ro], s44 offen\n"
        "buffer_store_dwordx4 v[0:3], %[vout1], %[ro], s44 offen\n"
        "s_mov_b32 s44, 0x640000\n"
        "buffer_store_dwordx4 v[0:3], %[vout0], %[ro], s44 offen\n"
        "buffer_store_dwordx4 v[0:3], %[vout1], %[ro], s44 offen\n"
        "s_mov_b32 s44, 0x660000\n"
        "buffer_store_dwordx4 v[0:3], %[vout0], %[ro], s44 offen\n"
        "buffer_store_dwordx4 v[0:3], %[vout1], %[ro], s44 offen\n"
        "s_mov_b32 s44, 0x800000\n"
        "buffer_store_dwordx4 v[0:3], %[vout0], %[ro], s44 offen\n"
        "buffer_store_dwordx4 v[0:3], %[vout1], %[ro], s44 offen\n"
        "s_mov_b32 s44, 0x820000\n"
        "buffer_store_dwordx4 v[0:3], %[vout0], %[ro], s44 offen\n"
        "buffer_store_dwordx4 v[0:3], %[vout1], %[ro], s44 offen\n"
        "s_mov_b32 s44, 0x840000\n"
        "buffer_store_dwordx4 v[0:3], %[vout0], %[ro], s44 offen\n"
        "buffer_store_dwordx4 v[0:3], %[vout1], %[ro], s44 offen\n"
        "s_mov_b32 s44, 0x860000\n"
        "buffer_store_dwordx4 v[0:3], %[vout0], %[ro], s44 offen\n"
        "buffer_store_dwordx4 v[0:3], %[vout1], %[ro], s44 offen\n"
        "s_mov_b32 s44, 0xa00000\n"
        "buffer_store_dwordx4 v[0:3], %[vout0], %[ro], s44 offen\n"
        "buffer_store_dwordx4 v[0:3], %[vout1], %[ro], s44 offen\n"
        "s_mov_b32 s44, 0xa20000\n"
        "buffer_store_dwordx4 v[0:3], %[vout0], %[ro], s44 offen\n"
        "buffer_store_dwordx4 v[0:3], %[vout1], %[ro], s44 offen\n"
        "s_mov_b32 s44, 0xa40000\n"
        "buffer_store_dwordx4 v[0:3], %[vout0], %[ro], s44 offen\n"
        "buffer_store_dwordx4 v[0:3], %[vout1], %[ro], s44 offen\n"
        "s_mov_b32 s44, 0xa60000\n"
        "buffer_store_dwordx4 v[0:3], %[vout0], %[ro], s44 offen\n"
        "buffer_store_dwordx4 v[0:3], %[vout1], %[ro], s44 offen\n"
        "s_mov_b32 s44, 0xc00000\n"
        "buffer_store_dwordx4 v[0:3], %[vout0], %[ro], s44 offen\n"
        "buffer_store_dwordx4 v[0:3], %[vout1], %[ro], s44 offen\n"
        "s_mov_b32 s44, 0xc20000\n"
        "buffer_store_dwordx4 v[0:3], %[vout0], %[ro], s44 offen\n"
        "buffer_store_dwordx4 v[0:3], %[vout1], %[ro], s44 offen\n"
        "s_mov_b32 s44, 0xc40000\n"
        "buffer_store_dwordx4 v[0:3], %[vout0], %[ro], s44 offen\n"
        "buffer_store_dwordx4 v[0:3], %[vout1], %[ro], s44 offen\n"
        "s_mov_b32 s44, 0xc60000\n"
        "buffer_store_dwordx4 v[0:3], %[vout0], %[ro], s44 offen\n"
        "buffer_store_dwordx4 v[0:3], %[vout1], %[ro], s44 offen\n"
        "s_mov_b32 s44, 0xe00000\n"
        "buffer_store_dwordx4 v[0:3], %[vout0], %[ro], s44 offen\n"
        "buffer_store_dwordx4 v[0:3], %[vout1], %[ro], s44 offen\n"
        "s_mov_b32 s44, 0xe20000\n"
        "buffer_store_dwordx4 v[0:3], %[vout0], %[ro], s44 offen\n"
        "buffer_store_dwordx4 v[0:3], %[vout1], %[ro], s44 offen\n"
        "s_mov_b32 s44, 0xe40000\n"
        "buffer_store_dwordx4 v[0:3], %[vout0], %[ro], s44 offen\n"
        "buffer_store_dwordx4 v[0:3], %[vout1], %[ro], s44 offen\n"
        "s_mov_b32 s44, 0xe60000\n"
        "buffer_store_dwordx4 v[0:3], %[vout0], %[ro], s44 offen\n"
        "buffer_store_dwordx4 v[0:3], %[vout1], %[ro], s44 offen\n"
        "L_done_%=:\n"
        : [vpd] "+v"(vpd), [vtk] "+v"(vtk)
        : [rw] "s"(rw), [rc] "s"(rc), [rs] "s"(rs), [ro] "s"(ro), [rx0] "s"(rx0), [srank] "s"(rank), [swv] "s"(wv),
          [sd0] "s"(sd[0]), [sd1] "s"(sd[1]), [sd2] "s"(sd[2]), [sd3] "s"(sd[3]), [sd4] "s"(sd[4]), [sd5] "s"(sd[5]), [sd6] "s"(sd[6]),
          [pd0] "s"(pd[0]), [pd1] "s"(pd[1]), [pd2] "s"(pd[2]), [pd3] "s"(pd[3]), [pd4] "s"(pd[4]), [pd5] "s"(pd[5]), [pd6] "s"(pd[6]),
          [vwoff] "v"(vwoff), [vxoff] "v"(vxoff), [vlw] "v"(vlw), [vlr0] "v"(vlr0), [vlr1] "v"(vlr1),
          [vps] "v"(vps), [vpf] "v"(vpf), [vst] "v"(vst), [vfl] "v"(vfl), [vout0] "v"(vout0), [vout1] "v"(vout1)
        : "v0", "v1", "v2", "v3", "v4", "v5", "v6", "v7", "v8", "v9", "v10", "v11", "v12", "v13", "v14", "v15", "v16", "v17", "v18", "v19", "v20", "v21", "v22", "v23", "v24", "v25", "v26", "v27", "v28", "v29", "v30", "v31", "v32", "v33", "v34", "v35", "v36", "v37", "v38", "v39", "v40", "v41", "v42", "v43", "v44", "v45", "v46", "v47", "v48", "v49", "v50", "v51", "v52", "v53", "v54", "v55", "v56", "v57", "v58", "v59", "v60", "v61", "v62", "v63", "v64", "v65", "v66", "v67", "v68", "v69", "v70", "v71", "v72", "v73", "v74", "v75", "v76", "v77", "v78", "v79", "v80", "v81", "v82", "v83", "v84", "v85", "v86", "v87", "v88", "v89", "v90", "v91", "v92", "v93", "v94", "v95", "v96", "v97", "v98", "v99", "v100", "v101", "v102", "v103", "v104", "v105", "v106", "v107", "v108", "v109", "v110", "v111", "v112", "v113", "v114", "v115", "v116", "v117", "v118", "v119", "v120", "v121", "v122", "v123", "v124", "v125", "v126", "v127", "v128", "v129", "v130", "v131", "v132", "v133", "v134", "v135", "v136", "v137", "v138", "v139", "v140", "v141", "v142", "v143", "v144", "v145", "v146", "v147", "v148", "v149", "v150", "v151", "v152", "v153", "v154", "v155", "v156", "v157", "v158", "v159", "v160", "v161", "v162", "v163", "v164", "v165", "v166", "v167", "v168", "v169", "v170", "v171", "v172", "v173", "v174", "v175", "v176", "v177", "v178", "v179", "v180", "v181", "v182", "v183", "v184", "v185", "v186", "v187", "v188", "v189", "v190", "v191", "v192", "v193", "v194", "v195", "v196", "v197", "v198", "v199", "v200", "v201", "v202", "v203", "v204", "v205", "v206", "v207", "v208", "v209", "v210", "v211", "v212", "v213", "v214", "v215", "v216", "v217", "v218", "v219", "v220", "v221", "v222", "v223", "v224", "v225", "v226", "v227", "v228", "v229", "v230", "v231", "v232", "v233", "v234", "v235", "v236", "v237", "v238", "v239", "v240", "v241", "v243", "a0", "a1", "a2", "a3", "a4", "a5", "a6", "a7", "a8", "a9", "a10", "a11", "a12", "a13", "a14", "a15", "a16", "a17", "a18", "a19", "a20", "a21", "a22", "a23", "a24", "a25", "a26", "a27", "a28", "a29", "a30", "a31", "a32", "a33", "a34", "a35", "a36", "a37", "a38", "a39", "a40", "a41", "a42", "a43", "a44", "a45", "a46", "a47", "a48", "a49", "a50", "a51", "a52", "a53", "a54", "a55", "a56", "a57", "a58", "a59", "a60", "a61", "a62", "a63", "a64", "a65", "a66", "a67", "a68", "a69", "a70", "a71", "a72", "a73", "a74", "a75", "a76", "a77", "a78", "a79", "a80", "a81", "a82", "a83", "a84", "a85", "a86", "a87", "a88", "a89", "a90", "a91", "a92", "a93", "a94", "a95", "a96", "a97", "a98", "a99", "a100", "a101", "a102", "a103", "a104", "a105", "a106", "a107", "a108", "a109", "a110", "a111", "a112", "a113", "a114", "a115", "a116", "a117", "a118", "a119", "a120", "a121", "a122", "a123", "a124", "a125", "a126", "a127", "a128", "a129", "a130", "a131", "a132", "a133", "a134", "a135", "a136", "a137", "a138", "a139", "a140", "a141", "a142", "a143", "a144", "a145", "a146", "a147", "a148", "a149", "a150", "a151", "a152", "a153", "a154", "a155", "a156", "a157", "a158", "a159", "a160", "a161", "a162", "a163", "a164", "a165", "a166", "a167", "a168", "a169", "a170", "a171", "a172", "a173", "a174", "a175", "a176", "a177", "a178", "a179", "a180", "a181", "a182", "a183", "a184", "a185", "a186", "a187", "a188", "a189", "a190", "a191", "a192", "a193", "a194", "a195", "a196", "a197", "a198", "a199", "a200", "a201", "a202", "a203", "a204", "a205", "a206", "a207", "a208", "a209", "a210", "a211", "a212", "a213", "a214", "a215", "a216", "a217", "a218", "a219", "a220", "a221", "a222", "a223", "a224", "a225", "a226", "a227", "a228", "a229", "a230", "a231", "a232", "a233", "a234", "a235", "a236", "a237", "a238", "a239", "a240", "a241", "a242", "a243", "a244", "a245", "a246", "a247", "a248", "a249", "a250", "a251", "a252", "a253", "a254", "a255", "s40", "s41", "s42", "s43", "s44", "s45", "s46", "s47", "s48", "s49", "s50", "s51", "s52", "s53", "s54", "s55", "s56", "s57", "s58", "s59", "s88", "s89", "vcc", "scc", "memory");
}

extern "C" void kernel_launch(int rank, int nranks, ncclComm_t comm,
                              hipStream_t stream,
                              void* const* d_in, const int* in_sizes, int n_in,
                              void* d_out, int out_size,
                              const int* in_dtypes, int out_dtype,
                              void* const* d_out_all,
                              void* const* d_ws_all, size_t ws_bytes) {
    if (nranks != NR || n_in != 2 || ws_bytes < WS_NEED || (unsigned)rank >= (unsigned)NR ||
        in_sizes[0] != NR * MB * KB || in_sizes[1] != KT * NT || out_size != MB * NT) {
        fprintf(stderr, "unexpected problem configuration\n");
        return;
    }
    Params p;
    p.x = (const float*)d_in[0];
    p.w = (const float*)d_in[1];
    p.out = (float*)d_out;
    for (int i = 0; i < NR; ++i) p.ws[i] = (char*)d_ws_all[i];
    p.rank = rank;
    hipLaunchKernelGGL(fused_a2a_gemm, dim3(NWG), dim3(NTHR), 49152, stream, p);
    HIP_CHECK(hipGetLastError());
}
```
